```python
import numpy as np

kernel_name = "gpu_gemm_rs_m4096_k4096_n16384_e5m2e5m2_dqsilu"


ABSMAX_RTOL = 0.05

COMPUTE_DTYPE = "float32"
EPILOGUE = "dequant_silu"
IN_QUANT = None
OUT_QUANT = None
IN_FP8 = "e5m2e5m2"
IN_DTYPES = {"x": "e5m2", "w_mat": "e5m2",
             "scale_x": "float32", "scale_w": "float32"}
WORLD_SIZE = 8

def _snap_fp8(x, mant_bits, max_abs, min_normal):
    a = np.abs(x.astype(np.float64))
    _m, _e = np.frexp(a)
    q = np.where(a >= min_normal, np.ldexp(1.0, _e - 1 - mant_bits),
                 min_normal * 2.0 ** -mant_bits)
    snapped = np.minimum(np.round(a / q) * q, max_abs)
    return (np.sign(x) * snapped).astype(np.float32)

def setup_inputs():

    x = _snap_fp8(np.random.randn(4096, 4096).astype(np.float32), 2, 57344.0, 2.0 ** -14)
    w_mat = _snap_fp8(np.random.randn(4096, 16384).astype(np.float32), 2, 57344.0, 2.0 ** -14)
    scale_x = (0.02 * (1.0 + np.random.rand(1))).astype(np.float32)
    scale_w = (0.02 * (1.0 + np.random.rand(1))).astype(np.float32)
    return {"x": x, "w_mat": w_mat, "scale_x": scale_x, "scale_w": scale_w}

def reference(x, w_mat, scale_x, scale_w):
    acc = (x @ w_mat).astype(np.float32)
    y = (acc * (scale_x * scale_w)).astype(np.float32)
    return (y / (1.0 + np.exp(-np.clip(y, -60.0, 60.0)))).astype(np.float32)
```

```cpp
#include <hip/hip_runtime.h>
#include <rccl/rccl.h>
#include <cstdio>
#include <cstdint>

#define HIP_CHECK(expr) do { hipError_t _e = (expr); if (_e != hipSuccess) { \
    fprintf(stderr, "HIP error %s at %s:%d\n", hipGetErrorString(_e), __FILE__, __LINE__); \
    return; } } while (0)

typedef unsigned u32;
typedef unsigned long long u64;
typedef unsigned u32x4 __attribute__((ext_vector_type(4)));
typedef int i32x8 __attribute__((ext_vector_type(8)));
typedef float f32x4 __attribute__((ext_vector_type(4)));
typedef __attribute__((address_space(1))) const void gvoid;
typedef __attribute__((address_space(3))) void lvoid;
typedef __attribute__((address_space(3))) volatile u32 lu32;

namespace {

constexpr int NR = 8;
constexpr int MM = 4096, KK = 4096, NN = 16384;
constexpr int KS = KK / NR;
constexpr int MS = MM / NR;
constexpr int F = 220;
constexpr int G = MS - F;
constexpr int XREM = 3 * F + 4 * G;
constexpr int XROWS = XREM + MS;
constexpr int XCH_REM = (XREM + 127) / 128;
constexpr int XCH_OWN = MS / 128;
constexpr int NSTEP = 32;

constexpr int GRID = 256, NSEND = 64, NTHR = 512;
#define SEND_WAVES 2

constexpr size_t WS_CNT = 0;
constexpr size_t WS_TMO = 32768;
constexpr size_t WS_WFLAG = 65536;
constexpr size_t WS_XREM = WS_WFLAG + 4096;
constexpr size_t WS_XOWN = WS_XREM + 128;
constexpr size_t WS_DONE = 81920;
constexpr size_t WS_DBG = 524288;
constexpr size_t WS_DATA = 1u << 20;
constexpr size_t W_BYTES = (size_t)NSTEP * 32 * 16384;
constexpr size_t XSLAB = (size_t)XROWS * 128;
constexpr size_t BUF_BYTES = 32u << 20;
static_assert(W_BYTES + NSTEP * XSLAB <= BUF_BYTES, "staging");
constexpr size_t WS_NEED = WS_DATA + 2 * BUF_BYTES;

constexpr int STAGE = 49152;
constexpr int LDS_CTL = 3 * STAGE;
constexpr u64 DEADLINE = 40000000ull;

constexpr int UX = XCH_REM * NR;
constexpr int UW = 128;
constexpr int UR = UX + UW;
constexpr int NUNITS = 4 * UR + 4 * XCH_OWN * NR;

struct Params {
    const uint8_t* x;
    const uint8_t* w;
    const float* sx;
    const float* sw;
    float* out[NR];
    uint8_t* ws[NR];
    int rank;
    int pad;
};

__device__ __forceinline__ void srow_map(int r, int srow, int& q, int& row) {
    const int s = (srow >= F) + (srow >= 2 * F);
    const int q1 = 2 * (((r >> 1) + 1 + s) & 3) + (r & 1);
    const int r1 = srow - s * F;
    const int v = srow - 3 * F;
    const int j = (v >= G) + (v >= 2 * G) + (v >= 3 * G);
    const int q2 = 2 * j + (1 - (r & 1));
    const int r2 = F + v - j * G;
    const bool c1 = srow < 3 * F, c2 = srow < XREM;
    q = c1 ? q1 : (c2 ? q2 : r);
    row = c1 ? r1 : (c2 ? r2 : srow - XREM);
}

__device__ __forceinline__ void st_sys(void* p, u32x4 v) {
    asm volatile("global_store_dwordx4 %0, %1, off sc0 sc1\n\ts_nop 1" :: "v"(p), "v"(v) : "memory");
}
#define WAIT_VM(n) asm volatile("s_waitcnt vmcnt(" #n ")" ::: "memory")

#define STAMP(k) do { } while (0)

__global__ void __launch_bounds__(NTHR, 2) fused_kernel(Params P) {
    __shared__ __attribute__((aligned(1024))) uint8_t lds[LDS_CTL + 64];
    const int tid = threadIdx.x;
    const int lane = tid & 63;
    const int wave = __builtin_amdgcn_readfirstlane(tid >> 6);
    const int b = blockIdx.x;
    const int rank = P.rank;
    uint8_t* const myws = P.ws[rank];
    u32* const tmo = (u32*)(myws + WS_TMO);

    STAMP(0);
    if (tid == 0) {
        u32 t = __hip_atomic_fetch_add((u32*)(myws + WS_CNT + (size_t)b * 128), 1u, __ATOMIC_RELAXED,
                                       __HIP_MEMORY_SCOPE_AGENT);
        *(lu32*)(lds + LDS_CTL) = t + 1;
    }
    __syncthreads();
    const u32 call = __builtin_amdgcn_readfirstlane(*(lu32*)(lds + LDS_CTL));
    const size_t bufoff = WS_DATA + (size_t)(call & 1) * BUF_BYTES;

    if (b < NSEND) {
        if (wave < SEND_WAVES) {
            for (int u = b + NSEND * wave; u < NUNITS; u += NSEND * SEND_WAVES) {
                int kc, v;
                bool isw = false, own = false;
                if (u < 4 * UR) {
                    kc = u / UR;
                    v = u - kc * UR;
                    if (v >= UX) { isw = true; v -= UX; }
                } else {
                    v = u - 4 * UR;
                    kc = v >> 5;
                    v &= 31;
                    own = true;
                }
                const int step = rank * 4 + kc;
                u32x4 in[16];
                if (isw) {
                    const int i8 = v >> 4;
                    const int blk = (v + 2 * rank) & 15;
                    const uint8_t* src = P.w + (size_t)(kc * 128 + 16 * (lane >> 3)) * NN + blk * 1024 + i8 * 128 + 16 * (lane & 7);
#pragma unroll
                    for (int j = 0; j < 16; ++j) in[j] = *(const u32x4*)(src + (size_t)j * NN);
                    u32x4 o[16];
#pragma unroll
                    for (int d = 0; d < 4; ++d)
#pragma unroll
                        for (int e = 0; e < 4; ++e) {
                            u32 t0 = __builtin_amdgcn_perm(in[4 * e + 1][d], in[4 * e][d], 0x05010400u);
                            u32 t1 = __builtin_amdgcn_perm(in[4 * e + 1][d], in[4 * e][d], 0x07030602u);
                            u32 t2 = __builtin_amdgcn_perm(in[4 * e + 3][d], in[4 * e + 2][d], 0x05010400u);
                            u32 t3 = __builtin_amdgcn_perm(in[4 * e + 3][d], in[4 * e + 2][d], 0x07030602u);
                            o[4 * d + 0][e] = __builtin_amdgcn_perm(t2, t0, 0x05040100u);
                            o[4 * d + 1][e] = __builtin_amdgcn_perm(t2, t0, 0x07060302u);
                            o[4 * d + 2][e] = __builtin_amdgcn_perm(t3, t1, 0x05040100u);
                            o[4 * d + 3][e] = __builtin_amdgcn_perm(t3, t1, 0x07060302u);
                        }
                    const int a = blk >> 2, bb = blk & 3;
                    const int re = 2 * a, ro = 2 * bb + 1;
                    const int cte = bb * 8 + i8, cto = a * 8 + i8;
                    uint8_t* we = P.ws[re & 7];
                    uint8_t* wo = P.ws[ro & 7];
                    uint8_t* de = we + bufoff + ((size_t)step * 32 + cte) * 16384;
                    uint8_t* dd = wo + bufoff + ((size_t)step * 32 + cto) * 16384;
#pragma unroll
                    for (int i = 0; i < 16; ++i) {
                        const int off = i * 1024 + ((lane ^ ((i >> 2) << 2)) << 4);
                        st_sys(de + off, o[i]);
                        st_sys(dd + off, o[i]);
                    }
                    WAIT_VM(0);
                    if (lane == 0) {
                        __hip_atomic_store((u32*)(we + WS_WFLAG) + cte * 32 + step, call, __ATOMIC_RELAXED, __HIP_MEMORY_SCOPE_SYSTEM);
                        __hip_atomic_store((u32*)(wo + WS_WFLAG) + cto * 32 + step, call, __ATOMIC_RELAXED, __HIP_MEMORY_SCOPE_SYSTEM);
                    }
                } else {
                    const int r = (rank + 1 + v) & 7;
                    const int s0 = (own ? XREM : 0) + (v >> 3) * 128 + (lane >> 3);
                    const int lim = own ? XROWS : XREM;
#pragma unroll
                    for (int j = 0; j < 16; ++j) {
                        int srow = s0 + 8 * j, q, row;
                        srow = srow < lim ? srow : lim - 1;
                        srow_map(r, srow, q, row);
                        in[j] = *(const u32x4*)(P.x + (size_t)(q * MS + row) * KS + kc * 128 + 16 * (lane & 7));
                    }
#pragma unroll
                    for (int j = 0; j < 16; ++j) asm volatile("" : "+v"(in[j]));
                    uint8_t* wr = P.ws[r];
                    uint8_t* dx = wr + bufoff + W_BYTES + (size_t)step * XSLAB + (size_t)s0 * 128 + 16 * (lane & 7);
#pragma unroll
                    for (int j = 0; j < 16; ++j)
                        if (s0 + 8 * j < lim) st_sys(dx + j * 1024, in[j]);
                    WAIT_VM(0);
                    if (lane == 0)
                        __hip_atomic_fetch_add((u32*)(wr + (own ? WS_XOWN : WS_XREM)) + step, 1u, __ATOMIC_RELAXED, __HIP_MEMORY_SCOPE_SYSTEM);
                }
            }
        }
        STAMP(1);
        __syncthreads();
    }

    const bool local = b < NSEND;
    const int bb2 = local ? b : b - NSEND;
    const int xcd = bb2 & 7, idx = bb2 >> 3;
    const int part = xcd >> 1;
    const int i8 = 2 * (idx & 3) + (xcd & 1);
    const int ct = part * 8 + i8;
    const int rt = idx >> 2;
    const int blk = (rank & 1) ? 4 * part + (rank >> 1) : 4 * (rank >> 1) + part;
    const int col0 = blk * 1024 + i8 * 128;
    const int skip_at = local ? 0 : 3 * F + part * G;
    const int skip_by = local ? XREM : G;

    const uint8_t* const wbuf = myws + bufoff + (size_t)ct * 16384;
    const uint8_t* const xbuf = myws + bufoff + W_BYTES;

    u32 aoff[4];
#pragma unroll
    for (int jj = 0; jj < 4; ++jj) {
        int i = rt * 256 + 32 * wave + 8 * jj + (lane >> 3);
        int srow = i < skip_at ? i : i + skip_by;
        aoff[jj] = (u32)srow * 128u + (u32)(((lane & 7) ^ ((4 * jj + (lane >> 4)) & 7)) << 4);
    }
    const u32 boff = (u32)wave * 2048u + (u32)lane * 16u;

    const u32* const pollp = lane < 32 ? (const u32*)(myws + WS_WFLAG) + ct * 32 + lane
                                       : (const u32*)(myws + (local ? WS_XOWN : WS_XREM)) + (lane - 32);
    const u32 pollneed = lane < 32 ? call : call * (u32)(local ? XCH_OWN : XCH_REM);
    u32 ready = 0;
    auto stepno = [&](int s) { return ((rank + s) & 7) * 4 + (s >> 3); };
    auto ensure = [&](int s) {
        const u32 bit = 1u << stepno(s);
        if (ready & bit) return;
        const u64 t0 = __builtin_amdgcn_s_memrealtime();
        for (;;) {
            const u32 v = __hip_atomic_load(pollp, __ATOMIC_RELAXED, __HIP_MEMORY_SCOPE_SYSTEM);
            const u64 m = __ballot(v >= pollneed);
            ready = (u32)m & (u32)(m >> 32);
            if (ready & bit) break;
            if (__hip_atomic_load(tmo, __ATOMIC_RELAXED, __HIP_MEMORY_SCOPE_AGENT) == call ||
                __builtin_amdgcn_s_memrealtime() - t0 > DEADLINE) {
                if (lane == 0) __hip_atomic_store(tmo, call, __ATOMIC_RELAXED, __HIP_MEMORY_SCOPE_AGENT);
                ready = ~0u;
                break;
            }
            __builtin_amdgcn_s_sleep(8);
        }
    };
    auto issue = [&](int s, int stage) {
        const int st = stepno(s);
        const uint8_t* xa = xbuf + (size_t)st * XSLAB;
        const uint8_t* wa = wbuf + (size_t)st * (32 * 16384);
        uint8_t* la = lds + stage * STAGE + wave * 4096;
#pragma unroll
        for (int jj = 0; jj < 4; ++jj)
            __builtin_amdgcn_global_load_lds((gvoid*)(xa + aoff[jj]), (lvoid*)(la + jj * 1024), 16, 0, 17);
        uint8_t* lb = lds + stage * STAGE + 32768 + wave * 2048;
        __builtin_amdgcn_global_load_lds((gvoid*)(wa + boff), (lvoid*)lb, 16, 0, 17);
        __builtin_amdgcn_global_load_lds((gvoid*)(wa + boff), (lvoid*)lb, 16, 1024, 17);
    };

    const int wm = wave >> 1, wn = wave & 1;
    const int c = lane & 15, q4 = lane >> 4;
    const int ra0 = wm * 8192 + c * 128 + ((q4 ^ (c >> 1)) << 4);
    const int ra1 = ra0 ^ 64;
    const int rb0 = 32768 + (c & 3) * 4096 + ((((q4 << 3) | (4 * wn + (c >> 2))) ^ ((c & 3) << 2)) << 4);

    f32x4 acc[4][4];
#pragma unroll
    for (int m = 0; m < 4; ++m)
#pragma unroll
        for (int t = 0; t < 4; ++t) acc[m][t] = f32x4{0.f, 0.f, 0.f, 0.f};

    if (wave == 0) { ensure(0); ensure(1); }
    __builtin_amdgcn_s_barrier();
    asm volatile("" ::: "memory");
    STAMP(2);
    issue(0, 0);
    issue(1, 1);
    int stage = 0;
    for (int s = 0; s < NSTEP; ++s) {
        if (wave == 0 && s + 2 < NSTEP) ensure(s + 2);
        if (s + 1 < NSTEP) WAIT_VM(6); else WAIT_VM(0);
        __builtin_amdgcn_s_barrier();
        asm volatile("" ::: "memory");
        if (s + 2 < NSTEP) issue(s + 2, stage == 0 ? 2 : stage - 1);
        const uint8_t* sb = lds + stage * STAGE;
        i32x8 A[4], B[4];
#pragma unroll
        for (int t = 0; t < 4; ++t) {
            u32x4 lo = *(const u32x4*)(sb + rb0 + t * 1024);
            u32x4 hi = *(const u32x4*)(sb + rb0 + t * 1024 + 512);
            B[t] = i32x8{(int)lo[0], (int)lo[1], (int)lo[2], (int)lo[3], (int)hi[0], (int)hi[1], (int)hi[2], (int)hi[3]};
        }
#pragma unroll
        for (int m = 0; m < 4; ++m) {
            u32x4 lo = *(const u32x4*)(sb + ra0 + m * 2048);
            u32x4 hi = *(const u32x4*)(sb + ra1 + m * 2048);
            A[m] = i32x8{(int)lo[0], (int)lo[1], (int)lo[2], (int)lo[3], (int)hi[0], (int)hi[1], (int)hi[2], (int)hi[3]};
        }
#pragma unroll
        for (int m = 0; m < 4; ++m)
#pragma unroll
            for (int t = 0; t < 4; ++t)
                acc[m][t] = __builtin_amdgcn_mfma_scale_f32_16x16x128_f8f6f4(A[m], B[t], acc[m][t], 1, 1, 0, 0, 0, 0);
        stage = stage == 2 ? 0 : stage + 1;
    }
    STAMP(3);

    const bool bad = __hip_atomic_load(tmo, __ATOMIC_RELAXED, __HIP_MEMORY_SCOPE_AGENT) == call;
    float scale = bad ? __builtin_nanf("") : P.sx[0] * P.sw[0];
    float* dst[4];
#pragma unroll
    for (int m = 0; m < 4; ++m) {
        const int i = rt * 256 + wm * 64 + m * 16 + 4 * q4;
        const int srow = i < skip_at ? i : i + skip_by;
        int q, row;
        srow_map(rank, srow, q, row);
        dst[m] = P.out[q & 7] + (size_t)row * NN + col0 + wn * 64 + 4 * c;
    }
    asm volatile("" : "+v"(dst[0]), "+v"(dst[1]), "+v"(dst[2]), "+v"(dst[3]), "+v"(scale));
#pragma unroll
    for (int m = 0; m < 4; ++m) {
#pragma unroll
        for (int r = 0; r < 4; ++r) {
            f32x4 y;
#pragma unroll
            for (int t = 0; t < 4; ++t) {
                const float v = acc[m][t][r] * scale;
                const float z = __builtin_amdgcn_fmed3f(v, -60.f, 60.f);
                const float e = __builtin_amdgcn_exp2f(z * -1.4426950408889634f);
                y[t] = v * __builtin_amdgcn_rcpf(1.0f + e);
            }
            st_sys(dst[m] + (size_t)r * NN, __builtin_bit_cast(u32x4, y));
        }
    }
    STAMP(4);
    WAIT_VM(0);
    STAMP(5);
    __syncthreads();
    if (!local) {
        if (tid < NR && tid != rank) {
            const int q = tid;
            int lo, n;
            if (((q ^ rank) & 1) == 0) {
                lo = ((((q >> 1) - (rank >> 1) - 1) & 3)) * F;
                n = F;
            } else {
                const int j = q >> 1;
                lo = 3 * F + (j > part ? j - 1 : j) * G;
                n = j == part ? 0 : G;
            }
            const int a0 = max(lo, rt * 256), a1 = min(lo + n, rt * 256 + 256);
            if (a1 > a0)
                __hip_atomic_fetch_add((u32*)(P.ws[q] + WS_DONE + rank * 128), (u32)(a1 - a0), __ATOMIC_RELAXED, __HIP_MEMORY_SCOPE_SYSTEM);
        }
    } else if (b == 0 && wave == 0) {
        const int q = lane & 7;
        const u32 need = q == rank ? 0u : call * (u32)(((q ^ rank) & 1) == 0 ? 32 * F : 24 * G);
        const u32* dp = (const u32*)(myws + WS_DONE + q * 128);
        const u64 t0 = __builtin_amdgcn_s_memrealtime();
        for (;;) {
            const u32 v = __hip_atomic_load(dp, __ATOMIC_RELAXED, __HIP_MEMORY_SCOPE_SYSTEM);
            if (__all(v >= need)) break;
            if (__hip_atomic_load(tmo, __ATOMIC_RELAXED, __HIP_MEMORY_SCOPE_AGENT) == call ||
                __builtin_amdgcn_s_memrealtime() - t0 > DEADLINE) {
                if (lane == 0) {
                    __hip_atomic_store(tmo, call, __ATOMIC_RELAXED, __HIP_MEMORY_SCOPE_AGENT);
                    P.out[rank][col0] = __builtin_nanf("");
                }
                break;
            }
            __builtin_amdgcn_s_sleep(2);
        }
        STAMP(6);
    }
}

}

extern "C" void kernel_launch(int rank, int nranks, ncclComm_t comm,
                              hipStream_t stream,
                              void* const* d_in, const int* in_sizes, int n_in,
                              void* d_out, int out_size,
                              const int* in_dtypes, int out_dtype,
                              void* const* d_out_all,
                              void* const* d_ws_all, size_t ws_bytes) {
    if (nranks != NR || (unsigned)rank >= (unsigned)NR || n_in != 4 || ws_bytes < WS_NEED ||
        in_sizes[0] != MM * KS || in_sizes[1] != KS * NN || out_size != MS * NN) {
        fprintf(stderr, "kernel_launch: unexpected problem shape\n");
        return;
    }
    Params P;
    P.x = (const uint8_t*)d_in[0];
    P.w = (const uint8_t*)d_in[1];
    P.sx = (const float*)d_in[2];
    P.sw = (const float*)d_in[3];
    for (int p = 0; p < NR; ++p) {
        P.out[p] = (float*)d_out_all[p];
        P.ws[p] = (uint8_t*)d_ws_all[p];
    }
    P.rank = rank;
    P.pad = 0;
    hipLaunchKernelGGL(fused_kernel, dim3(GRID), dim3(NTHR), 0, stream, P);
    HIP_CHECK(hipGetLastError());
}
```

```cpp
#include <hip/hip_runtime.h>
#include <rccl/rccl.h>
#include <cstdio>
#include <cstdint>

#define HIP_CHECK(expr) do { hipError_t _e = (expr); if (_e != hipSuccess) { \
    fprintf(stderr, "HIP error %s at %s:%d\n", hipGetErrorString(_e), __FILE__, __LINE__); \
    return; } } while (0)

typedef unsigned u32;
typedef unsigned long long u64;
typedef unsigned u32x4 __attribute__((ext_vector_type(4)));
typedef int i32x8 __attribute__((ext_vector_type(8)));
typedef float f32x4 __attribute__((ext_vector_type(4)));
typedef __attribute__((address_space(1))) const void gvoid;
typedef __attribute__((address_space(3))) void lvoid;
typedef __attribute__((address_space(3))) uint8_t lbyte;
typedef __attribute__((address_space(3))) u32 lu32;
typedef __attribute__((address_space(3))) unsigned short lu16;
typedef __attribute__((address_space(3))) u32x4 lu32x4;

namespace {

constexpr int NR = 8;
constexpr int MM = 4096, KK = 4096, NN = 16384;
constexpr int KS = KK / NR;
constexpr int MS = MM / NR;
constexpr int FG = 7, GG = 9;
constexpr int F = 32 * FG, G = 32 * GG;
static_assert(F + G == MS, "rows");
constexpr int XGREM = 3 * FG + 4 * GG;
constexpr int XREM = 32 * XGREM;
constexpr int XROWS = XREM + MS;
constexpr int NSTEP = 32;

constexpr int GRID = 256, NSEND = 64, NTHR = 512;
#define WPL 15
#define POLL_SLEEP 16
#define I8A 4

constexpr int SLOT = 6144 + 256 + 2048;
constexpr int LIST_MAX = 1022;
constexpr u32 H_DENSE = 1u << 11, H_RAW = 1u << 12, H_OVF = 1u << 13;
constexpr int SL_X = 0, SL_W = 256, SL_O = SL_W + 1024, SL_END = SL_O + 32 * 48 * 2;
constexpr int FL_X = 0, FL_W = 256, FL_O = FL_W + 128, FL_END = FL_O + 32 * 48 * 2;
static_assert(FL_END <= 4096, "flags");

constexpr size_t WS_CNT = 0;
constexpr size_t WS_TMO = 32768;
constexpr size_t WS_XREM = 65536;
constexpr size_t WS_XOWN = WS_XREM + 2048;
constexpr size_t WS_WCNT = 131072;
constexpr size_t WS_FLAG = 28u << 20;
constexpr size_t WS_DBG = 524288;
constexpr size_t WS_GW = 1u << 20;
constexpr size_t WS_GX = WS_GW + (size_t)NSTEP * 32 * 16384;
constexpr size_t XSLAB = (size_t)XROWS * 128;
constexpr size_t WS_PK = 32u << 20;
constexpr size_t PK_BYTES = 40u << 20;
static_assert(WS_GX + NSTEP * XSLAB <= WS_FLAG && WS_FLAG + NR * 4096 * 128 <= WS_PK, "staging");
static_assert((size_t)SL_END * SLOT <= PK_BYTES, "packed");
constexpr size_t WS_NEED = WS_PK + 2 * PK_BYTES;

constexpr int STAGE = 49152;
constexpr int LDS_CTL = 3 * STAGE;
constexpr int ESLOT = 6656;
constexpr int EWAVE = 2 * ESLOT + 2048;
constexpr u64 DEADLINE = 40000000ull;

constexpr int U1 = XGREM;
constexpr int U2 = U1 + 16 * I8A;
constexpr int U3 = U2 + XGREM;
constexpr int U4 = U3 + 16 * I8A;
constexpr int U5 = U4 + 32 * (8 - I8A);
constexpr int U6 = U5 + 32;

struct Params {
    const uint8_t* x;
    const uint8_t* w;
    const float* sx;
    const float* sw;
    float* out;
    uint8_t* ws[NR];
    int rank;
    int pad;
};

__device__ __forceinline__ void st_sys(void* p, u32x4 v) {
    asm volatile("global_store_dwordx4 %0, %1, off sc0 sc1\n\ts_nop 1" :: "v"(p), "v"(v) : "memory");
}
#define WAIT_VM(n) asm volatile("s_waitcnt vmcnt(" #n ")" ::: "memory")
#define CBAR() asm volatile("" ::: "memory")
#define DPP(v, ctl, rm) (u32)__builtin_amdgcn_update_dpp(0, (int)(v), ctl, rm, 0xf, false)

__device__ __forceinline__ u32 wave_scan_add(u32 v) {
    v += DPP(v, 0x111, 0xf); v += DPP(v, 0x112, 0xf); v += DPP(v, 0x114, 0xf); v += DPP(v, 0x118, 0xf);
    v += DPP(v, 0x142, 0xa); v += DPP(v, 0x143, 0xc);
    return v;
}
__device__ __forceinline__ u32 wave_sum(u32 v) { return __builtin_amdgcn_readlane(wave_scan_add(v), 63); }
__device__ __forceinline__ u32 wave_umax(u32 v) {
    v = max(v, DPP(v, 0x111, 0xf)); v = max(v, DPP(v, 0x112, 0xf)); v = max(v, DPP(v, 0x114, 0xf));
    v = max(v, DPP(v, 0x118, 0xf)); v = max(v, DPP(v, 0x142, 0xa)); v = max(v, DPP(v, 0x143, 0xc));
    return __builtin_amdgcn_readlane(v, 63);
}
__device__ __forceinline__ int block_of(int r, int part) {
    return (r & 1) ? 4 * part + (r >> 1) : 4 * (r >> 1) + part;
}
__device__ __forceinline__ float silu(float v) {
    const float z = __builtin_amdgcn_fmed3f(v, -60.f, 60.f);
    const float e = __builtin_amdgcn_exp2f(z * -1.4426950408889634f);
    return v * __builtin_amdgcn_rcpf(1.0f + e);
}

template <bool ACC>
__device__ __forceinline__ void put_unit(uint8_t* slot, const u32 (&pay)[24], u32 (&pl)[8], u32 hdr,
                                         lbyte* scr, int lane) {
#pragma unroll
    for (int k = 0; k < 6; ++k)
        st_sys(slot + k * 1024 + lane * 16, u32x4{pay[4 * k], pay[4 * k + 1], pay[4 * k + 2], pay[4 * k + 3]});
    u32 cnt = 0;
#pragma unroll
    for (int j = 0; j < 8; ++j) {
        u32 t = pl[j] | (pl[j] >> 1);
        if (ACC) { t |= t >> 2; t &= 0x10101010u; } else t &= 0x55555555u;
        cnt += __builtin_popcount(t);
    }
    const u32 inc = wave_scan_add(cnt);
    const u32 total = __builtin_amdgcn_readlane(inc, 63);
    if ((hdr & H_DENSE) || total > LIST_MAX) {
        st_sys(slot + 6400 + lane * 16, u32x4{pl[0], pl[1], pl[2], pl[3]});
        st_sys(slot + 6400 + 1024 + lane * 16, u32x4{pl[4], pl[5], pl[6], pl[7]});
        if (lane < 16) st_sys(slot + 6144 + lane * 16, u32x4{lane == 0 ? (hdr | H_DENSE) : 0u, 0u, 0u, 0u});
    } else {
        *(lu32*)(scr + lane * 4) = lane == 0 ? (hdr | total) : 0u;
        if (lane == 0) *(lu16*)(scr + 4 + 2 * total) = 0;
        CBAR();
        u32 pos = inc - cnt;
#pragma unroll
        for (int j = 0; j < 8; ++j) {
            const u32 dw = ((j >> 2) * 256 + lane * 4 + (j & 3)) << 6;
            u32 p = pl[j];
            while (p) {
                const u32 b = __builtin_ctz(p);
                u32 e;
                if (ACC) {
                    const u32 sh = (b & 24u) + 4u;
                    e = dw | ((b >> 3) << 4) | ((p >> sh) & 15u);
                    p &= ~(15u << sh);
                } else {
                    const u32 sh = b & 30u;
                    e = dw | (sh << 1) | ((p >> sh) & 3u);
                    p &= ~(3u << sh);
                }
                *(lu16*)(scr + 4 + 2 * pos) = (unsigned short)e;
                ++pos;
            }
        }
        CBAR();
        const u32 nb = max(4u + 2u * total, 256u);
        for (u32 o = lane * 16; o < nb; o += 1024) st_sys(slot + 6144 + o, *(lu32x4*)(scr + o));
    }
}

__device__ __forceinline__ u32 pick_base(u32 esum4  ) {
    const int b = ((int)esum4 - 33587) >> 13;
    return (u32)min(max(b, 0), 24);
}
__device__ __forceinline__ void enc_op(const u32x4* v, u32 base4, u32 (&pay)[24], u32 (&pl)[8]) {
#pragma unroll
    for (int j = 0; j < 8; ++j) {
        u32 c[4], x[4];
#pragma unroll
        for (int d = 0; d < 4; ++d) {
            const u32 u = (v[j][d] | 0x80808080u) - base4;
            c[d] = (u & 0x1F1F1F1Fu) | (v[j][d] & 0x80808080u);
            x[d] = v[j][d] ^ (c[d] + base4);
        }
        pl[j] = (x[0] >> 5) | (x[1] >> 3) | (x[2] >> 1) | (x[3] << 1);
        pay[3 * j + 0] = c[0] | ((c[3] & 0x03030303u) << 5);
        pay[3 * j + 1] = c[1] | ((c[3] & 0x0C0C0C0Cu) << 3);
        pay[3 * j + 2] = c[2] | ((c[3] & 0x10101010u) << 1) | ((c[3] & 0x80808080u) >> 1);
    }
}

struct Unit {
    u32 slot;
    u32 flag;
    u32 dst;
    u32 kind;
    u32 sig;
};
__device__ __forceinline__ Unit unit_of(int n, int r, int p, int nend) {
    Unit u;
    if (n >= nend) { u.slot = 0; u.flag = 0; u.dst = 0; u.kind = 3; u.sig = 0; return u; }
    if (n >= U6) {
        const bool same = ((r ^ p) & 1) == 0;
        const int v = n - U6, wn = v & 1, h = v >> 1;
        int i8, part, g, rg, row;
        if (same) {
            i8 = h / (4 * FG);
            const int t = h - i8 * (4 * FG);
            part = t / FG; g = t - part * FG;
            rg = FG * (((r >> 1) - (p >> 1) - 1) & 3) + g;
            row = 32 * g;
        } else {
            i8 = h / (3 * GG);
            const int t = h - i8 * (3 * GG);
            const int pi = t / GG, jr = r >> 1;
            g = t - pi * GG;
            part = pi + (pi >= jr);
            rg = 3 * FG + (jr > part ? jr - 1 : jr) * GG + g;
            row = F + 32 * g;
        }
        const int id = ((part * 8 + i8) * 48 + rg) * 2 + wn;
        u.slot = SL_O + id; u.flag = FL_O + id; u.kind = 2; u.sig = 0;
        u.dst = (u32)(row * NN + block_of(p, part) * 1024 + i8 * 128 + wn * 64) * 4u;
        return u;
    }
    const bool isx = n < U1 || (n >= U2 && n < U3) || n >= U5;
    if (isx) {
        int sg, kp;
        if (n < U1) { sg = n; kp = 0; }
        else if (n < U3) { sg = n - U2; kp = 1; }
        else { sg = XGREM + ((n - U5) & 15); kp = (n - U5) >> 4; }
        int gg;
        if (sg < 3 * FG) {
            const int s = sg / FG;
            gg = 16 * (2 * (((r >> 1) + 1 + s) & 3) + (r & 1)) + sg - s * FG;
        } else if (sg < XGREM) {
            const int j = (sg - 3 * FG) / GG;
            gg = 16 * (2 * j + 1 - (r & 1)) + FG + sg - 3 * FG - j * GG;
        } else {
            gg = 16 * r + sg - XGREM;
        }
        u.slot = SL_X + gg * 2 + kp; u.flag = FL_X + gg * 2 + kp; u.kind = 0;
        u.dst = (u32)(WS_GX + (size_t)(p * 4 + 2 * kp) * XSLAB) + (u32)sg * 4096u;
        u.sig = (u32)(sg < XGREM ? WS_XREM : WS_XOWN) + (u32)(p * 2 + kp) * 128u;
        return u;
    }
    int i8, kc, t;
    if (n < U2) { const int v = n - U1; i8 = v >> 4; kc = (v >> 3) & 1; t = v & 7; }
    else if (n < U4) { const int v = n - U3; i8 = v >> 4; kc = 2 + ((v >> 3) & 1); t = v & 7; }
    else { const int v = n - U4; i8 = I8A + (v >> 5); kc = (v >> 3) & 3; t = v & 7; }
    const int part = t >> 1, half = t & 1, ct = part * 8 + i8, step = p * 4 + kc;
    u.slot = SL_W + ((kc * 128 + block_of(r, part) * 8 + i8) * 2 + half);
    u.flag = FL_W + kc * 32 + ct;
    u.kind = 1 | (half << 2);
    u.dst = (u32)WS_GW + (u32)(step * 32 + ct) * 16384u + (u32)half * 8192u;
    u.sig = (u32)WS_WCNT + (u32)(ct * 32 + step) * 128u;
    return u;
}

#define STAMP(k) do { } while (0)

__global__ void __launch_bounds__(NTHR, 2) fused_kernel(Params P) {
    __shared__ __attribute__((aligned(1024))) uint8_t lds_[LDS_CTL + 64];
    lbyte* const lds = (lbyte*)lds_;
    const int tid = threadIdx.x;
    const int lane = tid & 63;
    const int wave = __builtin_amdgcn_readfirstlane(tid >> 6);
    const int b = blockIdx.x;
    const int rank = P.rank;
    uint8_t* const myws = P.ws[rank];
    u32* const tmo = (u32*)(myws + WS_TMO);

    STAMP(0);
    if (tid == 0) {
        u32 t = __hip_atomic_fetch_add((u32*)(myws + WS_CNT + (size_t)b * 128), 1u, __ATOMIC_RELAXED,
                                       __HIP_MEMORY_SCOPE_AGENT);
        *(lu32*)(lds + LDS_CTL) = t + 1;
    }
    __syncthreads();
    const u32 call = __builtin_amdgcn_readfirstlane(*(lu32*)(lds + LDS_CTL));
    const size_t pkoff = WS_PK + (size_t)(call & 1) * PK_BYTES;

    if (b < NSEND) {
        const int gw = b + NSEND * wave;
        if (gw >= 7 * WPL) return;
        const int p = (rank + 1 + gw % 7) & 7;
        const int nend = U6 + ((((rank ^ p) & 1) == 0) ? 64 * FG : 48 * GG);
        const uint8_t* const pk = P.ws[p] + pkoff;
        const u32* const flags = (const u32*)(myws + WS_FLAG) + p * 4096 * 32;
        lbyte* const el = lds + wave * EWAVE;
        lbyte* const plane = el + 2 * ESLOT;
        const u32 el_addr = (u32)(uintptr_t)el;
        float scale = P.sx[0] * P.sw[0];
        *(lu32x4*)(plane + lane * 16) = u32x4{0u, 0u, 0u, 0u};
        *(lu32x4*)(plane + 1024 + lane * 16) = u32x4{0u, 0u, 0u, 0u};
        auto spin = [&](u32 fl) {
            const u64 t0 = __builtin_amdgcn_s_memrealtime();
            while (__hip_atomic_load(flags + fl * 32, __ATOMIC_RELAXED, __HIP_MEMORY_SCOPE_SYSTEM) < call) {
                if (__hip_atomic_load(tmo, __ATOMIC_RELAXED, __HIP_MEMORY_SCOPE_AGENT) == call ||
                    __builtin_amdgcn_s_memrealtime() - t0 > DEADLINE) {
                    if (lane == 0) __hip_atomic_store(tmo, call, __ATOMIC_RELAXED, __HIP_MEMORY_SCOPE_AGENT);
                    scale = __builtin_nanf("");
                    break;
                }
                __builtin_amdgcn_s_sleep(2);
            }
            WAIT_VM(0);
        };
        const u32 vo0 = lane * 16, vo1 = lane * 16 + 4096;
        auto fetch = [&](const Unit& u, u32 nextflag, int sl) {
            const uint8_t* src = pk + (size_t)u.slot * SLOT;
            const u32 la = el_addr + sl * ESLOT;
            u32 keep;
            asm volatile("s_nop 4\n\ts_mov_b32 %0, m0\n\ts_mov_b32 m0, %4\n\ts_nop 0\n\t"
                         "global_load_lds_dwordx4 %1, %3 sc0 sc1\n\t"
                         "global_load_lds_dwordx4 %1, %3 offset:1024 sc0 sc1\n\t"
                         "global_load_lds_dwordx4 %1, %3 offset:2048 sc0 sc1\n\t"
                         "global_load_lds_dwordx4 %1, %3 offset:3072 sc0 sc1\n\t"
                         "s_mov_b32 m0, %5\n\ts_nop 0\n\t"
                         "global_load_lds_dwordx4 %2, %3 sc0 sc1\n\t"
                         "global_load_lds_dwordx4 %2, %3 offset:1024 sc0 sc1\n\t"
                         "s_mov_b32 m0, %0"
                         : "=&s"(keep) : "v"(vo0), "v"(vo1), "s"(src), "s"(la), "s"(la + 4096) : "memory");
            const uint8_t* a = lane == 16 ? (const uint8_t*)(flags + nextflag * 32) : src + 6144 + lane * 16;
            if (lane < (u.kind == 2 ? 8 : 16) || lane == 16)
                asm volatile("s_mov_b32 %0, m0\n\ts_mov_b32 m0, %2\n\ts_nop 0\n\t"
                             "global_load_lds_dwordx4 %1, off sc0 sc1\n\ts_mov_b32 m0, %0"
                             : "=&s"(keep) : "v"(a), "s"(la + 6144) : "memory");
        };
        auto signal = [&](u32 sig) {
            if (lane == 0) __hip_atomic_fetch_add((u32*)(myws + sig), 1u, __ATOMIC_RELAXED, __HIP_MEMORY_SCOPE_SYSTEM);
        };

        int n = gw / 7;
        Unit cur = unit_of(n, rank, p, nend), nxt = unit_of(n + WPL, rank, p, nend);
        spin(cur.flag);
        STAMP(2);
        WAIT_VM(0);
        fetch(cur, nxt.flag, 0);
        WAIT_VM(0);
        u32 pend = 0;
        int sl = 0;
        for (;;) {
            lbyte* const s = el + sl * ESLOT;
            const Unit nn = unit_of(n + 2 * WPL, rank, p, nend);
            if (nxt.kind != 3) {
                const u32 fv = __builtin_amdgcn_readfirstlane(*(lu32*)(s + 6400));
                if (fv < call) {
                    WAIT_VM(0);
                    if (pend) { signal(pend); pend = 0; }
                    spin(nxt.flag);
                }
                fetch(nxt, nn.flag, sl ^ 1);
            }
            const bool acc = cur.kind == 2;
            u32 le = *(lu32*)(s + 6144 + lane * 4);
            const u32 hdr = __builtin_amdgcn_readfirstlane(le);
            const u32 first = acc ? 32u : 64u;
            auto scatter = [&](u32 e2) {
#pragma unroll
                for (int h = 0; h < 2; ++h) {
                    const u32 e = h ? e2 >> 16 : e2 & 0xffffu;
                    const u32 val = acc ? (e & 15u) << (((e >> 1) & 24u) + 4u) : (e & 3u) << ((e >> 1) & 30u);
                    if (val) __hip_atomic_fetch_or((lu32*)(plane + ((e >> 4) & 0x7fcu)), val, __ATOMIC_RELAXED, __HIP_MEMORY_SCOPE_WAVEFRONT);
                }
            };
            if (hdr & H_DENSE) {
                const uint8_t* a = pk + (size_t)cur.slot * SLOT + 6400 + lane * 16;
                u32 keep;
                asm volatile("s_waitcnt lgkmcnt(0)\n\ts_mov_b32 %0, m0\n\ts_mov_b32 m0, %2\n\ts_nop 0\n\t"
                             "global_load_lds_dwordx4 %1, off sc0 sc1\n\t"
                             "global_load_lds_dwordx4 %1, off offset:1024 sc0 sc1\n\t"
                             "s_mov_b32 m0, %0\n\ts_waitcnt vmcnt(0)"
                             : "=&s"(keep) : "v"(a), "s"(el_addr + 2 * ESLOT) : "memory");
            } else {
                if (lane == 0 || (u32)lane >= first) le = 0;
                scatter(le);
                const u32 ndw = (2u + (hdr & 0x7ffu) + 1u) >> 1;
                for (u32 o = first; o < ndw; o += 64) {
                    u32 e2 = 0;
                    if (o + lane < ndw)
                        e2 = __hip_atomic_load((const u32*)(pk + (size_t)cur.slot * SLOT + 6144) + o + lane,
                                               __ATOMIC_RELAXED, __HIP_MEMORY_SCOPE_SYSTEM);
                    scatter(e2);
                }
            }
            CBAR();
            u32 pay[24], pl[8];
#pragma unroll
            for (int k = 0; k < 6; ++k) {
                const u32x4 t = *(lu32x4*)(s + k * 1024 + lane * 16);
                pay[4 * k] = t[0]; pay[4 * k + 1] = t[1]; pay[4 * k + 2] = t[2]; pay[4 * k + 3] = t[3];
            }
#pragma unroll
            for (int k = 0; k < 2; ++k) {
                const u32x4 t = *(lu32x4*)(plane + k * 1024 + lane * 16);
                pl[4 * k] = t[0]; pl[4 * k + 1] = t[1]; pl[4 * k + 2] = t[2]; pl[4 * k + 3] = t[3];
            }
            *(lu32x4*)(plane + lane * 16) = u32x4{0u, 0u, 0u, 0u};
            *(lu32x4*)(plane + 1024 + lane * 16) = u32x4{0u, 0u, 0u, 0u};

            bool sigd = false;
            if (pend) {
                if (nxt.kind != 3) WAIT_VM(7); else WAIT_VM(0);
                signal(pend);
                sigd = true;
            }
            pend = cur.sig;

            if (!acc) {
                const u32 base4 = (hdr >> 16) * 0x04040404u;
                const u32 half = cur.kind >> 2;
                const bool isw = cur.kind & 1;
                uint8_t* d0 = myws + cur.dst + ((lane ^ (isw ? half * 8 : 0)) << 4);
                uint8_t* d1 = myws + cur.dst + (isw ? 4096 : XSLAB) + ((lane ^ (isw ? half * 8 + 4 : 0)) << 4);
#pragma unroll
                for (int j = 0; j < 8; ++j) {
                    const u32 p0 = pay[3 * j], p1 = pay[3 * j + 1], p2 = pay[3 * j + 2];
                    const u32 c3 = ((p0 >> 5) & 0x03030303u) | ((p1 >> 3) & 0x0C0C0C0Cu) |
                                   ((p2 >> 1) & 0x10101010u) | ((p2 << 1) & 0x80808080u);
                    u32x4 v;
                    v[0] = ((p0 & 0x9F9F9F9Fu) + base4) ^ ((pl[j] << 5) & 0x60606060u);
                    v[1] = ((p1 & 0x9F9F9F9Fu) + base4) ^ ((pl[j] << 3) & 0x60606060u);
                    v[2] = ((p2 & 0x9F9F9F9Fu) + base4) ^ ((pl[j] << 1) & 0x60606060u);
                    v[3] = (c3 + base4) ^ ((pl[j] >> 1) & 0x60606060u);
                    st_sys((j < 4 ? d0 : d1) + (j & 3) * 1024, v);
                }
            } else {
                const float c1 = __builtin_bit_cast(float, (hdr >> 16) << 23);
                uint8_t* d0 = (uint8_t*)P.out + cur.dst + (size_t)(lane >> 4) * (4 * NN * 4) + (lane & 15) * 16;
                float a[32];
#pragma unroll
                for (int j = 0; j < 8; ++j) {
                    const u32 l0 = __builtin_amdgcn_perm(pay[3 * j + 2], pl[j], 0x05010400u);
                    const u32 l1 = __builtin_amdgcn_perm(pay[3 * j + 2], pl[j], 0x07030602u);
                    a[4 * j + 0] = __builtin_bit_cast(float, __builtin_amdgcn_perm(pay[3 * j], l0, 0x05040100u));
                    a[4 * j + 1] = __builtin_bit_cast(float, __builtin_amdgcn_perm(pay[3 * j], l0, 0x07060302u));
                    a[4 * j + 2] = __builtin_bit_cast(float, __builtin_amdgcn_perm(pay[3 * j + 1], l1, 0x05040100u));
                    a[4 * j + 3] = __builtin_bit_cast(float, __builtin_amdgcn_perm(pay[3 * j + 1], l1, 0x07060302u));
                }
                if (hdr & H_RAW) {
                } else if (hdr & H_OVF) {
#pragma unroll
                    for (int i = 0; i < 32; ++i) {
                        const int t = __builtin_bit_cast(int, a[i]);
                        const float fix = (t & 16) ? (t < 0 ? 4294967296.f : -4294967296.f) : 0.f;
                        a[i] = ((float)(t & ~16) + fix) * c1;
                    }
                } else {
#pragma unroll
                    for (int i = 0; i < 32; ++i) a[i] = (float)__builtin_bit_cast(int, a[i]) * c1;
                }
#pragma unroll
                for (int j = 0; j < 8; ++j) {
                    f32x4 y;
#pragma unroll
                    for (int t = 0; t < 4; ++t) y[t] = silu(a[4 * j + t] * scale);
                    st_sys(d0 + (size_t)((j >> 2) * 16 + (j & 3)) * (NN * 4), __builtin_bit_cast(u32x4, y));
                }
            }
            if (nxt.kind == 3) break;
            if (sigd) WAIT_VM(9); else WAIT_VM(8);
            cur = nxt; nxt = nn; n += WPL; sl ^= 1;
        }
        STAMP(4);
        WAIT_VM(0);
        if (pend) signal(pend);
        STAMP(5);
        return;
    }

    const int g = b - NSEND;
    lbyte* const scr = lds + wave * 4096;
    if (wave < 4) {
        const int task = g + 192 * wave;
        uint8_t* const pk = myws + pkoff;
        u32 pay[24], pl[8];
        if (task < 256) {
            const int gg = task >> 1, kp = task & 1;
            const uint8_t* src = P.x + (size_t)(gg * 32 + (lane >> 3)) * KS + kp * 256 + (lane & 7) * 16;
            u32x4 v[8];
#pragma unroll
            for (int j = 0; j < 8; ++j) v[j] = *(const u32x4*)(src + (j & 3) * 8 * KS + (j >> 2) * 128);
            u32 es = 0;
#pragma unroll
            for (int j = 0; j < 8; ++j) es = __builtin_amdgcn_sad_u8(v[j][j & 3] & 0x7C7C7C7Cu, 0u, es);
            const u32 base = pick_base(wave_sum(es));
            enc_op(v, base * 0x04040404u, pay, pl);
            put_unit<false>(pk + (size_t)(SL_X + task) * SLOT, pay, pl, base << 16, scr, lane);
            const int q = gg >> 4, lg = gg & 15;
            auto needs = [&](int r) { return r == q || (((r ^ q) & 1) == 0) == (lg < FG); };
            const bool self = needs(rank);
            if (self) {
                const int sg = q == rank ? XGREM + lg
                             : lg < FG ? FG * (((q >> 1) - (rank >> 1) - 1) & 3) + lg
                                       : 3 * FG + GG * (q >> 1) + lg - FG;
                uint8_t* d = myws + WS_GX + (size_t)(rank * 4 + 2 * kp) * XSLAB + (size_t)sg * 4096 + lane * 16;
#pragma unroll
                for (int j = 0; j < 8; ++j) st_sys(d + (j >> 2) * XSLAB + (j & 3) * 1024, v[j]);
            }
            WAIT_VM(0);
            if (lane < NR && lane != rank && needs(lane))
                __hip_atomic_store((u32*)(P.ws[lane] + WS_FLAG) + (rank * 4096 + FL_X + task) * 32, call, __ATOMIC_RELAXED, __HIP_MEMORY_SCOPE_SYSTEM);
            if (self && lane == 0)
                __hip_atomic_fetch_add((u32*)(myws + (q == rank ? WS_XOWN : WS_XREM)) + (rank * 2 + kp) * 32, 1u, __ATOMIC_RELAXED, __HIP_MEMORY_SCOPE_SYSTEM);
        } else {
            const int t = task - 256;
            const int i8 = t >> 6, kc = (t >> 4) & 3, blk = t & 15;
            const uint8_t* src = P.w + (size_t)(kc * 128 + 16 * (lane >> 3)) * NN + blk * 1024 + i8 * 128 + 16 * (lane & 7);
            u32x4 in[16], o[16];
#pragma unroll
            for (int j = 0; j < 16; ++j) in[j] = *(const u32x4*)(src + (size_t)j * NN);
            u32 es = 0;
#pragma unroll
            for (int j = 0; j < 8; ++j) es = __builtin_amdgcn_sad_u8(in[2 * j][j & 3] & 0x7C7C7C7Cu, 0u, es);
            const u32 base = pick_base(wave_sum(es));
#pragma unroll
            for (int d = 0; d < 4; ++d)
#pragma unroll
                for (int e = 0; e < 4; ++e) {
                    const u32 t0 = __builtin_amdgcn_perm(in[4 * e + 1][d], in[4 * e][d], 0x05010400u);
                    const u32 t1 = __builtin_amdgcn_perm(in[4 * e + 1][d], in[4 * e][d], 0x07030602u);
                    const u32 t2 = __builtin_amdgcn_perm(in[4 * e + 3][d], in[4 * e + 2][d], 0x05010400u);
                    const u32 t3 = __builtin_amdgcn_perm(in[4 * e + 3][d], in[4 * e + 2][d], 0x07030602u);
                    o[4 * d + 0][e] = __builtin_amdgcn_perm(t2, t0, 0x05040100u);
                    o[4 * d + 1][e] = __builtin_amdgcn_perm(t2, t0, 0x07060302u);
                    o[4 * d + 2][e] = __builtin_amdgcn_perm(t3, t1, 0x05040100u);
                    o[4 * d + 3][e] = __builtin_amdgcn_perm(t3, t1, 0x07060302u);
                }
            uint8_t* const slot = pk + (size_t)(SL_W + (kc * 128 + blk * 8 + i8) * 2) * SLOT;
            enc_op(o, base * 0x04040404u, pay, pl);
            put_unit<false>(slot, pay, pl, base << 16, scr, lane);
            enc_op(o + 8, base * 0x04040404u, pay, pl);
            put_unit<false>(slot + SLOT, pay, pl, base << 16, scr, lane);
            const int a = blk >> 2, bb = blk & 3;
            const int re = 2 * a, ro = 2 * bb + 1;
            const int cte = bb * 8 + i8, cto = a * 8 + i8;
            const bool self = rank == re || rank == ro;
            const int cts = rank == re ? cte : cto;
            if (self) {
                uint8_t* d = myws + WS_GW + (size_t)((rank * 4 + kc) * 32 + cts) * 16384;
#pragma unroll
                for (int i = 0; i < 16; ++i) st_sys(d + i * 1024 + ((lane ^ ((i >> 2) << 2)) << 4), o[i]);
            }
            WAIT_VM(0);
            if (lane < 2) {
                const int r = lane ? ro : re;
                if (r != rank)
                    __hip_atomic_store((u32*)(P.ws[r & 7] + WS_FLAG) + (rank * 4096 + FL_W + kc * 32 + (lane ? cto : cte)) * 32, call, __ATOMIC_RELAXED, __HIP_MEMORY_SCOPE_SYSTEM);
            }
            if (self && lane == 0)
                __hip_atomic_fetch_add((u32*)(myws + WS_WCNT) + (cts * 32 + rank * 4 + kc) * 32, 2u, __ATOMIC_RELAXED, __HIP_MEMORY_SCOPE_SYSTEM);
        }
    }
    STAMP(1);

    const int xcd = g & 7, idx = g >> 3;
    const int part = xcd >> 1;
    const int wm = wave >> 1, wn = wave & 1;
    const int c = lane & 15, q4 = lane >> 4;
    const int ra0 = wm * 8192 + c * 128 + ((q4 ^ (c >> 1)) << 4);
    const int ra1 = ra0 ^ 64;
    const int rb0 = 32768 + (c & 3) * 4096 + ((((q4 << 3) | (4 * wn + (c >> 2))) ^ ((c & 3) << 2)) << 4);
    const u32 boff = (u32)wave * 2048u + (u32)lane * 16u;
    const uint8_t* const xbuf = myws + WS_GX;

    for (int round = 0; round < (idx < 8 ? 2 : 1); ++round) {
        const bool local = round == 1;
        const int i8 = local ? idx : idx / 3;
        const int rt = local ? (xcd & 1) : 3 * (xcd & 1) + idx % 3;
        const int ct = part * 8 + i8;
        const uint8_t* const wbuf = myws + WS_GW + (size_t)ct * 16384;
        const int rgw = rt * 8 + wave;
        const int sgw = local ? XGREM + rgw : (rgw < 3 * FG + part * GG ? rgw : rgw + GG);
        u32 aoff[4];
#pragma unroll
        for (int jj = 0; jj < 4; ++jj)
            aoff[jj] = (u32)sgw * 4096u + (u32)(8 * jj + (lane >> 3)) * 128u +
                       (u32)(((lane & 7) ^ ((4 * jj + (lane >> 4)) & 7)) << 4);

        const u32* const wpoll = (const u32*)(myws + WS_WCNT) + (ct * 32 + (lane & 7) * 4) * 32;
        const u32* const xpoll = (const u32*)(myws + (local ? WS_XOWN : WS_XREM)) + (lane & 7) * 64;
        const u32 xneed = call * (local ? 16u : (u32)XGREM);
        u32 wready = 0, xready = 0;
        auto stepno = [&](int s) { return ((rank + s) & 7) * 4 + (s >> 3); };
        auto ensure = [&](int s) {
            const int kc = s >> 3, src = (rank + s) & 7;
            const u32 wbit = 1u << (8 * kc + src), xbit = 1u << (8 * (kc >> 1) + src);
            if ((wready & wbit) && (xready & xbit)) return;
            const u64 t0 = __builtin_amdgcn_s_memrealtime();
            for (;;) {
                if (!(wready & wbit)) {
                    u32 v = 0;
                    if (lane < 8) v = __hip_atomic_load(wpoll + kc * 32, __ATOMIC_RELAXED, __HIP_MEMORY_SCOPE_SYSTEM);
                    wready |= (u32)__ballot(v >= 2u * call) << (8 * kc);
                }
                if ((wready & wbit) && !(xready & xbit)) {
                    u32 v = 0;
                    if (lane < 8) v = __hip_atomic_load(xpoll + (kc >> 1) * 32, __ATOMIC_RELAXED, __HIP_MEMORY_SCOPE_SYSTEM);
                    xready |= (u32)__ballot(v >= xneed) << (8 * (kc >> 1));
                }
                if ((wready & wbit) && (xready & xbit)) break;
                if (__hip_atomic_load(tmo, __ATOMIC_RELAXED, __HIP_MEMORY_SCOPE_AGENT) == call ||
                    __builtin_amdgcn_s_memrealtime() - t0 > DEADLINE) {
                    if (lane == 0) __hip_atomic_store(tmo, call, __ATOMIC_RELAXED, __HIP_MEMORY_SCOPE_AGENT);
                    WAIT_VM(0);
                    wready = xready = ~0u;
                    break;
                }
                __builtin_amdgcn_s_sleep(POLL_SLEEP);
            }
        };
        auto issue = [&](int s, int stage) {
            const int st = stepno(s);
            const uint8_t* xa = xbuf + (size_t)st * XSLAB;
            const uint8_t* wa = wbuf + (size_t)st * (32 * 16384);
            lbyte* la = lds + stage * STAGE + wave * 4096;
#pragma unroll
            for (int jj = 0; jj < 4; ++jj)
                __builtin_amdgcn_global_load_lds((gvoid*)(xa + aoff[jj]), (lvoid*)(la + jj * 1024), 16, 0, 17);
            lbyte* lb = lds + stage * STAGE + 32768 + wave * 2048;
            __builtin_amdgcn_global_load_lds((gvoid*)(wa + boff), (lvoid*)lb, 16, 0, 17);
            __builtin_amdgcn_global_load_lds((gvoid*)(wa + boff), (lvoid*)lb, 16, 1024, 17);
        };

        f32x4 acc[4][4];
#pragma unroll
        for (int m = 0; m < 4; ++m)
#pragma unroll
            for (int t = 0; t < 4; ++t) acc[m][t] = f32x4{0.f, 0.f, 0.f, 0.f};

        if (wave == 0) { ensure(0); ensure(1); }
        asm volatile("s_waitcnt vmcnt(0) lgkmcnt(0)" ::: "memory");
        __builtin_amdgcn_s_barrier();
        asm volatile("" ::: "memory");
        issue(0, 0);
        issue(1, 1);
        int stage = 0;
        for (int s = 0; s < NSTEP; ++s) {
            if (wave == 0 && s + 2 < NSTEP) ensure(s + 2);
            if (s + 1 < NSTEP) WAIT_VM(6); else WAIT_VM(0);
            __builtin_amdgcn_s_barrier();
            asm volatile("" ::: "memory");
            if (s + 2 < NSTEP) issue(s + 2, stage == 0 ? 2 : stage - 1);
            const lbyte* sb = lds + stage * STAGE;
            i32x8 A[4], B[4];
#pragma unroll
            for (int t = 0; t < 4; ++t) {
                const u32x4 lo = *(const lu32x4*)(sb + rb0 + t * 1024);
                const u32x4 hi = *(const lu32x4*)(sb + rb0 + t * 1024 + 512);
                B[t] = i32x8{(int)lo[0], (int)lo[1], (int)lo[2], (int)lo[3], (int)hi[0], (int)hi[1], (int)hi[2], (int)hi[3]};
            }
#pragma unroll
            for (int m = 0; m < 4; ++m) {
                const u32x4 lo = *(const lu32x4*)(sb + ra0 + m * 2048);
                const u32x4 hi = *(const lu32x4*)(sb + ra1 + m * 2048);
                A[m] = i32x8{(int)lo[0], (int)lo[1], (int)lo[2], (int)lo[3], (int)hi[0], (int)hi[1], (int)hi[2], (int)hi[3]};
            }
#pragma unroll
            for (int m = 0; m < 4; ++m)
#pragma unroll
                for (int t = 0; t < 4; ++t)
                    acc[m][t] = __builtin_amdgcn_mfma_scale_f32_16x16x128_f8f6f4(A[m], B[t], acc[m][t], 1, 1, 0, 0, 0, 0);
            stage = stage == 2 ? 0 : stage + 1;
        }

        const bool bad = __hip_atomic_load(tmo, __ATOMIC_RELAXED, __HIP_MEMORY_SCOPE_AGENT) == call;
        if (local) {
            float scale = bad ? __builtin_nanf("") : P.sx[0] * P.sw[0];
            asm volatile("" : "+v"(scale));
            float* dst = P.out + (size_t)(rt * 256 + wm * 64 + 4 * q4) * NN + block_of(rank, part) * 1024 + i8 * 128 + wn * 64 + 4 * c;
#pragma unroll
            for (int m = 0; m < 4; ++m)
#pragma unroll
                for (int r = 0; r < 4; ++r) {
                    f32x4 y;
#pragma unroll
                    for (int t = 0; t < 4; ++t) y[t] = silu(acc[m][t][r] * scale);
                    st_sys(dst + (size_t)(m * 16 + r) * NN, __builtin_bit_cast(u32x4, y));
                }
            STAMP(7);
        } else {
#pragma unroll
            for (int mp = 0; mp < 2; ++mp) {
                u32 am = 0;
#pragma unroll
                for (int i = 0; i < 32; ++i)
                    am = max(am, __float_as_uint(acc[2 * mp + (i >> 4)][i & 3][(i >> 2) & 3]) & 0x7fffffffu);
                const int ex = min(max((int)(wave_umax(am) >> 23), 40), 200);
                const float up = __builtin_bit_cast(float, (u32)(280 - ex) << 23);
                const float dn = __builtin_bit_cast(float, (u32)(ex - 26) << 23);
                const u32 top = (u32)ex << 23;
                u32 T[32], miss = 0, low = 0;
                int vote = 0;
#pragma unroll
                for (int i = 0; i < 32; ++i) {
                    const float a = acc[2 * mp + (i >> 4)][i & 3][(i >> 2) & 3];
                    const int I = (int)(a * up);
                    miss |= __builtin_bit_cast(u32, (float)I * dn) ^ __builtin_bit_cast(u32, a);
                    low |= (u32)I;
                    vote += ((I >> 3) & 1) - ((__builtin_bit_cast(u32, a) & 0x7fffffffu) >= top);
                    T[i] = (u32)I;
                }
                u32 hdr;
                if (bad || __any(miss != 0 || am >= top + (1u << 23))) {
                    hdr = H_RAW | H_DENSE;
#pragma unroll
                    for (int i = 0; i < 32; ++i)
                        T[i] = bad ? 0x7fc00000u : __float_as_uint(acc[2 * mp + (i >> 4)][i & 3][(i >> 2) & 3]);
                } else if (__any(low & 1) || (int)wave_sum((u32)vote) > 0) {
                    hdr = H_OVF | ((u32)(ex - 31) << 16);
#pragma unroll
                    for (int i = 0; i < 32; ++i)
                        T[i] = (T[i] << 5) | ((T[i] + (1u << 26)) >= (1u << 27) ? 16u : 0u);
                } else {
                    hdr = (u32)(ex - 30) << 16;
#pragma unroll
                    for (int i = 0; i < 32; ++i) T[i] <<= 4;
                }
                u32 pay[24], pl[8];
#pragma unroll
                for (int j = 0; j < 8; ++j) {
                    pay[3 * j] = __builtin_amdgcn_perm(T[4 * j + 1], T[4 * j], 0x07060302u);
                    pay[3 * j + 1] = __builtin_amdgcn_perm(T[4 * j + 3], T[4 * j + 2], 0x07060302u);
                    const u32 l0 = __builtin_amdgcn_perm(T[4 * j + 1], T[4 * j], 0x05010400u);
                    const u32 l1 = __builtin_amdgcn_perm(T[4 * j + 3], T[4 * j + 2], 0x05010400u);
                    pl[j] = __builtin_amdgcn_perm(l1, l0, 0x05040100u);
                    pay[3 * j + 2] = __builtin_amdgcn_perm(l1, l0, 0x07060302u);
                }
                const int id = (ct * 48 + rt * 8 + wm * 2 + mp) * 2 + wn;
                put_unit<true>(myws + pkoff + (size_t)(SL_O + id) * SLOT, pay, pl, hdr, scr, lane);
            }
            WAIT_VM(0);
            if (lane < 2) {
                const int rg = rt * 8 + wm * 2 + lane;
                int q;
                if (rg < 3 * FG) {
                    q = 2 * (((rank >> 1) + 1 + rg / FG) & 3) + (rank & 1);
                } else {
                    const int jj = (rg - 3 * FG) / GG;
                    q = 2 * (jj + (jj >= part)) + 1 - (rank & 1);
                }
                __hip_atomic_store((u32*)(P.ws[q & 7] + WS_FLAG) + (rank * 4096 + FL_O + (ct * 48 + rg) * 2 + wn) * 32, call,
                                   __ATOMIC_RELAXED, __HIP_MEMORY_SCOPE_SYSTEM);
            }
            STAMP(4);
        }
    }
}

}

extern "C" void kernel_launch(int rank, int nranks, ncclComm_t comm,
                              hipStream_t stream,
                              void* const* d_in, const int* in_sizes, int n_in,
                              void* d_out, int out_size,
                              const int* in_dtypes, int out_dtype,
                              void* const* d_out_all,
                              void* const* d_ws_all, size_t ws_bytes) {
    if (nranks != NR || (unsigned)rank >= (unsigned)NR || n_in != 4 || ws_bytes < WS_NEED ||
        in_sizes[0] != MM * KS || in_sizes[1] != KS * NN || out_size != MS * NN) {
        fprintf(stderr, "kernel_launch: unexpected problem shape\n");
        return;
    }
    Params P;
    P.x = (const uint8_t*)d_in[0];
    P.w = (const uint8_t*)d_in[1];
    P.sx = (const float*)d_in[2];
    P.sw = (const float*)d_in[3];
    P.out = (float*)d_out;
    for (int p = 0; p < NR; ++p) P.ws[p] = (uint8_t*)d_ws_all[p];
    P.rank = rank;
    P.pad = 0;
    hipLaunchKernelGGL(fused_kernel, dim3(GRID), dim3(NTHR), 0, stream, P);
    HIP_CHECK(hipGetLastError());
}
```

```cpp
#include <hip/hip_runtime.h>
#include <rccl/rccl.h>
#include <cstdio>
#include <cstdint>

#define HIP_CHECK(expr) do { hipError_t _e = (expr); if (_e != hipSuccess) { \
    fprintf(stderr, "HIP error %s at %s:%d\n", hipGetErrorString(_e), __FILE__, __LINE__); \
    return; } } while (0)

typedef unsigned u32;
typedef unsigned long long u64;
typedef unsigned u32x4 __attribute__((ext_vector_type(4)));
typedef int i32x8 __attribute__((ext_vector_type(8)));
typedef float f32x4 __attribute__((ext_vector_type(4)));
typedef __attribute__((address_space(1))) const void gvoid;
typedef __attribute__((address_space(3))) void lvoid;
typedef __attribute__((address_space(3))) uint8_t lbyte;
typedef __attribute__((address_space(3))) u32 lu32;
typedef __attribute__((address_space(3))) unsigned short lu16;
typedef __attribute__((address_space(3))) u32x4 lu32x4;

namespace {

constexpr int NR = 8;
constexpr int MM = 4096, KK = 4096, NN = 16384;
constexpr int KS = KK / NR;
constexpr int MS = MM / NR;
constexpr int FG = 7, GG = 9;
constexpr int F = 32 * FG, G = 32 * GG;
static_assert(F + G == MS, "rows");
constexpr int XGREM = 3 * FG + 4 * GG;
constexpr int XREM = 32 * XGREM;
constexpr int XROWS = XREM + MS;
constexpr int NSTEP = 32;

constexpr int GRID = 256, NSEND = 64, NTHR = 512;
constexpr int NENG = 56;
#define WIN 9
#define LOOK 3
#define NWK 4
#define POLL_SLEEP 16
#define I8A 2

constexpr int SLOT = 6144 + 256 + 2048;
constexpr int LIST_MAX = 1022;
constexpr u32 H_DENSE = 1u << 11, H_RAW = 1u << 12, H_OVF = 1u << 13;
constexpr int SL_X = 0, SL_W = 256, SL_O = SL_W + 1024, SL_END = SL_O + 32 * 48 * 2;
constexpr int FL_X = 0, FL_W = 256, FL_O = FL_W + 128, FL_END = FL_O + 32 * 48 * 2;
static_assert(FL_END <= 4096, "flags");

constexpr size_t WS_CNT = 262144;
constexpr size_t WS_TMO = 32768;
constexpr size_t WS_TICK = 49152;
constexpr size_t WS_XREM = 65536;
constexpr size_t WS_XOWN = WS_XREM + 2048;
constexpr size_t WS_WCNT = 131072;
constexpr size_t WS_FLAG = 28u << 20;
constexpr size_t WS_DBG = 524288;
constexpr size_t WS_GW = 1u << 20;
constexpr size_t WS_GX = WS_GW + (size_t)NSTEP * 32 * 16384;
constexpr size_t XSLAB = (size_t)XROWS * 128;
constexpr size_t WS_PK = 32u << 20;
constexpr size_t PK_BYTES = 40u << 20;
static_assert(WS_GX + NSTEP * XSLAB <= WS_FLAG && WS_FLAG + NR * 4096 * 128 <= WS_PK, "staging");
static_assert((size_t)SL_END * SLOT <= PK_BYTES, "packed");
constexpr size_t WS_NEED = WS_PK + 2 * PK_BYTES;

constexpr int STAGE = 49152;
constexpr int LDS_CTL = 3 * STAGE;
constexpr int NS = 16;
constexpr int ESLOT = 6400;
constexpr int EL_PLANE = NS * ESLOT;
constexpr int EL_TAB = EL_PLANE + 8 * 2048;
constexpr int EL_GO = EL_TAB + 1024 * 16;
constexpr int EL_FULL = EL_GO + 64;
constexpr int EL_FREE = EL_FULL + 64;
constexpr int EL_ISS = EL_FREE + 64;
constexpr int EL_DEAD = EL_ISS + 4;
constexpr int EL_END = EL_ISS + 64;
static_assert(EL_END + 1024 <= LDS_CTL, "engine LDS");
constexpr u32 UNIT_END = 0xffffu;
constexpr u64 DEADLINE = 40000000ull;

constexpr int U1 = XGREM;
constexpr int U2 = U1 + 16 * I8A;
constexpr int U3 = U2 + XGREM;
constexpr int U4 = U3 + 16 * I8A;
constexpr int U5 = U4 + 32 * (8 - I8A);
constexpr int U6 = U5 + 32;

struct Params {
    const uint8_t* x;
    const uint8_t* w;
    const float* sx;
    const float* sw;
    float* out;
    uint8_t* ws[NR];
    int rank;
    int pad;
};

__device__ __forceinline__ void st_sys(void* p, u32x4 v) {
    asm volatile("global_store_dwordx4 %0, %1, off sc0 sc1\n\ts_nop 1" :: "v"(p), "v"(v) : "memory");
}
#define WAIT_VM(n) asm volatile("s_waitcnt vmcnt(" #n ")" ::: "memory")
#define CBAR() asm volatile("" ::: "memory")
#define DPP(v, ctl, rm) (u32)__builtin_amdgcn_update_dpp(0, (int)(v), ctl, rm, 0xf, false)

__device__ __forceinline__ u32 wave_scan_add(u32 v) {
    v += DPP(v, 0x111, 0xf); v += DPP(v, 0x112, 0xf); v += DPP(v, 0x114, 0xf); v += DPP(v, 0x118, 0xf);
    v += DPP(v, 0x142, 0xa); v += DPP(v, 0x143, 0xc);
    return v;
}
__device__ __forceinline__ u32 wave_sum(u32 v) { return __builtin_amdgcn_readlane(wave_scan_add(v), 63); }
__device__ __forceinline__ u32 wave_umax(u32 v) {
    v = max(v, DPP(v, 0x111, 0xf)); v = max(v, DPP(v, 0x112, 0xf)); v = max(v, DPP(v, 0x114, 0xf));
    v = max(v, DPP(v, 0x118, 0xf)); v = max(v, DPP(v, 0x142, 0xa)); v = max(v, DPP(v, 0x143, 0xc));
    return __builtin_amdgcn_readlane(v, 63);
}
__device__ __forceinline__ int block_of(int r, int part) {
    return (r & 1) ? 4 * part + (r >> 1) : 4 * (r >> 1) + part;
}
__device__ __forceinline__ float silu(float v) {
    const float z = __builtin_amdgcn_fmed3f(v, -60.f, 60.f);
    const float e = __builtin_amdgcn_exp2f(z * -1.4426950408889634f);
    return v * __builtin_amdgcn_rcpf(1.0f + e);
}

template <bool ACC>
__device__ __forceinline__ void put_unit(uint8_t* slot, const u32 (&pay)[24], const u32 (&pl)[8], u32 hdr,
                                         lbyte* scr, int lane) {
#pragma unroll
    for (int k = 0; k < 6; ++k)
        st_sys(slot + k * 1024 + lane * 16, u32x4{pay[4 * k], pay[4 * k + 1], pay[4 * k + 2], pay[4 * k + 3]});
    constexpr int NQ = ACC ? 4 : 8;
    u32 q[NQ], cnt = 0;
#pragma unroll
    for (int j = 0; j < NQ; ++j) {
        q[j] = ACC ? (pl[2 * j] >> 4) | pl[2 * j + 1] : pl[j];
        u32 t = q[j] | (q[j] >> 1);
        if (ACC) { t |= t >> 2; t &= 0x11111111u; } else t &= 0x55555555u;
        cnt += __builtin_popcount(t);
    }
    const u32 inc = wave_scan_add(cnt);
    const u32 total = __builtin_amdgcn_readlane(inc, 63);
    if ((hdr & H_DENSE) || total > LIST_MAX) {
        st_sys(slot + 6400 + lane * 16, u32x4{pl[0], pl[1], pl[2], pl[3]});
        st_sys(slot + 6400 + 1024 + lane * 16, u32x4{pl[4], pl[5], pl[6], pl[7]});
        if (lane < 16) st_sys(slot + 6144 + lane * 16, u32x4{lane == 0 ? (hdr | H_DENSE) : 0u, 0u, 0u, 0u});
    } else {
        *(lu32*)(scr + lane * 4) = lane == 0 ? (hdr | total) : 0u;
        if (lane == 0) *(lu16*)(scr + 4 + 2 * total) = 0;
        CBAR();
        lbyte* wp = scr + 4 + 2 * (inc - cnt);
#pragma unroll
        for (int j = 0; j < NQ; ++j) {
            const u32 dw = ACC ? ((j >> 1) << 14) | (lane << 8) | ((j & 1) << 7) : ((j >> 2) << 14) | (lane << 8) | ((j & 3) << 6);
            u32 p = q[j];
            while (p) {
                const u32 sh = __builtin_ctz(p) & (ACC ? 28u : 30u);
                const u32 val = (p >> sh) & (ACC ? 15u : 3u);
                *(lu16*)wp = (unsigned short)(dw | (sh << (ACC ? 2 : 1)) | val);
                p ^= val << sh;
                wp += 2;
            }
        }
        CBAR();
        const u32 nb = max(4u + 2u * total, ACC ? 128u : 256u);
        for (u32 o = lane * 16; o < nb; o += 1024) st_sys(slot + 6144 + o, *(lu32x4*)(scr + o));
    }
}

__device__ __forceinline__ u32 pick_base(u32 esum4  ) {
    const int b = ((int)esum4 - 33587) >> 13;
    return (u32)min(max(b, 0), 24);
}
__device__ __forceinline__ u32 bfi(u32 m, u32 a, u32 b) { return (a & m) | (b & ~m); }
__device__ __forceinline__ void enc_op(const u32x4* v, u32 base4, u32 (&pay)[24], u32 (&pl)[8]) {
#pragma unroll
    for (int j = 0; j < 8; ++j) {
        u32 u[4];
#pragma unroll
        for (int d = 0; d < 4; ++d) u[d] = (v[j][d] | 0x80808080u) - base4;
        const u32 u31 = u[3] << 1;
        pl[j] = bfi(0x0F0F0F0Fu, bfi(0x03030303u, u[0] >> 5, u[1] >> 3), bfi(0x30303030u, u[2] >> 1, u31));
        pay[3 * j + 0] = bfi(0x60606060u, u[3] << 5, bfi(0x7F7F7F7Fu, u[0], v[j][0]));
        pay[3 * j + 1] = bfi(0x60606060u, u[3] << 3, bfi(0x7F7F7F7Fu, u[1], v[j][1]));
        pay[3 * j + 2] = bfi(0x60606060u, bfi(0x20202020u, u31, v[j][3] >> 1), bfi(0x7F7F7F7Fu, u[2], v[j][2]));
    }
}
__device__ __forceinline__ u32x4 dec_op(u32 p0, u32 p1, u32 p2, u32 pl, u32 base4) {
    const u32 c3 = bfi(0x0F0F0F0Fu, bfi(0x03030303u, p0 >> 5, p1 >> 3), p2 >> 1);
    u32x4 v;
    v[0] = bfi(0x7F7F7F7Fu, (bfi(0x1F1F1F1Fu, p0, pl << 5) & 0x7F7F7F7Fu) + base4, p0);
    v[1] = bfi(0x7F7F7F7Fu, (bfi(0x1F1F1F1Fu, p1, pl << 3) & 0x7F7F7F7Fu) + base4, p1);
    v[2] = bfi(0x7F7F7F7Fu, (bfi(0x1F1F1F1Fu, p2, pl << 1) & 0x7F7F7F7Fu) + base4, p2);
    v[3] = bfi(0x7F7F7F7Fu, (bfi(0x1F1F1F1Fu, c3, pl >> 1) & 0x7F7F7F7Fu) + base4, p2 << 1);
    return v;
}

struct Unit {
    u32 slot;
    u32 flag;
    u32 dst;
    u32 kind;
    u32 sig;
};
__device__ __forceinline__ Unit unit_of(int n, int r, int p) {
    Unit u;
    if (n >= U6) {
        const bool same = ((r ^ p) & 1) == 0;
        const int v = n - U6, wn = v & 1, h = v >> 1;
        int i8, part, g, rg, row;
        if (same) {
            i8 = h / (4 * FG);
            const int t = h - i8 * (4 * FG);
            part = t / FG; g = t - part * FG;
            rg = FG * (((r >> 1) - (p >> 1) - 1) & 3) + g;
            row = 32 * g;
        } else {
            i8 = h / (3 * GG);
            const int t = h - i8 * (3 * GG);
            const int pi = t / GG, jr = r >> 1;
            g = t - pi * GG;
            part = pi + (pi >= jr);
            rg = 3 * FG + (jr > part ? jr - 1 : jr) * GG + g;
            row = F + 32 * g;
        }
        const int id = ((part * 8 + i8) * 48 + rg) * 2 + wn;
        u.slot = SL_O + id; u.flag = FL_O + id; u.kind = 2; u.sig = 0;
        u.dst = (u32)(row * NN + block_of(p, part) * 1024 + i8 * 128 + wn * 64) * 4u;
        return u;
    }
    const bool isx = n < U1 || (n >= U2 && n < U3) || n >= U5;
    if (isx) {
        int sg, kp;
        if (n < U1) { sg = n; kp = 0; }
        else if (n < U3) { sg = n - U2; kp = 1; }
        else { sg = XGREM + ((n - U5) & 15); kp = (n - U5) >> 4; }
        int gg;
        if (sg < 3 * FG) {
            const int s = sg / FG;
            gg = 16 * (2 * (((r >> 1) + 1 + s) & 3) + (r & 1)) + sg - s * FG;
        } else if (sg < XGREM) {
            const int j = (sg - 3 * FG) / GG;
            gg = 16 * (2 * j + 1 - (r & 1)) + FG + sg - 3 * FG - j * GG;
        } else {
            gg = 16 * r + sg - XGREM;
        }
        u.slot = SL_X + gg * 2 + kp; u.flag = FL_X + gg * 2 + kp; u.kind = 0;
        u.dst = (u32)(WS_GX + (size_t)(p * 4 + 2 * kp) * XSLAB) + (u32)sg * 4096u;
        u.sig = (u32)(sg < XGREM ? WS_XREM : WS_XOWN) + (u32)(p * 2 + kp) * 128u;
        return u;
    }
    int i8, kc, t;
    if (n < U2) { const int v = n - U1; i8 = v >> 4; kc = (v >> 3) & 1; t = v & 7; }
    else if (n < U4) { const int v = n - U3; i8 = v >> 4; kc = 2 + ((v >> 3) & 1); t = v & 7; }
    else { const int v = n - U4; i8 = I8A + (v >> 5); kc = (v >> 3) & 3; t = v & 7; }
    const int part = t >> 1, half = t & 1, ct = part * 8 + i8, step = p * 4 + kc;
    u.slot = SL_W + ((kc * 128 + block_of(r, part) * 8 + i8) * 2 + half);
    u.flag = FL_W + kc * 32 + ct;
    u.kind = 1 | (half << 2);
    u.dst = (u32)WS_GW + (u32)(step * 32 + ct) * 16384u + (u32)half * 8192u;
    u.sig = (u32)WS_WCNT + (u32)(ct * 32 + step) * 128u;
    return u;
}

#define STAMP(k) do { } while (0)

__global__ void __launch_bounds__(NTHR, 2) fused_kernel(Params P) {
    __shared__ __attribute__((aligned(1024))) uint8_t lds_[LDS_CTL];
    lbyte* const lds = (lbyte*)lds_;
    const int tid = threadIdx.x;
    const int lane = tid & 63;
    const int wave = __builtin_amdgcn_readfirstlane(tid >> 6);
    const int b = blockIdx.x;
    const int rank = P.rank;
    uint8_t* const myws = P.ws[rank];
    u32* const tmo = (u32*)(myws + WS_TMO);

    STAMP(0);
    u32* const cntp = (u32*)(myws + WS_CNT + (size_t)(b * 8 + wave) * 128);
    const u32 ncalls = __hip_atomic_load(cntp, __ATOMIC_RELAXED, __HIP_MEMORY_SCOPE_AGENT);
#define GET_CALL() \
    const u32 call = __builtin_amdgcn_readfirstlane(ncalls) + 1; \
    if (lane == 0) __hip_atomic_store(cntp, call, __ATOMIC_RELAXED, __HIP_MEMORY_SCOPE_AGENT); \
    const size_t pkoff = WS_PK + (size_t)(call & 1) * PK_BYTES

    if (b < NSEND) {
        if (b >= NENG) return;
        const int p = (rank + 1 + b % 7) & 7;
        const int nend = U6 + ((((rank ^ p) & 1) == 0) ? 64 * FG : 48 * GG);
        for (int n = tid; n < nend; n += NTHR) {
            const Unit u = unit_of(n, rank, p);
            *(lu32x4*)(lds + EL_TAB + n * 16) = u32x4{u.slot | (u.kind << 16), u.flag, u.dst, u.sig};
        }
        if (tid < (EL_END - EL_GO) / 4) *(lu32*)(lds + EL_GO + tid * 4) = 0;
        *(lu32x4*)(lds + EL_PLANE + tid * 32) = u32x4{0u, 0u, 0u, 0u};
        *(lu32x4*)(lds + EL_PLANE + tid * 32 + 16) = u32x4{0u, 0u, 0u, 0u};
        __syncthreads();
        if (wave >= 2 + NWK) return;
        GET_CALL();
        const uint8_t* const pk = P.ws[p] + pkoff;
        const u32 lds_addr = (u32)(uintptr_t)lds;
        auto lds_get = [&](int off) {
            return (u32)__builtin_amdgcn_readfirstlane(
                __hip_atomic_load((lu32*)(lds + off), __ATOMIC_RELAXED, __HIP_MEMORY_SCOPE_WORKGROUP));
        };
        auto lds_put = [&](int off, u32 v) {
            CBAR();
            if (lane == 0) __hip_atomic_store((lu32*)(lds + off), v, __ATOMIC_RELAXED, __HIP_MEMORY_SCOPE_WORKGROUP);
            CBAR();
        };

        if (wave == 0) {
            __builtin_amdgcn_s_setprio(3);
            WAIT_VM(0);
            const u32 vo0 = lane * 16, vo1 = lane * 16 + 4096;
            constexpr int LAG = (WIN - 1) / 7 + 1, JPUB = (WIN - 1) % 7;
            u32 pub = 0;
#define DMA1(J, VO, OFF) do { \
                asm volatile("s_waitcnt vmcnt(%0)" :: "i"(WIN - 1) : "memory"); \
                if (J == JPUB && k >= LAG && pub <= k - LAG) { lds_put(EL_FULL + (pub % NS) * 4, pub + 1); ++pub; } \
                asm volatile("s_mov_b32 m0, %2\n\ts_nop 0\n\tglobal_load_lds_dwordx4 %0, %1 offset:" #OFF " sc0 sc1" \
                             :: "v"(VO), "s"(src), "s"(J < 4 ? la : la + 4096) : "memory"); \
            } while (0)
            for (u32 k = 0;; ++k) {
                const int sl = k % NS;
                u32 go = lds_get(EL_GO + sl * 4);
                if ((go >> 16) != ((k + 1) & 0xffffu) || (go & 0xffffu) == UNIT_END) {
                    WAIT_VM(0);
                    for (; pub < k; ++pub) lds_put(EL_FULL + (pub % NS) * 4, pub + 1);
                    while ((go >> 16) != ((k + 1) & 0xffffu)) {
                        __builtin_amdgcn_s_sleep(1);
                        go = lds_get(EL_GO + sl * 4);
                    }
                    if ((go & 0xffffu) == UNIT_END) break;
                }
                lds_put(EL_ISS, k + 1);
                const u32 t0 = lds_get(EL_TAB + (go & 0xffffu) * 16);
                const uint8_t* src = pk + (size_t)(t0 & 0xffffu) * SLOT;
                const u32 la = lds_addr + sl * ESLOT;
                u32 keep;
                asm volatile("s_mov_b32 %0, m0\n\ts_nop 4" : "=s"(keep) :: "memory");
                DMA1(0, vo0, 0); DMA1(1, vo0, 1024); DMA1(2, vo0, 2048); DMA1(3, vo0, 3072);
                DMA1(4, vo1, 0); DMA1(5, vo1, 1024);
                asm volatile("s_waitcnt vmcnt(%0)" :: "i"(WIN - 1) : "memory");
                if (6 == JPUB && k >= LAG && pub <= k - LAG) { lds_put(EL_FULL + (pub % NS) * 4, pub + 1); ++pub; }
                if (lane < ((t0 >> 16) == 2 ? 8 : 16))
                    asm volatile("s_mov_b32 m0, %2\n\ts_nop 0\n\tglobal_load_lds_dwordx4 %0, %1 offset:2048 sc0 sc1"
                                 :: "v"(vo1), "s"(src), "s"(la + 4096) : "memory");
                asm volatile("s_mov_b32 m0, %0" :: "s"(keep) : "memory");
            }
            STAMP(4);
            return;
        }
        if (wave == 1) {
            const u32* const flags = (const u32*)(myws + WS_FLAG) + p * 4096 * 32;
            u32* const tick = (u32*)(myws + WS_TICK) + p * 32;
            const u32 npair = nend / 2, tbase = (call - 1) * npair - 8;
            u32 pair = b / 7;
            bool dead = false;
            u32 k = 0;
            for (; pair < npair; k += 2) {
                while (lds_get(EL_ISS) + LOOK < k) __builtin_amdgcn_s_sleep(1);
                u32 tk;
                const u32 fl = *(lu32*)(lds + EL_TAB + (2 * pair + (lane & 1)) * 16 + 4);
                u32 fv;
                asm volatile("s_mov_b64 exec, 1\n\tglobal_atomic_add %0, %2, %3, off sc0\n\ts_mov_b64 exec, 3\n\t"
                             "global_load_dword %1, %4, off sc0 sc1\n\ts_mov_b64 exec, -1\n\ts_waitcnt vmcnt(0)"
                             : "=&v"(tk), "=&v"(fv) : "v"(tick), "v"(1u), "v"(flags + fl * 32) : "memory");
                if (!dead && (__ballot(fv >= call) & 3) != 3) {
                    const u64 t0 = __builtin_amdgcn_s_memrealtime();
                    while (!__all(__hip_atomic_load(flags + fl * 32, __ATOMIC_RELAXED, __HIP_MEMORY_SCOPE_SYSTEM) >= call)) {
                        if (__hip_atomic_load(tmo, __ATOMIC_RELAXED, __HIP_MEMORY_SCOPE_AGENT) == call ||
                            __builtin_amdgcn_s_memrealtime() - t0 > DEADLINE) {
                            if (lane == 0) __hip_atomic_store(tmo, call, __ATOMIC_RELAXED, __HIP_MEMORY_SCOPE_AGENT);
                            dead = true;
                            lds_put(EL_DEAD, 1);
                            break;
                        }
                        __builtin_amdgcn_s_sleep(2);
                    }
                }
#pragma unroll
                for (u32 h = 0; h < 2; ++h) {
                    const u32 kk = k + h;
                    while (lds_get(EL_FREE + (kk % NS) * 4) + NS < kk + 1) __builtin_amdgcn_s_sleep(1);
                    lds_put(EL_GO + (kk % NS) * 4, ((kk + 1) << 16) | (2 * pair + h));
                }
                pair = __builtin_amdgcn_readfirstlane(tk) - tbase;
            }
            for (u32 kk = k; kk < k + NWK; ++kk) {
                while (lds_get(EL_FREE + (kk % NS) * 4) + NS < kk + 1) __builtin_amdgcn_s_sleep(1);
                lds_put(EL_GO + (kk % NS) * 4, ((kk + 1) << 16) | UNIT_END);
            }
            return;
        }
        lbyte* const plane = lds + EL_PLANE + wave * 2048;
        const float scale0 = P.sx[0] * P.sw[0];
        for (u32 k = wave - 2;; k += NWK) {
            const int sl = k % NS;
            lbyte* const s = lds + sl * ESLOT;
            u32 go;
            bool fin = false;
            while (lds_get(EL_FULL + sl * 4) != k + 1) {
                go = lds_get(EL_GO + sl * 4);
                if (go == (((k + 1) << 16) | UNIT_END)) { fin = true; break; }
                __builtin_amdgcn_s_sleep(1);
            }
            if (fin) break;
            CBAR();
            go = lds_get(EL_GO + sl * 4);
            const u32x4 te = *(lu32x4*)(lds + EL_TAB + (go & 0xffffu) * 16);
            u32 le = *(lu32*)(s + 6144 + lane * 4);
            u32 pay[24], pl[8];
#pragma unroll
            for (int j = 0; j < 6; ++j) {
                const u32x4 t = *(lu32x4*)(s + j * 1024 + lane * 16);
                pay[4 * j] = t[0]; pay[4 * j + 1] = t[1]; pay[4 * j + 2] = t[2]; pay[4 * j + 3] = t[3];
            }
            const float scale = lds_get(EL_DEAD) ? __builtin_nanf("") : scale0;
            lds_put(EL_FREE + sl * 4, k + 1);
            const u32 kind = __builtin_amdgcn_readfirstlane(te[0]) >> 16;
            const u32 dst = __builtin_amdgcn_readfirstlane(te[2]);
            const uint8_t* const src = pk + (size_t)(__builtin_amdgcn_readfirstlane(te[0]) & 0xffffu) * SLOT;
            const bool acc = kind == 2;
            const u32 hdr = __builtin_amdgcn_readfirstlane(le);
            const u32 first = acc ? 32u : 64u;
            auto scatter = [&](u32 e2) {
#pragma unroll
                for (int h = 0; h < 2; ++h) {
                    const u32 e = h ? e2 >> 16 : e2 & 0xffffu;
                    const u32 val = acc ? (e & 15u) << (((e >> 2) & 24u) + 4u) : (e & 3u) << ((e >> 1) & 30u);
                    const u32 ad = acc ? ((e >> 4) & 0x7f8u) | ((e >> 2) & 4u) : (e >> 4) & 0x7fcu;
                    if (val) __hip_atomic_fetch_or((lu32*)(plane + ad), val, __ATOMIC_RELAXED, __HIP_MEMORY_SCOPE_WAVEFRONT);
                }
            };
            if (hdr & H_DENSE) {
                u32 keep;
                asm volatile("s_waitcnt lgkmcnt(0)\n\ts_mov_b32 %0, m0\n\ts_mov_b32 m0, %2\n\ts_nop 0\n\t"
                             "global_load_lds_dwordx4 %1, off sc0 sc1\n\t"
                             "global_load_lds_dwordx4 %1, off offset:1024 sc0 sc1\n\t"
                             "s_mov_b32 m0, %0\n\ts_waitcnt vmcnt(0)"
                             : "=&s"(keep) : "v"(src + 6400 + lane * 16), "s"(lds_addr + EL_PLANE + wave * 2048) : "memory");
            } else {
                if (lane == 0 || (u32)lane >= first) le = 0;
                scatter(le);
                const u32 ndw = (2u + (hdr & 0x7ffu) + 1u) >> 1;
                for (u32 o = first; o < ndw; o += 64) {
                    u32 e2 = 0;
                    if (o + lane < ndw)
                        e2 = __hip_atomic_load((const u32*)(src + 6144) + o + lane, __ATOMIC_RELAXED, __HIP_MEMORY_SCOPE_SYSTEM);
                    scatter(e2);
                }
            }
            CBAR();
#pragma unroll
            for (int j = 0; j < 2; ++j) {
                const u32x4 t = *(lu32x4*)(plane + j * 1024 + lane * 16);
                pl[4 * j] = t[0]; pl[4 * j + 1] = t[1]; pl[4 * j + 2] = t[2]; pl[4 * j + 3] = t[3];
            }
            *(lu32x4*)(plane + lane * 16) = u32x4{0u, 0u, 0u, 0u};
            *(lu32x4*)(plane + 1024 + lane * 16) = u32x4{0u, 0u, 0u, 0u};

            if (!acc) {
                const u32 base4 = (hdr >> 16) * 0x04040404u;
                const u32 half = kind >> 2;
                const bool isw = kind & 1;
                uint8_t* d0 = myws + dst + ((lane ^ (isw ? half * 8 : 0)) << 4);
                uint8_t* d1 = myws + dst + (isw ? 4096 : XSLAB) + ((lane ^ (isw ? half * 8 + 4 : 0)) << 4);
#pragma unroll
                for (int j = 0; j < 8; ++j) {
                    const u32x4 v = dec_op(pay[3 * j], pay[3 * j + 1], pay[3 * j + 2], pl[j], base4);
                    st_sys((j < 4 ? d0 : d1) + (j & 3) * 1024, v);
                }
                WAIT_VM(0);
                if (lane == 0)
                    __hip_atomic_fetch_add((u32*)(myws + te[3]), 1u, __ATOMIC_RELAXED, __HIP_MEMORY_SCOPE_SYSTEM);
            } else {
                const float c1 = __builtin_bit_cast(float, (hdr >> 16) << 23);
                uint8_t* d0 = (uint8_t*)P.out + dst + (size_t)(lane >> 4) * (4 * NN * 4) + (lane & 15) * 16;
                float a[32];
#pragma unroll
                for (int j = 0; j < 8; ++j) {
                    const u32 l0 = __builtin_amdgcn_perm(pay[3 * j + 2], pl[j], 0x05010400u);
                    const u32 l1 = __builtin_amdgcn_perm(pay[3 * j + 2], pl[j], 0x07030602u);
                    a[4 * j + 0] = __uint_as_float(__builtin_amdgcn_perm(pay[3 * j], l0, 0x05040100u));
                    a[4 * j + 1] = __uint_as_float(__builtin_amdgcn_perm(pay[3 * j], l0, 0x07060302u));
                    a[4 * j + 2] = __uint_as_float(__builtin_amdgcn_perm(pay[3 * j + 1], l1, 0x05040100u));
                    a[4 * j + 3] = __uint_as_float(__builtin_amdgcn_perm(pay[3 * j + 1], l1, 0x07060302u));
                }
                if (hdr & H_RAW) {
                } else if (hdr & H_OVF) {
#pragma unroll
                    for (int i = 0; i < 32; ++i) {
                        const int t = __float_as_int(a[i]);
                        const float fix = (t & 16) ? (t < 0 ? 4294967296.f : -4294967296.f) : 0.f;
                        a[i] = ((float)(t & ~16) + fix) * c1;
                    }
                } else {
#pragma unroll
                    for (int i = 0; i < 32; ++i) a[i] = (float)__float_as_int(a[i]) * c1;
                }
#pragma unroll
                for (int j = 0; j < 8; ++j) {
                    f32x4 y;
#pragma unroll
                    for (int t = 0; t < 4; ++t) y[t] = silu(a[4 * j + t] * scale);
                    st_sys(d0 + (size_t)((j >> 2) * 16 + (j & 3)) * (NN * 4), __builtin_bit_cast(u32x4, y));
                }
            }
        }
        return;
    }

    const int g = b - NSEND;
    lbyte* const scr = lds + wave * 4096;
    const int task = g + 192 * wave;
    u32x4 in[16];
    if (task < 256) {
        const uint8_t* src = P.x + (size_t)((task >> 1) * 32 + (lane >> 3)) * KS + (task & 1) * 256 + (lane & 7) * 16;
#pragma unroll
        for (int j = 0; j < 8; ++j) in[j] = *(const u32x4*)(src + (j & 3) * 8 * KS + (j >> 2) * 128);
    } else if (task < 768) {
        const int t = task - 256;
        const uint8_t* src = P.w + (size_t)(((t >> 4) & 3) * 128 + 16 * (lane >> 3)) * NN + (t & 15) * 1024 + (t >> 6) * 128 + 16 * (lane & 7);
#pragma unroll
        for (int j = 0; j < 16; ++j) in[j] = *(const u32x4*)(src + (size_t)j * NN);
    }
    GET_CALL();
    if (task < 768) {
        uint8_t* const pk = myws + pkoff;
        u32 pay[24], pl[8];
        if (task < 256) {
            const int gg = task >> 1, kp = task & 1;
            u32x4* const v = in;
            u32 es = 0;
#pragma unroll
            for (int j = 0; j < 8; ++j) es = __builtin_amdgcn_sad_u8(v[j][j & 3] & 0x7C7C7C7Cu, 0u, es);
            const u32 base = pick_base(wave_sum(es));
            enc_op(v, base * 0x04040404u, pay, pl);
            put_unit<false>(pk + (size_t)(SL_X + task) * SLOT, pay, pl, base << 16, scr, lane);
            const int q = gg >> 4, lg = gg & 15;
            auto needs = [&](int r) { return r == q || (((r ^ q) & 1) == 0) == (lg < FG); };
            const bool self = needs(rank);
            if (self) {
                const int sg = q == rank ? XGREM + lg
                             : lg < FG ? FG * (((q >> 1) - (rank >> 1) - 1) & 3) + lg
                                       : 3 * FG + GG * (q >> 1) + lg - FG;
                uint8_t* d = myws + WS_GX + (size_t)(rank * 4 + 2 * kp) * XSLAB + (size_t)sg * 4096 + lane * 16;
#pragma unroll
                for (int j = 0; j < 8; ++j) st_sys(d + (j >> 2) * XSLAB + (j & 3) * 1024, v[j]);
            }
            WAIT_VM(0);
            if (lane < NR && lane != rank && needs(lane))
                __hip_atomic_store((u32*)(P.ws[lane] + WS_FLAG) + (rank * 4096 + FL_X + task) * 32, call, __ATOMIC_RELAXED, __HIP_MEMORY_SCOPE_SYSTEM);
            if (self && lane == 0)
                __hip_atomic_fetch_add((u32*)(myws + (q == rank ? WS_XOWN : WS_XREM)) + (rank * 2 + kp) * 32, 1u, __ATOMIC_RELAXED, __HIP_MEMORY_SCOPE_SYSTEM);
        } else {
            const int t = task - 256;
            const int i8 = t >> 6, kc = (t >> 4) & 3, blk = t & 15;
            u32x4 o[16];
            u32 es = 0;
#pragma unroll
            for (int j = 0; j < 8; ++j) es = __builtin_amdgcn_sad_u8(in[2 * j][j & 3] & 0x7C7C7C7Cu, 0u, es);
            const u32 base = pick_base(wave_sum(es));
#pragma unroll
            for (int d = 0; d < 4; ++d)
#pragma unroll
                for (int e = 0; e < 4; ++e) {
                    const u32 t0 = __builtin_amdgcn_perm(in[4 * e + 1][d], in[4 * e][d], 0x05010400u);
                    const u32 t1 = __builtin_amdgcn_perm(in[4 * e + 1][d], in[4 * e][d], 0x07030602u);
                    const u32 t2 = __builtin_amdgcn_perm(in[4 * e + 3][d], in[4 * e + 2][d], 0x05010400u);
                    const u32 t3 = __builtin_amdgcn_perm(in[4 * e + 3][d], in[4 * e + 2][d], 0x07030602u);
                    o[4 * d + 0][e] = __builtin_amdgcn_perm(t2, t0, 0x05040100u);
                    o[4 * d + 1][e] = __builtin_amdgcn_perm(t2, t0, 0x07060302u);
                    o[4 * d + 2][e] = __builtin_amdgcn_perm(t3, t1, 0x05040100u);
                    o[4 * d + 3][e] = __builtin_amdgcn_perm(t3, t1, 0x07060302u);
                }
            uint8_t* const slot = pk + (size_t)(SL_W + (kc * 128 + blk * 8 + i8) * 2) * SLOT;
            enc_op(o, base * 0x04040404u, pay, pl);
            put_unit<false>(slot, pay, pl, base << 16, scr, lane);
            enc_op(o + 8, base * 0x04040404u, pay, pl);
            put_unit<false>(slot + SLOT, pay, pl, base << 16, scr, lane);
            const int a = blk >> 2, bb = blk & 3;
            const int re = 2 * a, ro = 2 * bb + 1;
            const int cte = bb * 8 + i8, cto = a * 8 + i8;
            const bool self = rank == re || rank == ro;
            const int cts = rank == re ? cte : cto;
            if (self) {
                uint8_t* d = myws + WS_GW + (size_t)((rank * 4 + kc) * 32 + cts) * 16384;
#pragma unroll
                for (int i = 0; i < 16; ++i) st_sys(d + i * 1024 + ((lane ^ ((i >> 2) << 2)) << 4), o[i]);
            }
            WAIT_VM(0);
            if (lane < 2) {
                const int r = lane ? ro : re;
                if (r != rank)
                    __hip_atomic_store((u32*)(P.ws[r & 7] + WS_FLAG) + (rank * 4096 + FL_W + kc * 32 + (lane ? cto : cte)) * 32, call, __ATOMIC_RELAXED, __HIP_MEMORY_SCOPE_SYSTEM);
            }
            if (self && lane == 0)
                __hip_atomic_fetch_add((u32*)(myws + WS_WCNT) + (cts * 32 + rank * 4 + kc) * 32, 2u, __ATOMIC_RELAXED, __HIP_MEMORY_SCOPE_SYSTEM);
        }
    }
    STAMP(1);

    const int xcd = g & 7, idx = g >> 3;
    const int part = xcd >> 1;
    const int wm = wave >> 1, wn = wave & 1;
    const int c = lane & 15, q4 = lane >> 4;
    const int ra0 = wm * 8192 + c * 128 + ((q4 ^ (c >> 1)) << 4);
    const int ra1 = ra0 ^ 64;
    const int rb0 = 32768 + (c & 3) * 4096 + ((((q4 << 3) | (4 * wn + (c >> 2))) ^ ((c & 3) << 2)) << 4);
    const u32 boff = (u32)wave * 2048u + (u32)lane * 16u;
    const uint8_t* const xbuf = myws + WS_GX;

    for (int round = 0; round < (idx < 8 ? 2 : 1); ++round) {
        const bool local = round == 1;
        const int i8 = local ? idx : idx / 3;
        const int rt = local ? (xcd & 1) : 3 * (xcd & 1) + idx % 3;
        const int ct = part * 8 + i8;
        const uint8_t* const wbuf = myws + WS_GW + (size_t)ct * 16384;
        const int rgw = rt * 8 + wave;
        const int sgw = local ? XGREM + rgw : (rgw < 3 * FG + part * GG ? rgw : rgw + GG);
        u32 aoff[4];
#pragma unroll
        for (int jj = 0; jj < 4; ++jj)
            aoff[jj] = (u32)sgw * 4096u + (u32)(8 * jj + (lane >> 3)) * 128u +
                       (u32)(((lane & 7) ^ ((4 * jj + (lane >> 4)) & 7)) << 4);

        const u32* const wpoll = (const u32*)(myws + WS_WCNT) + (ct * 32 + (lane & 7) * 4) * 32;
        const u32* const xpoll = (const u32*)(myws + (local ? WS_XOWN : WS_XREM)) + (lane & 7) * 64;
        const u32 xneed = call * (local ? 16u : (u32)XGREM);
        u32 wready = 0, xready = 0;
        auto stepno = [&](int s) { return ((rank + s) & 7) * 4 + (s >> 3); };
        auto ensure = [&](int s) {
            const int kc = s >> 3, src = (rank + s) & 7;
            const u32 wbit = 1u << (8 * kc + src), xbit = 1u << (8 * (kc >> 1) + src);
            if ((wready & wbit) && (xready & xbit)) return;
            const u64 t0 = __builtin_amdgcn_s_memrealtime();
            for (;;) {
                if (!(wready & wbit)) {
                    u32 v = 0;
                    if (lane < 8) v = __hip_atomic_load(wpoll + kc * 32, __ATOMIC_RELAXED, __HIP_MEMORY_SCOPE_SYSTEM);
                    wready |= (u32)__ballot(v >= 2u * call) << (8 * kc);
                }
                if ((wready & wbit) && !(xready & xbit)) {
                    u32 v = 0;
                    if (lane < 8) v = __hip_atomic_load(xpoll + (kc >> 1) * 32, __ATOMIC_RELAXED, __HIP_MEMORY_SCOPE_SYSTEM);
                    xready |= (u32)__ballot(v >= xneed) << (8 * (kc >> 1));
                }
                if ((wready & wbit) && (xready & xbit)) break;
                if (__hip_atomic_load(tmo, __ATOMIC_RELAXED, __HIP_MEMORY_SCOPE_AGENT) == call ||
                    __builtin_amdgcn_s_memrealtime() - t0 > DEADLINE) {
                    if (lane == 0) __hip_atomic_store(tmo, call, __ATOMIC_RELAXED, __HIP_MEMORY_SCOPE_AGENT);
                    WAIT_VM(0);
                    wready = xready = ~0u;
                    break;
                }
                __builtin_amdgcn_s_sleep(POLL_SLEEP);
            }
        };
        auto issue = [&](int s, int stage) {
            const int st = stepno(s);
            const uint8_t* xa = xbuf + (size_t)st * XSLAB;
            const uint8_t* wa = wbuf + (size_t)st * (32 * 16384);
            lbyte* la = lds + stage * STAGE + wave * 4096;
#pragma unroll
            for (int jj = 0; jj < 4; ++jj)
                __builtin_amdgcn_global_load_lds((gvoid*)(xa + aoff[jj]), (lvoid*)(la + jj * 1024), 16, 0, 17);
            lbyte* lb = lds + stage * STAGE + 32768 + wave * 2048;
            __builtin_amdgcn_global_load_lds((gvoid*)(wa + boff), (lvoid*)lb, 16, 0, 17);
            __builtin_amdgcn_global_load_lds((gvoid*)(wa + boff), (lvoid*)lb, 16, 1024, 17);
        };

        f32x4 acc[4][4];
#pragma unroll
        for (int m = 0; m < 4; ++m)
#pragma unroll
            for (int t = 0; t < 4; ++t) acc[m][t] = f32x4{0.f, 0.f, 0.f, 0.f};

        auto rdfrag = [&](i32x8 (&A)[4], i32x8 (&B)[4], int stage) {
            const lbyte* sb = lds + stage * STAGE;
#pragma unroll
            for (int t = 0; t < 4; ++t) {
                const u32x4 al = *(const lu32x4*)(sb + ra0 + t * 2048);
                const u32x4 ah = *(const lu32x4*)(sb + ra1 + t * 2048);
                A[t] = i32x8{(int)al[0], (int)al[1], (int)al[2], (int)al[3], (int)ah[0], (int)ah[1], (int)ah[2], (int)ah[3]};
                const u32x4 bl = *(const lu32x4*)(sb + rb0 + t * 1024);
                const u32x4 bh = *(const lu32x4*)(sb + rb0 + t * 1024 + 512);
                B[t] = i32x8{(int)bl[0], (int)bl[1], (int)bl[2], (int)bl[3], (int)bh[0], (int)bh[1], (int)bh[2], (int)bh[3]};
            }
        };
        auto mma = [&](const i32x8 (&A)[4], const i32x8 (&B)[4]) {
#pragma unroll
            for (int m = 0; m < 4; ++m)
#pragma unroll
                for (int t = 0; t < 4; ++t)
                    acc[m][t] = __builtin_amdgcn_mfma_scale_f32_16x16x128_f8f6f4(A[m], B[t], acc[m][t], 1, 1, 0, 0, 0, 0);
        };
#define KSTEP(XA, XB, YA, YB, ISSUE, NVM) do { \
            if (ISSUE && wave == 0) ensure(s + 3); \
            asm volatile("s_waitcnt vmcnt(" #NVM ") lgkmcnt(0)" ::: "memory"); \
            __builtin_amdgcn_s_barrier(); \
            asm volatile("" ::: "memory"); \
            __builtin_amdgcn_sched_barrier(0); \
            const int nst = stage == 2 ? 0 : stage + 1; \
            rdfrag(YA, YB, nst); \
            if (ISSUE) issue(s + 3, stage); \
            mma(XA, XB); \
            _Pragma("unroll") for (int z = 0; z < 8; ++z) { \
                __builtin_amdgcn_sched_group_barrier(0x8, 1, 0); \
                __builtin_amdgcn_sched_group_barrier(0x100, 2, 0); \
            } \
            _Pragma("unroll") for (int z = 0; z < 8; ++z) { \
                __builtin_amdgcn_sched_group_barrier(0x8, 1, 0); \
                __builtin_amdgcn_sched_group_barrier(0x10, 1, 0); \
            } \
            __builtin_amdgcn_sched_barrier(0); \
            stage = nst; ++s; \
        } while (0)

        if (wave == 0) { ensure(0); ensure(1); ensure(2); }
        asm volatile("s_waitcnt vmcnt(0) lgkmcnt(0)" ::: "memory");
        __builtin_amdgcn_s_barrier();
        asm volatile("" ::: "memory");
        issue(0, 0);
        issue(1, 1);
        issue(2, 2);
        WAIT_VM(12);
        __builtin_amdgcn_s_barrier();
        asm volatile("" ::: "memory");
        i32x8 A0[4], B0[4], A1[4], B1[4];
        rdfrag(A0, B0, 0);
        int stage = 0, s = 0;
        while (s < NSTEP - 4) {
            KSTEP(A0, B0, A1, B1, true, 6);
            KSTEP(A1, B1, A0, B0, true, 6);
        }
        KSTEP(A0, B0, A1, B1, true, 6);
        KSTEP(A1, B1, A0, B0, false, 6);
        KSTEP(A0, B0, A1, B1, false, 0);
        mma(A1, B1);

        const bool bad = __hip_atomic_load(tmo, __ATOMIC_RELAXED, __HIP_MEMORY_SCOPE_AGENT) == call;
        int ln = lane;
        asm volatile("" : "+v"(ln));
        if (local) {
            float scale = bad ? __builtin_nanf("") : P.sx[0] * P.sw[0];
            asm volatile("" : "+v"(scale));
            float* dst = P.out + (size_t)(rt * 256 + wm * 64 + 4 * (ln >> 4)) * NN + block_of(rank, part) * 1024 + i8 * 128 + wn * 64 + 4 * (ln & 15);
#pragma unroll
            for (int m = 0; m < 4; ++m)
#pragma unroll
                for (int r = 0; r < 4; ++r) {
                    f32x4 y;
#pragma unroll
                    for (int t = 0; t < 4; ++t) y[t] = silu(acc[m][t][r] * scale);
                    st_sys(dst + (size_t)(m * 16 + r) * NN, __builtin_bit_cast(u32x4, y));
                }
            STAMP(7);
        } else {
#pragma unroll
            for (int mp = 0; mp < 2; ++mp) {
                float amf = 0.f;
#pragma unroll
                for (int i = 0; i < 32; ++i) amf = fmaxf(amf, fabsf(acc[2 * mp + (i >> 4)][i & 3][(i >> 2) & 3]));
                const u32 am = wave_umax(__float_as_uint(amf));
                const int ex = min(max((int)(am >> 23), 40), 200);
                const float up = __builtin_bit_cast(float, (u32)(280 - ex) << 23);
                const float dn = __builtin_bit_cast(float, (u32)(ex - 26) << 23);
                u32 T[32], miss = 0, low = 0;
                int vote = 0;
#pragma unroll
                for (int i = 0; i < 32; ++i) {
                    const float a = acc[2 * mp + (i >> 4)][i & 3][(i >> 2) & 3];
                    const int I = (int)(a * up);
                    miss |= __float_as_uint((float)I * dn) ^ __float_as_uint(a);
                    low |= (u32)I;
                    if ((i & 3) == 0) vote += ((I >> 3) & 1) - (int)(((u32)I + (1u << 26)) >= (1u << 27));
                    T[i] = (u32)I;
                }
                u32 hdr;
                if (bad || (int)(am >> 23) > ex || __any(miss != 0)) {
                    hdr = H_RAW | H_DENSE;
#pragma unroll
                    for (int i = 0; i < 32; ++i)
                        T[i] = bad ? 0x7fc00000u : __float_as_uint(acc[2 * mp + (i >> 4)][i & 3][(i >> 2) & 3]);
                } else if (__any(low & 1) || (int)wave_sum((u32)vote) > 0) {
                    hdr = H_OVF | ((u32)(ex - 31) << 16);
#pragma unroll
                    for (int i = 0; i < 32; ++i)
                        T[i] = (T[i] << 5) | ((T[i] + (1u << 26)) >= (1u << 27) ? 16u : 0u);
                } else {
                    hdr = (u32)(ex - 30) << 16;
#pragma unroll
                    for (int i = 0; i < 32; ++i) T[i] <<= 4;
                }
                u32 pay[24], pl[8];
#pragma unroll
                for (int j = 0; j < 8; ++j) {
                    pay[3 * j] = __builtin_amdgcn_perm(T[4 * j + 1], T[4 * j], 0x07060302u);
                    pay[3 * j + 1] = __builtin_amdgcn_perm(T[4 * j + 3], T[4 * j + 2], 0x07060302u);
                    const u32 l0 = __builtin_amdgcn_perm(T[4 * j + 1], T[4 * j], 0x05010400u);
                    const u32 l1 = __builtin_amdgcn_perm(T[4 * j + 3], T[4 * j + 2], 0x05010400u);
                    pl[j] = __builtin_amdgcn_perm(l1, l0, 0x05040100u);
                    pay[3 * j + 2] = __builtin_amdgcn_perm(l1, l0, 0x07060302u);
                }
                const int id = (ct * 48 + rt * 8 + wm * 2 + mp) * 2 + wn;
                put_unit<true>(myws + pkoff + (size_t)(SL_O + id) * SLOT, pay, pl, hdr, scr, ln);
            }
            WAIT_VM(0);
            if (ln < 2) {
                const int rg = rt * 8 + wm * 2 + ln;
                int q;
                if (rg < 3 * FG) {
                    q = 2 * (((rank >> 1) + 1 + rg / FG) & 3) + (rank & 1);
                } else {
                    const int jj = (rg - 3 * FG) / GG;
                    q = 2 * (jj + (jj >= part)) + 1 - (rank & 1);
                }
                __hip_atomic_store((u32*)(P.ws[q & 7] + WS_FLAG) + (rank * 4096 + FL_O + (ct * 48 + rg) * 2 + wn) * 32, call,
                                   __ATOMIC_RELAXED, __HIP_MEMORY_SCOPE_SYSTEM);
            }
            STAMP(4);
        }
    }
}

}

extern "C" void kernel_launch(int rank, int nranks, ncclComm_t comm,
                              hipStream_t stream,
                              void* const* d_in, const int* in_sizes, int n_in,
                              void* d_out, int out_size,
                              const int* in_dtypes, int out_dtype,
                              void* const* d_out_all,
                              void* const* d_ws_all, size_t ws_bytes) {
    if (nranks != NR || (unsigned)rank >= (unsigned)NR || n_in != 4 || ws_bytes < WS_NEED ||
        in_sizes[0] != MM * KS || in_sizes[1] != KS * NN || out_size != MS * NN) {
        fprintf(stderr, "kernel_launch: unexpected problem shape\n");
        return;
    }
    Params P;
    P.x = (const uint8_t*)d_in[0];
    P.w = (const uint8_t*)d_in[1];
    P.sx = (const float*)d_in[2];
    P.sw = (const float*)d_in[3];
    P.out = (float*)d_out;
    for (int p = 0; p < NR; ++p) P.ws[p] = (uint8_t*)d_ws_all[p];
    P.rank = rank;
    P.pad = 0;
    hipLaunchKernelGGL(fused_kernel, dim3(GRID), dim3(NTHR), 0, stream, P);
    HIP_CHECK(hipGetLastError());
}
```

```cpp
#include <hip/hip_runtime.h>
#include <rccl/rccl.h>
#include <cstdio>
#include <cstdint>

#define HIP_CHECK(expr) do { hipError_t _e = (expr); if (_e != hipSuccess) { \
    fprintf(stderr, "HIP error %s at %s:%d\n", hipGetErrorString(_e), __FILE__, __LINE__); \
    return; } } while (0)

typedef unsigned u32;
typedef unsigned long long u64;
typedef unsigned u32x4 __attribute__((ext_vector_type(4)));
typedef unsigned u32x2 __attribute__((ext_vector_type(2)));
typedef int i32x8 __attribute__((ext_vector_type(8)));
typedef float f32x4 __attribute__((ext_vector_type(4)));
typedef __attribute__((address_space(1))) const void gvoid;
typedef __attribute__((address_space(3))) void lvoid;
typedef __attribute__((address_space(3))) uint8_t lbyte;
typedef __attribute__((address_space(3))) u32 lu32;
typedef __attribute__((address_space(3))) unsigned short lu16;
typedef __attribute__((address_space(3))) u32x4 lu32x4;

namespace {

constexpr int NR = 8;
constexpr int MM = 4096, KK = 4096, NN = 16384;
constexpr int KS = KK / NR;
constexpr int MS = MM / NR;
constexpr int FG = 7, GG = 9;
constexpr int F = 32 * FG, G = 32 * GG;
static_assert(F + G == MS, "rows");
constexpr int XGREM = 3 * FG + 4 * GG;
constexpr int XREM = 32 * XGREM;
constexpr int XROWS = XREM + MS;
constexpr int NSTEP = 32;

constexpr int GRID = 256, NSEND = 64, NTHR = 512;
constexpr int NENG = 56;
#define WIN 8
#define LOOK 3
#define NPUSH 22
#define PUSH_WIN 3
#define W_DELAY 400
#define PRE_STAG 80
#define PRE_START 1700
#define LATE_STAG 100
#define PRE_WIN 32
#define LOC_WIN 32
#define RAW_WIN 32
#define POLL_ALLW 0
#define CODE_PF 4
#define POLL_SLEEP 16
#define I8A 3

constexpr int SLOT = 8192;
constexpr u32 H_DENSE = 1u << 7;
constexpr u32 OP_MAIN = 6144, ACC_MAIN = 5888;
constexpr int SL_X = 0, SL_W = 256, SL_O = SL_W + 1024, SL_END = SL_O + 32 * 48 * 2;
constexpr int FL_X = 0, FL_W = 256, FL_O = FL_W + 256, FL_END = FL_O + 32 * 48 * 2;
static_assert(FL_END <= 4096, "flags");

constexpr size_t WS_CNT = 262144;
constexpr size_t WS_ZERO = 0;
constexpr size_t WS_TMO = 32768;
constexpr size_t WS_TICK = 49152;
constexpr size_t WS_XREM = 65536;
constexpr size_t WS_XOWN = WS_XREM + 2048;
constexpr size_t WS_PCNT = WS_XOWN + 2048;
constexpr size_t WS_WCNT = 27u << 20;
constexpr size_t WS_FLAG = 28u << 20;
constexpr size_t WS_DBG = 524288;
constexpr size_t WS_GW = 1u << 20;
constexpr size_t WS_GX = WS_GW + (size_t)NSTEP * 32 * 16384;
constexpr size_t XSLAB = (size_t)XROWS * 128;
constexpr size_t WS_PK = 32u << 20;
constexpr size_t PK_BYTES = 40u << 20;
static_assert(WS_GX + NSTEP * XSLAB <= WS_WCNT && WS_WCNT + 128 * 32 * 128 <= WS_FLAG && WS_FLAG + NR * 4096 * 128 <= WS_PK, "staging");
static_assert((size_t)SL_END * SLOT <= PK_BYTES, "packed");
constexpr size_t GX_ODD = WS_PK + 2 * PK_BYTES - WS_GX;
constexpr size_t WS_GW2 = 124u << 20;
static_assert(WS_GX + GX_ODD + NSTEP * XSLAB <= WS_GW2, "staging");
constexpr size_t WS_NEED = WS_GW2 + (size_t)3 * 128 * 16384;

constexpr int STAGE = 49152;
constexpr int LDS_CTL = 3 * STAGE;
constexpr int NS = 12;
constexpr int NPAIR = 3;
constexpr int EL_PLANE = NS * SLOT;
constexpr int EL_TAB = EL_PLANE + 8 * 1024;
constexpr int EL_GO = EL_TAB + 1024 * 16;
constexpr int EL_FULL = EL_GO + 128;
constexpr int EL_FREE = EL_FULL + 64;
constexpr int EL_ISS = EL_FREE + 128;
constexpr int EL_DEAD = EL_ISS + 4;
constexpr int EL_END = EL_ISS + 64;
static_assert(EL_END + 1024 <= LDS_CTL && NS % NPAIR == 0 && NS % 2 == 0, "engine LDS");
constexpr u32 UNIT_END = 0xffffu;
constexpr u64 DEADLINE = 40000000ull;

static_assert(NPUSH % 2 == 0 && 7 * NPUSH <= 256 && NPUSH <= XGREM - GG, "push");

struct Params {
    const uint8_t* x;
    const uint8_t* w;
    const float* sx;
    const float* sw;
    float* out;
    uint8_t* ws[NR];
    int rank;
    int pad;
};

__device__ __forceinline__ void st_sys(void* p, u32x4 v) {
    asm volatile("global_store_dwordx4 %0, %1, off sc0 sc1\n\ts_nop 1" :: "v"(p), "v"(v) : "memory");
}
#define WAIT_VM(n) asm volatile("s_waitcnt vmcnt(" #n ")" ::: "memory")
#define THROTTLE(n) asm volatile("s_waitcnt vmcnt(%0)" :: "i"((n) - 1) : "memory")
#define CBAR() asm volatile("" ::: "memory")
#define DPP(v, ctl, rm) (u32)__builtin_amdgcn_update_dpp(0, (int)(v), ctl, rm, 0xf, false)

__device__ __forceinline__ u32 wave_scan_add(u32 v) {
    v += DPP(v, 0x111, 0xf); v += DPP(v, 0x112, 0xf); v += DPP(v, 0x114, 0xf); v += DPP(v, 0x118, 0xf);
    v += DPP(v, 0x142, 0xa); v += DPP(v, 0x143, 0xc);
    return v;
}
__device__ __forceinline__ u32 wave_sum(u32 v) { return __builtin_amdgcn_readlane(wave_scan_add(v), 63); }
__device__ __forceinline__ u32 wave_umax(u32 v) {
    v = max(v, DPP(v, 0x111, 0xf)); v = max(v, DPP(v, 0x112, 0xf)); v = max(v, DPP(v, 0x114, 0xf));
    v = max(v, DPP(v, 0x118, 0xf)); v = max(v, DPP(v, 0x142, 0xa)); v = max(v, DPP(v, 0x143, 0xc));
    return __builtin_amdgcn_readlane(v, 63);
}
__device__ __forceinline__ int block_of(int r, int part) {
    return (r & 1) ? 4 * part + (r >> 1) : 4 * (r >> 1) + part;
}
__device__ __forceinline__ float silu(float v) {
    const float z = __builtin_amdgcn_fmed3f(v, -60.f, 60.f);
    const float e = __builtin_amdgcn_exp2f(z * -1.4426950408889634f);
    return v * __builtin_amdgcn_rcpf(1.0f + e);
}

template <bool ACC>
__device__ __forceinline__ void write_list(const u32 (&q)[8], const u32* nz, u32 pos, u32 total,
                                           lbyte* lst, lbyte* cp, int lane) {
    *(lu32x4*)(cp + lane * 16) = u32x4{q[0], q[1], q[2], q[3]};
    *(lu32x4*)(cp + 1024 + lane * 16) = u32x4{q[4], q[5], q[6], q[7]};
    lbyte* wp = lst + 2 * pos;
#pragma unroll
    for (int k = 0; k < (ACC ? 1 : 4); ++k) {
        const u32 tag = (u32)(k * 64 + lane) << 5;
        for (u32 m = nz[k]; m; m &= m - 1) {
            *(lu16*)wp = (unsigned short)(tag | __builtin_ctz(m));
            wp += 2;
        }
    }
    CBAR();
    for (u32 i = lane; i < total; i += 64) {
        const u32 t = *(lu16*)(lst + 2 * i);
        u32 e;
        if (ACC) {
            e = ((t & 4u) << 13) | ((t >> 5) << 9) | ((t & 3u) << 7) | ((t & 24u) << 2);
            e |= *(lbyte*)(cp + (e >> 5));
        } else {
            e = ((t & 0x1000u) << 2) | ((t & 0x7e0u) << 3) | ((t & 0x800u) >> 4) | ((t & 1u) << 6) | ((t & 30u) << 1);
            e |= (*(lu32*)(cp + ((e >> 4) & 0x7fcu)) >> (t & 30u)) & 3u;
        }
        *(lu16*)(lst + 2 * i) = (unsigned short)e;
    }
    CBAR();
}
__device__ __forceinline__ void st_flag(uint8_t* ws, int src, int flag, u32 call, u32 hdr) {
    __hip_atomic_store((u64*)(ws + WS_FLAG) + (size_t)(src * 4096 + flag) * 16 + (call & 1), ((u64)hdr << 32) | call,
                       __ATOMIC_RELAXED, __HIP_MEMORY_SCOPE_SYSTEM);
}

__device__ __forceinline__ u32 pick_base(u32 esum4  ) {
    const int b = ((int)esum4 - 33587) >> 13;
    return (u32)min(max(b, 0), 24);
}
__device__ __forceinline__ u32 bfi(u32 m, u32 a, u32 b) { return (a & m) | (b & ~m); }
__device__ __forceinline__ void enc_op(const u32x4* v, u32 base4, u32 (&pay)[24], u32 (&pl)[8]) {
#pragma unroll
    for (int j = 0; j < 8; ++j) {
        u32 u[4];
#pragma unroll
        for (int d = 0; d < 4; ++d) u[d] = (v[j][d] | 0x80808080u) - base4;
        const u32 u31 = u[3] << 1;
        pl[j] = bfi(0x0F0F0F0Fu, bfi(0x03030303u, u[0] >> 5, u[1] >> 3), bfi(0x30303030u, u[2] >> 1, u31));
        pay[3 * j + 0] = bfi(0x60606060u, u[3] << 5, bfi(0x7F7F7F7Fu, u[0], v[j][0]));
        pay[3 * j + 1] = bfi(0x60606060u, u[3] << 3, bfi(0x7F7F7F7Fu, u[1], v[j][1]));
        pay[3 * j + 2] = bfi(0x60606060u, bfi(0x20202020u, u31, v[j][3] >> 1), bfi(0x7F7F7F7Fu, u[2], v[j][2]));
    }
}
__device__ __forceinline__ u32x4 dec_op(u32 p0, u32 p1, u32 p2, u32 pl, u32 base4) {
    const u32 c3 = bfi(0x0F0F0F0Fu, bfi(0x03030303u, p0 >> 5, p1 >> 3), p2 >> 1);
    u32x4 v;
    v[0] = bfi(0x7F7F7F7Fu, (bfi(0x1F1F1F1Fu, p0, pl << 5) & 0x7F7F7F7Fu) + base4, p0);
    v[1] = bfi(0x7F7F7F7Fu, (bfi(0x1F1F1F1Fu, p1, pl << 3) & 0x7F7F7F7Fu) + base4, p1);
    v[2] = bfi(0x7F7F7F7Fu, (bfi(0x1F1F1F1Fu, p2, pl << 1) & 0x7F7F7F7Fu) + base4, p2);
    v[3] = bfi(0x7F7F7F7Fu, (bfi(0x1F1F1F1Fu, c3, pl >> 1) & 0x7F7F7F7Fu) + base4, p2 << 1);
    return v;
}

__device__ __forceinline__ int group_of(int r, int sg) {
    if (sg < 3 * FG) {
        const int s = sg / FG;
        return 16 * (2 * (((r >> 1) + 1 + s) & 3) + (r & 1)) + sg - s * FG;
    }
    if (sg < XGREM) {
        const int j = (sg - 3 * FG) / GG;
        return 16 * (2 * j + 1 - (r & 1)) + FG + sg - 3 * FG - j * GG;
    }
    return 16 * r + sg - XGREM;
}

__device__ __forceinline__ int sent_group(int r, int p, int i) {
    const bool same = ((r ^ p) & 1) == 0;
    const int at = same ? FG * (((p >> 1) - (r >> 1) - 1) & 3) : 3 * FG + GG * (p >> 1);
    return i < at ? i : i + (same ? FG : GG);
}

struct Unit {
    u32 slot;
    u32 flag;
    u32 dst;
    u32 kind;
    u32 aux;
};
__device__ __forceinline__ Unit unit_of(int n, int r, int p) {
    const bool same = ((r ^ p) & 1) == 0;
    const int nx = XGREM - (same ? FG : GG);
    const int U1 = nx - NPUSH;
    const int U2 = U1 + 16 * I8A;
    const int U3 = U2 + nx;
    const int U4 = U3 + 16 * I8A;
    const int U5 = U4 + 32 * (8 - I8A);
    const int U6 = U5 + 32;
    Unit u;
    if (n >= U6) {
        const int v = n - U6, wn = v & 1, h = v >> 1;
        int i8, part, rg, lg;
        if (same) {
            i8 = h / (4 * FG);
            const int t = h - i8 * (4 * FG);
            part = t / FG; lg = t - part * FG;
            rg = FG * (((r >> 1) - (p >> 1) - 1) & 3) + lg;
        } else {
            i8 = h / (3 * GG);
            const int t = h - i8 * (3 * GG);
            const int pi = t / GG, jr = r >> 1, g = t - pi * GG;
            part = pi + (pi >= jr);
            rg = 3 * FG + (jr > part ? jr - 1 : jr) * GG + g;
            lg = FG + g;
        }
        const int id = ((part * 8 + i8) * 48 + rg) * 2 + wn, gct = block_of(p, part) * 8 + i8;
        u.slot = SL_O + id; u.flag = FL_O + id; u.kind = 2;
        u.dst = (u32)(lg * 32 * NN + gct * 128 + wn * 64) * 4u;
        u.aux = 0;
        return u;
    }
    if (n < U1 || (n >= U2 && n < U3) || n >= U5) {
        int sg, kp;
        if (n < U1) { sg = sent_group(r, p, n + NPUSH); kp = 0; }
        else if (n < U3) { sg = sent_group(r, p, n - U2); kp = 1; }
        else { sg = XGREM + ((n - U5) & 15); kp = (n - U5) >> 4; }
        const int gg = group_of(r, sg);
        u.slot = SL_X + gg * 2 + kp; u.flag = FL_X + gg * 2 + kp; u.kind = 0;
        u.dst = (u32)(WS_GX + (size_t)(p * 4 + 2 * kp) * XSLAB) + (u32)sg * 4096u;
        u.aux = (u32)(sg < XGREM ? WS_XREM : WS_XOWN) + (u32)(p * 2 + kp) * 128u;
        return u;
    }
    int i8, kc, t;
    if (n < U2) { const int v = n - U1; i8 = v >> 4; kc = (v >> 3) & 1; t = v & 7; }
    else if (n < U4) { const int v = n - U3; i8 = v >> 4; kc = 2 + ((v >> 3) & 1); t = v & 7; }
    else { const int v = n - U4; i8 = I8A + (v >> 5); kc = (v >> 3) & 3; t = v & 7; }
    const int part = t >> 1, half = t & 1, ct = part * 8 + i8, step = p * 4 + kc;
    u.slot = SL_W + ((kc * 128 + block_of(r, part) * 8 + i8) * 2 + half);
    u.flag = FL_W + (kc * 32 + ct) * 2 + half;
    u.kind = 1 | (half << 2);
    u.dst = (u32)WS_GW + (u32)(step * 32 + ct) * 16384u + (u32)half * 8192u;
    u.aux = (u32)WS_WCNT + (u32)(ct * 32 + step) * 128u;
    return u;
}

#define STAMP(k) do { } while (0)

__global__ void __launch_bounds__(NTHR, 2) fused_kernel(Params P) {
    __shared__ __attribute__((aligned(1024))) uint8_t lds_[LDS_CTL];
    lbyte* const lds = (lbyte*)lds_;
    const int tid = threadIdx.x;
    const int lane = tid & 63;
    const int wave = __builtin_amdgcn_readfirstlane(tid >> 6);
    const int b = blockIdx.x;
    if (b >= NENG && b < NSEND) {
        u64 pc;
        asm volatile("s_getpc_b64 %0" : "=s"(pc));
        const uint8_t* cb = (const uint8_t*)(pc & ~255ull) + tid * 16;
        u32x4 sink = {0u, 0u, 0u, 0u};
#pragma unroll
        for (int j = 0; j < CODE_PF; ++j) sink |= *(const __attribute__((address_space(1))) u32x4*)(cb + j * 8192);
        asm volatile("" :: "v"(sink));
    }
    const int rank = P.rank;
    uint8_t* const myws = P.ws[rank];
    u32* const tmo = (u32*)(myws + WS_TMO);

    STAMP(0);
    u32* const cntp = (u32*)(myws + WS_CNT + (size_t)(b * 8 + wave) * 128);
#define LOAD_CALL() \
    const u32 ncalls = __hip_atomic_load(cntp, __ATOMIC_RELAXED, __HIP_MEMORY_SCOPE_AGENT)
#define GET_CALL() \
    const u32 call = __builtin_amdgcn_readfirstlane(ncalls) + 1; \
    const size_t pkoff = WS_PK + (size_t)(call & 1) * PK_BYTES
#define PUT_CALL() \
    if (lane == 0) __hip_atomic_store(cntp, call, __ATOMIC_RELAXED, __HIP_MEMORY_SCOPE_AGENT)

    if (b < NSEND) {
        if (b >= NENG) return;
        LOAD_CALL();
        const int p = (rank + 1 + b % 7) & 7;
        const bool same = ((rank ^ p) & 1) == 0;
        const int nop = 2 * (XGREM - (same ? FG : GG)) - NPUSH + 256 + 32;
        const int nend = nop + (same ? 64 * FG : 48 * GG);
        for (int n = tid; n < nend; n += NTHR) {
            const Unit u = unit_of(n, rank, p);
            *(lu32x4*)(lds + EL_TAB + n * 16) = u32x4{u.slot | (u.kind << 16), u.flag, u.dst, u.aux};
        }
        if (tid < (EL_END - EL_GO) / 4) *(lu32*)(lds + EL_GO + tid * 4) = 0;
        *(lu32x4*)(lds + EL_PLANE + tid * 16) = u32x4{0u, 0u, 0u, 0u};
        __syncthreads();
        GET_CALL();
        PUT_CALL();
        auto lds_get = [&](int off) {
            return (u32)__builtin_amdgcn_readfirstlane(
                __hip_atomic_load((lu32*)(lds + off), __ATOMIC_RELAXED, __HIP_MEMORY_SCOPE_WORKGROUP));
        };
        auto lds_put = [&](int off, u32 v) {
            CBAR();
            if (lane == 0) __hip_atomic_store((lu32*)(lds + off), v, __ATOMIC_RELAXED, __HIP_MEMORY_SCOPE_WORKGROUP);
            CBAR();
        };

        if (wave == 0) {
            __builtin_amdgcn_s_setprio(3);
            WAIT_VM(0);
            const uint8_t* const pk = P.ws[p] + pkoff;
            const u32 lds_addr = (u32)(uintptr_t)lds;
            const u32 vo0 = lane * 16, vo1 = lane * 16 + 4096;
            u32 pub = 0, psl = 0;
            u32 issued = 0;
            u32 end0 = 0, end1 = 0;
            static_assert(WIN - 1 <= 11, "at most two whole units may be in flight behind the current one");
#define DMA1(VO, M0OFF, OFF) do { \
                asm volatile("s_waitcnt vmcnt(%0)" :: "i"(WIN - 1) : "memory"); \
                if (pub < k && end0 + (WIN - 1) <= issued) { \
                    lds_put(EL_FULL + psl * 4, ++pub); psl = psl == NS - 1 ? 0 : psl + 1; end0 = end1; } \
                if ((u32)lane < (lines - ((M0OFF + OFF) >> 7)) * 8u) \
                    asm volatile("s_mov_b32 m0, %2\n\ts_nop 0\n\tglobal_load_lds_dwordx4 %0, %1 offset:" #OFF " sc0 sc1" \
                                 :: "v"(VO), "s"(src), "s"(la + M0OFF) : "memory"); \
                ++issued; \
            } while (0)
            u32 sl = 0;
            for (u32 k = 0;; ++k) {
                u32 go = lds_get(EL_GO + sl * 8);
                if ((go >> 16) != k + 1 || (go & 0xffffu) == UNIT_END) {
                    WAIT_VM(0);
                    while (pub < k) { lds_put(EL_FULL + psl * 4, ++pub); psl = psl == NS - 1 ? 0 : psl + 1; }
                    while ((go >> 16) != k + 1) {
                        __builtin_amdgcn_s_sleep(1);
                        go = lds_get(EL_GO + sl * 8);
                    }
                    if ((go & 0xffffu) == UNIT_END) break;
                }
                lds_put(EL_ISS, k + 1);
                const u32 lines = min(max(lds_get(EL_GO + sl * 8 + 4) & 127u, 41u), 64u);
                const uint8_t* src = pk + (size_t)(lds_get(EL_TAB + (go & 0xffffu) * 16) & 0xffffu) * SLOT;
                const u32 la = lds_addr + sl * SLOT;
                u32 keep;
                asm volatile("s_mov_b32 %0, m0\n\ts_nop 4" : "=s"(keep) :: "memory");
                DMA1(vo0, 0, 0); DMA1(vo0, 0, 1024); DMA1(vo0, 0, 2048); DMA1(vo0, 0, 3072);
                DMA1(vo1, 4096, 0); DMA1(vo1, 4096, 1024);
                if (lines > 48) DMA1(vo1, 4096, 2048);
                if (lines > 56) DMA1(vo1, 4096, 3072);
                asm volatile("s_mov_b32 m0, %0" :: "s"(keep) : "memory");
                if (pub + 2 <= k) {
                    WAIT_VM(0);
                    while (pub < k) { lds_put(EL_FULL + psl * 4, ++pub); psl = psl == NS - 1 ? 0 : psl + 1; }
                }
                if (pub == k) end0 = issued; else end1 = issued;
                sl = sl == NS - 1 ? 0 : sl + 1;
            }
            STAMP(4);
            return;
        }
        if (wave == 1) {
            const u64* const flags = (const u64*)(myws + WS_FLAG) + (size_t)p * 4096 * 16 + (call & 1);
            u32* const tick = (u32*)(myws + WS_TICK) + p * 32;
            const u32 npair = nend / 2, tbase = (call - 1) * npair - 8;
            u32 pair = b / 7;
            bool dead = false, partials = false;
            auto give_up = [&](u64 t0) {
                if (__hip_atomic_load(tmo, __ATOMIC_RELAXED, __HIP_MEMORY_SCOPE_AGENT) == call ||
                    __builtin_amdgcn_s_memrealtime() - t0 > DEADLINE) {
                    if (lane == 0) __hip_atomic_store(tmo, call, __ATOMIC_RELAXED, __HIP_MEMORY_SCOPE_AGENT);
                    dead = true;
                    lds_put(EL_DEAD, 1);
                    return true;
                }
                __builtin_amdgcn_s_sleep(2);
                return false;
            };
            auto post = [&](u32 kk, u32 sl, u32 unit, u32 hdr) {
                while (min(lds_get(EL_FREE + sl * 8), lds_get(EL_FREE + sl * 8 + 4)) + NS < kk + 1) __builtin_amdgcn_s_sleep(1);
                lds_put(EL_GO + sl * 8 + 4, hdr);
                lds_put(EL_GO + sl * 8, ((kk + 1) << 16) | unit);
            };
            u32 k = 0, sl = 0;
            for (; pair < npair; k += 2) {
                while (lds_get(EL_ISS) + LOOK < k) __builtin_amdgcn_s_sleep(1);
                const u64* const fp = flags + (size_t)*(lu32*)(lds + EL_TAB + (2 * pair + (lane & 1)) * 16 + 4) * 16;
                u32 tk;
                u64 fv;
                asm volatile("s_mov_b64 exec, 1\n\tglobal_atomic_add %0, %2, %3, off sc0\n\ts_mov_b64 exec, 3\n\t"
                             "global_load_dwordx2 %1, %4, off sc0 sc1\n\ts_mov_b64 exec, -1\n\ts_waitcnt vmcnt(0)"
                             : "=&v"(tk), "=&v"(fv) : "v"(tick), "v"(1u), "v"(fp) : "memory");
                if (!dead && (__ballot((u32)fv >= call) & 3) != 3) {
                    const u64 t0 = __builtin_amdgcn_s_memrealtime();
                    for (;;) {
                        fv = __hip_atomic_load(fp, __ATOMIC_RELAXED, __HIP_MEMORY_SCOPE_SYSTEM);
                        if (__all((u32)fv >= call) || give_up(t0)) break;
                    }
                }
                if (2 * (int)pair >= nop && !partials) {
                    partials = true;
                    const u64 t0 = __builtin_amdgcn_s_memrealtime();
                    while (!dead && __hip_atomic_load((const u32*)(myws + WS_PCNT), __ATOMIC_RELAXED, __HIP_MEMORY_SCOPE_SYSTEM) < 96u * call)
                        if (give_up(t0)) break;
                }
                const u32 hi = (u32)(fv >> 32);
                post(k, sl, 2 * pair, __builtin_amdgcn_readlane(hi, 0));
                post(k + 1, sl + 1, 2 * pair + 1, __builtin_amdgcn_readlane(hi, 1));
                sl = sl == NS - 2 ? 0 : sl + 2;
                pair = __builtin_amdgcn_readfirstlane(tk) - tbase;
            }
            for (u32 kk = k; kk < k + NPAIR; ++kk) {
                post(kk, sl, UNIT_END, 0);
                sl = sl == NS - 1 ? 0 : sl + 1;
            }
            return;
        }
        lbyte* const plane = lds + EL_PLANE + wave * 1024;
        const float scale0 = P.sx[0] * P.sw[0];
        const u32 hf = wave & 1;
        u32 sl = (wave - 2) >> 1;
        for (u32 k = sl;; k += NPAIR) {
            lbyte* const s = lds + sl * SLOT;
            u32 go;
            while (((go = lds_get(EL_GO + sl * 8)) >> 16) != k + 1) __builtin_amdgcn_s_sleep(1);
            if ((go & 0xffffu) == UNIT_END) break;
            CBAR();
            const u32 hdr = lds_get(EL_GO + sl * 8 + 4);
            const u32x4 te = *(lu32x4*)(lds + EL_TAB + (go & 0xffffu) * 16);
            const u32 kind = __builtin_amdgcn_readfirstlane(te[0]) >> 16;
            const u32 dst = __builtin_amdgcn_readfirstlane(te[2]);
            const u32 aux = __builtin_amdgcn_readfirstlane(te[3]);
            const bool acc = kind == 2;
            u32x4 pa[4];
            uint8_t* const od = (uint8_t*)P.out + dst + (size_t)(hf * 16 + (lane >> 4) * 4) * (NN * 4) + (lane & 15) * 16;
            if (acc) {
                asm volatile("global_load_dwordx4 %0, %4, off sc0 sc1\n\tglobal_load_dwordx4 %1, %5, off sc0 sc1\n\t"
                             "global_load_dwordx4 %2, %6, off sc0 sc1\n\tglobal_load_dwordx4 %3, %7, off sc0 sc1\n\t"
                             "s_waitcnt vmcnt(0)"
                             : "=&v"(pa[0]), "=&v"(pa[1]), "=&v"(pa[2]), "=&v"(pa[3])
                             : "v"(od), "v"(od + NN * 4), "v"(od + NN * 8), "v"(od + NN * 12) : "memory");
            }
            while (lds_get(EL_FULL + sl * 4) != k + 1) __builtin_amdgcn_s_sleep(1);
            CBAR();
            if (!(hdr & H_DENSE)) {
                const u32 cnt = min((hdr >> 8) & 0x7ffu, (SLOT - (acc ? ACC_MAIN : OP_MAIN)) / 2);
                for (u32 o = lane; 2 * o < cnt; o += 64) {
                    const u32 e2 = *(lu32*)(s + (acc ? ACC_MAIN : OP_MAIN) + o * 4);
#pragma unroll
                    for (int h = 0; h < 2; ++h) {
                        const u32 e = h ? e2 >> 16 : e2 & 0xffffu;
                        const u32 val = acc ? (e & 31u) << ((e >> 2) & 24u) : (e & 3u) << ((e >> 1) & 30u);
                        const u32 ad = acc ? e >> 5 : e >> 4;
                        if (2 * o + h < cnt && ((ad >> 10) & 1u) == hf)
                            __hip_atomic_fetch_or((lu32*)(plane + (ad & 0x3fcu)), val, __ATOMIC_RELAXED, __HIP_MEMORY_SCOPE_WAVEFRONT);
                    }
                }
            }
            CBAR();
            if (!acc) {
                u32 pay[12];
#pragma unroll
                for (int j = 0; j < 3; ++j) {
                    const u32x4 t = *(lu32x4*)(s + hf * 3072 + j * 1024 + lane * 16);
                    pay[4 * j] = t[0]; pay[4 * j + 1] = t[1]; pay[4 * j + 2] = t[2]; pay[4 * j + 3] = t[3];
                }
                const u32x4 pl = *(lu32x4*)(((hdr & H_DENSE) ? s + OP_MAIN + hf * 1024 : plane) + lane * 16);
                *(lu32x4*)(plane + lane * 16) = u32x4{0u, 0u, 0u, 0u};
                lds_put(EL_FREE + sl * 8 + hf * 4, k + 1);
                const u32 base4 = ((hdr >> 19) & 31u) * 0x04040404u;
                const bool isw = kind & 1;
                uint8_t* d = myws + dst + (isw ? hf * 4096 : (call & 1) * GX_ODD + hf * XSLAB) +
                             ((lane ^ (isw ? (kind >> 2) * 8 + hf * 4 : 0)) << 4);
#pragma unroll
                for (int j = 0; j < 4; ++j)
                    st_sys(d + j * 1024, dec_op(pay[3 * j], pay[3 * j + 1], pay[3 * j + 2], pl[j], base4));
                WAIT_VM(0);
                if (lane == 0)
                    __hip_atomic_fetch_add((u32*)(myws + aux), 1u, __ATOMIC_RELAXED, __HIP_MEMORY_SCOPE_SYSTEM);
            } else {
                const float scale = lds_get(EL_DEAD) ? __builtin_nanf("") : scale0;
                float a[16];
                if (hdr & H_DENSE) {
#pragma unroll
                    for (int j = 0; j < 4; ++j) {
                        const u32x4 t = *(lu32x4*)(s + hf * 4096 + j * 1024 + lane * 16);
#pragma unroll
                        for (int t4 = 0; t4 < 4; ++t4) a[4 * j + t4] = __uint_as_float(t[t4]);
                    }
                    lds_put(EL_FREE + sl * 8 + hf * 4, k + 1);
                } else {
                    const u32x4 h0 = *(lu32x4*)(s + hf * 2048 + lane * 16), h1 = *(lu32x4*)(s + hf * 2048 + 1024 + lane * 16);
                    u32x4 mid = *(lu32x4*)(s + 4096 + lane * 16);
                    const u32x4 q = *(lu32x4*)(plane + lane * 16);
                    *(lu32x4*)(plane + lane * 16) = u32x4{0u, 0u, 0u, 0u};
                    if (hf) {
                        const u32 m4 = *(lu32*)(s + 5120 + lane * 4), m5 = *(lu32*)(s + 5376 + lane * 4), m6 = *(lu32*)(s + 5632 + lane * 4);
                        u32 m7 = (mid[0] << 1) & 0x02020202u;
                        m7 |= (mid[1] << 2) & 0x04040404u; m7 |= (mid[2] << 3) & 0x08080808u; m7 |= (mid[3] << 4) & 0x10101010u;
                        m7 |= (m4 << 5) & 0x20202020u; m7 |= (m5 << 6) & 0x40404040u; m7 |= (m6 << 7) & 0x80808080u;
                        mid = u32x4{m4, m5, m6, m7};
                    }
                    lds_put(EL_FREE + sl * 8 + hf * 4, k + 1);
                    const float c1 = __uint_as_float(((hdr >> 19) & 255u) << 23);
#pragma unroll
                    for (int j = 0; j < 4; ++j) {
                        const u32 mi = bfi(0x01010101u, q[j] >> 2, mid[j]);
                        const u32 lo = (q[j] & 0x03030303u) | ((q[j] << 3) & 0xC0C0C0C0u);
                        const u32 l0 = __builtin_amdgcn_perm(mi, lo, 0x05010400u), l1 = __builtin_amdgcn_perm(mi, lo, 0x07030602u);
                        const u32 ha = j < 2 ? h0[2 * j] : h1[2 * j - 4], hb = j < 2 ? h0[2 * j + 1] : h1[2 * j - 3];
                        const u32 T[4] = {__builtin_amdgcn_perm(ha, l0, 0x05040100u), __builtin_amdgcn_perm(ha, l0, 0x07060302u),
                                          __builtin_amdgcn_perm(hb, l1, 0x05040100u), __builtin_amdgcn_perm(hb, l1, 0x07060302u)};
#pragma unroll
                        for (int t4 = 0; t4 < 4; ++t4) a[4 * j + t4] = (float)(((int)T[t4] >> 2) ^ (int)(T[t4] << 30)) * c1;
                    }
                }
#pragma unroll
                for (int j = 0; j < 4; ++j) {
                    f32x4 y;
#pragma unroll
                    for (int t4 = 0; t4 < 4; ++t4) {
                        const u32 pb = pa[j][t4];
                        y[t4] = silu((a[4 * j + t4] + __uint_as_float(pb)) * scale);
                    }
                    st_sys(od + (size_t)j * (NN * 4), __builtin_bit_cast(u32x4, y));
                }
            }
            sl = sl >= NS - NPAIR ? sl + NPAIR - NS : sl + NPAIR;
        }
        return;
    }

    const int g = b - NSEND;
    lbyte* const scr = lds + wave * 6144;
    constexpr int WEARLY = 64 * I8A, GLATE = 64 + 8 * I8A;
    static_assert(I8A <= 4, "tasks");
    const u64 tstart0 = __builtin_amdgcn_s_memrealtime();
    for (int phase = wave < 4 ? 0 : 1; phase < (g >= GLATE ? 2 : 1); ++phase) {
    const int task = phase ? 512 + WEARLY + g - GLATE + (192 - GLATE) * (wave ^ 4) : g + 192 * wave;
    if (!phase && task >= 512 + WEARLY) continue;
    LOAD_CALL();
    int l7 = lane & 7;
    asm volatile("" : "+v"(l7));
    uint8_t* const wsl = P.ws[l7];
    const bool ispush = task < 256, isx = task < 512;
    const int pdst = (rank + 1 + task % 7) & 7;
    const int psg = sent_group(pdst, rank, task / 7);
    const int gg = ispush ? group_of(pdst, psg) : (task - 256) >> 1;
    const int kp = ispush ? 0 : task & 1;
    const int wt = task - 512, wq = wt >> 5;
    const int half = (wt >> 3) & 1, blk = (wt & 7) | ((wt >> 1) & 8);
    const int kc = wq < 2 * I8A ? wq & 1 : wq < 4 * I8A ? 2 + (wq & 1) : wq & 3;
    const int wi8 = wq < 2 * I8A ? wq >> 1 : wq < 4 * I8A ? (wq >> 1) - I8A : wq >> 2;
    u32x4 v[8];
    if (isx) {
        if (!ispush || task < 7 * NPUSH) {
            const uint8_t* src = P.x + (size_t)(gg * 32 + (lane >> 3)) * KS + kp * 256 + (lane & 7) * 16;
#pragma unroll
            for (int j = 0; j < 8; ++j) v[j] = *(const u32x4*)(src + (j & 3) * 8 * KS + (j >> 2) * 128);
        }
    } else {
        if (phase) {
            while (__builtin_amdgcn_s_memrealtime() - tstart0 < (u64)(W_DELAY + LATE_STAG * (wave ^ 4))) __builtin_amdgcn_s_sleep(8);
        }
        const uint8_t* src = P.w + (size_t)(kc * 128 + 16 * (lane >> 3)) * NN + blk * 1024 + wi8 * 128 + 16 * (lane & 7) + 8 * half;
        u32x2 in[16];
#pragma unroll
        for (int j = 0; j < 16; ++j) in[j] = *(const u32x2*)(src + (size_t)j * NN);
#pragma unroll
        for (int d = 0; d < 2; ++d)
#pragma unroll
            for (int e = 0; e < 4; ++e) {
                const u32 t0 = __builtin_amdgcn_perm(in[4 * e + 1][d], in[4 * e][d], 0x05010400u);
                const u32 t1 = __builtin_amdgcn_perm(in[4 * e + 1][d], in[4 * e][d], 0x07030602u);
                const u32 t2 = __builtin_amdgcn_perm(in[4 * e + 3][d], in[4 * e + 2][d], 0x05010400u);
                const u32 t3 = __builtin_amdgcn_perm(in[4 * e + 3][d], in[4 * e + 2][d], 0x07030602u);
                v[4 * d + 0][e] = __builtin_amdgcn_perm(t2, t0, 0x05040100u);
                v[4 * d + 1][e] = __builtin_amdgcn_perm(t2, t0, 0x07060302u);
                v[4 * d + 2][e] = __builtin_amdgcn_perm(t3, t1, 0x05040100u);
                v[4 * d + 3][e] = __builtin_amdgcn_perm(t3, t1, 0x07060302u);
            }
    }
    GET_CALL();
    if (ispush) {
        if (task < 7 * NPUSH) {
            uint8_t* const wd = P.ws[pdst];
            uint8_t* d = wd + WS_GX + (call & 1) * GX_ODD + (size_t)(rank * 4) * XSLAB + (size_t)psg * 4096 + lane * 16;
#pragma unroll
            for (int j = 0; j < 8; ++j) asm volatile("" : "+v"(v[j]));
#pragma unroll
            for (int j = 0; j < 8; ++j) {
                asm volatile("s_waitcnt vmcnt(%0)" :: "i"(PUSH_WIN - 1) : "memory");
                st_sys(d + (j >> 2) * XSLAB + (j & 3) * 1024, v[j]);
            }
            WAIT_VM(0);
            if (lane == 0)
                __hip_atomic_fetch_add((u32*)(wd + WS_XREM) + rank * 2 * 32, 2u, __ATOMIC_RELAXED, __HIP_MEMORY_SCOPE_SYSTEM);
        }
    } else {
        const int q = gg >> 4, lg = gg & 15;
        auto needs = [&](int r) {
            return isx ? r == q || (((r ^ q) & 1) == 0) == (lg < FG) : (r >> 1) == ((r & 1) ? blk & 3 : blk >> 2);
        };
        auto ctile = [&](int r) { return ((r & 1) ? blk >> 2 : blk & 3) * 8 + wi8; };
        if (!isx || q != rank) {
            u32 es = 0;
#pragma unroll
            for (int j = 0; j < 8; ++j) es = __builtin_amdgcn_sad_u8(v[j][j & 3] & 0x7C7C7C7Cu, 0u, es);
            const u32 base = pick_base(wave_sum(es));
            u32 pay[24], pl[8];
            enc_op(v, base * 0x04040404u, pay, pl);
            uint8_t* const slot = myws + pkoff + (size_t)(isx ? SL_X + gg * 2 + kp : SL_W + (kc * 128 + blk * 8 + wi8) * 2 + half) * SLOT;
#pragma unroll
            for (int j = 0; j < 6; ++j)
                st_sys(slot + j * 1024 + lane * 16, u32x4{pay[4 * j], pay[4 * j + 1], pay[4 * j + 2], pay[4 * j + 3]});
            u32 nz[4], cnt = 0;
#pragma unroll
            for (int j = 0; j < 4; ++j) {
                nz[j] = bfi(0x55555555u, pl[2 * j] | (pl[2 * j] >> 1), pl[2 * j + 1] | (pl[2 * j + 1] << 1));
                cnt += __builtin_popcount(nz[j]);
            }
            const u32 inc = wave_scan_add(cnt);
            const u32 total = __builtin_amdgcn_readlane(inc, 63);
            u32 hdr = base << 19;
            if (total > (SLOT - 128 - OP_MAIN) / 2) {
                st_sys(slot + OP_MAIN + lane * 16, u32x4{pl[0], pl[1], pl[2], pl[3]});
                st_sys(slot + OP_MAIN + 1024 + lane * 16, u32x4{pl[4], pl[5], pl[6], pl[7]});
                hdr |= 64u | H_DENSE;
            } else {
                write_list<false>(pl, nz, inc - cnt, total, scr, scr + 4096, lane);
                for (u32 o = lane * 16; o < 2 * total; o += 1024) st_sys(slot + OP_MAIN + o, *(lu32x4*)(scr + o));
                hdr |= ((OP_MAIN + 2 * total + 127) >> 7) | (total << 8);
            }
            WAIT_VM(0);
            if (lane < NR && lane != rank && needs(lane))
                st_flag(wsl, rank, isx ? FL_X + gg * 2 + kp : FL_W + (kc * 32 + ctile(lane)) * 2 + half, call, hdr);
        }
        const bool mine = needs(rank);
        if (mine || !isx) {
            const int sg = q == rank ? XGREM + lg
                         : lg < FG ? FG * (((q >> 1) - (rank >> 1) - 1) & 3) + lg
                                   : 3 * FG + GG * (q >> 1) + lg - FG;
            const int fb = (rank & 1) ? 3 * (blk >> 2) + (blk & 3) - ((blk & 3) > (rank >> 1))
                                      : blk - (blk >= 2 * rank ? 4 : 0);
            const int ft = fb * 8 + wi8;
            uint8_t* const db = isx ? myws + WS_GX + (call & 1) * GX_ODD + (size_t)(rank * 4 + 2 * kp) * XSLAB + (size_t)sg * 4096
                              : mine ? myws + WS_GW + (size_t)((rank * 4 + kc) * 32 + ctile(rank)) * 16384 + half * 8192
                                     : myws + WS_GW2 + (size_t)(((ft >> 5) * 4 + kc) * 32 + (ft & 31)) * 16384 + half * 8192;
            uint8_t* d0 = db + ((lane ^ (isx ? 0 : half * 8)) << 4);
            uint8_t* d1 = db + (isx ? XSLAB : 4096) + ((lane ^ (isx ? 0 : half * 8 + 4)) << 4);
#pragma unroll
            for (int j = 0; j < 8; ++j) asm volatile("" : "+v"(v[j]));
#pragma unroll
            for (int j = 0; j < 8; ++j) { THROTTLE(RAW_WIN); st_sys((j < 4 ? d0 : d1) + (j & 3) * 1024, v[j]); }
            WAIT_VM(0);
            if (lane == 0)
                __hip_atomic_fetch_add(isx ? (u32*)(myws + (q == rank ? WS_XOWN : WS_XREM)) + (rank * 2 + kp) * 32
                                           : (u32*)(myws + WS_WCNT) + ((mine ? ctile(rank) : 32 + ft) * 32 + rank * 4 + kc) * 32,
                                       2u, __ATOMIC_RELAXED, __HIP_MEMORY_SCOPE_SYSTEM);
        }
    }
    }
    LOAD_CALL();
    GET_CALL();
    PUT_CALL();
    STAMP(1);

    const int xcd = g & 7, idx = g >> 3;
    const int part = xcd >> 1;
    const int wm = wave >> 1, wn = wave & 1;
    const int c = lane & 15, q4 = lane >> 4;
    const int ra0 = wm * 8192 + c * 128 + ((q4 ^ (c >> 1)) << 4);
    const int ra1 = ra0 ^ 64;
    const int rb0 = 32768 + (c & 3) * 4096 + ((((q4 << 3) | (4 * wn + (c >> 2))) ^ ((c & 3) << 2)) << 4);
    const u32 boff = (u32)wave * 2048u + (u32)lane * 16u;
    const u32 aoff0 = (u32)(lane >> 3) * 128u + (u32)(((lane & 7) ^ (lane >> 4)) << 4), aoff1 = aoff0 ^ 64u;

    for (int job = idx >= 12 ? 0 : 2; job < (idx < 8 ? 4 : 3); ++job) {
        const bool pre = job < 2, local = job == 3, ownrows = job != 2;
        const int ft = g - 96;
        const int i8 = pre ? ft & 7 : local ? idx : idx / 3;
        const int rt = pre ? job : local ? (xcd & 1) : 3 * (xcd & 1) + idx % 3;
        const int ct = pre ? 32 + ft : part * 8 + i8;
        const int nstep = pre ? 4 : NSTEP, ssh = pre ? 3 : 0;
        const u32 wbase = pre ? (u32)WS_GW2 + (u32)(((ft >> 5) - rank) * 128 + (ft & 31)) * 16384u : (u32)WS_GW + (u32)ct * 16384u;
        const int rgw = rt * 8 + wave;
        const int sgw = ownrows ? XGREM + rgw : (rgw < 3 * FG + part * GG ? rgw : rgw + GG);
        const int skipsrc = ownrows ? -1 : group_of(rank, sgw) >> 4;
        const u32 xbase = (u32)(WS_GX + (call & 1) * GX_ODD) + (u32)sgw * 4096u;

        const int psrc = lane < 32 ? lane >> 2 : (lane >> 1) & 7;
        const u32* const pollp = lane < 32 ? (const u32*)(myws + WS_WCNT) + (ct * 32 + lane) * 32
                                           : (const u32*)(myws + (ownrows ? WS_XOWN : WS_XREM)) + (lane & 15) * 32;
        const u32 pneed = call * (lane < 32 ? 4u : ownrows ? 32u : psrc == rank ? 2u * XGREM
                                  : ((psrc ^ rank) & 1) ? 2u * (XGREM - GG) : 2u * (XGREM - FG));
        u64 ready = 0;
        auto stepno = [&](int s) { s <<= ssh; return ((rank + s) & 7) * 4 + (s >> 3); };
        auto ensure = [&](int s) {
            const int st = stepno(s);
            const u64 want = (1ull << st) | (1ull << (32 + (st >> 1)));
            if ((ready & want) == want) return;
            const u64 t0 = __builtin_amdgcn_s_memrealtime();
            for (;;) {
                u64 rd = pre ? (0xfull << (4 * rank)) | (3ull << (32 + 2 * rank))
                       : !((ready >> st) & 1) ? (POLL_ALLW ? 0xffffffffull : 0x11111111ull << (st & 3))
                                              : 0x5555ull << (32 + ((st >> 1) & 1));
                rd &= ~ready;
                u32 v = 0;
                if ((rd >> lane) & 1) v = __hip_atomic_load(pollp, __ATOMIC_RELAXED, __HIP_MEMORY_SCOPE_SYSTEM);
                const u64 got = __ballot(v >= pneed);
                ready |= got;
                if ((ready & want) == want) break;
                if ((got >> st) & 1) continue;
                if (__hip_atomic_load(tmo, __ATOMIC_RELAXED, __HIP_MEMORY_SCOPE_AGENT) == call ||
                    __builtin_amdgcn_s_memrealtime() - t0 > DEADLINE) {
                    if (lane == 0) __hip_atomic_store(tmo, call, __ATOMIC_RELAXED, __HIP_MEMORY_SCOPE_AGENT);
                    WAIT_VM(0);
                    ready = ~0ull;
                    break;
                }
                __builtin_amdgcn_s_sleep(POLL_SLEEP);
            }
        };
        u32 xo, wo;
        auto locate = [&](int s) {
            const u32 st = stepno(s);
            xo = xbase + st * (u32)XSLAB;
            xo = (int)(st >> 2) == skipsrc ? (u32)WS_ZERO : xo;
            wo = wbase + st * (32u * 16384u);
            asm volatile("" : "+s"(xo), "+s"(wo));
        };
        auto issue = [&](int stage) {
            const uint8_t* xa = myws + xo;
            const uint8_t* wa = myws + wo;
            lbyte* la = lds + stage * STAGE + wave * 4096;
            __builtin_amdgcn_global_load_lds((gvoid*)(xa + aoff0), (lvoid*)la, 16, 0, 17);
            __builtin_amdgcn_global_load_lds((gvoid*)(xa + aoff1), (lvoid*)la, 16, 1024, 17);
            __builtin_amdgcn_global_load_lds((gvoid*)(xa + aoff0), (lvoid*)la, 16, 2048, 17);
            __builtin_amdgcn_global_load_lds((gvoid*)(xa + aoff1), (lvoid*)la, 16, 3072, 17);
            lbyte* lb = lds + stage * STAGE + 32768 + wave * 2048;
            __builtin_amdgcn_global_load_lds((gvoid*)(wa + boff), (lvoid*)lb, 16, 0, 17);
            __builtin_amdgcn_global_load_lds((gvoid*)(wa + boff), (lvoid*)lb, 16, 1024, 17);
        };

        f32x4 acc[4][4];
#pragma unroll
        for (int m = 0; m < 4; ++m)
#pragma unroll
            for (int t = 0; t < 4; ++t) acc[m][t] = f32x4{0.f, 0.f, 0.f, 0.f};

        auto rdfrag = [&](i32x8 (&A)[4], i32x8 (&B)[4], int stage) {
            const lbyte* sb = lds + stage * STAGE;
#pragma unroll
            for (int t = 0; t < 4; ++t) {
                const u32x4 al = *(const lu32x4*)(sb + ra0 + t * 2048);
                const u32x4 ah = *(const lu32x4*)(sb + ra1 + t * 2048);
                A[t] = i32x8{(int)al[0], (int)al[1], (int)al[2], (int)al[3], (int)ah[0], (int)ah[1], (int)ah[2], (int)ah[3]};
                const u32x4 bl = *(const lu32x4*)(sb + rb0 + t * 1024);
                const u32x4 bh = *(const lu32x4*)(sb + rb0 + t * 1024 + 512);
                B[t] = i32x8{(int)bl[0], (int)bl[1], (int)bl[2], (int)bl[3], (int)bh[0], (int)bh[1], (int)bh[2], (int)bh[3]};
            }
        };
        auto mma = [&](const i32x8 (&A)[4], const i32x8 (&B)[4]) {
#pragma unroll
            for (int m = 0; m < 4; ++m)
#pragma unroll
                for (int t = 0; t < 4; ++t)
                    acc[m][t] = __builtin_amdgcn_mfma_scale_f32_16x16x128_f8f6f4(A[m], B[t], acc[m][t], 1, 1, 0, 0, 0, 0);
        };
#define KSTEP(XA, XB, YA, YB, ISSUE, NVM) do { \
            if (ISSUE && wave == 0) ensure(s + 3); \
            if (ISSUE) locate(s + 3); \
            asm volatile("s_waitcnt vmcnt(" #NVM ") lgkmcnt(0)" ::: "memory"); \
            __builtin_amdgcn_s_barrier(); \
            asm volatile("" ::: "memory"); \
            __builtin_amdgcn_sched_barrier(0); \
            const int nst = stage == 2 ? 0 : stage + 1; \
            rdfrag(YA, YB, nst); \
            if (ISSUE) issue(stage); \
            mma(XA, XB); \
            _Pragma("unroll") for (int z = 0; z < 8; ++z) { \
                __builtin_amdgcn_sched_group_barrier(0x8, 1, 0); \
                __builtin_amdgcn_sched_group_barrier(0x100, 2, 0); \
            } \
            _Pragma("unroll") for (int z = 0; z < 8; ++z) { \
                __builtin_amdgcn_sched_group_barrier(0x8, 1, 0); \
                __builtin_amdgcn_sched_group_barrier(0x10, 1, 0); \
            } \
            __builtin_amdgcn_sched_barrier(0); \
            stage = nst; ++s; \
        } while (0)

        if (job == 0 && wave == 0) {
            while (__builtin_amdgcn_s_memrealtime() - tstart0 < (u64)(PRE_START + (idx - 12) * PRE_STAG)) __builtin_amdgcn_s_sleep(8);
        }
        if (wave == 0) { ensure(0); ensure(1); ensure(2); }
        asm volatile("s_waitcnt vmcnt(0) lgkmcnt(0)" ::: "memory");
        __builtin_amdgcn_s_barrier();
        asm volatile("" ::: "memory");
        locate(0); issue(0);
        locate(1); issue(1);
        locate(2); issue(2);
        WAIT_VM(12);
        __builtin_amdgcn_s_barrier();
        asm volatile("" ::: "memory");
        i32x8 A0[4], B0[4], A1[4], B1[4];
        rdfrag(A0, B0, 0);
        int stage = 0, s = 0;
        while (s < nstep - 4) {
            KSTEP(A0, B0, A1, B1, true, 6);
            KSTEP(A1, B1, A0, B0, true, 6);
        }
        KSTEP(A0, B0, A1, B1, true, 6);
        KSTEP(A1, B1, A0, B0, false, 6);
        KSTEP(A0, B0, A1, B1, false, 0);
        mma(A1, B1);

        const bool bad = __hip_atomic_load(tmo, __ATOMIC_RELAXED, __HIP_MEMORY_SCOPE_AGENT) == call;
        int ln = lane;
        asm volatile("" : "+v"(ln));
        if (local) {
            float scale = bad ? __builtin_nanf("") : P.sx[0] * P.sw[0];
            asm volatile("" : "+v"(scale));
            float* dst = P.out + (size_t)(rt * 256 + wm * 64 + 4 * (ln >> 4)) * NN + block_of(rank, part) * 1024 + i8 * 128 + wn * 64 + 4 * (ln & 15);
#pragma unroll
            for (int m = 0; m < 4; ++m)
#pragma unroll
                for (int r = 0; r < 4; ++r) {
                    f32x4 y;
#pragma unroll
                    for (int t = 0; t < 4; ++t) y[t] = silu(acc[m][t][r] * scale);
                    THROTTLE(LOC_WIN);
                    st_sys(dst + (size_t)(m * 16 + r) * NN, __builtin_bit_cast(u32x4, y));
                }
            STAMP(7);
        } else if (pre) {
            const int fb = ft >> 3;
            const int blkf = (rank & 1) ? 4 * (fb / 3) + fb % 3 + (fb % 3 >= (rank >> 1)) : fb + (fb >= 2 * rank ? 4 : 0);
            float* d = P.out + (size_t)(rt * 256 + wm * 64 + 4 * (ln >> 4)) * NN + blkf * 1024 + i8 * 128 + wn * 64 + 4 * (ln & 15);
#pragma unroll
            for (int m = 0; m < 4; ++m)
#pragma unroll
                for (int r = 0; r < 4; ++r) {
                    f32x4 y = {acc[m][0][r], acc[m][1][r], acc[m][2][r], acc[m][3][r]};
                    if (bad) y[0] = __builtin_nanf("");
                    THROTTLE(PRE_WIN);
                    st_sys(d + (size_t)(m * 16 + r) * NN, __builtin_bit_cast(u32x4, y));
                }
            if (job == 1) {
                WAIT_VM(0);
                __builtin_amdgcn_s_barrier();
                if (tid == 0) __hip_atomic_fetch_add((u32*)(myws + WS_PCNT), 1u, __ATOMIC_RELAXED, __HIP_MEMORY_SCOPE_SYSTEM);
                STAMP(7);
            }
        } else {
            float amf = 0.f;
#pragma unroll
            for (int i = 0; i < 64; ++i) amf = fmaxf(amf, fabsf(acc[i >> 4][i & 3][(i >> 2) & 3]));
            const u32 am = wave_umax(__float_as_uint(amf));
            const int ex = min(max((int)(am >> 23), 40), 200);
            u32 votes = 0;
#pragma unroll
            for (int i = 0; i < 8; ++i) {
                const u32 I = (u32)(int)(acc[i >> 1][i & 3][i & 1] * __uint_as_float((u32)(280 - ex) << 23));
                votes += (((I + (1u << 25)) & 0xfc000007u) != 0) + ((((I + (1u << 26)) & 0xf800000fu) != 0) << 10) +
                         (((I & 0x1fu) != 0) << 20);
            }
            votes = wave_sum(votes);
            const u32 n0 = votes & 1023u, n1 = (votes >> 10) & 1023u, n2 = votes >> 20;
            const int dsc = n2 <= n1 && n2 <= n0 ? 2 : n1 <= n0 ? 1 : 0;
            const float up = __uint_as_float((u32)(280 - ex - dsc) << 23);
            u32 hdrs[2];
#pragma unroll
            for (int mp = 0; mp < 2; ++mp) {
#define ACCV(i) acc[2 * mp + ((i) >> 4)][(i) & 3][((i) >> 2) & 3]
                uint8_t* const slot = myws + pkoff + (size_t)(SL_O + (ct * 48 + rt * 8 + wm * 2 + mp) * 2 + wn) * SLOT;
                u32 hi[16], mid[8], q[8], nz = 0, miss = 0;
#pragma unroll
                for (int j = 0; j < 8; ++j) {
                    u32 t[4];
#pragma unroll
                    for (int t4 = 0; t4 < 4; ++t4) {
                        const float f = ACCV(4 * j + t4) * up;
                        const u32 I = (u32)(int)f;
                        miss |= __float_as_uint(__builtin_amdgcn_fractf(f));
                        t[t4] = __builtin_amdgcn_alignbit(I, I ^ (u32)__builtin_amdgcn_sbfe((int)I, 0, 26), 26);
                    }
                    hi[2 * j] = __builtin_amdgcn_perm(t[1], t[0], 0x07060302u);
                    hi[2 * j + 1] = __builtin_amdgcn_perm(t[3], t[2], 0x07060302u);
                    const u32 l0 = __builtin_amdgcn_perm(t[1], t[0], 0x05010400u), l1 = __builtin_amdgcn_perm(t[3], t[2], 0x05010400u);
                    const u32 lo = __builtin_amdgcn_perm(l1, l0, 0x05040100u);
                    mid[j] = __builtin_amdgcn_perm(l1, l0, 0x07060302u);
                    q[j] = bfi(0x07070707u, bfi(0x03030303u, lo, mid[j] << 2), (lo >> 3) & 0x18181818u);
                    const u32 nb = q[j] + 0x1F1F1F1Fu;
                    nz = bfi(0x01010101u << j, j < 5 ? nb >> (5 - j) : nb << (j - 5), nz);
                }
                const u32 cnt = __builtin_popcount(nz);
                const u32 inc = wave_scan_add(cnt);
                const u32 total = __builtin_amdgcn_readlane(inc, 63);
                if (bad || (int)(am >> 23) > ex || __any(miss != 0) || total > (SLOT - 128 - ACC_MAIN) / 2) {
#pragma unroll
                    for (int j = 0; j < 8; ++j) {
                        f32x4 y = {ACCV(4 * j), ACCV(4 * j + 1), ACCV(4 * j + 2), ACCV(4 * j + 3)};
                        if (bad) y[0] = __builtin_nanf("");
                        st_sys(slot + j * 1024 + ln * 16, __builtin_bit_cast(u32x4, y));
                    }
                    hdrs[mp] = 64u | H_DENSE;
                } else {
#pragma unroll
                    for (int j = 0; j < 7; ++j) mid[j] = bfi(0x01010101u, mid[7] >> (j + 1), mid[j]);
#pragma unroll
                    for (int j = 0; j < 4; ++j)
                        st_sys(slot + j * 1024 + ln * 16, u32x4{hi[4 * j], hi[4 * j + 1], hi[4 * j + 2], hi[4 * j + 3]});
                    st_sys(slot + 4096 + ln * 16, u32x4{mid[0], mid[1], mid[2], mid[3]});
                    *(lu32*)(scr + ln * 4) = mid[4];
                    *(lu32*)(scr + 256 + ln * 4) = mid[5];
                    *(lu32*)(scr + 512 + ln * 4) = mid[6];
                    write_list<true>(q, &nz, inc - cnt, total, scr + 768, scr + 4096, ln);
                    for (u32 o = ln * 16; o < 768 + 2 * total; o += 1024) st_sys(slot + 5120 + o, *(lu32x4*)(scr + o));
                    CBAR();
                    hdrs[mp] = ((ACC_MAIN + 2 * total + 127) >> 7) | (total << 8) | ((u32)(ex + dsc - 30) << 19);
                }
            }
            WAIT_VM(0);
            if (ln < 2) {
                const int rg = rt * 8 + wm * 2 + ln;
                int q;
                if (rg < 3 * FG) {
                    q = 2 * (((rank >> 1) + 1 + rg / FG) & 3) + (rank & 1);
                } else {
                    const int jj = (rg - 3 * FG) / GG;
                    q = 2 * (jj + (jj >= part)) + 1 - (rank & 1);
                }
                st_flag(P.ws[q & 7], rank, FL_O + (ct * 48 + rg) * 2 + wn, call, ln ? hdrs[1] : hdrs[0]);
            }
            STAMP(4);
        }
    }
}

}

extern "C" void kernel_launch(int rank, int nranks, ncclComm_t comm,
                              hipStream_t stream,
                              void* const* d_in, const int* in_sizes, int n_in,
                              void* d_out, int out_size,
                              const int* in_dtypes, int out_dtype,
                              void* const* d_out_all,
                              void* const* d_ws_all, size_t ws_bytes) {
    if (nranks != NR || (unsigned)rank >= (unsigned)NR || n_in != 4 || ws_bytes < WS_NEED ||
        in_sizes[0] != MM * KS || in_sizes[1] != KS * NN || out_size != MS * NN) {
        fprintf(stderr, "kernel_launch: unexpected problem shape\n");
        return;
    }
    Params P;
    P.x = (const uint8_t*)d_in[0];
    P.w = (const uint8_t*)d_in[1];
    P.sx = (const float*)d_in[2];
    P.sw = (const float*)d_in[3];
    P.out = (float*)d_out;
    for (int p = 0; p < NR; ++p) P.ws[p] = (uint8_t*)d_ws_all[p];
    P.rank = rank;
    P.pad = 0;
    hipLaunchKernelGGL(fused_kernel, dim3(GRID), dim3(NTHR), 0, stream, P);
    HIP_CHECK(hipGetLastError());
}
```

```cpp
#include <hip/hip_runtime.h>
#include <rccl/rccl.h>
#include <cstdio>
#include <cstdint>

#define HIP_CHECK(expr) do { hipError_t _e = (expr); if (_e != hipSuccess) { \
    fprintf(stderr, "HIP error %s at %s:%d\n", hipGetErrorString(_e), __FILE__, __LINE__); \
    return; } } while (0)

typedef unsigned u32;
typedef unsigned long long u64;
typedef unsigned u32x4 __attribute__((ext_vector_type(4)));
typedef unsigned u32x2 __attribute__((ext_vector_type(2)));
typedef int i32x8 __attribute__((ext_vector_type(8)));
typedef float f32x4 __attribute__((ext_vector_type(4)));
typedef __attribute__((address_space(1))) const void gvoid;
typedef __attribute__((address_space(3))) void lvoid;
typedef __attribute__((address_space(3))) uint8_t lbyte;
typedef __attribute__((address_space(3))) u32 lu32;
typedef __attribute__((address_space(3))) unsigned short lu16;
typedef __attribute__((address_space(3))) u32x4 lu32x4;
typedef __attribute__((address_space(3))) u32x2 lu32x2;

namespace {

constexpr int NR = 8;
constexpr int MM = 4096, KK = 4096, NN = 16384;
constexpr int KS = KK / NR;
constexpr int MS = MM / NR;
constexpr int FG = 7, GG = 9;
constexpr int F = 32 * FG, G = 32 * GG;
static_assert(F + G == MS, "rows");
constexpr int XGREM = 3 * FG + 4 * GG;
constexpr int XREM = 32 * XGREM;
constexpr int XROWS = XREM + MS;
constexpr int NSTEP = 32;

constexpr int GRID = 256, NSEND = 64, NTHR = 512;
constexpr int NENG = 56;
#define WIN 7
#define WIN_OP 8
#define WIN_TAIL 12
#define TAIL_UNITS 0
#define BATCH 5
#define LOOK 3
#define NPUSH 20
#define PUSH_WIN 3
#define W_DELAY 500
#define PRE_STAG 60
#define PRE_START 1700
#define LATE_STAG 100
#define LOC_WIN 32
#define POLL_ALLW 0
#define PEEK_AHEAD 6
#define W_DELAY0 200
#define NT_PRE 0
#define NT_OUT 0
#define CODE_PF 6
#define POLL_SLEEP 16
#define I8A 3

constexpr int SLOT = 8192;
constexpr u32 H_RAW = 1u << 7;
constexpr u32 ACC_MAIN = 5888;
constexpr int SL_X = 0, SL_W = 256, SL_O = SL_W + 1024, SL_END = SL_O + 32 * 48 * 2;
constexpr int FL_X = 0, FL_W = 256, FL_O = FL_W + 256, FL_END = FL_O + 32 * 48 * 2;
static_assert(FL_END <= 4096, "flags");

constexpr size_t WS_CNT = 262144;
constexpr size_t WS_ZERO = 0;
constexpr size_t WS_TMO = 32768;
constexpr size_t WS_TICK = 49152;
constexpr size_t WS_XREM = 65536;
constexpr size_t WS_XOWN = WS_XREM + 2048;
constexpr size_t WS_PCNT = WS_XOWN + 2048;
constexpr size_t WS_WCNT = 27u << 20;
constexpr size_t WS_FLAG = 28u << 20;
constexpr size_t WS_DBG = 524288;
constexpr size_t WS_GW = 1u << 20;
constexpr size_t WS_GX = WS_GW + (size_t)NSTEP * 32 * 16384;
constexpr size_t XSLAB = (size_t)XROWS * 128;
constexpr size_t WS_PK = 32u << 20;
constexpr size_t PK_BYTES = 40u << 20;
static_assert(WS_GX + NSTEP * XSLAB <= WS_WCNT && WS_WCNT + 128 * 32 * 128 <= WS_FLAG && WS_FLAG + NR * 4096 * 128 <= WS_PK, "staging");
static_assert((size_t)SL_END * SLOT <= PK_BYTES, "packed");
constexpr size_t GX_ODD = WS_PK + 2 * PK_BYTES - WS_GX;
constexpr size_t WS_GW2 = 124u << 20;
static_assert(WS_GX + GX_ODD + NSTEP * XSLAB <= WS_GW2, "staging");
constexpr size_t WS_NEED = WS_GW2 + (size_t)3 * 128 * 16384;

constexpr int STAGE = 49152;
constexpr int LDS_CTL = 3 * STAGE;
constexpr int LDS_LUT = LDS_CTL + 256;
constexpr int NS = 12;
constexpr int NPAIR = 3;
constexpr int EL_PLANE = NS * SLOT;
constexpr int EL_BUF = EL_PLANE + 8 * 1024;
constexpr int EL_TAB = EL_BUF + 8 * 2304;
constexpr int EL_GO = EL_TAB + 1024 * 16;
constexpr int EL_FULL = EL_GO + 256;
constexpr int EL_FREE = EL_FULL + 64;
constexpr int EL_ISS = EL_FREE + 128;
constexpr int EL_DEAD = EL_ISS + 4;
constexpr int EL_END = EL_ISS + 64;
static_assert(EL_END + 1024 <= LDS_CTL && NS % NPAIR == 0 && NS % 2 == 0, "engine LDS");
constexpr u32 UNIT_END = 0xffffu;
constexpr u64 DEADLINE = 40000000ull;

static_assert(7 * NPUSH <= 256 && NPUSH <= XGREM - GG, "push");

struct Params {
    const uint8_t* x;
    uint8_t* myws;
    int rank;
    int pad;
    const uint8_t* w;
    const float* sx;
    const float* sw;
    float* out;
    uint8_t* ws[NR];
};

__device__ __forceinline__ void st_sys(void* p, u32x4 v) {
    asm volatile("global_store_dwordx4 %0, %1, off sc0 sc1\n\ts_nop 1" :: "v"(p), "v"(v) : "memory");
}
template <int M> __device__ __forceinline__ void st_mode(void* p, u32x4 v) {
    if (M == 0) asm volatile("global_store_dwordx4 %0, %1, off sc0 sc1\n\ts_nop 1" :: "v"(p), "v"(v) : "memory");
    if (M == 1) asm volatile("global_store_dwordx4 %0, %1, off sc0 sc1 nt\n\ts_nop 1" :: "v"(p), "v"(v) : "memory");
    if (M == 2) asm volatile("global_store_dwordx4 %0, %1, off nt\n\ts_nop 1" :: "v"(p), "v"(v) : "memory");
    if (M == 3) asm volatile("global_store_dwordx4 %0, %1, off sc1\n\ts_nop 1" :: "v"(p), "v"(v) : "memory");
}
#define WAIT_VM(n) asm volatile("s_waitcnt vmcnt(" #n ")" ::: "memory")
#define THROTTLE(n) asm volatile("s_waitcnt vmcnt(%0)" :: "i"((n) - 1) : "memory")
#define CBAR() asm volatile("" ::: "memory")
#define DPP(v, ctl, rm) (u32)__builtin_amdgcn_update_dpp(0, (int)(v), ctl, rm, 0xf, false)

__device__ __forceinline__ u32 wave_scan_add(u32 v) {
    v += DPP(v, 0x111, 0xf); v += DPP(v, 0x112, 0xf); v += DPP(v, 0x114, 0xf); v += DPP(v, 0x118, 0xf);
    v += DPP(v, 0x142, 0xa); v += DPP(v, 0x143, 0xc);
    return v;
}
__device__ __forceinline__ u32 wave_sum(u32 v) { return __builtin_amdgcn_readlane(wave_scan_add(v), 63); }
__device__ __forceinline__ u32 wave_umax(u32 v) {
    v = max(v, DPP(v, 0x111, 0xf)); v = max(v, DPP(v, 0x112, 0xf)); v = max(v, DPP(v, 0x114, 0xf));
    v = max(v, DPP(v, 0x118, 0xf)); v = max(v, DPP(v, 0x142, 0xa)); v = max(v, DPP(v, 0x143, 0xc));
    return __builtin_amdgcn_readlane(v, 63);
}
__device__ __forceinline__ int block_of(int r, int part) {
    return (r & 1) ? 4 * part + (r >> 1) : 4 * (r >> 1) + part;
}
__device__ __forceinline__ float silu(float v) {
    const float z = __builtin_amdgcn_fmed3f(v, -60.f, 60.f);
    const float e = __builtin_amdgcn_exp2f(z * -1.4426950408889634f);
    return v * __builtin_amdgcn_rcpf(1.0f + e);
}

__device__ __forceinline__ void write_list(const u32 (&q)[8], u32 nz, u32 pos, u32 total,
                                           lbyte* lst, lbyte* cp, int lane) {
    *(lu32x4*)(cp + lane * 16) = u32x4{q[0], q[1], q[2], q[3]};
    *(lu32x4*)(cp + 1024 + lane * 16) = u32x4{q[4], q[5], q[6], q[7]};
    lbyte* wp = lst + 2 * pos;
    for (u32 m = nz; m; m &= m - 1) {
        *(lu16*)wp = (unsigned short)(((u32)lane << 5) | __builtin_ctz(m));
        wp += 2;
    }
    CBAR();
    for (u32 i = lane; i < total; i += 64) {
        const u32 t = *(lu16*)(lst + 2 * i);
        u32 e = ((t & 4u) << 13) | ((t >> 5) << 9) | ((t & 3u) << 7) | ((t & 24u) << 2);
        e |= *(lbyte*)(cp + (e >> 5));
        *(lu16*)(lst + 2 * i) = (unsigned short)e;
    }
    CBAR();
}
__device__ __forceinline__ void st_flag(uint8_t* ws, int src, int flag, int half, u32 call, u32 hdr) {
    __hip_atomic_store((u64*)(ws + WS_FLAG) + (size_t)(src * 4096 + flag) * 16 + (call & 1) * 2 + half,
                       ((u64)hdr << 32) | call, __ATOMIC_RELAXED, __HIP_MEMORY_SCOPE_SYSTEM);
}
__device__ __forceinline__ u32 bfi(u32 m, u32 a, u32 b) { return (a & m) | (b & ~m); }

__device__ __forceinline__ void op_lut_init(lbyte* lut, int m) {
    u32 e = 0, c = 0x0c0c0c0cu, n = 0;
#pragma unroll
    for (u32 i = 0; i < 4; ++i) {
        const bool on = (m >> i) & 1;
        e |= (on ? n : 0x0cu) << (8 * i);
        if (on) c = (c & ~(0xffu << (8 * n))) | (i << (8 * n));
        n += on;
    }
    *(lu32*)(lut + m * 4) = e;
    *(lu32*)(lut + 64 + m * 4) = c;
}
__device__ __forceinline__ u32 and_or(u32 a, u32 m, u32 b) {
    u32 r; asm("v_and_or_b32 %0, %1, %2, %3" : "=v"(r) : "v"(a), "s"(m), "v"(b)); return r;
}
__device__ __forceinline__ u32 bfi_s(u32 m, u32 a, u32 b) {
    u32 r; asm("v_bfi_b32 %0, %1, %2, %3" : "=v"(r) : "s"(m), "v"(a), "v"(b)); return r;
}
__device__ __forceinline__ u32 ff_count(u32 ff, u32 acc) { return (u32)__builtin_amdgcn_sdot4((int)ff, -1, (int)acc, true); }
__device__ __forceinline__ u32 ff_lut(u32 ff, u32 lut) {
    return *(const lu32*)(uintptr_t)(u32)__builtin_amdgcn_sdot4((int)ff, (int)0xe0f0f8fcu, (int)lut, true);
}
__device__ __forceinline__ u32 ge13(u32 x) { return __builtin_amdgcn_perm(0u, 0u, x); }
__device__ __forceinline__ void lds_or(u32 p, u32 v) {
    __hip_atomic_fetch_or((lu32*)(uintptr_t)p, v, __ATOMIC_RELAXED, __HIP_MEMORY_SCOPE_WAVEFRONT);
}
__device__ __forceinline__ u32 chunk_starts(const u32 (&c)[4], u32 (&p)[4], u32 at) {
    const u32 a = c[0] | (c[1] << 16), b = c[2] | (c[3] << 16);
    const u32 ia = wave_scan_add(a), ib = wave_scan_add(b);
    const u32 ta = __builtin_amdgcn_readlane(ia, 63), tb = __builtin_amdgcn_readlane(ib, 63);
    const u32 t1 = ta & 0xffffu, t2 = t1 + (ta >> 16), t3 = t2 + (tb & 0xffffu);
    p[0] = ((ia - a) & 0xffffu) + at;
    p[1] = ((ia - a) >> 16) + (at + t1);
    p[2] = ((ib - b) & 0xffffu) + (at + t2);
    p[3] = ((ib - b) >> 16) + (at + t3);
    return t3 + (tb >> 16);
}
__device__ __forceinline__ u32 op_pack(const u32x4* v, u32 base4, uint8_t* blk, lbyte* scr, const lbyte* lutp, int lane) {
    lbyte* const side = scr + 2560;
    const u32 lut = (u32)(uintptr_t)lutp + 64, eb = (u32)(uintptr_t)scr + 16;
    const u32x4 z4 = {0u, 0u, 0u, 0u};
    *(lu32x4*)(scr + lane * 16) = z4;
    *(lu32x4*)(scr + 1024 + lane * 16) = z4;
    *(lu32x4*)(scr + 2048 + lane * 16) = z4;
    *(lu32x4*)(scr + 3072 + lane * 16) = z4;
    u32 ub[16], ef[16], sel[16], N[8], S[2] = {0u, 0u}, cnt[4] = {0u, 0u, 0u, 0u}, pos[4];
#pragma unroll
    for (int i = 0; i < 16; ++i) {
        const u32 d = v[i >> 2][i & 3];
        ub[i] = (d | 0x80808080u) - base4;
        ef[i] = ge13(and_or(ub[i], 0x7C7C7C7Cu, 0x01010101u));
        sel[i] = ff_lut(ef[i], lut);
        const u32 x = and_or(ef[i], 0x0C0C0C0Cu, ub[i]);
        N[i >> 1] = (i & 1) ? bfi_s(0x0F0F0F0Fu, N[i >> 1], x << 4) : x;
        S[i >> 3] = and_or(d >> (7 - (i & 7)), 0x01010101u << (i & 7), S[i >> 3]);
        cnt[i >> 2] = ff_count(ef[i], cnt[i >> 2]);
    }
    const u32 n2 = chunk_starts(cnt, pos, eb - 1), n2w = (n2 + 15) >> 4;
    bool raw = n2 > 2048;
    u32 n3 = 0;
    if (!raw) {
        st_sys(blk + lane * 16, u32x4{N[0], N[1], N[2], N[3]});
        st_sys(blk + 1024 + lane * 16, u32x4{N[4], N[5], N[6], N[7]});
        CBAR();
        *(lu32x2*)(side + lane * 8) = u32x2{S[0], S[1]};
#pragma unroll
        for (int i = 0; i < 16; ++i) {
            const u32 c = __builtin_amdgcn_perm(0u, ub[i], sel[i]), p = pos[i >> 2];
            lu32* const q = (lu32*)(uintptr_t)(p & ~3u);
            __hip_atomic_fetch_or(q, __builtin_amdgcn_alignbyte(c, 0u, ~p), __ATOMIC_RELAXED, __HIP_MEMORY_SCOPE_WAVEFRONT);
            __hip_atomic_fetch_or(q + 1, __builtin_amdgcn_alignbyte(0u, c, ~p), __ATOMIC_RELAXED, __HIP_MEMORY_SCOPE_WAVEFRONT);
            pos[i >> 2] = ff_count(ef[i], p);
        }
        CBAR();
        const u32 l3 = (u32)(uintptr_t)side + 512 + n2w * 4;
        u32 bad = 0;
        for (u32 w0 = 0; w0 < n2w; w0 += 64) {
            const u32x4 B = *(lu32x4*)(scr + 16 + (w0 + lane) * 16);
            u32 W = 0, c3 = 0;
            u64 str = 0;
#pragma unroll
            for (int k = 0; k < 4; ++k) {
                const u32 u = (((B[k] & 0x7C7C7C7Cu) + 0x08080808u) >> 2) & 0x1F1F1F1Fu;
                const u32 code = __builtin_amdgcn_perm(0x83038200u, 0x80008100u, u) & 0x03030303u;
                W |= __builtin_amdgcn_udot4(code, 0x40100401u, 0u, false) << (8 * k);
                bad |= (u + 0x78787878u) & ~(u + 0x6E6E6E6Eu);
                const u32 e = __builtin_amdgcn_perm(0u, 0xff000000u, code);
                const u32 sym = u & (and_or(ge13(u), 0x0E0E0E0Eu, 0x01010101u));
                const u32 c = __builtin_amdgcn_perm(0u, sym, ff_lut(e, lut));
                str |= (u64)__builtin_amdgcn_perm(0u, c | (c >> 4), 0x0c0c0200u) << c3;
                c3 = (u32)__builtin_amdgcn_sdot4((int)e, (int)0xfcfcfcfcu, (int)c3, true);
            }
            const u32 i3 = wave_scan_add(c3);
            const u32 o3 = n3 + i3 - c3;
            n3 += __builtin_amdgcn_readlane(i3, 63);
            if (n2w * 4 + (n3 >> 3) > 1000) { raw = true; break; }
            if (w0 + lane < n2w) *(lu32*)(side + 512 + (w0 + lane) * 4) = W;
            const u64 lo = (u64)(u32)str << (o3 & 31u), hi = (str >> 32) << (o3 & 31u);
            const u32 p = l3 + ((o3 >> 5) << 2);
            if (c3) lds_or(p, (u32)lo);
            if ((o3 & 31u) + c3 > 32) {
                lds_or(p + 4, (u32)(lo >> 32) | (u32)hi);
                lds_or(p + 8, (u32)(hi >> 32));
            }
        }
        raw = raw || __any(bad & 0x80808080u);
        CBAR();
    }
    if (raw) {
        WAIT_VM(0);
#pragma unroll
        for (int j = 0; j < 4; ++j) st_sys(blk + j * 1024 + lane * 16, v[j]);
        return 31u;
    }
    const u32 nb = 512 + n2w * 4 + ((n3 + 7) >> 3);
    for (u32 o = lane * 16; o < nb; o += 1024) st_sys(blk + 2048 + o, *(lu32x4*)(side + o));
    CBAR();
    return ((2048 + nb - 1) >> 7) | (n2w << 5);
}
template <class Done>
__device__ __forceinline__ void op_unpack(u32x4* v, u32 base4, u32 n2w, const lbyte* blk, lbyte* buf, const lbyte* lutp,
                                          int lane, Done done) {
    const u32 lut = (u32)(uintptr_t)lutp;
    const u32x4 N0 = *(const lu32x4*)(blk + lane * 16), N1 = *(const lu32x4*)(blk + 1024 + lane * 16);
    const u32x2 S = *(const lu32x2*)(blk + 2048 + lane * 8);
    const lbyte* const l3 = blk + 2560 + n2w * 4;
    u32 n3 = 0;
    for (u32 w0 = 0; w0 < n2w; w0 += 64) {
        u32 W = *(const lu32*)(blk + 2560 + (w0 + lane) * 4);
        if (w0 + lane >= n2w) W = 0;
        const u32 c3t = __builtin_popcount(W & (W >> 1) & 0x55555555u);
        const u32 i3 = wave_scan_add(c3t);
        const u32 o3 = n3 + i3 - c3t;
        n3 += __builtin_amdgcn_readlane(i3, 63);
        const lbyte* const p = l3 + ((o3 >> 3) << 2);
        const u32 d0 = *(const lu32*)p, d1 = *(const lu32*)(p + 4), d2 = *(const lu32*)(p + 8);
        const u64 str = ((u64)__builtin_amdgcn_alignbit(d2, d1, o3 << 2) << 32) | __builtin_amdgcn_alignbit(d1, d0, o3 << 2);
        u32x4 B;
        u32 c3 = 0;
#pragma unroll
        for (int k = 0; k < 4; ++k) {
            const u32 f = (W >> (8 * k)) & 255u;
            const u32 t = (f | (f << 12)) & 0x000F000Fu;
            const u32 code = (t | (t << 6)) & 0x03030303u;
            const u32 e = __builtin_amdgcn_perm(0u, 0xff000000u, code);
            const u32 x = __builtin_amdgcn_perm(0u, (u32)(str >> c3), 0x01010000u);
            const u32 s4 = bfi_s(0x00FF00FFu, x << 2, x >> 2) & 0x3C3C3C3Cu;
            const u32 u4 = s4 + and_or(ge13(s4 + 0x05050505u), 0x28282828u, 0x04040404u);
            B[k] = __builtin_amdgcn_perm(0u, u4, ff_lut(e, lut)) | __builtin_amdgcn_perm(0u, 0x0000706Cu, code);
            c3 = (u32)__builtin_amdgcn_sdot4((int)e, (int)0xfcfcfcfcu, (int)c3, true);
        }
        *(lu32x4*)(buf + (w0 + lane) * 16) = B;
    }
    done();
    u32 nib[16], ef[16], sel[16], cnt[4] = {0u, 0u, 0u, 0u}, pos[4];
#pragma unroll
    for (int i = 0; i < 16; ++i) {
        const u32 n = i < 8 ? N0[i >> 1] : N1[(i >> 1) - 4];
        nib[i] = ((i & 1) ? n >> 4 : n) & 0x0F0F0F0Fu;
        ef[i] = ge13(nib[i] | 0x01010101u);
        sel[i] = ff_lut(ef[i], lut);
        cnt[i >> 2] = ff_count(ef[i], cnt[i >> 2]);
    }
    chunk_starts(cnt, pos, (u32)(uintptr_t)buf);
    CBAR();
    u32 ps[16], q0[16], q1[16];
#pragma unroll
    for (int i = 0; i < 16; ++i) {
        const int j = (i & 3) * 4 + (i >> 2);
        ps[j] = (j & 3) ? ff_count(ef[j - 1], ps[j - 1]) : pos[j >> 2];
        const lu32* const q = (const lu32*)(uintptr_t)(ps[j] & ~3u);
        q0[j] = q[0]; q1[j] = q[1];
    }
    __builtin_amdgcn_sched_barrier(0);
#pragma unroll
    for (int i = 0; i < 16; ++i) {
        const u32 y = nib[i] + base4 + __builtin_amdgcn_perm(0u, __builtin_amdgcn_alignbyte(q1[i], q0[i], ps[i]), sel[i]);
        v[i >> 2][i & 3] = bfi_s(0x7F7F7F7Fu, y, S[i >> 3] << (7 - (i & 7)));
    }
    CBAR();
}

__device__ __forceinline__ int group_of(int r, int sg) {
    if (sg < 3 * FG) {
        const int s = sg / FG;
        return 16 * (2 * (((r >> 1) + 1 + s) & 3) + (r & 1)) + sg - s * FG;
    }
    if (sg < XGREM) {
        const int j = (sg - 3 * FG) / GG;
        return 16 * (2 * j + 1 - (r & 1)) + FG + sg - 3 * FG - j * GG;
    }
    return 16 * r + sg - XGREM;
}

__device__ __forceinline__ int sent_group(int r, int p, int i) {
    const bool same = ((r ^ p) & 1) == 0;
    const int at = same ? FG * (((p >> 1) - (r >> 1) - 1) & 3) : 3 * FG + GG * (p >> 1);
    return i < at ? i : i + (same ? FG : GG);
}

struct Unit {
    u32 slot;
    u32 flag;
    u32 dst;
    u32 kind;
    u32 aux;
};
__device__ __forceinline__ Unit unit_of(int n, int r, int p) {
    const bool same = ((r ^ p) & 1) == 0;
    const int nx = XGREM - (same ? FG : GG);
    const int U1 = nx - NPUSH;
    const int U2 = U1 + 16 * I8A;
    const int U3 = U2 + nx;
    const int U4 = U3 + 16 * I8A;
    const int U5 = U4 + 32 * (8 - I8A);
    const int U6 = U5 + 32;
    Unit u;
    if (n >= U6) {
        const int v = n - U6, wn = v & 1, h = v >> 1;
        int i8, part, rg, lg;
        if (same) {
            i8 = h / (4 * FG);
            const int t = h - i8 * (4 * FG);
            part = t / FG; lg = t - part * FG;
            rg = FG * (((r >> 1) - (p >> 1) - 1) & 3) + lg;
        } else {
            i8 = h / (3 * GG);
            const int t = h - i8 * (3 * GG);
            const int pi = t / GG, jr = r >> 1, g = t - pi * GG;
            part = pi + (pi >= jr);
            rg = 3 * FG + (jr > part ? jr - 1 : jr) * GG + g;
            lg = FG + g;
        }
        const int id = ((part * 8 + i8) * 48 + rg) * 2 + wn, gct = block_of(p, part) * 8 + i8;
        u.slot = SL_O + id; u.flag = FL_O + id; u.kind = 2;
        u.dst = (u32)(lg * 32 * NN + gct * 128 + wn * 64) * 4u;
        u.aux = 0;
        return u;
    }
    if (n < U1 || (n >= U2 && n < U3) || n >= U5) {
        int sg, kp;
        if (n < U1) { sg = sent_group(r, p, n + NPUSH); kp = 0; }
        else if (n < U3) { sg = sent_group(r, p, n - U2); kp = 1; }
        else { sg = XGREM + ((n - U5) & 15); kp = (n - U5) >> 4; }
        const int gg = group_of(r, sg);
        u.slot = SL_X + gg * 2 + kp; u.flag = FL_X + gg * 2 + kp; u.kind = 0;
        u.dst = (u32)(WS_GX + (size_t)(p * 4 + 2 * kp) * XSLAB) + (u32)sg * 4096u;
        u.aux = (u32)(sg < XGREM ? WS_XREM : WS_XOWN) + (u32)(p * 2 + kp) * 128u;
        return u;
    }
    int i8, kc, t;
    if (n < U2) { const int v = n - U1; i8 = v >> 4; kc = (v >> 3) & 1; t = v & 7; }
    else if (n < U4) { const int v = n - U3; i8 = v >> 4; kc = 2 + ((v >> 3) & 1); t = v & 7; }
    else { const int v = n - U4; i8 = I8A + (v >> 5); kc = (v >> 3) & 3; t = v & 7; }
    const int part = t >> 1, half = t & 1, ct = part * 8 + i8, step = p * 4 + kc;
    u.slot = SL_W + ((kc * 128 + block_of(r, part) * 8 + i8) * 2 + half);
    u.flag = FL_W + (kc * 32 + ct) * 2 + half;
    u.kind = 1 | (half << 2);
    u.dst = (u32)WS_GW + (u32)(step * 32 + ct) * 16384u + (u32)half * 8192u;
    u.aux = (u32)WS_WCNT + (u32)(ct * 32 + step) * 128u;
    return u;
}

#define STAMP(k) do { } while (0)

__global__ void __launch_bounds__(NTHR, 2) fused_kernel(Params P) {
    __shared__ __attribute__((aligned(1024))) uint8_t lds_[LDS_CTL + 384];
    lbyte* const lds = (lbyte*)lds_;
    const int tid = threadIdx.x;
    const int lane = tid & 63;
    const int wave = __builtin_amdgcn_readfirstlane(tid >> 6);
    const int b = blockIdx.x;
    if (b >= NENG && b < NSEND) {
        u64 pc;
        asm volatile("s_getpc_b64 %0" : "=s"(pc));
        const uint8_t* cb = (const uint8_t*)(pc & ~255ull) + tid * 16;
        u32x4 sink = {0u, 0u, 0u, 0u};
#pragma unroll
        for (int j = 0; j < CODE_PF; ++j) sink |= *(const __attribute__((address_space(1))) u32x4*)(cb + j * 8192);
        asm volatile("" :: "v"(sink));
    }
    const int rank = P.rank;
    uint8_t* const myws = P.myws;
    const uint8_t* const xin = P.x;
    asm volatile("" :: "s"(rank), "s"(myws), "s"(xin));
    u32* const tmo = (u32*)(myws + WS_TMO);

    STAMP(0);
    u32* const cntp = (u32*)(myws + WS_CNT + (size_t)(b * 8 + wave) * 128);
#define LOAD_CALL() \
    const u32 ncalls = __hip_atomic_load(cntp, __ATOMIC_RELAXED, __HIP_MEMORY_SCOPE_AGENT)
#define GET_CALL() \
    const u32 call = __builtin_amdgcn_readfirstlane(ncalls) + 1; \
    const size_t pkoff = WS_PK + (size_t)(call & 1) * PK_BYTES
#define PUT_CALL() \
    if (lane == 0) __hip_atomic_store(cntp, call, __ATOMIC_RELAXED, __HIP_MEMORY_SCOPE_AGENT)

    if (b < NSEND) {
        if (b >= NENG) return;
        LOAD_CALL();
        const int p = (rank + 1 + b % 7) & 7;
        const bool same = ((rank ^ p) & 1) == 0;
        const int nop = 2 * (XGREM - (same ? FG : GG)) - NPUSH + 256 + 32;
        const int nend = nop + (same ? 64 * FG : 48 * GG);
        for (int n = tid; n < nend; n += NTHR) {
            const Unit u = unit_of(n, rank, p);
            *(lu32x4*)(lds + EL_TAB + n * 16) = u32x4{u.slot | (u.kind << 16), u.flag, u.dst, u.aux};
        }
        if (tid < (EL_END - EL_GO) / 4) *(lu32*)(lds + EL_GO + tid * 4) = 0;
        *(lu32x4*)(lds + EL_PLANE + tid * 16) = u32x4{0u, 0u, 0u, 0u};
        if (tid < 16) op_lut_init(lds + LDS_LUT, tid);
        __syncthreads();
        GET_CALL();
        PUT_CALL();
        auto lds_get = [&](int off) {
            return (u32)__builtin_amdgcn_readfirstlane(
                __hip_atomic_load((lu32*)(lds + off), __ATOMIC_RELAXED, __HIP_MEMORY_SCOPE_WORKGROUP));
        };
        auto lds_put = [&](int off, u32 v) {
            CBAR();
            if (lane == 0) __hip_atomic_store((lu32*)(lds + off), v, __ATOMIC_RELAXED, __HIP_MEMORY_SCOPE_WORKGROUP);
            CBAR();
        };

        if (wave == 0) {
            __builtin_amdgcn_s_setprio(3);
            WAIT_VM(0);
            const uint8_t* const pk = P.ws[p] + pkoff;
            const u32 lds_addr = (u32)(uintptr_t)lds;
            const u32 vo0 = lane * 16, vo1 = lane * 16 + 4096;
            u32 pub = 0, psl = 0;
            u32 issued = 0;
            u32 end0 = 0, end1 = 0;
            static_assert(WIN <= 12 && WIN_OP <= 12 && WIN_TAIL <= 12, "at most two whole units may be in flight behind the current one");
#define DMA1(W, LINES, VO, M0OFF, OFF) do { \
                asm volatile("s_waitcnt vmcnt(%0)" :: "i"(W - 1) : "memory"); \
                if (pub < k && end0 + (W - 1) <= issued) { \
                    lds_put(EL_FULL + psl * 4, ++pub); psl = psl == NS - 1 ? 0 : psl + 1; end0 = end1; } \
                if ((u32)lane < (LINES - (OFF >> 7)) * 8u) \
                    asm volatile("s_mov_b32 m0, %2\n\ts_nop 0\n\tglobal_load_lds_dwordx4 %0, %1 offset:" #OFF " sc0 sc1" \
                                 :: "v"(VO), "s"(src), "s"(la + M0OFF) : "memory"); \
                ++issued; \
            } while (0)
            u32 sl = 0;
            for (u32 k = 0;; ++k) {
                u32 go = lds_get(EL_GO + sl * 16);
                if ((go >> 16) != k + 1 || (go & 0xffffu) == UNIT_END) {
                    WAIT_VM(0);
                    while (pub < k) { lds_put(EL_FULL + psl * 4, ++pub); psl = psl == NS - 1 ? 0 : psl + 1; }
                    while ((go >> 16) != k + 1) {
                        __builtin_amdgcn_s_sleep(1);
                        go = lds_get(EL_GO + sl * 16);
                    }
                    if ((go & 0xffffu) == UNIT_END) break;
                }
                lds_put(EL_ISS, k + 1);
                const int un = go & 0xffffu;
                const u32 h0 = lds_get(EL_GO + sl * 16 + 4), h1 = lds_get(EL_GO + sl * 16 + 8);
                const u32 l0 = un < nop ? max((h0 & 31u) + 1, 17u) : 32u;
                const u32 l1 = un < nop ? max((h1 & 31u) + 1, 17u) : min(max(h0 & 127u, 41u), 64u) - 32u;
                const uint8_t* src = pk + (size_t)(lds_get(EL_TAB + (go & 0xffffu) * 16) & 0xffffu) * SLOT;
                const u32 la = lds_addr + sl * SLOT;
                u32 keep;
                asm volatile("s_mov_b32 %0, m0\n\ts_nop 4" : "=s"(keep) :: "memory");
#define DMAU(W) do { \
                DMA1(W, l0, vo0, 0, 0); DMA1(W, l0, vo0, 0, 1024); DMA1(W, l0, vo0, 0, 2048); \
                if (l0 > 24) DMA1(W, l0, vo0, 0, 3072); \
                DMA1(W, l1, vo1, 4096, 0); DMA1(W, l1, vo1, 4096, 1024); \
                if (l1 > 16) DMA1(W, l1, vo1, 4096, 2048); \
                if (l1 > 24) DMA1(W, l1, vo1, 4096, 3072); } while (0)
                if (un < nop) DMAU(WIN_OP);
                else if (un < nend - TAIL_UNITS) DMAU(WIN);
                else DMAU(WIN_TAIL);
                asm volatile("s_mov_b32 m0, %0" :: "s"(keep) : "memory");
                if (pub + 2 <= k) {
                    WAIT_VM(0);
                    while (pub < k) { lds_put(EL_FULL + psl * 4, ++pub); psl = psl == NS - 1 ? 0 : psl + 1; }
                }
                if (pub == k) end0 = issued; else end1 = issued;
                sl = sl == NS - 1 ? 0 : sl + 1;
            }
            STAMP(4);
            return;
        }
        if (wave == 1) {
            const u64* const flags = (const u64*)(myws + WS_FLAG) + (size_t)p * 4096 * 16 + (call & 1) * 2;
            u32* const tick = (u32*)(myws + WS_TICK) + p * 32;
            static_assert(BATCH % 2 == 1 && BATCH <= 7, "a bijection needs BATCH odd");
            constexpr int TC = 8 * (BATCH - 1) / BATCH;
            const u32 ntick = (nend + BATCH - 1) / BATCH + TC, tbase = (call - 1) * (ntick - 8) - 16;
            u32 tick_no = b / 7;
            bool dead = false, partials = false;
            auto give_up = [&](u64 t0) {
                if (__hip_atomic_load(tmo, __ATOMIC_RELAXED, __HIP_MEMORY_SCOPE_AGENT) == call ||
                    __builtin_amdgcn_s_memrealtime() - t0 > DEADLINE) {
                    if (lane == 0) __hip_atomic_store(tmo, call, __ATOMIC_RELAXED, __HIP_MEMORY_SCOPE_AGENT);
                    dead = true;
                    lds_put(EL_DEAD, 1);
                    return true;
                }
                __builtin_amdgcn_s_sleep(2);
                return false;
            };
            auto post = [&](u32 kk, u32 sl, u32 unit, u32 hdr, u32 hdr1) {
                while (min(lds_get(EL_FREE + sl * 8), lds_get(EL_FREE + sl * 8 + 4)) + NS < kk + 1) __builtin_amdgcn_s_sleep(1);
                lds_put(EL_GO + sl * 16 + 4, hdr);
                lds_put(EL_GO + sl * 16 + 8, hdr1);
                lds_put(EL_GO + sl * 16, ((kk + 1) << 16) | unit);
            };
            u32 k = 0, sl = 0;
            while (tick_no < ntick) {
                const int u0 = BATCH * ((int)tick_no - TC);
                while (lds_get(EL_ISS) + LOOK < k) __builtin_amdgcn_s_sleep(1);
                const int ul = min(max(u0 + 8 * min((lane >> 1) & 7, BATCH - 1), 0), nend - 1);
                const u64* const fp = flags + (size_t)*(lu32*)(lds + EL_TAB + ul * 16 + 4) * 16 + (ul < nop ? lane & 1 : 0);
                u32 tk;
                u64 fv;
                asm volatile("s_mov_b64 exec, 1\n\tglobal_atomic_add %0, %2, %3, off sc0\n\ts_mov_b64 exec, %5\n\t"
                             "global_load_dwordx2 %1, %4, off sc0 sc1\n\ts_mov_b64 exec, -1\n\ts_waitcnt vmcnt(0)"
                             : "=&v"(tk), "=&v"(fv) : "v"(tick), "v"(tick_no < 8 ? 0u : 1u), "v"(fp), "i"((1 << (2 * BATCH)) - 1) : "memory");
                for (int j = 0; j < BATCH; ++j) {
                    const int un = u0 + 8 * j;
                    if (un < 0 || un >= nend) continue;
                    if (!dead && ((__ballot((u32)fv >= call) >> (2 * j)) & 3) != 3) {
                        while (lds_get(EL_ISS) + LOOK < k) __builtin_amdgcn_s_sleep(1);
                        const u64 t0 = __builtin_amdgcn_s_memrealtime();
                        for (u32 it = 1;; ++it) {
                            if (lane >= 2 * j && lane < 2 * BATCH) fv = __hip_atomic_load(fp, __ATOMIC_RELAXED, __HIP_MEMORY_SCOPE_SYSTEM);
                            if (((__ballot((u32)fv >= call) >> (2 * j)) & 3) == 3 || ((it & 7) == 0 && give_up(t0))) break;
                            __builtin_amdgcn_s_sleep(1);
                        }
                    }
                    if (un >= nop && !partials) {
                        partials = true;
                        const u64 t0 = __builtin_amdgcn_s_memrealtime();
                        while (!dead && __hip_atomic_load((const u32*)(myws + WS_PCNT), __ATOMIC_RELAXED, __HIP_MEMORY_SCOPE_SYSTEM) < 96u * call)
                            if (give_up(t0)) break;
                    }
                    const u32 hi = (u32)(fv >> 32);
                    post(k, sl, un, __builtin_amdgcn_readlane(hi, 2 * j), __builtin_amdgcn_readlane(hi, 2 * j + 1));
                    sl = sl == NS - 1 ? 0 : sl + 1;
                    ++k;
                }
                tick_no = tick_no < 8 ? tick_no + 8 : __builtin_amdgcn_readfirstlane(tk) - tbase;
            }
            for (u32 kk = k; kk < k + NPAIR; ++kk) {
                post(kk, sl, UNIT_END, 0, 0);
                sl = sl == NS - 1 ? 0 : sl + 1;
            }
            return;
        }
        lbyte* const plane = lds + EL_PLANE + wave * 1024;
        const float scale0 = P.sx[0] * P.sw[0];
        const u32 hf = wave & 1;
        u32 owed = 0;
        auto settle = [&]() {
            if (!owed) return;
            WAIT_VM(0);
            if (lane == 0) __hip_atomic_fetch_add((u32*)(myws + owed), 1u, __ATOMIC_RELAXED, __HIP_MEMORY_SCOPE_SYSTEM);
            owed = 0;
        };
        u32 sl = (wave - 2) >> 1;
        for (u32 k = sl;; k += NPAIR) {
            lbyte* const s = lds + sl * SLOT;
            u32 go;
            while (((go = lds_get(EL_GO + sl * 16)) >> 16) != k + 1) { settle(); __builtin_amdgcn_s_sleep(1); }
            if ((go & 0xffffu) == UNIT_END) break;
            CBAR();
            const u32 hdr = lds_get(EL_GO + sl * 16 + 4 + hf * 4);
            const u32x4 te = *(lu32x4*)(lds + EL_TAB + (go & 0xffffu) * 16);
            const u32 kind = __builtin_amdgcn_readfirstlane(te[0]) >> 16;
            const u32 dst = __builtin_amdgcn_readfirstlane(te[2]);
            const u32 aux = __builtin_amdgcn_readfirstlane(te[3]);
            const bool acc = kind == 2;
            u32x4 pa[4];
            uint8_t* const od = (uint8_t*)P.out + dst + (size_t)(hf * 16 + (lane >> 4) * 4) * (NN * 4) + (lane & 15) * 16;
            if (acc) {
                settle();
                asm volatile("global_load_dwordx4 %0, %4, off sc0 sc1\n\tglobal_load_dwordx4 %1, %5, off sc0 sc1\n\t"
                             "global_load_dwordx4 %2, %6, off sc0 sc1\n\tglobal_load_dwordx4 %3, %7, off sc0 sc1\n\t"
                             "s_waitcnt vmcnt(0)"
                             : "=&v"(pa[0]), "=&v"(pa[1]), "=&v"(pa[2]), "=&v"(pa[3])
                             : "v"(od), "v"(od + NN * 4), "v"(od + NN * 8), "v"(od + NN * 12) : "memory");
            }
            const float scale = lds_get(EL_DEAD) ? __builtin_nanf("") : scale0;
            while (lds_get(EL_FULL + sl * 4) != k + 1) { settle(); __builtin_amdgcn_s_sleep(1); }
            CBAR();
            if (!acc) {
                u32x4 v[4];
                const lbyte* const blk = s + hf * 4096;
                if ((hdr & 31u) == 31u) {
#pragma unroll
                    for (int j = 0; j < 4; ++j) v[j] = *(const lu32x4*)(blk + j * 1024 + lane * 16);
                    lds_put(EL_FREE + sl * 8 + hf * 4, k + 1);
                } else {
                    op_unpack(v, ((hdr >> 13) & 31u) * 0x04040404u, min((hdr >> 5) & 255u, 128u), blk,
                              lds + EL_BUF + wave * 2304, lds + LDS_LUT, lane,
                              [&] { lds_put(EL_FREE + sl * 8 + hf * 4, k + 1); });
                }
                const bool isw = kind & 1;
                uint8_t* d = myws + dst + (isw ? hf * 4096 : (call & 1) * GX_ODD + hf * XSLAB) +
                             ((lane ^ (isw ? (kind >> 2) * 8 + hf * 4 : 0)) << 4);
                settle();
#pragma unroll
                for (int j = 0; j < 4; ++j) st_sys(d + j * 1024, v[j]);
                owed = aux;
            } else {
                float a[16], c1 = 1.0f;
                if (hdr & H_RAW) {
#pragma unroll
                    for (int j = 0; j < 4; ++j) {
                        const u32x4 t = *(lu32x4*)(s + hf * 4096 + j * 1024 + lane * 16);
#pragma unroll
                        for (int t4 = 0; t4 < 4; ++t4) a[4 * j + t4] = __uint_as_float(t[t4]);
                    }
                    lds_put(EL_FREE + sl * 8 + hf * 4, k + 1);
                } else {
                    const u32x4 h0 = *(lu32x4*)(s + hf * 2048 + lane * 16), h1 = *(lu32x4*)(s + hf * 2048 + 1024 + lane * 16);
                    u32x4 mid = *(lu32x4*)(s + 4096 + lane * 16);
                    const u32 cnt = min((hdr >> 8) & 0x7ffu, (SLOT - ACC_MAIN) / 2);
                    for (u32 o = lane; 2 * o < cnt; o += 64) {
                        const u32 e2 = *(lu32*)(s + ACC_MAIN + o * 4);
#pragma unroll
                        for (int h = 0; h < 2; ++h) {
                            const u32 e = h ? e2 >> 16 : e2 & 0xffffu;
                            if (2 * o + h < cnt && (e >> 15) == hf) *(plane + ((e >> 5) & 0x3ffu)) = (uint8_t)(e & 31u);
                        }
                    }
                    CBAR();
                    const u32x4 q = *(lu32x4*)(plane + lane * 16);
                    *(lu32x4*)(plane + lane * 16) = u32x4{0u, 0u, 0u, 0u};
                    if (hf) {
                        const u32 m4 = *(lu32*)(s + 5120 + lane * 4), m5 = *(lu32*)(s + 5376 + lane * 4), m6 = *(lu32*)(s + 5632 + lane * 4);
                        u32 m7 = (mid[0] << 1) & 0x02020202u;
                        m7 |= (mid[1] << 2) & 0x04040404u; m7 |= (mid[2] << 3) & 0x08080808u; m7 |= (mid[3] << 4) & 0x10101010u;
                        m7 |= (m4 << 5) & 0x20202020u; m7 |= (m5 << 6) & 0x40404040u; m7 |= (m6 << 7) & 0x80808080u;
                        mid = u32x4{m4, m5, m6, m7};
                    }
                    lds_put(EL_FREE + sl * 8 + hf * 4, k + 1);
                    c1 = __uint_as_float(((hdr >> 19) & 255u) << 23);
#pragma unroll
                    for (int j = 0; j < 4; ++j) {
                        const u32 mi = bfi(0x01010101u, q[j] >> 2, mid[j]);
                        const u32 lo = (q[j] & 0x03030303u) | ((q[j] << 3) & 0xC0C0C0C0u);
                        const u32 l0 = __builtin_amdgcn_perm(mi, lo, 0x05010400u), l1 = __builtin_amdgcn_perm(mi, lo, 0x07030602u);
                        const u32 ha = j < 2 ? h0[2 * j] : h1[2 * j - 4], hb = j < 2 ? h0[2 * j + 1] : h1[2 * j - 3];
                        const u32 T[4] = {__builtin_amdgcn_perm(ha, l0, 0x05040100u), __builtin_amdgcn_perm(ha, l0, 0x07060302u),
                                          __builtin_amdgcn_perm(hb, l1, 0x05040100u), __builtin_amdgcn_perm(hb, l1, 0x07060302u)};
#pragma unroll
                        for (int t4 = 0; t4 < 4; ++t4) a[4 * j + t4] = (float)(((int)T[t4] >> 2) ^ (int)(T[t4] << 30));
                    }
                }
#pragma unroll
                for (int j = 0; j < 4; ++j) {
                    f32x4 y;
#pragma unroll
                    for (int t4 = 0; t4 < 4; ++t4) {
                        const u32 pb = pa[j][t4];
                        y[t4] = silu(fmaf(a[4 * j + t4], c1, __uint_as_float(pb)) * scale);
                    }
                    st_mode<NT_OUT>(od + (size_t)j * (NN * 4), __builtin_bit_cast(u32x4, y));
                }
            }
            sl = sl >= NS - NPAIR ? sl + NPAIR - NS : sl + NPAIR;
        }
        settle();
        return;
    }

    const int g = b - NSEND;
    lbyte* const scr = lds + wave * 6144;
    constexpr int WFIRST = 64 * I8A, GLATE = 64 + 16 * I8A;
    static_assert(I8A >= 2 && I8A <= 4, "tasks");
    const int task0 = g + 192 * wave;
    const int pdst = (rank + 1 + task0 % 7) & 7;
    const int psg = sent_group(pdst, rank, task0 / 7);
    const int gg = task0 < 256 ? group_of(pdst, psg) : task0 & 127;
    u32x4 v[8];
    if (task0 < 7 * NPUSH || (task0 >= 256 && task0 < 768)) {
        const uint8_t* src = xin + (size_t)(gg * 32 + (lane >> 3)) * KS + (task0 < 256 ? 0 : (task0 - 256) & ~127) + (lane & 7) * 16;
#pragma unroll
        for (int j = 0; j < 4; ++j) v[j] = *(const u32x4*)(src + j * 8 * KS);
        if (task0 < 256) {
#pragma unroll
            for (int j = 0; j < 4; ++j) v[4 + j] = *(const u32x4*)(src + j * 8 * KS + 128);
        }
    }
    LOAD_CALL();
    uint8_t* const wsl = P.ws[lane & 7];
    __builtin_amdgcn_sched_barrier(0);
    const u64 tstart0 = __builtin_amdgcn_s_memrealtime();
    lbyte* const lut = scr + 4096;
    if (lane < 16) op_lut_init(lut, lane);
    CBAR();
    if (task0 < 7 * NPUSH) {
        uint8_t* const wd = P.ws[pdst];
        uint8_t* d = wd + WS_GX + (size_t)(rank * 4) * XSLAB + (size_t)psg * 4096 + lane * 16;
        GET_CALL();
        d += (call & 1) * GX_ODD;
#pragma unroll
        for (int j = 0; j < 8; ++j) asm volatile("" : "+v"(v[j]));
#pragma unroll
        for (int j = 0; j < 8; ++j) {
            THROTTLE(PUSH_WIN);
            st_sys(d + (j >> 2) * XSLAB + (j & 3) * 1024, v[j]);
        }
        WAIT_VM(0);
        if (lane == 0)
            __hip_atomic_fetch_add((u32*)(wd + WS_XREM) + rank * 2 * 32, 2u, __ATOMIC_RELAXED, __HIP_MEMORY_SCOPE_SYSTEM);
    }
    const int l7 = lane & 7;
    bool isx = task0 >= 256 && task0 < 768, send = true, keep = true, tell = false;
    int h0 = 0, h1 = 2, flag = 0;
    u32 dbo = 0, swz = 0, slo = 0, cno = 0;
    if (isx) {
        const int kc = (task0 - 256) >> 7, q = gg >> 4, lg = gg & 15;
        const int fl = (lg >= FG) ^ (q & 1);
        h0 = kc & 1; h1 = h0 + 1;
        send = q != rank;
        keep = q == rank || (rank & 1) == fl;
        tell = send && l7 != rank && (l7 == q || (l7 & 1) == fl);
        flag = FL_X + gg * 2 + (kc >> 1);
        const int sg = q == rank ? XGREM + lg
                     : lg < FG ? FG * (((q >> 1) - (rank >> 1) - 1) & 3) + lg
                               : 3 * FG + GG * (q >> 1) + lg - FG;
        dbo = (u32)WS_GX + (u32)(rank * 4 + kc) * (u32)XSLAB + (u32)(sg - h0) * 4096u;
        slo = (u32)(SL_X + gg * 2 + (kc >> 1)) * (u32)SLOT;
        cno = (u32)(q == rank ? WS_XOWN : WS_XREM) + (u32)(rank * 2 + (kc >> 1)) * 128u;
        asm volatile("" : "+s"(dbo), "+s"(slo), "+s"(cno), "+v"(flag));
    }
    for (int phase = task0 >= 256 && task0 < 768 + 3 * WFIRST ? 0 : 1; phase < (g >= GLATE ? 2 : 1); ++phase) {
        if (phase) isx = false;
        if (!isx) {
            int wt = phase ? 2 * WFIRST + g - GLATE + (192 - GLATE) * (7 - wave) : task0 - 768 - WFIRST;
            h0 = 0; h1 = 2;
            if (wt < 0) { wt += WFIRST; h1 = 1; } else if (wt < WFIRST) h0 = 1;
            const int wq = wt >> 5;
            const int half = (wt >> 3) & 1, blk = (wt & 7) | ((wt >> 1) & 8);
            const int kc = wq < 2 * I8A ? wq & 1 : wq < 4 * I8A ? 2 + (wq & 1) : wq & 3;
            const int wi8 = wq < 2 * I8A ? wq >> 1 : wq < 4 * I8A ? (wq >> 1) - I8A : wq >> 2;
            if (wt >= WFIRST) {
                while (__builtin_amdgcn_s_memrealtime() - tstart0 < (u64)(phase ? W_DELAY + LATE_STAG * (7 - wave) : W_DELAY0))
                    __builtin_amdgcn_s_sleep(8);
            }
            const uint8_t* src = P.w + (size_t)(kc * 128 + 16 * (lane >> 3)) * NN + blk * 1024 + wi8 * 128 + 16 * l7 + 8 * half;
            u32x2 in[16];
#pragma unroll
            for (int j = 0; j < 16; ++j) in[j] = *(const u32x2*)(src + (size_t)j * NN);
            __builtin_amdgcn_sched_barrier(0);
            auto needs = [&](int r) { return (r >> 1) == ((r & 1) ? blk & 3 : blk >> 2); };
            auto ctile = [&](int r) { return ((r & 1) ? blk >> 2 : blk & 3) * 8 + wi8; };
            send = keep = true;
            tell = l7 != rank && needs(l7);
            flag = FL_W + (kc * 32 + ctile(l7)) * 2 + half;
            const bool mine = needs(rank);
            const int fb = (rank & 1) ? 3 * (blk >> 2) + (blk & 3) - ((blk & 3) > (rank >> 1))
                                      : blk - (blk >= 2 * rank ? 4 : 0);
            const int ft = fb * 8 + wi8;
            dbo = half * 8192u + (mine ? (u32)WS_GW + (u32)((rank * 4 + kc) * 32 + ctile(rank)) * 16384u
                                       : (u32)WS_GW2 + (u32)(((ft >> 5) * 4 + kc) * 32 + (ft & 31)) * 16384u);
            swz = half * 8;
            slo = (u32)(SL_W + (kc * 128 + blk * 8 + wi8) * 2 + half) * (u32)SLOT;
            cno = (u32)WS_WCNT + (u32)((mine ? ctile(rank) : 32 + ft) * 32 + rank * 4 + kc) * 128u;
            asm volatile("" : "+s"(dbo), "+s"(slo), "+s"(cno), "+v"(flag));
#pragma unroll
            for (int d = 0; d < 2; ++d)
#pragma unroll
                for (int e = 0; e < 4; ++e) {
                    const u32 t0 = __builtin_amdgcn_perm(in[4 * e + 1][d], in[4 * e][d], 0x05010400u);
                    const u32 t1 = __builtin_amdgcn_perm(in[4 * e + 1][d], in[4 * e][d], 0x07030602u);
                    const u32 t2 = __builtin_amdgcn_perm(in[4 * e + 3][d], in[4 * e + 2][d], 0x05010400u);
                    const u32 t3 = __builtin_amdgcn_perm(in[4 * e + 3][d], in[4 * e + 2][d], 0x07030602u);
                    v[4 * d + 0][e] = __builtin_amdgcn_perm(t2, t0, 0x05040100u);
                    v[4 * d + 1][e] = __builtin_amdgcn_perm(t2, t0, 0x07060302u);
                    v[4 * d + 2][e] = __builtin_amdgcn_perm(t3, t1, 0x05040100u);
                    v[4 * d + 3][e] = __builtin_amdgcn_perm(t3, t1, 0x07060302u);
                }
            if (h0) {
#pragma unroll
                for (int j = 0; j < 4; ++j) v[j] = v[4 + j];
            }
        }
        GET_CALL();
#pragma unroll
        for (int j = 0; j < 8; ++j) asm volatile("" : "+v"(v[j]));
        uint8_t* const db = myws + dbo + (isx ? (call & 1) * GX_ODD : 0);
        uint8_t* const slot = myws + pkoff + slo;
        u32 es = 0;
#pragma unroll
        for (int j = 0; j < 8; ++j) es = __builtin_amdgcn_sad_u8(v[j >> 1][(j & 1) * 2] & 0x7C7C7C7Cu, 0u, es);
        const u32 base = (u32)min(max(((int)wave_sum(es) - 1475) >> 13, 0), 29);
        u32 hdr = 0;
        for (int hh = h0; hh < h1; ++hh) {
            if (send) {
                const u32 h = op_pack(v, base * 0x04040404u, slot + hh * 4096, scr, lut, lane) | (base << 13);
                if ((lane >> 3) == hh) hdr = h;
            }
            if (keep) {
                uint8_t* d = db + hh * 4096 + ((lane ^ (isx ? 0 : swz + hh * 4)) << 4);
#pragma unroll
                for (int j = 0; j < 4; ++j) st_sys(d + j * 1024, v[j]);
            }
#pragma unroll
            for (int j = 0; j < 4; ++j) v[j] = v[4 + j];
        }
        WAIT_VM(0);
        if (tell && (lane >> 3) >= h0 && (lane >> 3) < h1) st_flag(wsl, rank, flag, lane >> 3, call, hdr);
        if (keep && lane == 0)
            __hip_atomic_fetch_add((u32*)(myws + cno), (u32)(h1 - h0), __ATOMIC_RELAXED, __HIP_MEMORY_SCOPE_SYSTEM);
    }
    GET_CALL();
    PUT_CALL();
    STAMP(1);

    const int xcd = g & 7, idx = g >> 3;
    const int part = xcd >> 1;
    const int wm = wave >> 1, wn = wave & 1;
    const int c = lane & 15, q4 = lane >> 4;
    const int ra0 = wm * 8192 + c * 128 + ((q4 ^ (c >> 1)) << 4);
    const int ra1 = ra0 ^ 64;
    const int rb0 = 32768 + (c & 3) * 4096 + ((((q4 << 3) | (4 * wn + (c >> 2))) ^ ((c & 3) << 2)) << 4);
    const u32 boff = (u32)wave * 2048u + (u32)lane * 16u;
    const u32 la0 = (u32)(uintptr_t)lds + wave * 4096, lb0 = (u32)(uintptr_t)lds + 32768 + wave * 2048;
    const u32 aoff0 = (u32)(lane >> 3) * 128u + (u32)(((lane & 7) ^ (lane >> 4)) << 4), aoff1 = aoff0 ^ 64u;

    for (int job = idx >= 12 ? 0 : 2; job < (idx < 8 ? 4 : 3); ++job) {
        const bool pre = job < 2, local = job == 3, ownrows = job != 2;
        const int ft = g - 96;
        const int i8 = pre ? ft & 7 : local ? idx : idx / 3;
        const int rt = pre ? job : local ? (xcd & 1) : 3 * (xcd & 1) + idx % 3;
        const int ct = pre ? 32 + ft : part * 8 + i8;
        const int nstep = pre ? 4 : NSTEP;
        const u32 wbase = pre ? (u32)WS_GW2 + (u32)(((ft >> 5) - rank) * 128 + (ft & 31)) * 16384u : (u32)WS_GW + (u32)ct * 16384u;
        const int rgw = rt * 8 + wave;
        const int sgw = ownrows ? XGREM + rgw : (rgw < 3 * FG + part * GG ? rgw : rgw + GG);
        const int skipsrc = ownrows ? -1 : group_of(rank, sgw) >> 4;
        const u32 xbase = (u32)(WS_GX + (call & 1) * GX_ODD) + (u32)sgw * 4096u;

        const int psrc = lane < 32 ? lane >> 2 : (lane >> 1) & 7;
        const u32* const pollp = lane < 32 ? (const u32*)(myws + WS_WCNT) + (ct * 32 + lane) * 32
                                           : (const u32*)(myws + (ownrows ? WS_XOWN : WS_XREM)) + (lane & 15) * 32;
        const u32 pneed = call * (lane < 32 ? 4u : ownrows ? 32u : psrc == rank ? 2u * XGREM
                                  : ((psrc ^ rank) & 1) ? 2u * (XGREM - GG) : 2u * (XGREM - FG));
        u64 ready = job == 1 ? ~0ull : 0;
        const int tq = ((lane & 31) - 4) / 7;
        const u32 vst = (lane & 31) < 4 ? rank * 4 + (lane & 3) : ((rank + (lane & 31) - 3 - 7 * tq) & 7) * 4 + tq;
        auto stepno = [&](int s) { return (int)__builtin_amdgcn_readlane(vst, s); };
        auto ensure = [&](int s) {
            const int st = stepno(s);
            const u64 want = (1ull << st) | (1ull << (32 + (st >> 1)));
            if ((ready & want) == want) return;
            const u64 t0 = __builtin_amdgcn_s_memrealtime();
            for (;;) {
                u64 rd = (st >> 2) == rank ? (0xfull << (4 * rank)) | (3ull << (32 + 2 * rank))
                       : !((ready >> st) & 1) ? (POLL_ALLW ? 0xffffffffull : 0x11111111ull << (st & 3))
                                              : 0x5555ull << (32 + ((st >> 1) & 1));
                rd &= ~ready;
                u32 v = 0;
                if ((rd >> lane) & 1) v = __hip_atomic_load(pollp, __ATOMIC_RELAXED, __HIP_MEMORY_SCOPE_SYSTEM);
                const u64 got = __ballot(v >= pneed);
                ready |= got;
                if ((ready & want) == want) break;
                if ((got >> st) & 1) continue;
                if (__hip_atomic_load(tmo, __ATOMIC_RELAXED, __HIP_MEMORY_SCOPE_AGENT) == call ||
                    __builtin_amdgcn_s_memrealtime() - t0 > DEADLINE) {
                    if (lane == 0) __hip_atomic_store(tmo, call, __ATOMIC_RELAXED, __HIP_MEMORY_SCOPE_AGENT);
                    WAIT_VM(0);
                    ready = ~0ull;
                    break;
                }
                __builtin_amdgcn_s_sleep(POLL_SLEEP);
            }
        };
        bool polling = wave == 0;
        int age = 0;
        u64 asked = 0;
        auto peek_issue = [&](int s) {
            if (age) { ++age; return; }
            const int st = stepno(min(s + 3 + PEEK_AHEAD, nstep - 1));
            const u64 want = (1ull << st) | (1ull << (32 + (st >> 1)));
            if ((ready & want) == want) return;
            const u64 rw = ready & ~(0xfull << (4 * rank));
            asked = (0xffffffffull | ((rw & 0x33333333ull) ? 0x5555ull << 32 : 0) | ((rw & 0xccccccccull) ? 0xaaaaull << 32 : 0)) & ~ready;
            if (!asked) return;
            if ((asked >> lane) & 1) {
                u32 keep;
                asm volatile("s_mov_b32 %0, m0\n\ts_mov_b32 m0, %2\n\ts_nop 0\n\tglobal_load_lds_dword %1, off sc0 sc1\n\ts_mov_b32 m0, %0"
                             : "=&s"(keep) : "v"(pollp), "s"((u32)(uintptr_t)(lds + LDS_CTL)) : "memory");
            }
            age = 1;
        };
        auto peek_read = [&]() {
            if (age != 3) return;
            WAIT_VM(6);
            ready |= __ballot(*(volatile lu32*)(lds + LDS_CTL + lane * 4) >= pneed) & asked;
            age = 0;
        };
        const u32 vxo = (int)(vst >> 2) == skipsrc ? (u32)WS_ZERO : xbase + vst * (u32)XSLAB;
        const u32 vwo = wbase + vst * (32u * 16384u);
        const uint8_t *xa, *wa;
        auto locate = [&](int s) {
            xa = myws + (u32)__builtin_amdgcn_readlane(vxo, s);
            wa = myws + (u32)__builtin_amdgcn_readlane(vwo, s);
        };
#define DMA(M0V, VOFF, SRC, OFF) \
        asm volatile("s_mov_b32 m0, %2\n\ts_nop 0\n\tglobal_load_lds_dwordx4 %0, %1 offset:" #OFF " sc0 sc1" \
                     :: "v"(VOFF), "s"(SRC), "s"(M0V) : "memory")
        auto issue = [&](u32 so) {
            DMA(so + la0, aoff0, xa, 0); DMA(so + la0, aoff1, xa, 1024); DMA(so + la0, aoff0, xa, 2048); DMA(so + la0, aoff1, xa, 3072);
            DMA(so + lb0, boff, wa, 0); DMA(so + lb0, boff, wa, 1024);
        };

        f32x4 acc[4][4];
#pragma unroll
        for (int m = 0; m < 4; ++m)
#pragma unroll
            for (int t = 0; t < 4; ++t) acc[m][t] = f32x4{0.f, 0.f, 0.f, 0.f};

        auto rd1 = [&](const lbyte* p0, const lbyte* p1) {
            const u32x4 l = *(const lu32x4*)p0, h = *(const lu32x4*)p1;
            return i32x8{(int)l[0], (int)l[1], (int)l[2], (int)l[3], (int)h[0], (int)h[1], (int)h[2], (int)h[3]};
        };
#define MM(A, B, m, t) acc[m][t] = __builtin_amdgcn_mfma_scale_f32_16x16x128_f8f6f4(A[m], B[t], acc[m][t], 1, 1, 0, 0, 0, 0)
#define SB() __builtin_amdgcn_sched_barrier(0)
#define GTRACE() do { } while (0)
#define KSTEP(XA, XB, YA, YB, ISSUE, NVM) do { \
            if (ISSUE && polling) { peek_read(); ensure(s + 3); } \
            if (ISSUE && age) asm volatile("s_waitcnt vmcnt(7) lgkmcnt(0)" ::: "memory"); \
            else asm volatile("s_waitcnt vmcnt(" #NVM ") lgkmcnt(0)" ::: "memory"); \
            __builtin_amdgcn_s_barrier(); \
            asm volatile("" ::: "memory"); \
            SB(); MM(XA, XB, 0, 0); SB(); YA[0] = rd1(pa0, pa1); \
            SB(); MM(XA, XB, 0, 1); SB(); YB[0] = rd1(pb, pb + 512); \
            SB(); MM(XA, XB, 0, 2); SB(); if (ISSUE) DMA(ma, aoff0, xa, 0); \
            SB(); MM(XA, XB, 0, 3); SB(); YB[1] = rd1(pb + 1024, pb + 1536); \
            SB(); MM(XA, XB, 1, 0); SB(); if (ISSUE) DMA(ma, aoff1, xa, 1024); \
            SB(); MM(XA, XB, 1, 1); SB(); YB[2] = rd1(pb + 2048, pb + 2560); \
            SB(); MM(XA, XB, 1, 2); SB(); if (ISSUE) DMA(ma, aoff0, xa, 2048); \
            SB(); MM(XA, XB, 1, 3); SB(); YB[3] = rd1(pb + 3072, pb + 3584); \
            SB(); MM(XA, XB, 2, 0); SB(); if (ISSUE) DMA(ma, aoff1, xa, 3072); \
            SB(); MM(XA, XB, 2, 1); SB(); YA[1] = rd1(pa0 + 2048, pa1 + 2048); \
            SB(); MM(XA, XB, 2, 2); SB(); if (ISSUE) DMA(mb, boff, wa, 0); \
            SB(); MM(XA, XB, 2, 3); SB(); YA[2] = rd1(pa0 + 4096, pa1 + 4096); \
            SB(); MM(XA, XB, 3, 0); SB(); if (ISSUE) DMA(mb, boff, wa, 1024); \
            YA[3] = rd1(pa0 + 6144, pa1 + 6144); \
            SB(); MM(XA, XB, 3, 1); SB(); \
            ++s; \
            ma = sn + la0; mb = sn + lb0; \
            sn = sn == 2 * STAGE ? 0 : sn + STAGE; \
            pa0 = lds + sn + ra0; pa1 = lds + sn + ra1; pb = lds + sn + rb0; \
            if (ISSUE) locate(s + 3); \
            asm volatile("" : "+s"(ma), "+s"(mb), "+s"(xa), "+s"(wa), "+v"(pa0), "+v"(pa1), "+v"(pb)); \
            SB(); MM(XA, XB, 3, 2); MM(XA, XB, 3, 3); SB(); \
            if (ISSUE && polling) { peek_issue(s - 1); polling = age || (~ready << 16); } \
            GTRACE(); \
        } while (0)

        if (job == 0 && wave == 0) {
            while (__builtin_amdgcn_s_memrealtime() - tstart0 < (u64)(PRE_START + (idx - 12) * PRE_STAG)) __builtin_amdgcn_s_sleep(8);
        }
        if (wave == 0) { ensure(0); ensure(1); ensure(2); }
        asm volatile("s_waitcnt lgkmcnt(0)" ::: "memory");
        __builtin_amdgcn_s_barrier();
        asm volatile("" ::: "memory");
        asm volatile("s_nop 4");
        locate(0); issue(0);
        locate(1); issue(STAGE);
        locate(2); issue(2 * STAGE);
        locate(3);
        WAIT_VM(12);
        __builtin_amdgcn_s_barrier();
        asm volatile("" ::: "memory");
        i32x8 A0[4], B0[4], A1[4], B1[4];
#pragma unroll
        for (int t = 0; t < 4; ++t) {
            A0[t] = rd1(lds + ra0 + t * 2048, lds + ra1 + t * 2048);
            B0[t] = rd1(lds + rb0 + t * 1024, lds + rb0 + t * 1024 + 512);
        }
        int s = 0;
        u32 sn = STAGE, ma = la0, mb = lb0;
        const lbyte *pa0 = lds + STAGE + ra0, *pa1 = lds + STAGE + ra1, *pb = lds + STAGE + rb0;
        while (s < nstep - 4) {
            KSTEP(A0, B0, A1, B1, true, 6);
            KSTEP(A1, B1, A0, B0, true, 6);
        }
        KSTEP(A0, B0, A1, B1, true, 6);
        KSTEP(A1, B1, A0, B0, false, 6);
        KSTEP(A0, B0, A1, B1, false, 0);
#pragma unroll
        for (int m = 0; m < 4; ++m)
#pragma unroll
            for (int t = 0; t < 4; ++t) MM(A1, B1, m, t);

        const bool bad = !pre && __hip_atomic_load(tmo, __ATOMIC_RELAXED, __HIP_MEMORY_SCOPE_AGENT) == call;
        int ln = lane;
        asm volatile("" : "+v"(ln));
        if (local) {
            float scale = bad ? __builtin_nanf("") : P.sx[0] * P.sw[0];
            asm volatile("" : "+v"(scale));
            float* dst = P.out + (size_t)(rt * 256 + wm * 64 + 4 * (ln >> 4)) * NN + block_of(rank, part) * 1024 + i8 * 128 + wn * 64 + 4 * (ln & 15);
#pragma unroll
            for (int m = 0; m < 4; ++m)
#pragma unroll
                for (int r = 0; r < 4; ++r) {
                    f32x4 y;
#pragma unroll
                    for (int t = 0; t < 4; ++t) y[t] = silu(acc[m][t][r] * scale);
                    THROTTLE(LOC_WIN);
                    st_mode<NT_OUT>(dst + (size_t)(m * 16 + r) * NN, __builtin_bit_cast(u32x4, y));
                }
            STAMP(7);
        } else if (pre) {
            const int fb = ft >> 3;
            const int blkf = (rank & 1) ? 4 * (fb / 3) + fb % 3 + (fb % 3 >= (rank >> 1)) : fb + (fb >= 2 * rank ? 4 : 0);
            float* d = P.out + (size_t)(rt * 256 + wm * 64 + 4 * (ln >> 4)) * NN + blkf * 1024 + i8 * 128 + wn * 64 + 4 * (ln & 15);
#pragma unroll
            for (int m = 0; m < 4; ++m)
#pragma unroll
                for (int r = 0; r < 4; ++r) {
                    const f32x4 y = {acc[m][0][r], acc[m][1][r], acc[m][2][r], acc[m][3][r]};
                    st_mode<NT_PRE>(d + (size_t)(m * 16 + r) * NN, __builtin_bit_cast(u32x4, y));
                }
            if (job == 1) {
                WAIT_VM(0);
                __builtin_amdgcn_s_barrier();
                if (tid == 0) __hip_atomic_fetch_add((u32*)(myws + WS_PCNT), 1u, __ATOMIC_RELAXED, __HIP_MEMORY_SCOPE_SYSTEM);
                STAMP(7);
            }
        } else {
            float amf = 0.f;
#pragma unroll
            for (int i = 0; i < 64; ++i) amf = fmaxf(amf, fabsf(acc[i >> 4][i & 3][(i >> 2) & 3]));
            const u32 am = wave_umax(__float_as_uint(amf));
            const int ex = min(max((int)(am >> 23), 40), 200);
            u32 votes = 0;
#pragma unroll
            for (int i = 0; i < 8; ++i) {
                const u32 I = (u32)(int)(acc[i >> 1][i & 3][i & 1] * __uint_as_float((u32)(280 - ex) << 23));
                votes += (((I + (1u << 25)) & 0xfc000007u) != 0) + ((((I + (1u << 26)) & 0xf800000fu) != 0) << 10) +
                         (((I & 0x1fu) != 0) << 20);
            }
            votes = wave_sum(votes);
            const u32 n0 = votes & 1023u, n1 = (votes >> 10) & 1023u, n2 = votes >> 20;
            const int dsc = n2 <= n1 && n2 <= n0 ? 2 : n1 <= n0 ? 1 : 0;
            const float up = __uint_as_float((u32)(280 - ex - dsc) << 23);
            u32 hdrs[2];
#pragma unroll
            for (int mp = 0; mp < 2; ++mp) {
#define ACCV(i) acc[2 * mp + ((i) >> 4)][(i) & 3][((i) >> 2) & 3]
                uint8_t* const slot = myws + pkoff + (size_t)(SL_O + (ct * 48 + rt * 8 + wm * 2 + mp) * 2 + wn) * SLOT;
                u32 hi[16], mid[8], q[8], nz = 0, miss = 0;
#pragma unroll
                for (int j = 0; j < 8; ++j) {
                    u32 t[4];
#pragma unroll
                    for (int t4 = 0; t4 < 4; ++t4) {
                        const float f = ACCV(4 * j + t4) * up;
                        const u32 I = (u32)(int)f;
                        miss |= __float_as_uint(__builtin_amdgcn_fractf(f));
                        t[t4] = __builtin_amdgcn_alignbit(I, I ^ (u32)__builtin_amdgcn_sbfe((int)I, 0, 26), 26);
                    }
                    hi[2 * j] = __builtin_amdgcn_perm(t[1], t[0], 0x07060302u);
                    hi[2 * j + 1] = __builtin_amdgcn_perm(t[3], t[2], 0x07060302u);
                    const u32 l0 = __builtin_amdgcn_perm(t[1], t[0], 0x05010400u), l1 = __builtin_amdgcn_perm(t[3], t[2], 0x05010400u);
                    const u32 lo = __builtin_amdgcn_perm(l1, l0, 0x05040100u);
                    mid[j] = __builtin_amdgcn_perm(l1, l0, 0x07060302u);
                    q[j] = bfi(0x07070707u, bfi(0x03030303u, lo, mid[j] << 2), (lo >> 3) & 0x18181818u);
                    const u32 nb = q[j] + 0x1F1F1F1Fu;
                    nz = bfi(0x01010101u << j, j < 5 ? nb >> (5 - j) : nb << (j - 5), nz);
                }
                const u32 cnt = __builtin_popcount(nz);
                const u32 inc = wave_scan_add(cnt);
                const u32 total = __builtin_amdgcn_readlane(inc, 63);
                if (bad || (int)(am >> 23) > ex || __any(miss != 0) || total > (SLOT - 128 - ACC_MAIN) / 2) {
#pragma unroll
                    for (int j = 0; j < 8; ++j) {
                        f32x4 y = {ACCV(4 * j), ACCV(4 * j + 1), ACCV(4 * j + 2), ACCV(4 * j + 3)};
                        if (bad) y[0] = __builtin_nanf("");
                        st_sys(slot + j * 1024 + ln * 16, __builtin_bit_cast(u32x4, y));
                    }
                    hdrs[mp] = 64u | H_RAW;
                } else {
#pragma unroll
                    for (int j = 0; j < 7; ++j) mid[j] = bfi(0x01010101u, mid[7] >> (j + 1), mid[j]);
#pragma unroll
                    for (int j = 0; j < 4; ++j)
                        st_sys(slot + j * 1024 + ln * 16, u32x4{hi[4 * j], hi[4 * j + 1], hi[4 * j + 2], hi[4 * j + 3]});
                    st_sys(slot + 4096 + ln * 16, u32x4{mid[0], mid[1], mid[2], mid[3]});
                    *(lu32*)(scr + ln * 4) = mid[4];
                    *(lu32*)(scr + 256 + ln * 4) = mid[5];
                    *(lu32*)(scr + 512 + ln * 4) = mid[6];
                    write_list(q, nz, inc - cnt, total, scr + 768, scr + 4096, ln);
                    for (u32 o = ln * 16; o < 768 + 2 * total; o += 1024) st_sys(slot + 5120 + o, *(lu32x4*)(scr + o));
                    CBAR();
                    hdrs[mp] = ((ACC_MAIN + 2 * total + 127) >> 7) | (total << 8) | ((u32)(ex + dsc - 30) << 19);
                }
            }
            WAIT_VM(0);
            if (ln < 2) {
                const int rg = rt * 8 + wm * 2 + ln;
                int q;
                if (rg < 3 * FG) {
                    q = 2 * (((rank >> 1) + 1 + rg / FG) & 3) + (rank & 1);
                } else {
                    const int jj = (rg - 3 * FG) / GG;
                    q = 2 * (jj + (jj >= part)) + 1 - (rank & 1);
                }
                st_flag(P.ws[q & 7], rank, FL_O + (ct * 48 + rg) * 2 + wn, 0, call, ln ? hdrs[1] : hdrs[0]);
            }
            STAMP(4);
        }
    }
}

}

extern "C" void kernel_launch(int rank, int nranks, ncclComm_t comm,
                              hipStream_t stream,
                              void* const* d_in, const int* in_sizes, int n_in,
                              void* d_out, int out_size,
                              const int* in_dtypes, int out_dtype,
                              void* const* d_out_all,
                              void* const* d_ws_all, size_t ws_bytes) {
    if (nranks != NR || (unsigned)rank >= (unsigned)NR || n_in != 4 || ws_bytes < WS_NEED ||
        in_sizes[0] != MM * KS || in_sizes[1] != KS * NN || out_size != MS * NN) {
        fprintf(stderr, "kernel_launch: unexpected problem shape\n");
        return;
    }
    Params P;
    P.x = (const uint8_t*)d_in[0];
    P.w = (const uint8_t*)d_in[1];
    P.sx = (const float*)d_in[2];
    P.sw = (const float*)d_in[3];
    P.out = (float*)d_out;
    for (int p = 0; p < NR; ++p) P.ws[p] = (uint8_t*)d_ws_all[p];
    P.myws = P.ws[rank];
    P.rank = rank;
    P.pad = 0;
    hipLaunchKernelGGL(fused_kernel, dim3(GRID), dim3(NTHR), 0, stream, P);
    HIP_CHECK(hipGetLastError());
}
```

```cpp
#include <hip/hip_runtime.h>
#include <rccl/rccl.h>
#include <cstdio>
#include <cstdint>

#define HIP_CHECK(expr) do { hipError_t _e = (expr); if (_e != hipSuccess) { \
    fprintf(stderr, "HIP error %s at %s:%d\n", hipGetErrorString(_e), __FILE__, __LINE__); \
    return; } } while (0)

typedef unsigned u32;
typedef unsigned long long u64;
typedef unsigned u32x4 __attribute__((ext_vector_type(4)));
typedef unsigned u32x2 __attribute__((ext_vector_type(2)));
typedef int i32x8 __attribute__((ext_vector_type(8)));
typedef float f32x4 __attribute__((ext_vector_type(4)));
typedef __attribute__((address_space(1))) const void gvoid;
typedef __attribute__((address_space(3))) void lvoid;
typedef __attribute__((address_space(3))) uint8_t lbyte;
typedef __attribute__((address_space(3))) u32 lu32;
typedef __attribute__((address_space(3))) unsigned short lu16;
typedef __attribute__((address_space(3))) u32x4 lu32x4;
typedef __attribute__((address_space(3))) u32x2 lu32x2;

namespace {

constexpr int NR = 8;
constexpr int MM = 4096, KK = 4096, NN = 16384;
constexpr int KS = KK / NR;
constexpr int MS = MM / NR;
constexpr int FG = 7, GG = 9;
constexpr int F = 32 * FG, G = 32 * GG;
static_assert(F + G == MS, "rows");
constexpr int XGREM = 3 * FG + 4 * GG;
constexpr int XREM = 32 * XGREM;
constexpr int XROWS = XREM + MS;
constexpr int NSTEP = 32;

constexpr int GRID = 256, NSEND = 64, NTHR = 512;
constexpr int NENG = 56;
#define WIN 7
#define WIN_OP 8
#define WIN_TAIL 12
#define TAIL_UNITS 0
#define BATCH 5
#define NFWD 3
#define PP_PACE 1
#define W_ORDER 0x652410
#define WB 2
#define X_DELAY 0
#define MON_SLEEP 4
#define LOOK 3
#define NPUSH 5
#define PUSH_WIN 8
#define W_DELAY 500
#define PRE_STAG 140
#define PRE_START 1700
#define LATE_STAG 100
#define LOC_WIN 32
#define PEEK_AHEAD 6
#define W_DELAY0 200
#define NT_PRE 0
#define NT_OUT 0
#define POLL_SLEEP 16
#define I8A 3

constexpr int SLOT = 8192;
constexpr int NPP = 24;
constexpr u32 H_RAW = 1u << 7;
constexpr u32 ACC_MAIN = 5888;
constexpr int SL_X = 0, SL_W = 256, SL_O = SL_W + 1024, SL_F = SL_O + 32 * 48 * 2, SL_P = SL_F + 128, SL_END = SL_P + NR * NPP;
constexpr int FL_X = 0, FL_W = 256, FL_O = FL_W + 256, FL_F = FL_O + 32 * 48 * 2, FL_P = FL_F + 32, FL_END = FL_P + NPP;
static_assert(FL_END <= 4096, "flags");

constexpr size_t WS_CNT = 262144;
constexpr size_t WS_ZERO = 0;
constexpr size_t WS_TMO = 32768;
constexpr size_t WS_TICK = 49152;
constexpr size_t WS_XREM = 65536;
constexpr size_t WS_XOWN = WS_XREM + 2048;
constexpr size_t X_DONE = 8192;
constexpr size_t WS_PCNT = WS_XOWN + 2048;
constexpr size_t WS_WCNT = 27u << 20;
constexpr size_t WS_FLAG = 28u << 20;
constexpr size_t WS_DBG = 524288;
constexpr size_t WS_GW = 1u << 20;
constexpr size_t WS_GX = WS_GW + (size_t)NSTEP * 32 * 16384;
constexpr size_t XSLAB = (size_t)XROWS * 128;
constexpr size_t WS_PK = 32u << 20;
constexpr size_t PK_BYTES = 40u << 20;
static_assert(WS_GX + NSTEP * XSLAB <= WS_WCNT && WS_WCNT + 128 * 32 * 128 <= WS_FLAG && WS_FLAG + NR * 4096 * 128 <= WS_PK, "staging");
static_assert((size_t)SL_END * SLOT <= PK_BYTES, "packed");
constexpr size_t GX_ODD = WS_PK + 2 * PK_BYTES - WS_GX;
constexpr size_t WS_GW2 = 124u << 20;
static_assert(WS_GX + GX_ODD + NSTEP * XSLAB <= WS_GW2, "staging");
constexpr size_t WS_NEED = WS_GW2 + (size_t)3 * 128 * 16384;

constexpr int STAGE = 49152;
constexpr int LDS_CTL = 3 * STAGE;
constexpr int LDS_LUT = LDS_CTL + 256;
constexpr int NS = 12;
constexpr int NPAIR = 3;
constexpr int EL_PLANE = NS * SLOT;
constexpr int EL_BUF = EL_PLANE + 8 * 1024;
constexpr int EL_TAB = EL_BUF + 8 * 2304;
constexpr int EL_GO = EL_TAB + 1024 * 16;
constexpr int EL_FULL = EL_GO + 256;
constexpr int EL_FREE = EL_FULL + 64;
constexpr int EL_ISS = EL_FREE + 128;
constexpr int EL_DEAD = EL_ISS + 4;
constexpr int EL_END = EL_ISS + 64;
static_assert(EL_END + 1024 <= LDS_CTL && NS % NPAIR == 0 && NS % 2 == 0, "engine LDS");
constexpr u32 UNIT_END = 0xffffu;
constexpr u64 DEADLINE = 40000000ull;

static_assert(7 * NPUSH <= 112 && NPUSH + NPP <= XGREM - GG, "push");

struct Params {
    const uint8_t* x;
    uint8_t* myws;
    int rank;
    int pad;
    const uint8_t* w;
    const float* sx;
    const float* sw;
    float* out;
    uint8_t* ws[NR];
};

__device__ __forceinline__ void st_sys(void* p, u32x4 v) {
    asm volatile("global_store_dwordx4 %0, %1, off sc0 sc1\n\ts_nop 1" :: "v"(p), "v"(v) : "memory");
}
template <int M> __device__ __forceinline__ void st_mode(void* p, u32x4 v) {
    if (M == 0) asm volatile("global_store_dwordx4 %0, %1, off sc0 sc1\n\ts_nop 1" :: "v"(p), "v"(v) : "memory");
    if (M == 1) asm volatile("global_store_dwordx4 %0, %1, off sc0 sc1 nt\n\ts_nop 1" :: "v"(p), "v"(v) : "memory");
    if (M == 2) asm volatile("global_store_dwordx4 %0, %1, off nt\n\ts_nop 1" :: "v"(p), "v"(v) : "memory");
    if (M == 3) asm volatile("global_store_dwordx4 %0, %1, off sc1\n\ts_nop 1" :: "v"(p), "v"(v) : "memory");
}
#define WAIT_VM(n) asm volatile("s_waitcnt vmcnt(" #n ")" ::: "memory")
#define THROTTLE(n) asm volatile("s_waitcnt vmcnt(%0)" :: "i"((n) - 1) : "memory")
#define CBAR() asm volatile("" ::: "memory")
#define DPP(v, ctl, rm) (u32)__builtin_amdgcn_update_dpp(0, (int)(v), ctl, rm, 0xf, false)

__device__ __forceinline__ u32 wave_scan_add(u32 v) {
    v += DPP(v, 0x111, 0xf); v += DPP(v, 0x112, 0xf); v += DPP(v, 0x114, 0xf); v += DPP(v, 0x118, 0xf);
    v += DPP(v, 0x142, 0xa); v += DPP(v, 0x143, 0xc);
    return v;
}
__device__ __forceinline__ u32 wave_sum(u32 v) { return __builtin_amdgcn_readlane(wave_scan_add(v), 63); }
__device__ __forceinline__ u32 wave_umax(u32 v) {
    v = max(v, DPP(v, 0x111, 0xf)); v = max(v, DPP(v, 0x112, 0xf)); v = max(v, DPP(v, 0x114, 0xf));
    v = max(v, DPP(v, 0x118, 0xf)); v = max(v, DPP(v, 0x142, 0xa)); v = max(v, DPP(v, 0x143, 0xc));
    return __builtin_amdgcn_readlane(v, 63);
}
__device__ __forceinline__ int block_of(int r, int part) {
    return (r & 1) ? 4 * part + (r >> 1) : 4 * (r >> 1) + part;
}
__device__ __forceinline__ float silu(float v) {
    const float z = __builtin_amdgcn_fmed3f(v, -60.f, 60.f);
    const float e = __builtin_amdgcn_exp2f(z * -1.4426950408889634f);
    return v * __builtin_amdgcn_rcpf(1.0f + e);
}

constexpr u32 LIST_MAX = 960;
__device__ __forceinline__ u32 plane_addr(u32 t) { return ((t & 4u) << 8) | ((t >> 5) << 4) | ((t & 3u) << 2) | ((t >> 3) & 3u); }
__device__ __forceinline__ u32 write_list(const u32 (&q)[8], u32 nz, u32 pos, u32 total,
                                          lbyte* lst, lbyte* tmp, int lane) {
    *(lu32x4*)(tmp + lane * 16) = u32x4{q[0], q[1], q[2], q[3]};
    *(lu32x4*)(tmp + 1024 + lane * 16) = u32x4{q[4], q[5], q[6], q[7]};
    for (u32 o = lane * 16; o < 2 * total + 32; o += 1024) *(lu32x4*)(lst + o) = u32x4{0xc0c0c0c0u, 0xc0c0c0c0u, 0xc0c0c0c0u, 0xc0c0c0c0u};
    lbyte* const pl = tmp + 2048;
    if (lane == 0) *(lu16*)(pl + 2) = 0xffffu;
    lbyte* wp = pl + 4 + 2 * pos;
    for (u32 m = nz; m; m &= m - 1) {
        *(lu16*)wp = (unsigned short)(((u32)lane << 5) | __builtin_ctz(m));
        wp += 2;
    }
    CBAR();
    u32 nb = 0;
    for (u32 i0 = 0; i0 < total; i0 += 128) {
        const u32 i = i0 + 2 * lane;
        const u32 pa = *(lu32*)(pl + 2 * i), pb = *(lu32*)(pl + 2 * i + 4);
        const u32 t[3] = {pa >> 16, pb & 0xffffu, pb >> 16};
        u32 n[2], e[2], val[2];
#pragma unroll
        for (int h = 0; h < 2; ++h) {
            val[h] = *(tmp + plane_addr(t[h + 1] & 2047u));
            const u32 gap = (t[h + 1] - t[h] - 1) & 0xffffu, ty = min(val[h] - 3u, 2u);
            e[h] = (ty << 6) | (gap & 63u);
            n[h] = i + h < total ? (gap >> 6) + 1 + (ty >> 1) : 0;
        }
        const u32 inc = wave_scan_add(n[0] + n[1]);
        lbyte* const end = lst + nb + inc;
#pragma unroll
        for (int h = 0; h < 2; ++h) {
            lbyte* const le = end - (h ? 0 : n[1]) - 1;
            if (n[h]) {
                *(le - (e[h] >> 7)) = (uint8_t)e[h];
                if (e[h] >> 7) *le = (uint8_t)(0xe0u | val[h]);
            }
        }
        nb += __builtin_amdgcn_readlane(inc, 63);
    }
    CBAR();
    return nb;
}
__device__ __forceinline__ void st_flag(uint8_t* ws, int src, int flag, int half, u32 call, u32 hdr) {
    __hip_atomic_store((u64*)(ws + WS_FLAG) + (size_t)(src * 4096 + flag) * 16 + (call & 1) * 2 + half,
                       ((u64)hdr << 32) | call, __ATOMIC_RELAXED, __HIP_MEMORY_SCOPE_SYSTEM);
}
__device__ __forceinline__ void count_in(uint8_t* cnt, u32 n) {
    __hip_atomic_fetch_add((u32*)cnt, n, __ATOMIC_RELAXED, __HIP_MEMORY_SCOPE_SYSTEM);
}
__device__ __forceinline__ u32 bfi(u32 m, u32 a, u32 b) { return (a & m) | (b & ~m); }

__device__ __forceinline__ void op_lut_init(lbyte* lut, int m) {
    u32 e = 0, c = 0x0c0c0c0cu, n = 0;
#pragma unroll
    for (u32 i = 0; i < 4; ++i) {
        const bool on = (m >> i) & 1;
        e |= (on ? n : 0x0cu) << (8 * i);
        if (on) c = (c & ~(0xffu << (8 * n))) | (i << (8 * n));
        n += on;
    }
    *(lu32*)(lut + m * 4) = e;
    *(lu32*)(lut + 64 + m * 4) = c;
}
__device__ __forceinline__ u32 and_or(u32 a, u32 m, u32 b) {
    u32 r; asm("v_and_or_b32 %0, %1, %2, %3" : "=v"(r) : "v"(a), "s"(m), "v"(b)); return r;
}
__device__ __forceinline__ u32 bfi_s(u32 m, u32 a, u32 b) {
    u32 r; asm("v_bfi_b32 %0, %1, %2, %3" : "=v"(r) : "s"(m), "v"(a), "v"(b)); return r;
}
__device__ __forceinline__ u32 ff_count(u32 ff, u32 acc) { return (u32)__builtin_amdgcn_sdot4((int)ff, -1, (int)acc, true); }
__device__ __forceinline__ u32 ff_lut(u32 ff, u32 lut) {
    return *(const lu32*)(uintptr_t)(u32)__builtin_amdgcn_sdot4((int)ff, (int)0xe0f0f8fcu, (int)lut, true);
}
__device__ __forceinline__ u32 ge13(u32 x) { return __builtin_amdgcn_perm(0u, 0u, x); }
__device__ __forceinline__ void lds_or(u32 p, u32 v) {
    __hip_atomic_fetch_or((lu32*)(uintptr_t)p, v, __ATOMIC_RELAXED, __HIP_MEMORY_SCOPE_WAVEFRONT);
}
__device__ __forceinline__ u32 chunk_starts(const u32 (&c)[4], u32 (&p)[4], u32 at) {
    const u32 a = c[0] | (c[1] << 16), b = c[2] | (c[3] << 16);
    const u32 ia = wave_scan_add(a), ib = wave_scan_add(b);
    const u32 ta = __builtin_amdgcn_readlane(ia, 63), tb = __builtin_amdgcn_readlane(ib, 63);
    const u32 t1 = ta & 0xffffu, t2 = t1 + (ta >> 16), t3 = t2 + (tb & 0xffffu);
    p[0] = ((ia - a) & 0xffffu) + at;
    p[1] = ((ia - a) >> 16) + (at + t1);
    p[2] = ((ib - b) & 0xffffu) + (at + t2);
    p[3] = ((ib - b) >> 16) + (at + t3);
    return t3 + (tb >> 16);
}
__device__ __forceinline__ u32 op_pack(const u32x4* v, u32 base4, uint8_t* blk, bool paced, lbyte* scr, const lbyte* lutp, int lane) {
    lbyte* const side = scr + 2560;
    const u32 lut = (u32)(uintptr_t)lutp + 64, eb = (u32)(uintptr_t)scr + 16;
    const u32x4 z4 = {0u, 0u, 0u, 0u};
    *(lu32x4*)(scr + lane * 16) = z4;
    *(lu32x4*)(scr + 1024 + lane * 16) = z4;
    *(lu32x4*)(scr + 2048 + lane * 16) = z4;
    *(lu32x4*)(scr + 3072 + lane * 16) = z4;
    u32 ub[16], ef[16], sel[16], N[8], S[2] = {0u, 0u}, cnt[4] = {0u, 0u, 0u, 0u}, pos[4];
#pragma unroll
    for (int i = 0; i < 16; ++i) {
        const u32 d = v[i >> 2][i & 3];
        ub[i] = (d | 0x80808080u) - base4;
        ef[i] = ge13(and_or(ub[i], 0x7C7C7C7Cu, 0x01010101u));
        sel[i] = ff_lut(ef[i], lut);
        const u32 x = and_or(ef[i], 0x0C0C0C0Cu, ub[i]);
        N[i >> 1] = (i & 1) ? bfi_s(0x0F0F0F0Fu, N[i >> 1], x << 4) : x;
        S[i >> 3] = and_or(d >> (7 - (i & 7)), 0x01010101u << (i & 7), S[i >> 3]);
        cnt[i >> 2] = ff_count(ef[i], cnt[i >> 2]);
    }
    const u32 n2 = chunk_starts(cnt, pos, eb - 1), n2w = (n2 + 15) >> 4;
    bool raw = n2 > 2048;
    u32 n3 = 0;
    if (!raw) {
        st_sys(blk + lane * 16, u32x4{N[0], N[1], N[2], N[3]});
        if (!paced) st_sys(blk + 1024 + lane * 16, u32x4{N[4], N[5], N[6], N[7]});
        CBAR();
        *(lu32x2*)(side + lane * 8) = u32x2{S[0], S[1]};
#pragma unroll
        for (int i = 0; i < 16; ++i) {
            const u32 c = __builtin_amdgcn_perm(0u, ub[i], sel[i]), p = pos[i >> 2];
            lu32* const q = (lu32*)(uintptr_t)(p & ~3u);
            __hip_atomic_fetch_or(q, __builtin_amdgcn_alignbyte(c, 0u, ~p), __ATOMIC_RELAXED, __HIP_MEMORY_SCOPE_WAVEFRONT);
            __hip_atomic_fetch_or(q + 1, __builtin_amdgcn_alignbyte(0u, c, ~p), __ATOMIC_RELAXED, __HIP_MEMORY_SCOPE_WAVEFRONT);
            pos[i >> 2] = ff_count(ef[i], p);
        }
        CBAR();
        const u32 l3 = (u32)(uintptr_t)side + 512 + n2w * 4;
        u32 bad = 0;
        for (u32 w0 = 0; w0 < n2w; w0 += 64) {
            const u32x4 B = *(lu32x4*)(scr + 16 + (w0 + lane) * 16);
            u32 W = 0, c3 = 0;
            u64 str = 0;
#pragma unroll
            for (int k = 0; k < 4; ++k) {
                const u32 u = (((B[k] & 0x7C7C7C7Cu) + 0x08080808u) >> 2) & 0x1F1F1F1Fu;
                const u32 code = __builtin_amdgcn_perm(0x83038200u, 0x80008100u, u) & 0x03030303u;
                W |= __builtin_amdgcn_udot4(code, 0x40100401u, 0u, false) << (8 * k);
                bad |= (u + 0x78787878u) & ~(u + 0x6E6E6E6Eu);
                const u32 e = __builtin_amdgcn_perm(0u, 0xff000000u, code);
                const u32 sym = u & (and_or(ge13(u), 0x0E0E0E0Eu, 0x01010101u));
                const u32 c = __builtin_amdgcn_perm(0u, sym, ff_lut(e, lut));
                str |= (u64)__builtin_amdgcn_perm(0u, c | (c >> 4), 0x0c0c0200u) << c3;
                c3 = (u32)__builtin_amdgcn_sdot4((int)e, (int)0xfcfcfcfcu, (int)c3, true);
            }
            const u32 i3 = wave_scan_add(c3);
            const u32 o3 = n3 + i3 - c3;
            n3 += __builtin_amdgcn_readlane(i3, 63);
            if (n2w * 4 + (n3 >> 3) > 1000) { raw = true; break; }
            if (w0 + lane < n2w) *(lu32*)(side + 512 + (w0 + lane) * 4) = W;
            const u64 lo = (u64)(u32)str << (o3 & 31u), hi = (str >> 32) << (o3 & 31u);
            const u32 p = l3 + ((o3 >> 5) << 2);
            if (c3) lds_or(p, (u32)lo);
            if ((o3 & 31u) + c3 > 32) {
                lds_or(p + 4, (u32)(lo >> 32) | (u32)hi);
                lds_or(p + 8, (u32)(hi >> 32));
            }
        }
        raw = raw || __any(bad & 0x80808080u);
        CBAR();
    }
    if (raw) {
        WAIT_VM(0);
#pragma unroll
        for (int j = 0; j < 4; ++j) st_sys(blk + j * 1024 + lane * 16, v[j]);
        return 31u;
    }
    if (paced) {
        WAIT_VM(0);
        st_sys(blk + 1024 + lane * 16, u32x4{N[4], N[5], N[6], N[7]});
        if (PP_PACE) WAIT_VM(0);
    }
    const u32 nb = 512 + n2w * 4 + ((n3 + 7) >> 3);
    for (u32 o = lane * 16; o < nb; o += 1024) st_sys(blk + 2048 + o, *(lu32x4*)(side + o));
    CBAR();
    return ((2048 + nb - 1) >> 7) | (n2w << 5);
}
template <class Done>
__device__ __forceinline__ void op_unpack(u32x4* v, u32 base4, u32 n2w, const lbyte* blk, lbyte* buf, const lbyte* lutp,
                                          int lane, Done done) {
    const u32 lut = (u32)(uintptr_t)lutp;
    const u32x4 N0 = *(const lu32x4*)(blk + lane * 16), N1 = *(const lu32x4*)(blk + 1024 + lane * 16);
    const u32x2 S = *(const lu32x2*)(blk + 2048 + lane * 8);
    const lbyte* const l3 = blk + 2560 + n2w * 4;
    u32 n3 = 0;
    for (u32 w0 = 0; w0 < n2w; w0 += 64) {
        u32 W = *(const lu32*)(blk + 2560 + (w0 + lane) * 4);
        if (w0 + lane >= n2w) W = 0;
        const u32 c3t = __builtin_popcount(W & (W >> 1) & 0x55555555u);
        const u32 i3 = wave_scan_add(c3t);
        const u32 o3 = n3 + i3 - c3t;
        n3 += __builtin_amdgcn_readlane(i3, 63);
        const lbyte* const p = l3 + ((o3 >> 3) << 2);
        const u32 d0 = *(const lu32*)p, d1 = *(const lu32*)(p + 4), d2 = *(const lu32*)(p + 8);
        const u64 str = ((u64)__builtin_amdgcn_alignbit(d2, d1, o3 << 2) << 32) | __builtin_amdgcn_alignbit(d1, d0, o3 << 2);
        u32x4 B;
        u32 c3 = 0;
#pragma unroll
        for (int k = 0; k < 4; ++k) {
            const u32 f = (W >> (8 * k)) & 255u;
            const u32 t = (f | (f << 12)) & 0x000F000Fu;
            const u32 code = (t | (t << 6)) & 0x03030303u;
            const u32 e = __builtin_amdgcn_perm(0u, 0xff000000u, code);
            const u32 x = __builtin_amdgcn_perm(0u, (u32)(str >> c3), 0x01010000u);
            const u32 s4 = bfi_s(0x00FF00FFu, x << 2, x >> 2) & 0x3C3C3C3Cu;
            const u32 u4 = s4 + and_or(ge13(s4 + 0x05050505u), 0x28282828u, 0x04040404u);
            B[k] = __builtin_amdgcn_perm(0u, u4, ff_lut(e, lut)) | __builtin_amdgcn_perm(0u, 0x0000706Cu, code);
            c3 = (u32)__builtin_amdgcn_sdot4((int)e, (int)0xfcfcfcfcu, (int)c3, true);
        }
        *(lu32x4*)(buf + (w0 + lane) * 16) = B;
    }
    done();
    u32 nib[16], ef[16], sel[16], cnt[4] = {0u, 0u, 0u, 0u}, pos[4];
#pragma unroll
    for (int i = 0; i < 16; ++i) {
        const u32 n = i < 8 ? N0[i >> 1] : N1[(i >> 1) - 4];
        nib[i] = ((i & 1) ? n >> 4 : n) & 0x0F0F0F0Fu;
        ef[i] = ge13(nib[i] | 0x01010101u);
        sel[i] = ff_lut(ef[i], lut);
        cnt[i >> 2] = ff_count(ef[i], cnt[i >> 2]);
    }
    chunk_starts(cnt, pos, (u32)(uintptr_t)buf);
    CBAR();
    u32 ps[16], q0[16], q1[16];
#pragma unroll
    for (int i = 0; i < 16; ++i) {
        const int j = (i & 3) * 4 + (i >> 2);
        ps[j] = (j & 3) ? ff_count(ef[j - 1], ps[j - 1]) : pos[j >> 2];
        const lu32* const q = (const lu32*)(uintptr_t)(ps[j] & ~3u);
        q0[j] = q[0]; q1[j] = q[1];
    }
    __builtin_amdgcn_sched_barrier(0);
#pragma unroll
    for (int i = 0; i < 16; ++i) {
        const u32 y = nib[i] + base4 + __builtin_amdgcn_perm(0u, __builtin_amdgcn_alignbyte(q1[i], q0[i], ps[i]), sel[i]);
        v[i >> 2][i & 3] = bfi_s(0x7F7F7F7Fu, y, S[i >> 3] << (7 - (i & 7)));
    }
    CBAR();
}

__device__ __forceinline__ int group_of(int r, int sg) {
    if (sg < 3 * FG) {
        const int s = sg / FG;
        return 16 * (2 * (((r >> 1) + 1 + s) & 3) + (r & 1)) + sg - s * FG;
    }
    if (sg < XGREM) {
        const int j = (sg - 3 * FG) / GG;
        return 16 * (2 * j + 1 - (r & 1)) + FG + sg - 3 * FG - j * GG;
    }
    return 16 * r + sg - XGREM;
}

__device__ __forceinline__ int sent_group(int r, int p, int i) {
    const bool same = ((r ^ p) & 1) == 0;
    const int at = same ? FG * (((p >> 1) - (r >> 1) - 1) & 3) : 3 * FG + GG * (p >> 1);
    return i < at ? i : i + (same ? FG : GG);
}

struct Unit {
    u32 slot;
    u32 flag;
    u32 dst;
    u32 kind;
    u32 aux;
};
constexpr int NF = NFWD;
__device__ __forceinline__ int operand_units(bool same) {
    return 2 * (XGREM - (same ? FG : GG)) - NPUSH - NPP + 256 + 32 + (same ? -4 * NF : 3 * NF);
}
__device__ __forceinline__ Unit unit_of(int n, int r, int p) {
    const bool same = ((r ^ p) & 1) == 0;
    const int nx = XGREM - (same ? FG : GG);
    const int U1 = nx - NPUSH - NPP;
    const int U2 = U1 + 16 * WB;
    const int U3 = U2 + nx;
    const int U4 = U3 + 16 * (2 * I8A - WB);
    const int UA = U4 + 32 * (5 - I8A);
    const int UB = UA + (same ? 0 : 4 * NF);
    const int U5 = UB + 96 - 4 * NF;
    const int UC = U5 + 32;
    const int U6 = UC + (same ? 0 : 3 * NF);
    Unit u;
    if (n >= U6) {
        const int v = n - U6, wn = v & 1, h = v >> 1;
        int i8, part, rg0, j;
        if (same) {
            i8 = h / (4 * FG);
            const int t = h - i8 * (4 * FG);
            part = t & 3; j = t >> 2;
            rg0 = FG * (((r >> 1) - (p >> 1) - 1) & 3);
        } else {
            i8 = h / (3 * GG);
            const int t = h - i8 * (3 * GG), jr = r >> 1;
            j = t / 3;
            part = t - 3 * j + (t - 3 * j >= jr);
            rg0 = 3 * FG + (jr > part ? jr - 1 : jr) * GG;
        }
        const int e0 = rg0 & 1, ne = ((same ? FG : GG) - e0 + 1) >> 1;
        const int g = j < ne ? e0 + 2 * j : 1 - e0 + 2 * (j - ne);
        const int rg = rg0 + g, lg = same ? g : FG + g;
        const int id = ((part * 8 + i8) * 48 + rg) * 2 + wn, gct = block_of(p, part) * 8 + i8;
        u.slot = SL_O + id; u.flag = FL_O + id; u.kind = 2;
        u.dst = (u32)(lg * 32 * NN + gct * 128 + wn * 64) * 4u;
        u.aux = i8;
        return u;
    }
    if (n < U1 || (n >= U2 && n < U3) || (n >= U5 && n < UC)) {
        int sg, kp;
        if (n < U1) { sg = sent_group(r, p, n + NPUSH + NPP); kp = 0; }
        else if (n < U3) { sg = sent_group(r, p, n - U2); kp = 1; }
        else { sg = XGREM + ((n - U5) & 15); kp = (n - U5) >> 4; }
        const int gg = group_of(r, sg);
        u.slot = SL_X + gg * 2 + kp; u.flag = FL_X + gg * 2 + kp; u.kind = 0;
        u.dst = (u32)(WS_GX + (size_t)(p * 4 + 2 * kp) * XSLAB) + (u32)sg * 4096u;
        u.aux = (u32)(sg < XGREM ? WS_XREM : WS_XOWN) + (u32)(p * 2 + kp) * 128u;
        return u;
    }
    int i8, kh, part, src = p;
    if (n < U4) {
        const int v = n < U2 ? n - U1 : n - U3 + 16 * WB, o = (W_ORDER >> (v >> 4 << 2)) & 15;
        i8 = o & 3; kh = (o & 4) + ((v >> 2) & 3); part = v & 3;
    }
    else if (n < UA) { const int v = n - U4; i8 = I8A + (v >> 5); kh = (v >> 2) & 7; part = v & 3; }
    else if (n < UB) { const int v = n - UA; i8 = 7; kh = 8 - NF + (v >> 2); part = v & 3; }
    else if (n < U5) { const int v = n - UB; i8 = 5 + (v >> 5); kh = (v >> 2) & 7; part = v & 3; }
    else {
        const int v = n - UC, f = v / 3;
        i8 = 7; kh = 8 - NF + f; part = p >> 1;
        src = 2 * (((r >> 1) + 1 + v - 3 * f) & 3) + (r & 1);
    }
    const int kc = kh >> 1, half = kh & 1, ct = part * 8 + i8, step = src * 4 + kc;
    u.slot = SL_W + ((kc * 128 + block_of(r, part) * 8 + i8) * 2 + half);
    u.flag = FL_W + (kc * 32 + ct) * 2 + half;
    u.kind = 1 | (half << 2);
    if (n >= UA && n < UB && part != (p >> 1)) u.kind |= 8 | ((kh & 3) << 4) | (part << 6);
    if (n >= UC) { u.slot = SL_F + ((r >> 1) * 8 + src) * 4 + (kh & 3); u.flag = FL_F + src * 4 + (kh & 3); }
    u.dst = (u32)WS_GW + (u32)(step * 32 + ct) * 16384u + (u32)half * 8192u;
    u.aux = (u32)WS_WCNT + (u32)(ct * 32 + step) * 128u;
    return u;
}

#define STAMP(k) do { } while (0)

__global__ void __launch_bounds__(NTHR, 2) fused_kernel(Params P) {
    __shared__ __attribute__((aligned(1024))) uint8_t lds_[LDS_CTL + 384];
    lbyte* const lds = (lbyte*)lds_;
    const int tid = threadIdx.x;
    const int lane = tid & 63;
    const int wave = __builtin_amdgcn_readfirstlane(tid >> 6);
    const int b = blockIdx.x;
    const int rank = P.rank;
    uint8_t* const myws = P.myws;
    const uint8_t* const xin = P.x;
    asm volatile("" :: "s"(rank), "s"(myws), "s"(xin));
    u32* const tmo = (u32*)(myws + WS_TMO);

    STAMP(0);
    u32* const cntp = (u32*)(myws + WS_CNT + (size_t)(b * 8 + wave) * 128);
#define LOAD_CALL() \
    const u32 ncalls = __hip_atomic_load(cntp, __ATOMIC_RELAXED, __HIP_MEMORY_SCOPE_AGENT)
#define GET_CALL() \
    const u32 call = __builtin_amdgcn_readfirstlane(ncalls) + 1; \
    const size_t pkoff = WS_PK + (size_t)(call & 1) * PK_BYTES
#define PUT_CALL() \
    if (lane == 0) __hip_atomic_store(cntp, call, __ATOMIC_RELAXED, __HIP_MEMORY_SCOPE_AGENT)

    if (b < NSEND) {
        if (b >= NENG) {
            if (b != NENG || wave != 0) return;
            LOAD_CALL();
            GET_CALL();
            PUT_CALL();
            const int src = (lane >> 1) & 7;
            const u32 need = call * (lane >= 16 ? 32u : src == rank ? 2u * XGREM : 2u * (XGREM - (((src ^ rank) & 1) ? GG : FG)));
            uint8_t* const cnt = myws + WS_XREM + (lane & 31) * 128;
            const u64 t0 = __builtin_amdgcn_s_memrealtime();
            bool open = lane < 32;
            for (u32 it = 1; __any(open); ++it) {
                if (open && __hip_atomic_load((const u32*)cnt, __ATOMIC_RELAXED, __HIP_MEMORY_SCOPE_SYSTEM) >= need) {
                    __hip_atomic_store((u32*)(cnt + X_DONE), call, __ATOMIC_RELAXED, __HIP_MEMORY_SCOPE_SYSTEM);
                    open = false;
                }
                if ((it & 7) == 0 && (__hip_atomic_load(tmo, __ATOMIC_RELAXED, __HIP_MEMORY_SCOPE_AGENT) == call ||
                                      __builtin_amdgcn_s_memrealtime() - t0 > DEADLINE)) {
                    if (lane == 0) __hip_atomic_store(tmo, call, __ATOMIC_RELAXED, __HIP_MEMORY_SCOPE_AGENT);
                    break;
                }
                __builtin_amdgcn_s_sleep(MON_SLEEP);
            }
            return;
        }
        LOAD_CALL();
        const int p = (rank + 1 + b % 7) & 7;
        const bool same = ((rank ^ p) & 1) == 0;
        const int nop = operand_units(same);
        const int nend = nop + (same ? 64 * FG : 48 * GG);
        for (int n = tid; n < nend + NPP; n += NTHR) {
            const Unit u = unit_of(n < nend ? n : n - nend - NPP, rank, p);
            *(lu32x4*)(lds + EL_TAB + n * 16) = u32x4{u.slot | (u.kind << 16), u.flag, u.dst, u.aux};
        }
        if (tid < (EL_END - EL_GO) / 4) {
            const int fo = EL_GO + tid * 4 - EL_FREE;
            *(lu32*)(lds + EL_GO + tid * 4) = fo >= (NS - NPAIR) * 8 && fo < NS * 8 ? (u32)-NS : 0u;
        }
        *(lu32x4*)(lds + EL_PLANE + tid * 16) = u32x4{0u, 0u, 0u, 0u};
        if (tid < 16) op_lut_init(lds + LDS_LUT, tid);
        __syncthreads();
        GET_CALL();
        PUT_CALL();
        auto lds_get = [&](int off) {
            return (u32)__builtin_amdgcn_readfirstlane(
                __hip_atomic_load((lu32*)(lds + off), __ATOMIC_RELAXED, __HIP_MEMORY_SCOPE_WORKGROUP));
        };
        auto lds_put = [&](int off, u32 v) {
            CBAR();
            if (lane == 0) __hip_atomic_store((lu32*)(lds + off), v, __ATOMIC_RELAXED, __HIP_MEMORY_SCOPE_WORKGROUP);
            CBAR();
        };

        if (wave == 0) {
            __builtin_amdgcn_s_setprio(3);
            WAIT_VM(0);
            const uint8_t* const pk = P.ws[p] + pkoff;
            const u32 lds_addr = (u32)(uintptr_t)lds;
            const u32 vo0 = lane * 16, vo1 = lane * 16 + 4096;
            u32 pub = 0, psl = 0;
            u32 issued = 0;
            u32 end0 = 0, end1 = 0;
            static_assert(WIN <= 12 && WIN_OP <= 12 && WIN_TAIL <= 12, "at most two whole units may be in flight behind the current one");
#define DMA1(W, LINES, VO, M0OFF, OFF) do { \
                asm volatile("s_waitcnt vmcnt(%0)" :: "i"(W - 1) : "memory"); \
                if (pub < k && end0 + (W - 1) <= issued) { \
                    lds_put(EL_FULL + psl * 4, ++pub); psl = psl == NS - 1 ? 0 : psl + 1; end0 = end1; } \
                if ((u32)lane < (LINES - (OFF >> 7)) * 8u) \
                    asm volatile("s_mov_b32 m0, %2\n\ts_nop 0\n\tglobal_load_lds_dwordx4 %0, %1 offset:" #OFF " sc0 sc1" \
                                 :: "v"(VO), "s"(src), "s"(la + M0OFF) : "memory"); \
                ++issued; \
            } while (0)
            u32 sl = 0;
            for (u32 k = 0;; ++k) {
                u32 go = lds_get(EL_GO + sl * 16);
                if ((go >> 16) != k + 1 || (go & 0xffffu) == UNIT_END) {
                    WAIT_VM(0);
                    while (pub < k) { lds_put(EL_FULL + psl * 4, ++pub); psl = psl == NS - 1 ? 0 : psl + 1; }
                    while ((go >> 16) != k + 1) {
                        __builtin_amdgcn_s_sleep(1);
                        go = lds_get(EL_GO + sl * 16);
                    }
                    if ((go & 0xffffu) == UNIT_END) break;
                }
                lds_put(EL_ISS, k + 1);
                const int un = go & 0xffffu;
                const u32 h0 = lds_get(EL_GO + sl * 16 + 4), h1 = lds_get(EL_GO + sl * 16 + 8);
                const u32 l0 = un < nop ? max((h0 & 31u) + 1, 17u) : 32u;
                const u32 l1 = un < nop ? max((h1 & 31u) + 1, 17u) : min(max(h0 & 127u, 41u), 64u) - 32u;
                const uint8_t* src = pk + (size_t)(lds_get(EL_TAB + (go & 0xffffu) * 16) & 0xffffu) * SLOT;
                const u32 la = lds_addr + sl * SLOT;
                u32 keep;
                asm volatile("s_mov_b32 %0, m0\n\ts_nop 4" : "=s"(keep) :: "memory");
#define DMAU(W) do { \
                DMA1(W, l0, vo0, 0, 0); DMA1(W, l0, vo0, 0, 1024); DMA1(W, l0, vo0, 0, 2048); \
                if (l0 > 24) DMA1(W, l0, vo0, 0, 3072); \
                DMA1(W, l1, vo1, 4096, 0); DMA1(W, l1, vo1, 4096, 1024); \
                if (l1 > 16) DMA1(W, l1, vo1, 4096, 2048); \
                if (l1 > 24) DMA1(W, l1, vo1, 4096, 3072); } while (0)
                if (un < nop) DMAU(WIN_OP);
                else if (un < nend - TAIL_UNITS) DMAU(WIN);
                else DMAU(WIN_TAIL);
                asm volatile("s_mov_b32 m0, %0" :: "s"(keep) : "memory");
                if (pub + 2 <= k) {
                    WAIT_VM(0);
                    while (pub < k) { lds_put(EL_FULL + psl * 4, ++pub); psl = psl == NS - 1 ? 0 : psl + 1; }
                }
                if (pub == k) end0 = issued; else end1 = issued;
                sl = sl == NS - 1 ? 0 : sl + 1;
            }
            STAMP(4);
            return;
        }
        if (wave == 1) {
            const u64* const flags = (const u64*)(myws + WS_FLAG) + (size_t)p * 4096 * 16 + (call & 1) * 2;
            u32* const tick = (u32*)(myws + WS_TICK) + p * 32;
            static_assert(BATCH % 2 == 1 && BATCH <= 7, "a bijection needs BATCH odd");
            constexpr int TC = 8 * (BATCH - 1) / BATCH;
            const u32 ntick = (nend + BATCH - 1) / BATCH + TC, tbase = (call - 1) * (ntick - 8) - 16;
            u32 tick_no = b / 7;
            bool dead = false;
            u32 partials = 0;
            auto give_up = [&](u64 t0) {
                if (__hip_atomic_load(tmo, __ATOMIC_RELAXED, __HIP_MEMORY_SCOPE_AGENT) == call ||
                    __builtin_amdgcn_s_memrealtime() - t0 > DEADLINE) {
                    if (lane == 0) __hip_atomic_store(tmo, call, __ATOMIC_RELAXED, __HIP_MEMORY_SCOPE_AGENT);
                    dead = true;
                    lds_put(EL_DEAD, 1);
                    return true;
                }
                __builtin_amdgcn_s_sleep(2);
                return false;
            };
            auto post = [&](u32 kk, u32 sl, u32 unit, u32 hdr, u32 hdr1) {
                while (min(lds_get(EL_FREE + sl * 8) + NS, lds_get(EL_FREE + sl * 8 + 4) + NS) < kk + 1) __builtin_amdgcn_s_sleep(1);
                lds_put(EL_GO + sl * 16 + 4, hdr);
                lds_put(EL_GO + sl * 16 + 8, hdr1);
                lds_put(EL_GO + sl * 16, ((kk + 1) << 16) | unit);
            };
            u32 k = 0, sl = 0;
            while (tick_no < ntick) {
                const int u0 = BATCH * ((int)tick_no - TC);
                while (lds_get(EL_ISS) + LOOK < k) __builtin_amdgcn_s_sleep(1);
                const int ul = min(max(u0 + 8 * min((lane >> 1) & 7, BATCH - 1), 0), nend - 1);
                const u64* const fp = flags + (size_t)*(lu32*)(lds + EL_TAB + ul * 16 + 4) * 16 + (ul < nop ? lane & 1 : 0);
                u32 tk;
                u64 fv;
                asm volatile("s_mov_b64 exec, 1\n\tglobal_atomic_add %0, %2, %3, off sc0\n\ts_mov_b64 exec, %5\n\t"
                             "global_load_dwordx2 %1, %4, off sc0 sc1\n\ts_mov_b64 exec, -1\n\ts_waitcnt vmcnt(0)"
                             : "=&v"(tk), "=&v"(fv) : "v"(tick), "v"(tick_no < 8 ? 0u : 1u), "v"(fp), "i"((1 << (2 * BATCH)) - 1) : "memory");
                for (int j = 0; j < BATCH; ++j) {
                    const int un = u0 + 8 * j;
                    if (un < 0 || un >= nend) continue;
                    if (!dead && ((__ballot((u32)fv >= call) >> (2 * j)) & 3) != 3) {
                        while (lds_get(EL_ISS) + LOOK < k) __builtin_amdgcn_s_sleep(1);
                        const u64 t0 = __builtin_amdgcn_s_memrealtime();
                        for (u32 it = 1;; ++it) {
                            if (lane >= 2 * j && lane < 2 * BATCH) fv = __hip_atomic_load(fp, __ATOMIC_RELAXED, __HIP_MEMORY_SCOPE_SYSTEM);
                            if (((__ballot((u32)fv >= call) >> (2 * j)) & 3) == 3 || ((it & 7) == 0 && give_up(t0))) break;
                            __builtin_amdgcn_s_sleep(1);
                        }
                    }
                    if (un >= nop && partials != 255u) {
                        const u32 bit = 1u << lds_get(EL_TAB + un * 16 + 12);
                        if (!(partials & bit)) {
                            while (lds_get(EL_ISS) + LOOK < k) __builtin_amdgcn_s_sleep(1);
                            const u64 t0 = __builtin_amdgcn_s_memrealtime();
                            while (!dead) {
                                partials = (u32)__ballot(__hip_atomic_load((const u32*)(myws + WS_PCNT) + (lane & 7) * 32, __ATOMIC_RELAXED,
                                                                           __HIP_MEMORY_SCOPE_SYSTEM) >= 12u * call) & 255u;
                                if ((partials & bit) || give_up(t0)) break;
                            }
                        }
                    }
                    const u32 hi = (u32)(fv >> 32);
                    post(k, sl, un, __builtin_amdgcn_readlane(hi, 2 * j), __builtin_amdgcn_readlane(hi, 2 * j + 1));
                    sl = sl == NS - 1 ? 0 : sl + 1;
                    ++k;
                }
                tick_no = tick_no < 8 ? tick_no + 8 : __builtin_amdgcn_readfirstlane(tk) - tbase;
            }
            for (u32 kk = k; kk < k + NPAIR; ++kk) {
                post(kk, sl, UNIT_END, 0, 0);
                sl = sl == NS - 1 ? 0 : sl + 1;
            }
            return;
        }
        lbyte* const plane = lds + EL_PLANE + wave * 1024;
        const float scale0 = P.sx[0] * P.sw[0];
        const u32 hf = wave & 1;
        const __amdgpu_buffer_rsrc_t outrs = __builtin_amdgcn_make_buffer_rsrc(P.out, 0, MS * NN * 4, 0x00020000);
        const u32 olane = (hf * 16 + (lane >> 4) * 4) * (NN * 4) + (lane & 15) * 16;
        u32 owed = 0;
        auto settle = [&]() {
            if (!owed) return;
            WAIT_VM(0);
            if (lane == 0) count_in(myws + owed, 1u);
            owed = 0;
        };
        u32 sl = ((wave - 2) >> 1) + NS - NPAIR;
        for (u32 k = ((wave - 2) >> 1) - NPAIR;; k += NPAIR) {
            lbyte* const s = lds + sl * SLOT;
            const bool pushed = (int)k < 0;
            const u32 fin = pushed ? 0u : k + 1;
            u32 go, hdr;
            if (pushed) {
                const u32 j = b / 7 * NPAIR + k + NPAIR;
                go = nend + j;
                const u64* const fp = (const u64*)(myws + WS_FLAG) + (size_t)(p * 4096 + FL_P + j) * 16 + (call & 1) * 2 + hf;
                const u64 t0 = __builtin_amdgcn_s_memrealtime();
                u64 fv;
                for (u32 it = 1; (u32)(fv = __hip_atomic_load(fp, __ATOMIC_RELAXED, __HIP_MEMORY_SCOPE_SYSTEM)) < call; ++it) {
                    if ((it & 7) == 0 && (__hip_atomic_load(tmo, __ATOMIC_RELAXED, __HIP_MEMORY_SCOPE_AGENT) == call ||
                                          __builtin_amdgcn_s_memrealtime() - t0 > DEADLINE)) {
                        if (lane == 0) __hip_atomic_store(tmo, call, __ATOMIC_RELAXED, __HIP_MEMORY_SCOPE_AGENT);
                        lds_put(EL_DEAD, 1);
                        break;
                    }
                    __builtin_amdgcn_s_sleep(4);
                }
                hdr = __builtin_amdgcn_readfirstlane((u32)(fv >> 32));
                asm volatile("s_mov_b32 m0, %1\n\ts_nop 0\n\tglobal_load_lds_dwordx4 %0, off sc0 sc1\n\t"
                             "global_load_lds_dwordx4 %0, off offset:1024 sc0 sc1\n\tglobal_load_lds_dwordx4 %0, off offset:2048 sc0 sc1\n\t"
                             "global_load_lds_dwordx4 %0, off offset:3072 sc0 sc1\n\ts_waitcnt vmcnt(0)"
                             :: "v"(myws + pkoff + (size_t)(SL_P + p * NPP + j) * SLOT + hf * 4096 + lane * 16),
                                "s"((u32)(uintptr_t)s + hf * 4096) : "memory");
            } else {
                while (((go = lds_get(EL_GO + sl * 16)) >> 16) != k + 1) { settle(); __builtin_amdgcn_s_sleep(1); }
                if ((go & 0xffffu) == UNIT_END) break;
                CBAR();
                hdr = lds_get(EL_GO + sl * 16 + 4 + hf * 4);
            }
            const u32x4 te = *(lu32x4*)(lds + EL_TAB + (go & 0xffffu) * 16);
            const u32 kind = __builtin_amdgcn_readfirstlane(te[0]) >> 16;
            const u32 dst = __builtin_amdgcn_readfirstlane(te[2]);
            const u32 aux = __builtin_amdgcn_readfirstlane(te[3]);
            const bool acc = kind == 2;
            u32x4 pa[4];
            uint8_t* const od = (uint8_t*)P.out + dst + olane;
            if (acc) {
                settle();
#pragma unroll
                for (int j = 0; j < 4; ++j)
                    pa[j] = __builtin_bit_cast(u32x4, __builtin_amdgcn_raw_buffer_load_b128(outrs, olane, dst + j * (NN * 4), 17));
            }
            const float scale = lds_get(EL_DEAD) ? __builtin_nanf("") : scale0;
            while (lds_get(EL_FULL + sl * 4) != fin) { settle(); __builtin_amdgcn_s_sleep(1); }
            CBAR();
            if (!acc) {
                u32x4 v[4];
                const lbyte* const blk = s + hf * 4096;
                if (kind & 8) {
                    const u32 fp = (kind >> 6) & 3u, ff = (kind >> 4) & 3u;
                    uint8_t* const fs = myws + pkoff + (size_t)(SL_F + (fp * 8 + p) * 4 + ff) * SLOT + hf * 4096;
                    for (u32 o = lane * 16; o < ((hdr & 31u) + 1) * 128; o += 1024) st_sys(fs + o, *(const lu32x4*)(blk + o));
                    WAIT_VM(0);
                    if (lane == 0) st_flag(P.ws[2 * fp + (p & 1)], rank, FL_F + p * 4 + ff, hf, call, hdr);
                }
                if ((hdr & 31u) == 31u) {
#pragma unroll
                    for (int j = 0; j < 4; ++j) v[j] = *(const lu32x4*)(blk + j * 1024 + lane * 16);
                    lds_put(EL_FREE + sl * 8 + hf * 4, fin);
                } else {
                    op_unpack(v, ((hdr >> 13) & 31u) * 0x04040404u, min((hdr >> 5) & 255u, 128u), blk,
                              lds + EL_BUF + wave * 2304, lds + LDS_LUT, lane,
                              [&] { lds_put(EL_FREE + sl * 8 + hf * 4, fin); });
                }
                const bool isw = kind & 1;
                uint8_t* d = myws + dst + (isw ? hf * 4096 : (call & 1) * GX_ODD + hf * XSLAB) +
                             ((lane ^ (isw ? (kind & 4u) * 2 + hf * 4 : 0)) << 4);
                settle();
#pragma unroll
                for (int j = 0; j < 4; ++j) st_sys(d + j * 1024, v[j]);
                owed = aux;
            } else {
                float a[16], c1 = 1.0f;
                if (hdr & H_RAW) {
#pragma unroll
                    for (int j = 0; j < 4; ++j) {
                        const u32x4 t = *(lu32x4*)(s + hf * 4096 + j * 1024 + lane * 16);
#pragma unroll
                        for (int t4 = 0; t4 < 4; ++t4) a[4 * j + t4] = __uint_as_float(t[t4]);
                    }
                    lds_put(EL_FREE + sl * 8 + hf * 4, fin);
                } else {
                    const u32x4 h0 = *(lu32x4*)(s + hf * 2048 + lane * 16), h1 = *(lu32x4*)(s + hf * 2048 + 1024 + lane * 16);
                    u32x4 mid = *(lu32x4*)(s + 4096 + lane * 16);
                    const u32 nb = (hdr >> 8) & 0x7ffu;
                    u32 at = ~0u;
                    for (u32 i0 = 0; i0 < nb; i0 += 128) {
                        const u32 i = i0 + 2 * lane;
                        const u32 w2 = *(lu16*)(s + ACC_MAIN + i);
                        u32 d[2], val[2];
#pragma unroll
                        for (int h = 0; h < 2; ++h) {
                            const u32 e = i + h < nb ? (w2 >> (8 * h)) & 255u : 0xe0u;
                            d[h] = e < 0xc0u ? (e & 63u) + 1 : e < 0xe0u ? 64u : 0u;
                            val[h] = e < 0x80u ? 3u + (e >> 6) : e >= 0xe0u ? e & 31u : 0u;
                        }
                        const u32 inc = wave_scan_add(d[0] + d[1]);
#pragma unroll
                        for (int h = 0; h < 2; ++h) {
                            const u32 t = at + inc - (h ? 0 : d[1]);
                            if (val[h] && ((t >> 2) & 1u) == hf) *(plane + (plane_addr(t) & 0x3ffu)) = (uint8_t)val[h];
                        }
                        at += __builtin_amdgcn_readlane(inc, 63);
                    }
                    CBAR();
                    const u32x4 q = *(lu32x4*)(plane + lane * 16);
                    *(lu32x4*)(plane + lane * 16) = u32x4{0u, 0u, 0u, 0u};
                    if (hf) {
                        const u32 m4 = *(lu32*)(s + 5120 + lane * 4), m5 = *(lu32*)(s + 5376 + lane * 4), m6 = *(lu32*)(s + 5632 + lane * 4);
                        u32 m7 = (mid[0] << 1) & 0x02020202u;
                        m7 |= (mid[1] << 2) & 0x04040404u; m7 |= (mid[2] << 3) & 0x08080808u; m7 |= (mid[3] << 4) & 0x10101010u;
                        m7 |= (m4 << 5) & 0x20202020u; m7 |= (m5 << 6) & 0x40404040u; m7 |= (m6 << 7) & 0x80808080u;
                        mid = u32x4{m4, m5, m6, m7};
                    }
                    lds_put(EL_FREE + sl * 8 + hf * 4, fin);
                    c1 = __uint_as_float(((hdr >> 19) & 255u) << 23);
#pragma unroll
                    for (int j = 0; j < 4; ++j) {
                        const u32 mi = bfi(0x01010101u, q[j] >> 2, mid[j]);
                        const u32 lo = (q[j] & 0x03030303u) | ((q[j] << 3) & 0xC0C0C0C0u);
                        const u32 l0 = __builtin_amdgcn_perm(mi, lo, 0x05010400u), l1 = __builtin_amdgcn_perm(mi, lo, 0x07030602u);
                        const u32 ha = j < 2 ? h0[2 * j] : h1[2 * j - 4], hb = j < 2 ? h0[2 * j + 1] : h1[2 * j - 3];
                        const u32 T[4] = {__builtin_amdgcn_perm(ha, l0, 0x05040100u), __builtin_amdgcn_perm(ha, l0, 0x07060302u),
                                          __builtin_amdgcn_perm(hb, l1, 0x05040100u), __builtin_amdgcn_perm(hb, l1, 0x07060302u)};
#pragma unroll
                        for (int t4 = 0; t4 < 4; ++t4) a[4 * j + t4] = (float)(((int)T[t4] >> 2) ^ (int)(T[t4] << 30));
                    }
                }
#pragma unroll
                for (int j = 0; j < 4; ++j) asm volatile("" : "+v"(pa[j]));
#pragma unroll
                for (int j = 0; j < 4; ++j) {
                    f32x4 y;
#pragma unroll
                    for (int t4 = 0; t4 < 4; ++t4) {
                        const u32 pb = pa[j][t4];
                        y[t4] = silu(fmaf(a[4 * j + t4], c1, __uint_as_float(pb)) * scale);
                    }
                    st_mode<NT_OUT>(od + (size_t)j * (NN * 4), __builtin_bit_cast(u32x4, y));
                }
            }
            sl = sl >= NS - NPAIR ? sl + NPAIR - NS : sl + NPAIR;
        }
        settle();
        return;
    }

    const int g = b - NSEND;
    lbyte* const scr = lds + wave * 6144;
    constexpr int WFIRST = 64 * I8A, GLATE = 64 + 16 * I8A;
    constexpr int T_P = 112, T_X = T_P + 14 * NPP, T_W = T_X + 512;
    static_assert(I8A >= 2 && T_W + 3 * WFIRST <= 1536, "tasks");
    const int task0 = g + 192 * wave;
    const int pdst = (rank + 1 + task0 % 7) & 7;
    const int pj = task0 / 7 - T_P / 7;
    const int psg = sent_group(pdst, rank, task0 < T_P ? task0 / 7 : NPUSH + (pj >> 1));
    const int gg = task0 < T_X ? group_of(pdst, psg) : (task0 - T_X) & 127;
    const int xkc = task0 < T_P ? 0 : task0 < T_X ? pj & 1 : (task0 - T_X) >> 7;
    u32x4 v[8];
    if (task0 < 7 * NPUSH || (task0 >= T_P && task0 < T_W)) {
        const uint8_t* src = xin + (size_t)(gg * 32 + (lane >> 3)) * KS + xkc * 128 + (lane & 7) * 16;
#pragma unroll
        for (int j = 0; j < 4; ++j) v[j] = *(const u32x4*)(src + j * 8 * KS);
        if (task0 < T_P) {
#pragma unroll
            for (int j = 0; j < 4; ++j) v[4 + j] = *(const u32x4*)(src + j * 8 * KS + 128);
        }
    }
    LOAD_CALL();
    uint8_t* const wsl = P.ws[lane & 7];
    __builtin_amdgcn_sched_barrier(0);
    const u64 tstart0 = __builtin_amdgcn_s_memrealtime();
    lbyte* const lut = scr + 4096;
    if (lane < 16) op_lut_init(lut, lane);
    CBAR();
    if (task0 < 7 * NPUSH) {
        uint8_t* const wd = P.ws[pdst];
        uint8_t* d = wd + WS_GX + (size_t)(rank * 4) * XSLAB + (size_t)psg * 4096 + lane * 16;
        GET_CALL();
        d += (call & 1) * GX_ODD;
#pragma unroll
        for (int j = 0; j < 8; ++j) asm volatile("" : "+v"(v[j]));
#pragma unroll
        for (int j = 0; j < 8; ++j) {
            THROTTLE(PUSH_WIN);
            st_sys(d + (j >> 2) * XSLAB + (j & 3) * 1024, v[j]);
        }
        WAIT_VM(0);
        if (lane == 0)
            count_in(wd + WS_XREM + rank * 256, 2u);
    }
    const int l7 = lane & 7;
    bool isx = task0 >= T_P && task0 < T_W, send = true, keep = true, tell = false;
    int h0 = 0, h1 = 2, flag = 0;
    u32 dbo = 0, swz = 0, slo = 0, cno = 0;
    uint8_t* slotws = myws;
    if (task0 >= T_P && task0 < T_X) {
        h0 = xkc; h1 = h0 + 1;
        keep = false;
        tell = l7 == pdst;
        flag = FL_P + (pj >> 1);
        slo = (u32)(SL_P + rank * NPP + (pj >> 1)) * (u32)SLOT;
        slotws = P.ws[pdst];
    } else if (isx) {
        const int kc = xkc, q = gg >> 4, lg = gg & 15;
        const int fl = (lg >= FG) ^ (q & 1);
        h0 = kc & 1; h1 = h0 + 1;
        send = q != rank;
        keep = q == rank || (rank & 1) == fl;
        tell = send && l7 != rank && (l7 == q || (l7 & 1) == fl);
        flag = FL_X + gg * 2 + (kc >> 1);
        const int sg = q == rank ? XGREM + lg
                     : lg < FG ? FG * (((q >> 1) - (rank >> 1) - 1) & 3) + lg
                               : 3 * FG + GG * (q >> 1) + lg - FG;
        dbo = (u32)WS_GX + (u32)(rank * 4 + kc) * (u32)XSLAB + (u32)(sg - h0) * 4096u;
        slo = (u32)(SL_X + gg * 2 + (kc >> 1)) * (u32)SLOT;
        cno = (u32)(q == rank ? WS_XOWN : WS_XREM) + (u32)(rank * 2 + (kc >> 1)) * 128u;
        asm volatile("" : "+s"(dbo), "+s"(slo), "+s"(cno), "+v"(flag));
        if (kc >= 2) { while (__builtin_amdgcn_s_memrealtime() - tstart0 < X_DELAY) __builtin_amdgcn_s_sleep(8); }
    }
    for (int phase = task0 >= T_P && task0 < T_W + 3 * WFIRST ? 0 : 1; phase < (g >= GLATE ? 2 : 1); ++phase) {
        if (phase) { isx = false; slotws = myws; }
        if (!isx) {
            int wt = phase ? 2 * WFIRST + g - GLATE + (192 - GLATE) * (7 - wave) : task0 - T_W - WFIRST;
            h0 = 0; h1 = 2;
            if (wt < 0) { wt += WFIRST; h1 = 1; } else if (wt < WFIRST) h0 = 1;
            const int wq = wt >> 5;
            const int half = (wt >> 3) & 1, blk = (wt & 7) | ((wt >> 1) & 8);
            const int kc = wq < 2 * I8A ? wq & 1 : wq < 4 * I8A ? 2 + (wq & 1) : wq & 3;
            const int wi8 = wq < 2 * I8A ? wq >> 1 : wq < 4 * I8A ? (wq >> 1) - I8A : wq >> 2;
            if (wt >= WFIRST) {
                while (__builtin_amdgcn_s_memrealtime() - tstart0 < (u64)(phase ? W_DELAY + LATE_STAG * (7 - wave) : W_DELAY0))
                    __builtin_amdgcn_s_sleep(8);
            }
            const uint8_t* src = P.w + (size_t)(kc * 128 + 16 * (lane >> 3)) * NN + blk * 1024 + wi8 * 128 + 16 * l7 + 8 * half;
            u32x2 in[16];
#pragma unroll
            for (int j = 0; j < 16; ++j) in[j] = *(const u32x2*)(src + (size_t)j * NN);
            __builtin_amdgcn_sched_barrier(0);
            auto needs = [&](int r) { return (r >> 1) == ((r & 1) ? blk & 3 : blk >> 2); };
            auto ctile = [&](int r) { return ((r & 1) ? blk >> 2 : blk & 3) * 8 + wi8; };
            send = keep = true;
            tell = l7 != rank && needs(l7);
            flag = FL_W + (kc * 32 + ctile(l7)) * 2 + half;
            const bool mine = needs(rank);
            const int fb = (rank & 1) ? 3 * (blk >> 2) + (blk & 3) - ((blk & 3) > (rank >> 1))
                                      : blk - (blk >= 2 * rank ? 4 : 0);
            const int ft = fb * 8 + wi8;
            dbo = half * 8192u + (mine ? (u32)WS_GW + (u32)((rank * 4 + kc) * 32 + ctile(rank)) * 16384u
                                       : (u32)WS_GW2 + (u32)(((ft >> 5) * 4 + kc) * 32 + (ft & 31)) * 16384u);
            swz = half * 8;
            slo = (u32)(SL_W + (kc * 128 + blk * 8 + wi8) * 2 + half) * (u32)SLOT;
            cno = (u32)WS_WCNT + (u32)((mine ? ctile(rank) : 32 + ft) * 32 + rank * 4 + kc) * 128u;
            asm volatile("" : "+s"(dbo), "+s"(slo), "+s"(cno), "+v"(flag));
#pragma unroll
            for (int d = 0; d < 2; ++d)
#pragma unroll
                for (int e = 0; e < 4; ++e) {
                    const u32 t0 = __builtin_amdgcn_perm(in[4 * e + 1][d], in[4 * e][d], 0x05010400u);
                    const u32 t1 = __builtin_amdgcn_perm(in[4 * e + 1][d], in[4 * e][d], 0x07030602u);
                    const u32 t2 = __builtin_amdgcn_perm(in[4 * e + 3][d], in[4 * e + 2][d], 0x05010400u);
                    const u32 t3 = __builtin_amdgcn_perm(in[4 * e + 3][d], in[4 * e + 2][d], 0x07030602u);
                    v[4 * d + 0][e] = __builtin_amdgcn_perm(t2, t0, 0x05040100u);
                    v[4 * d + 1][e] = __builtin_amdgcn_perm(t2, t0, 0x07060302u);
                    v[4 * d + 2][e] = __builtin_amdgcn_perm(t3, t1, 0x05040100u);
                    v[4 * d + 3][e] = __builtin_amdgcn_perm(t3, t1, 0x07060302u);
                }
            if (h0) {
#pragma unroll
                for (int j = 0; j < 4; ++j) v[j] = v[4 + j];
            }
        }
        GET_CALL();
#pragma unroll
        for (int j = 0; j < 8; ++j) asm volatile("" : "+v"(v[j]));
        uint8_t* const db = myws + dbo + (isx ? (call & 1) * GX_ODD : 0);
        uint8_t* const slot = slotws + pkoff + slo;
        u32 es = 0;
#pragma unroll
        for (int j = 0; j < 8; ++j) es = __builtin_amdgcn_sad_u8(v[j >> 1][(j & 1) * 2] & 0x7C7C7C7Cu, 0u, es);
        const u32 base = (u32)min(max(((int)wave_sum(es) - 1475) >> 13, 0), 29);
        u32 hdr = 0;
        for (int hh = h0; hh < h1; ++hh) {
            if (send) {
                const u32 h = op_pack(v, base * 0x04040404u, slot + hh * 4096, slotws != myws, scr, lut, lane) | (base << 13);
                if ((lane >> 3) == hh) hdr = h;
            }
            if (keep) {
                uint8_t* d = db + hh * 4096 + ((lane ^ (isx ? 0 : swz + hh * 4)) << 4);
#pragma unroll
                for (int j = 0; j < 4; ++j) st_sys(d + j * 1024, v[j]);
            }
#pragma unroll
            for (int j = 0; j < 4; ++j) v[j] = v[4 + j];
        }
        WAIT_VM(0);
        if (tell && (lane >> 3) >= h0 && (lane >> 3) < h1) st_flag(wsl, rank, flag, lane >> 3, call, hdr);
        if (keep && lane == 0)
            count_in(myws + cno, (u32)(h1 - h0));
    }
    GET_CALL();
    PUT_CALL();
    STAMP(1);

    const int xcd = g & 7, idx = g >> 3;
    const int part = xcd >> 1;
    const int wm = wave >> 1, wn = wave & 1;
    const int c = lane & 15, q4 = lane >> 4;
    const int ra0 = wm * 8192 + c * 128 + ((q4 ^ (c >> 1)) << 4);
    const int ra1 = ra0 ^ 64;
    const int rb0 = 32768 + (c & 3) * 4096 + ((((q4 << 3) | (4 * wn + (c >> 2))) ^ ((c & 3) << 2)) << 4);
    const u32 boff = (u32)wave * 2048u + (u32)lane * 16u;
    const u32 la0 = (u32)(uintptr_t)lds + wave * 4096, lb0 = (u32)(uintptr_t)lds + 32768 + wave * 2048;
    const u32 aoff0 = (u32)(lane >> 3) * 128u + (u32)(((lane & 7) ^ (lane >> 4)) << 4), aoff1 = aoff0 ^ 64u;

    for (int job = idx >= 12 ? 0 : 2; job < (idx < 8 ? 4 : 3); ++job) {
        const bool pre = job < 2, local = job == 3, ownrows = job != 2;
        const int ft = (g - 96) % 12 * 8 + (g - 96) / 12;
        const int i8 = pre ? ft & 7 : local ? idx : idx / 3;
        const int rt = pre ? job : local ? (xcd & 1) : 3 * (xcd & 1) + idx % 3;
        const int ct = pre ? 32 + ft : part * 8 + i8;
        const int nstep = pre ? 4 : NSTEP;
        const u32 wbase = pre ? (u32)WS_GW2 + (u32)(((ft >> 5) - rank) * 128 + (ft & 31)) * 16384u : (u32)WS_GW + (u32)ct * 16384u;
        const int rgw = rt * 8 + wave;
        const int sgw = ownrows ? XGREM + rgw : (rgw < 3 * FG + part * GG ? rgw : rgw + GG);
        const int skipsrc = ownrows ? -1 : group_of(rank, sgw) >> 4;
        const u32 xbase = (u32)(WS_GX + (call & 1) * GX_ODD) + (u32)sgw * 4096u;

        const u32* const pollp = lane < 32 ? (const u32*)(myws + WS_WCNT) + (ct * 32 + lane) * 32
                                           : (const u32*)(myws + X_DONE + (ownrows ? WS_XOWN : WS_XREM)) + (lane & 15) * 32;
        const u32 pneed = lane < 32 ? 4u * call : call;
        u64 ready = job == 1 ? ~0ull : 0;
        const int tq = ((lane & 31) - 4) / 7;
        const u32 vst = (lane & 31) < 4 ? rank * 4 + (lane & 3) : ((rank + (lane & 31) - 3 - 7 * tq) & 7) * 4 + tq;
        auto stepno = [&](int s) { return (int)__builtin_amdgcn_readlane(vst, s); };
        auto ensure = [&](int s) {
            const int st = stepno(s);
            const u64 want = (1ull << st) | (1ull << (32 + (st >> 1)));
            if ((ready & want) == want) return;
            const u64 t0 = __builtin_amdgcn_s_memrealtime();
            const u64 about = (st >> 2) == rank ? (0xfull << (4 * rank)) | (3ull << (32 + 2 * rank))
                                                : (0x11111111ull << (st & 3)) | (0x5555ull << (32 + ((st >> 1) & 1)));
            for (u32 it = 1;; ++it) {
                u32 v = 0;
                if (((about & ~ready) >> lane) & 1) v = __hip_atomic_load(pollp, __ATOMIC_RELAXED, __HIP_MEMORY_SCOPE_SYSTEM);
                ready |= __ballot(v >= pneed);
                if ((ready & want) == want) break;
                if ((it & 7) == 0 && (__hip_atomic_load(tmo, __ATOMIC_RELAXED, __HIP_MEMORY_SCOPE_AGENT) == call ||
                                      __builtin_amdgcn_s_memrealtime() - t0 > DEADLINE)) {
                    if (lane == 0) __hip_atomic_store(tmo, call, __ATOMIC_RELAXED, __HIP_MEMORY_SCOPE_AGENT);
                    WAIT_VM(0);
                    ready = ~0ull;
                    break;
                }
                __builtin_amdgcn_s_sleep(POLL_SLEEP);
            }
        };
        bool polling = wave == 0;
        int age = 0;
        u64 asked = 0;
        auto peek_issue = [&](int s) {
            if (age) { ++age; return; }
            const int st = stepno(min(s + 3 + PEEK_AHEAD, nstep - 1));
            const u64 want = (1ull << st) | (1ull << (32 + (st >> 1)));
            if ((ready & want) == want) return;
            asked = 0xffffffffffffull & ~ready;
            if ((asked >> lane) & 1) {
                u32 keep;
                asm volatile("s_mov_b32 %0, m0\n\ts_mov_b32 m0, %2\n\ts_nop 0\n\tglobal_load_lds_dword %1, off sc0 sc1\n\ts_mov_b32 m0, %0"
                             : "=&s"(keep) : "v"(pollp), "s"((u32)(uintptr_t)(lds + LDS_CTL)) : "memory");
            }
            age = 1;
        };
        auto peek_read = [&]() {
            if (age != 3) return;
            WAIT_VM(6);
            ready |= __ballot(*(volatile lu32*)(lds + LDS_CTL + lane * 4) >= pneed) & asked;
            age = 0;
        };
        const u32 vxo = (int)(vst >> 2) == skipsrc ? (u32)WS_ZERO : xbase + vst * (u32)XSLAB;
        const u32 vwo = wbase + vst * (32u * 16384u);
        const uint8_t *xa, *wa;
        auto locate = [&](int s) {
            xa = myws + (u32)__builtin_amdgcn_readlane(vxo, s);
            wa = myws + (u32)__builtin_amdgcn_readlane(vwo, s);
        };
#define DMA(M0V, VOFF, SRC, OFF) \
        asm volatile("s_mov_b32 m0, %2\n\ts_nop 0\n\tglobal_load_lds_dwordx4 %0, %1 offset:" #OFF " sc0 sc1" \
                     :: "v"(VOFF), "s"(SRC), "s"(M0V) : "memory")
        auto issue = [&](u32 so) {
            DMA(so + la0, aoff0, xa, 0); DMA(so + la0, aoff1, xa, 1024); DMA(so + la0, aoff0, xa, 2048); DMA(so + la0, aoff1, xa, 3072);
            DMA(so + lb0, boff, wa, 0); DMA(so + lb0, boff, wa, 1024);
        };

        f32x4 acc[4][4];
#pragma unroll
        for (int m = 0; m < 4; ++m)
#pragma unroll
            for (int t = 0; t < 4; ++t) acc[m][t] = f32x4{0.f, 0.f, 0.f, 0.f};

        auto rd1 = [&](const lbyte* p0, const lbyte* p1) {
            const u32x4 l = *(const lu32x4*)p0, h = *(const lu32x4*)p1;
            return i32x8{(int)l[0], (int)l[1], (int)l[2], (int)l[3], (int)h[0], (int)h[1], (int)h[2], (int)h[3]};
        };
#define MM(A, B, m, t) acc[m][t] = __builtin_amdgcn_mfma_scale_f32_16x16x128_f8f6f4(A[m], B[t], acc[m][t], 1, 1, 0, 0, 0, 0)
#define SB() __builtin_amdgcn_sched_barrier(0)
#define GTRACE() do { } while (0)
#define KSTEP(XA, XB, YA, YB, ISSUE, NVM) do { \
            if (ISSUE && polling) { peek_read(); ensure(s + 3); } \
            if (ISSUE && age) asm volatile("s_waitcnt vmcnt(7) lgkmcnt(0)" ::: "memory"); \
            else asm volatile("s_waitcnt vmcnt(" #NVM ") lgkmcnt(0)" ::: "memory"); \
            __builtin_amdgcn_s_barrier(); \
            asm volatile("" ::: "memory"); \
            SB(); MM(XA, XB, 0, 0); SB(); YA[0] = rd1(pa0, pa1); \
            SB(); MM(XA, XB, 0, 1); SB(); YB[0] = rd1(pb, pb + 512); \
            SB(); MM(XA, XB, 0, 2); SB(); if (ISSUE) DMA(ma, aoff0, xa, 0); \
            SB(); MM(XA, XB, 0, 3); SB(); YB[1] = rd1(pb + 1024, pb + 1536); \
            SB(); MM(XA, XB, 1, 0); SB(); if (ISSUE) DMA(ma, aoff1, xa, 1024); \
            SB(); MM(XA, XB, 1, 1); SB(); YB[2] = rd1(pb + 2048, pb + 2560); \
            SB(); MM(XA, XB, 1, 2); SB(); if (ISSUE) DMA(ma, aoff0, xa, 2048); \
            SB(); MM(XA, XB, 1, 3); SB(); YB[3] = rd1(pb + 3072, pb + 3584); \
            SB(); MM(XA, XB, 2, 0); SB(); if (ISSUE) DMA(ma, aoff1, xa, 3072); \
            SB(); MM(XA, XB, 2, 1); SB(); YA[1] = rd1(pa0 + 2048, pa1 + 2048); \
            SB(); MM(XA, XB, 2, 2); SB(); if (ISSUE) DMA(mb, boff, wa, 0); \
            SB(); MM(XA, XB, 2, 3); SB(); YA[2] = rd1(pa0 + 4096, pa1 + 4096); \
            SB(); MM(XA, XB, 3, 0); SB(); if (ISSUE) DMA(mb, boff, wa, 1024); \
            YA[3] = rd1(pa0 + 6144, pa1 + 6144); \
            SB(); MM(XA, XB, 3, 1); SB(); \
            ++s; \
            ma = sn + la0; mb = sn + lb0; \
            sn = sn == 2 * STAGE ? 0 : sn + STAGE; \
            pa0 = lds + sn + ra0; pa1 = lds + sn + ra1; pb = lds + sn + rb0; \
            if (ISSUE) locate(s + 3); \
            asm volatile("" : "+s"(ma), "+s"(mb), "+s"(xa), "+s"(wa), "+v"(pa0), "+v"(pa1), "+v"(pb)); \
            SB(); MM(XA, XB, 3, 2); MM(XA, XB, 3, 3); SB(); \
            if (ISSUE && polling) { peek_issue(s - 1); polling = age || (~ready << 16); } \
            GTRACE(); \
        } while (0)

        if (job == 0 && wave == 0) {
            while (__builtin_amdgcn_s_memrealtime() - tstart0 < (u64)(PRE_START + (idx - 12) * PRE_STAG)) __builtin_amdgcn_s_sleep(8);
        }
        if (wave == 0) { ensure(0); ensure(1); ensure(2); }
        asm volatile("s_waitcnt lgkmcnt(0)" ::: "memory");
        __builtin_amdgcn_s_barrier();
        asm volatile("" ::: "memory");
        asm volatile("s_nop 4");
        locate(0); issue(0);
        locate(1); issue(STAGE);
        locate(2); issue(2 * STAGE);
        locate(3);
        WAIT_VM(12);
        __builtin_amdgcn_s_barrier();
        asm volatile("" ::: "memory");
        i32x8 A0[4], B0[4], A1[4], B1[4];
#pragma unroll
        for (int t = 0; t < 4; ++t) {
            A0[t] = rd1(lds + ra0 + t * 2048, lds + ra1 + t * 2048);
            B0[t] = rd1(lds + rb0 + t * 1024, lds + rb0 + t * 1024 + 512);
        }
        int s = 0;
        u32 sn = STAGE, ma = la0, mb = lb0;
        const lbyte *pa0 = lds + STAGE + ra0, *pa1 = lds + STAGE + ra1, *pb = lds + STAGE + rb0;
        while (s < nstep - 4) {
            KSTEP(A0, B0, A1, B1, true, 6);
            KSTEP(A1, B1, A0, B0, true, 6);
        }
        KSTEP(A0, B0, A1, B1, true, 6);
        KSTEP(A1, B1, A0, B0, false, 6);
        KSTEP(A0, B0, A1, B1, false, 0);
        u32 tmov = __hip_atomic_load(tmo, __ATOMIC_RELAXED, __HIP_MEMORY_SCOPE_AGENT);
        SB();
#pragma unroll
        for (int m = 0; m < 4; ++m)
#pragma unroll
            for (int t = 0; t < 4; ++t) MM(A1, B1, m, t);

        int ln = lane;
        asm volatile("" : "+v"(ln));
        if (local) {
            float scale = tmov == call ? __builtin_nanf("") : P.sx[0] * P.sw[0];
            asm volatile("" : "+v"(scale));
            float* dst = P.out + (size_t)(rt * 256 + wm * 64 + 4 * (ln >> 4)) * NN + block_of(rank, part) * 1024 + i8 * 128 + wn * 64 + 4 * (ln & 15);
#pragma unroll
            for (int m = 0; m < 4; ++m)
#pragma unroll
                for (int r = 0; r < 4; ++r) {
                    f32x4 y;
#pragma unroll
                    for (int t = 0; t < 4; ++t) y[t] = silu(acc[m][t][r] * scale);
                    THROTTLE(LOC_WIN);
                    st_mode<NT_OUT>(dst + (size_t)(m * 16 + r) * NN, __builtin_bit_cast(u32x4, y));
                }
            STAMP(7);
        } else if (pre) {
            const int fb = ft >> 3;
            const int blkf = (rank & 1) ? 4 * (fb / 3) + fb % 3 + (fb % 3 >= (rank >> 1)) : fb + (fb >= 2 * rank ? 4 : 0);
            float* d = P.out + (size_t)(rt * 256 + wm * 64 + 4 * (ln >> 4)) * NN + blkf * 1024 + i8 * 128 + wn * 64 + 4 * (ln & 15);
#pragma unroll
            for (int m = 0; m < 4; ++m)
#pragma unroll
                for (int r = 0; r < 4; ++r) {
                    const f32x4 y = {acc[m][0][r], acc[m][1][r], acc[m][2][r], acc[m][3][r]};
                    st_mode<NT_PRE>(d + (size_t)(m * 16 + r) * NN, __builtin_bit_cast(u32x4, y));
                }
            if (job == 1) {
                WAIT_VM(0);
                __builtin_amdgcn_s_barrier();
                if (tid == 0) __hip_atomic_fetch_add((u32*)(myws + WS_PCNT) + i8 * 32, 1u, __ATOMIC_RELAXED, __HIP_MEMORY_SCOPE_SYSTEM);
                STAMP(7);
            }
        } else {
            const int rgl = rt * 8 + wm * 2 + (ln & 1);
            const int jj = (rgl - 3 * FG) / GG;
            u64 qws = (u64)P.ws[(rgl < 3 * FG ? 2 * (((rank >> 1) + 1 + rgl / FG) & 3) + (rank & 1)
                                              : 2 * (jj + (jj >= part)) + 1 - (rank & 1)) & 7];
            auto publish = [&](int mp, u32 hdr) {
                WAIT_VM(0);
                if (ln == mp) st_flag((uint8_t*)(__attribute__((address_space(1))) uint8_t*)qws, rank, FL_O + (ct * 48 + rgl) * 2 + wn, 0, call, hdr);
            };
            float amf = 0.f;
#pragma unroll
            for (int i = 0; i < 64; ++i) amf = fmaxf(amf, fabsf(acc[i >> 4][i & 3][(i >> 2) & 3]));
            const u32 am = wave_umax(__float_as_uint(amf));
            const int ex = min(max((int)(am >> 23), 40), 200);
            u32 votes = 0;
#pragma unroll
            for (int i = 0; i < 8; ++i) {
                const u32 I = (u32)(int)(acc[i >> 1][i & 3][i & 1] * __uint_as_float((u32)(280 - ex) << 23));
                votes += (((I + (1u << 25)) & 0xfc000007u) != 0) + ((((I + (1u << 26)) & 0xf800000fu) != 0) << 10) +
                         (((I & 0x1fu) != 0) << 20);
            }
            votes = wave_sum(votes);
            const u32 n0 = votes & 1023u, n1 = (votes >> 10) & 1023u, n2 = votes >> 20;
            const int dsc = n2 <= n1 && n2 <= n0 ? 2 : n1 <= n0 ? 1 : 0;
            const float up = __uint_as_float((u32)(280 - ex - dsc) << 23);
            u32 hdrs[2];
            asm volatile("" : "+v"(qws), "+v"(tmov));
            const bool bad = __builtin_amdgcn_readfirstlane(tmov) == call;
#pragma unroll
            for (int mp = 0; mp < 2; ++mp) {
#define ACCV(i) acc[2 * mp + ((i) >> 4)][(i) & 3][((i) >> 2) & 3]
                uint8_t* const slot = myws + pkoff + (size_t)(SL_O + (ct * 48 + rt * 8 + wm * 2 + mp) * 2 + wn) * SLOT;
                u32 hi[16], mid[8], q[8], nz = 0, miss = 0;
#pragma unroll
                for (int j = 0; j < 8; ++j) {
                    if (mp == 1 && j == 3) { SB(); publish(0, hdrs[0]); SB(); }
                    u32 t[4];
#pragma unroll
                    for (int t4 = 0; t4 < 4; ++t4) {
                        const float f = ACCV(4 * j + t4) * up;
                        const u32 I = (u32)(int)f;
                        miss |= __float_as_uint(__builtin_amdgcn_fractf(f));
                        t[t4] = __builtin_amdgcn_alignbit(I, I ^ (u32)__builtin_amdgcn_sbfe((int)I, 0, 26), 26);
                    }
                    hi[2 * j] = __builtin_amdgcn_perm(t[1], t[0], 0x07060302u);
                    hi[2 * j + 1] = __builtin_amdgcn_perm(t[3], t[2], 0x07060302u);
                    const u32 l0 = __builtin_amdgcn_perm(t[1], t[0], 0x05010400u), l1 = __builtin_amdgcn_perm(t[3], t[2], 0x05010400u);
                    const u32 lo = __builtin_amdgcn_perm(l1, l0, 0x05040100u);
                    mid[j] = __builtin_amdgcn_perm(l1, l0, 0x07060302u);
                    q[j] = bfi(0x07070707u, bfi(0x03030303u, lo, mid[j] << 2), (lo >> 3) & 0x18181818u);
                    const u32 nb = q[j] + 0x1F1F1F1Fu;
                    nz = bfi(0x01010101u << j, j < 5 ? nb >> (5 - j) : nb << (j - 5), nz);
                }
                const u32 cnt = __builtin_popcount(nz);
                const u32 inc = wave_scan_add(cnt);
                const u32 total = __builtin_amdgcn_readlane(inc, 63);
                if (bad || (int)(am >> 23) > ex || __any(miss != 0) || total > LIST_MAX) {
#pragma unroll
                    for (int j = 0; j < 8; ++j) {
                        f32x4 y = {ACCV(4 * j), ACCV(4 * j + 1), ACCV(4 * j + 2), ACCV(4 * j + 3)};
                        if (bad) y[0] = __builtin_nanf("");
                        st_sys(slot + j * 1024 + ln * 16, __builtin_bit_cast(u32x4, y));
                    }
                    hdrs[mp] = 64u | H_RAW;
                } else {
#pragma unroll
                    for (int j = 0; j < 7; ++j) mid[j] = bfi(0x01010101u, mid[7] >> (j + 1), mid[j]);
#pragma unroll
                    for (int j = 0; j < 4; ++j)
                        st_sys(slot + j * 1024 + ln * 16, u32x4{hi[4 * j], hi[4 * j + 1], hi[4 * j + 2], hi[4 * j + 3]});
                    st_sys(slot + 4096 + ln * 16, u32x4{mid[0], mid[1], mid[2], mid[3]});
                    *(lu32*)(scr + ln * 4) = mid[4];
                    *(lu32*)(scr + 256 + ln * 4) = mid[5];
                    *(lu32*)(scr + 512 + ln * 4) = mid[6];
                    const u32 nb = write_list(q, nz, inc - cnt, total, scr + 768, lds + 2 * STAGE + wave * 4096, ln);
                    for (u32 o = ln * 16; o < 768 + nb; o += 1024) st_sys(slot + 5120 + o, *(lu32x4*)(scr + o));
                    CBAR();
                    hdrs[mp] = ((ACC_MAIN + nb + 127) >> 7) | (nb << 8) | ((u32)(ex + dsc - 30) << 19);
                }
            }
            publish(1, hdrs[1]);
            STAMP(4);
        }
    }
}

}

extern "C" void kernel_launch(int rank, int nranks, ncclComm_t comm,
                              hipStream_t stream,
                              void* const* d_in, const int* in_sizes, int n_in,
                              void* d_out, int out_size,
                              const int* in_dtypes, int out_dtype,
                              void* const* d_out_all,
                              void* const* d_ws_all, size_t ws_bytes) {
    if (nranks != NR || (unsigned)rank >= (unsigned)NR || n_in != 4 || ws_bytes < WS_NEED ||
        in_sizes[0] != MM * KS || in_sizes[1] != KS * NN || out_size != MS * NN) {
        fprintf(stderr, "kernel_launch: unexpected problem shape\n");
        return;
    }
    Params P;
    P.x = (const uint8_t*)d_in[0];
    P.w = (const uint8_t*)d_in[1];
    P.sx = (const float*)d_in[2];
    P.sw = (const float*)d_in[3];
    P.out = (float*)d_out;
    for (int p = 0; p < NR; ++p) P.ws[p] = (uint8_t*)d_ws_all[p];
    P.myws = P.ws[rank];
    P.rank = rank;
    P.pad = 0;
    hipLaunchKernelGGL(fused_kernel, dim3(GRID), dim3(NTHR), 0, stream, P);
    HIP_CHECK(hipGetLastError());
}
```

```cpp
#include <hip/hip_runtime.h>
#include <rccl/rccl.h>
#include <cstdio>
#include <cstdint>

#define HIP_CHECK(expr) do { hipError_t _e = (expr); if (_e != hipSuccess) { \
    fprintf(stderr, "HIP error %s at %s:%d\n", hipGetErrorString(_e), __FILE__, __LINE__); \
    return; } } while (0)

typedef unsigned u32;
typedef unsigned long long u64;
typedef unsigned u32x4 __attribute__((ext_vector_type(4)));
typedef unsigned u32x2 __attribute__((ext_vector_type(2)));
typedef int i32x8 __attribute__((ext_vector_type(8)));
typedef float f32x4 __attribute__((ext_vector_type(4)));
typedef __attribute__((address_space(1))) const void gvoid;
typedef __attribute__((address_space(3))) void lvoid;
typedef __attribute__((address_space(3))) uint8_t lbyte;
typedef __attribute__((address_space(3))) u32 lu32;
typedef __attribute__((address_space(3))) unsigned short lu16;
typedef __attribute__((address_space(3))) u32x4 lu32x4;
typedef __attribute__((address_space(3))) u32x2 lu32x2;

namespace {

constexpr int NR = 8;
constexpr int MM = 4096, KK = 4096, NN = 16384;
constexpr int KS = KK / NR;
constexpr int MS = MM / NR;
constexpr int FG = 7, GG = 9;
constexpr int F = 32 * FG, G = 32 * GG;
static_assert(F + G == MS, "rows");
constexpr int XGREM = 3 * FG + 4 * GG;
constexpr int XREM = 32 * XGREM;
constexpr int XROWS = XREM + MS;
constexpr int NSTEP = 32;

constexpr int GRID = 256, NSEND = 64, NTHR = 512;
constexpr int NENG = 56;
#define WIN 6
#define WIN_OP 7
#define STARVE 8
#define BATCH 5
#define NFWD 3
#define PP_PACE 1
#define W_ORDER 0x652410
#define WB 2
#define MON_SLEEP 4
#define X0_PRIO 2
#define SC_SLEEP 4
#define WK_SLEEP 1
#define WK_PRIO 2
#define LOOK 4
#define NPUSH 5
#define PUSH_WIN 8
#define W_DELAY 500
#define PRE_STAG 140
#define PRE_START 1700
#define LATE_STAG 160
#define LOC_WIN 32
#define PEEK_AHEAD 6
#define W_DELAY0 200
#define NT_PRE 0
#define NT_OUT 0
#define POLL_SLEEP 16
#define I8A 3

constexpr int SLOT = 8192;
constexpr int NPP = 24;
constexpr u32 H_RAW = 1u << 7;
constexpr u32 ACC_MAIN = 5888;
constexpr int SL_X = 0, SL_W = 256, SL_O = SL_W + 1024, SL_F = SL_O + 32 * 48 * 2, SL_P = SL_F + 128, SL_END = SL_P + NR * NPP;
constexpr int FL_X = 0, FL_X1 = 128, FL_W = FL_X1 + 16, FL_L = FL_W + 256, FL_O = FL_L + 32, FL_F = FL_O + 32 * 6 * 2,
              FL_P = FL_F + 32, FL_END = FL_P + NPP;
constexpr u32 FW_SPLIT = 1u << 16, FW_WIDE = 1u << 17;
static_assert(FL_END <= 4096, "flags");

constexpr size_t WS_CNT = 262144;
constexpr size_t WS_ZERO = 0;
constexpr size_t WS_TMO = 32768;
constexpr size_t WS_TICK = 49152;
constexpr size_t WS_XREM = 65536;
constexpr size_t WS_XOWN = WS_XREM + 2048;
constexpr size_t X_DONE = 8192;
constexpr size_t WS_PCNT = WS_XOWN + 2048;
constexpr size_t WS_WCNT = 27u << 20;
constexpr size_t WS_FLAG = 28u << 20;
constexpr size_t WS_DBG = 524288;
constexpr size_t WS_GW = 1u << 20;
constexpr size_t WS_GX = WS_GW + (size_t)NSTEP * 32 * 16384;
constexpr size_t XSLAB = (size_t)XROWS * 128;
constexpr size_t WS_PK = 32u << 20;
constexpr size_t PK_BYTES = 40u << 20;
static_assert(WS_GX + NSTEP * XSLAB <= WS_WCNT && WS_WCNT + 128 * 32 * 128 <= WS_FLAG && WS_FLAG + NR * 4096 * 128 <= WS_PK, "staging");
static_assert((size_t)SL_END * SLOT <= PK_BYTES, "packed");
constexpr size_t GX_ODD = WS_PK + 2 * PK_BYTES - WS_GX;
constexpr size_t WS_GW2 = 124u << 20;
static_assert(WS_GX + GX_ODD + NSTEP * XSLAB <= WS_GW2, "staging");
constexpr size_t WS_NEED = WS_GW2 + (size_t)3 * 128 * 16384;

constexpr int STAGE = 49152;
constexpr int LDS_CTL = 3 * STAGE;
constexpr int LDS_LUT = LDS_CTL + 256;
constexpr int LDS_PUB = LDS_LUT + 128;
constexpr int NS = 12;
constexpr int NPAIR = 3;
constexpr int EL_PLANE = NS * SLOT;
constexpr int EL_BUF = EL_PLANE + 8 * 1024;
constexpr int EL_TAB = EL_BUF + 8 * 2304;
constexpr int EL_GO = EL_TAB + 1024 * 16;
constexpr int EL_FREE = EL_GO + 256;
constexpr int EL_ISS = EL_FREE + 128;
constexpr int EL_FULL = EL_ISS + 4;
constexpr int EL_HEAD = EL_ISS + 8;
constexpr int EL_DEAD = EL_ISS + 12;
constexpr int EL_END = EL_ISS + 64;
static_assert(EL_END + 1024 <= LDS_CTL && NS % NPAIR == 0 && NS % 2 == 0, "engine LDS");
constexpr u32 UNIT_END = 0xffffu;
constexpr u64 DEADLINE = 40000000ull;

static_assert(7 * NPUSH <= 112 && NPUSH + NPP <= XGREM - GG, "push");

struct Params {
    const uint8_t* x;
    uint8_t* myws;
    int rank;
    int pad;
    const uint8_t* w;
    const float* sx;
    const float* sw;
    float* out;
    uint8_t* ws[NR];
};

__device__ __forceinline__ void st_sys(void* p, u32x4 v) {
    asm volatile("global_store_dwordx4 %0, %1, off sc0 sc1\n\ts_nop 1" :: "v"(p), "v"(v) : "memory");
}
template <int M> __device__ __forceinline__ void st_mode(void* p, u32x4 v) {
    if (M == 0) asm volatile("global_store_dwordx4 %0, %1, off sc0 sc1\n\ts_nop 1" :: "v"(p), "v"(v) : "memory");
    if (M == 1) asm volatile("global_store_dwordx4 %0, %1, off sc0 sc1 nt\n\ts_nop 1" :: "v"(p), "v"(v) : "memory");
    if (M == 2) asm volatile("global_store_dwordx4 %0, %1, off nt\n\ts_nop 1" :: "v"(p), "v"(v) : "memory");
    if (M == 3) asm volatile("global_store_dwordx4 %0, %1, off sc1\n\ts_nop 1" :: "v"(p), "v"(v) : "memory");
}
#define WAIT_VM(n) asm volatile("s_waitcnt vmcnt(" #n ")" ::: "memory")
#define THROTTLE(n) asm volatile("s_waitcnt vmcnt(%0)" :: "i"((n) - 1) : "memory")
#define CBAR() asm volatile("" ::: "memory")
#define DPP(v, ctl, rm) (u32)__builtin_amdgcn_update_dpp(0, (int)(v), ctl, rm, 0xf, false)

__device__ __forceinline__ u32 wave_scan_add(u32 v) {
    v += DPP(v, 0x111, 0xf); v += DPP(v, 0x112, 0xf); v += DPP(v, 0x114, 0xf); v += DPP(v, 0x118, 0xf);
    v += DPP(v, 0x142, 0xa); v += DPP(v, 0x143, 0xc);
    return v;
}
__device__ __forceinline__ u32 wave_sum(u32 v) { return __builtin_amdgcn_readlane(wave_scan_add(v), 63); }
__device__ __forceinline__ u32 wave_umax(u32 v) {
    v = max(v, DPP(v, 0x111, 0xf)); v = max(v, DPP(v, 0x112, 0xf)); v = max(v, DPP(v, 0x114, 0xf));
    v = max(v, DPP(v, 0x118, 0xf)); v = max(v, DPP(v, 0x142, 0xa)); v = max(v, DPP(v, 0x143, 0xc));
    return __builtin_amdgcn_readlane(v, 63);
}
__device__ __forceinline__ int block_of(int r, int part) {
    return (r & 1) ? 4 * part + (r >> 1) : 4 * (r >> 1) + part;
}
__device__ __forceinline__ float silu(float v) {
    const float z = __builtin_amdgcn_fmed3f(v, -60.f, 60.f);
    const float e = __builtin_amdgcn_exp2f(z * -1.4426950408889634f);
    return v * __builtin_amdgcn_rcpf(1.0f + e);
}

constexpr u32 LIST_MAX = 960;
__device__ __forceinline__ u32 plane_addr(u32 t) { return ((t & 4u) << 8) | ((t >> 5) << 4) | ((t & 3u) << 2) | ((t >> 3) & 3u); }
__device__ __forceinline__ u32 write_list(const u32 (&q)[8], u32 nz, u32 pos, u32 total,
                                          lbyte* lst, lbyte* tmp, int lane) {
    *(lu32x4*)(tmp + lane * 16) = u32x4{q[0], q[1], q[2], q[3]};
    *(lu32x4*)(tmp + 1024 + lane * 16) = u32x4{q[4], q[5], q[6], q[7]};
    for (u32 o = lane * 16; o < 2 * total + 32; o += 1024) *(lu32x4*)(lst + o) = u32x4{0xc0c0c0c0u, 0xc0c0c0c0u, 0xc0c0c0c0u, 0xc0c0c0c0u};
    lbyte* const pl = tmp + 2048;
    if (lane == 0) *(lu16*)(pl + 2) = 0xffffu;
    lbyte* wp = pl + 4 + 2 * pos;
    for (u32 m = nz; m; m &= m - 1) {
        *(lu16*)wp = (unsigned short)(((u32)lane << 5) | __builtin_ctz(m));
        wp += 2;
    }
    CBAR();
    u32 nb = 0;
    for (u32 i0 = 0; i0 < total; i0 += 128) {
        const u32 i = i0 + 2 * lane;
        const u32 pa = *(lu32*)(pl + 2 * i), pb = *(lu32*)(pl + 2 * i + 4);
        const u32 t[3] = {pa >> 16, pb & 0xffffu, pb >> 16};
        u32 n[2], e[2], val[2];
#pragma unroll
        for (int h = 0; h < 2; ++h) {
            val[h] = *(tmp + plane_addr(t[h + 1] & 2047u));
            const u32 gap = (t[h + 1] - t[h] - 1) & 0xffffu, ty = min(val[h] - 3u, 2u);
            e[h] = (ty << 6) | (gap & 63u);
            n[h] = i + h < total ? (gap >> 6) + 1 + (ty >> 1) : 0;
        }
        const u32 inc = wave_scan_add(n[0] + n[1]);
        lbyte* const end = lst + nb + inc;
#pragma unroll
        for (int h = 0; h < 2; ++h) {
            lbyte* const le = end - (h ? 0 : n[1]) - 1;
            if (n[h]) {
                *(le - (e[h] >> 7)) = (uint8_t)e[h];
                if (e[h] >> 7) *le = (uint8_t)(0xe0u | val[h]);
            }
        }
        nb += __builtin_amdgcn_readlane(inc, 63);
    }
    CBAR();
    return nb;
}
__device__ __forceinline__ void st_flags8(lbyte* pub, int word, u32 hdr, uint8_t* ws, bool to, int src, int line,
                                          u32 call, int lane) {
    u32 n = 0;
    if (lane == 0) {
        *(lu32*)(pub + word * 4) = hdr;
        CBAR();
        n = __hip_atomic_fetch_add((lu32*)(pub + 32), 1u, __ATOMIC_RELAXED, __HIP_MEMORY_SCOPE_WORKGROUP);
    }
    if (__builtin_amdgcn_readfirstlane(n) != 7) return;
    CBAR();
    const u32x2 h = *(lu32x2*)(pub + (lane & 3) * 8);
    if (to) st_sys(ws + WS_FLAG + (size_t)(src * 4096 + line) * 128 + (call & 1) * 64 + (lane & 3) * 16, u32x4{call, h[0], call, h[1]});
}
__device__ __forceinline__ void st_flag(uint8_t* ws, int src, int flag, int half, u32 call, u32 hdr) {
    __hip_atomic_store((u64*)(ws + WS_FLAG) + (size_t)(src * 4096 + flag) * 16 + (call & 1) * 2 + half,
                       ((u64)hdr << 32) | call, __ATOMIC_RELAXED, __HIP_MEMORY_SCOPE_SYSTEM);
}
__device__ __forceinline__ void count_in(uint8_t* cnt, u32 n) {
    __hip_atomic_fetch_add((u32*)cnt, n, __ATOMIC_RELAXED, __HIP_MEMORY_SCOPE_SYSTEM);
}
__device__ __forceinline__ u32 bfi(u32 m, u32 a, u32 b) { return (a & m) | (b & ~m); }

__device__ __forceinline__ void op_lut_init(lbyte* lut, int m) {
    u32 e = 0, c = 0x0c0c0c0cu, n = 0;
#pragma unroll
    for (u32 i = 0; i < 4; ++i) {
        const bool on = (m >> i) & 1;
        e |= (on ? n : 0x0cu) << (8 * i);
        if (on) c = (c & ~(0xffu << (8 * n))) | (i << (8 * n));
        n += on;
    }
    *(lu32*)(lut + m * 4) = e;
    *(lu32*)(lut + 64 + m * 4) = c;
}
__device__ __forceinline__ u32 and_or(u32 a, u32 m, u32 b) {
    u32 r; asm("v_and_or_b32 %0, %1, %2, %3" : "=v"(r) : "v"(a), "s"(m), "v"(b)); return r;
}
__device__ __forceinline__ u32 bfi_s(u32 m, u32 a, u32 b) {
    u32 r; asm("v_bfi_b32 %0, %1, %2, %3" : "=v"(r) : "s"(m), "v"(a), "v"(b)); return r;
}
__device__ __forceinline__ u32 ff_count(u32 ff, u32 acc) { return (u32)__builtin_amdgcn_sdot4((int)ff, -1, (int)acc, true); }
__device__ __forceinline__ u32 ff_lut(u32 ff, u32 lut) {
    return *(const lu32*)(uintptr_t)(u32)__builtin_amdgcn_sdot4((int)ff, (int)0xe0f0f8fcu, (int)lut, true);
}
__device__ __forceinline__ u32 ge13(u32 x) { return __builtin_amdgcn_perm(0u, 0u, x); }
__device__ __forceinline__ void lds_or(u32 p, u32 v) {
    __hip_atomic_fetch_or((lu32*)(uintptr_t)p, v, __ATOMIC_RELAXED, __HIP_MEMORY_SCOPE_WAVEFRONT);
}
__device__ __forceinline__ u32 chunk_starts(const u32 (&c)[4], u32 (&p)[4], u32 at) {
    const u32 a = c[0] | (c[1] << 16), b = c[2] | (c[3] << 16);
    const u32 ia = wave_scan_add(a), ib = wave_scan_add(b);
    const u32 ta = __builtin_amdgcn_readlane(ia, 63), tb = __builtin_amdgcn_readlane(ib, 63);
    const u32 t1 = ta & 0xffffu, t2 = t1 + (ta >> 16), t3 = t2 + (tb & 0xffffu);
    p[0] = ((ia - a) & 0xffffu) + at;
    p[1] = ((ia - a) >> 16) + (at + t1);
    p[2] = ((ib - b) & 0xffffu) + (at + t2);
    p[3] = ((ib - b) >> 16) + (at + t3);
    return t3 + (tb >> 16);
}
__device__ __forceinline__ u32 op_pack(const u32x4* v, u32 base4, uint8_t* blk, bool paced, lbyte* scr, const lbyte* lutp, int lane) {
    lbyte* const side = scr + 2560;
    const u32 lut = (u32)(uintptr_t)lutp + 64, eb = (u32)(uintptr_t)scr + 16;
    const u32x4 z4 = {0u, 0u, 0u, 0u};
    *(lu32x4*)(scr + lane * 16) = z4;
    *(lu32x4*)(scr + 1024 + lane * 16) = z4;
    *(lu32x4*)(scr + 2048 + lane * 16) = z4;
    *(lu32x4*)(scr + 3072 + lane * 16) = z4;
    u32 ub[16], ef[16], sel[16], N[8], S[2] = {0u, 0u}, cnt[4] = {0u, 0u, 0u, 0u}, pos[4];
#pragma unroll
    for (int i = 0; i < 16; ++i) {
        const u32 d = v[i >> 2][i & 3];
        ub[i] = (d | 0x80808080u) - base4;
        ef[i] = ge13(and_or(ub[i], 0x7C7C7C7Cu, 0x01010101u));
        sel[i] = ff_lut(ef[i], lut);
        const u32 x = and_or(ef[i], 0x0C0C0C0Cu, ub[i]);
        N[i >> 1] = (i & 1) ? bfi_s(0x0F0F0F0Fu, N[i >> 1], x << 4) : x;
        S[i >> 3] = and_or(d >> (7 - (i & 7)), 0x01010101u << (i & 7), S[i >> 3]);
        cnt[i >> 2] = ff_count(ef[i], cnt[i >> 2]);
    }
    const u32 n2 = chunk_starts(cnt, pos, eb - 1), n2w = (n2 + 15) >> 4;
    bool raw = n2 > 2048;
    u32 n3 = 0;
    if (!raw) {
        st_sys(blk + lane * 16, u32x4{N[0], N[1], N[2], N[3]});
        if (!paced) st_sys(blk + 1024 + lane * 16, u32x4{N[4], N[5], N[6], N[7]});
        CBAR();
        *(lu32x2*)(side + lane * 8) = u32x2{S[0], S[1]};
#pragma unroll
        for (int i = 0; i < 16; ++i) {
            const u32 c = __builtin_amdgcn_perm(0u, ub[i], sel[i]), p = pos[i >> 2];
            lu32* const q = (lu32*)(uintptr_t)(p & ~3u);
            __hip_atomic_fetch_or(q, __builtin_amdgcn_alignbyte(c, 0u, ~p), __ATOMIC_RELAXED, __HIP_MEMORY_SCOPE_WAVEFRONT);
            __hip_atomic_fetch_or(q + 1, __builtin_amdgcn_alignbyte(0u, c, ~p), __ATOMIC_RELAXED, __HIP_MEMORY_SCOPE_WAVEFRONT);
            pos[i >> 2] = ff_count(ef[i], p);
        }
        CBAR();
        const u32 l3 = (u32)(uintptr_t)side + 512 + n2w * 4;
        u32 bad = 0;
        for (u32 w0 = 0; w0 < n2w; w0 += 64) {
            const u32x4 B = *(lu32x4*)(scr + 16 + (w0 + lane) * 16);
            u32 W = 0, c3 = 0;
            u64 str = 0;
#pragma unroll
            for (int k = 0; k < 4; ++k) {
                const u32 u = (((B[k] & 0x7C7C7C7Cu) + 0x08080808u) >> 2) & 0x1F1F1F1Fu;
                const u32 code = __builtin_amdgcn_perm(0x83038200u, 0x80008100u, u) & 0x03030303u;
                W |= __builtin_amdgcn_udot4(code, 0x40100401u, 0u, false) << (8 * k);
                bad |= (u + 0x78787878u) & ~(u + 0x6E6E6E6Eu);
                const u32 e = __builtin_amdgcn_perm(0u, 0xff000000u, code);
                const u32 sym = u & (and_or(ge13(u), 0x0E0E0E0Eu, 0x01010101u));
                const u32 c = __builtin_amdgcn_perm(0u, sym, ff_lut(e, lut));
                str |= (u64)__builtin_amdgcn_perm(0u, c | (c >> 4), 0x0c0c0200u) << c3;
                c3 = (u32)__builtin_amdgcn_sdot4((int)e, (int)0xfcfcfcfcu, (int)c3, true);
            }
            const u32 i3 = wave_scan_add(c3);
            const u32 o3 = n3 + i3 - c3;
            n3 += __builtin_amdgcn_readlane(i3, 63);
            if (n2w * 4 + (n3 >> 3) > 1000) { raw = true; break; }
            if (w0 + lane < n2w) *(lu32*)(side + 512 + (w0 + lane) * 4) = W;
            const u64 lo = (u64)(u32)str << (o3 & 31u), hi = (str >> 32) << (o3 & 31u);
            const u32 p = l3 + ((o3 >> 5) << 2);
            if (c3) lds_or(p, (u32)lo);
            if ((o3 & 31u) + c3 > 32) {
                lds_or(p + 4, (u32)(lo >> 32) | (u32)hi);
                lds_or(p + 8, (u32)(hi >> 32));
            }
        }
        raw = raw || __any(bad & 0x80808080u);
        CBAR();
    }
    if (raw) {
        WAIT_VM(0);
#pragma unroll
        for (int j = 0; j < 4; ++j) st_sys(blk + j * 1024 + lane * 16, v[j]);
        return 31u;
    }
    if (paced) {
        WAIT_VM(0);
        st_sys(blk + 1024 + lane * 16, u32x4{N[4], N[5], N[6], N[7]});
        if (PP_PACE) WAIT_VM(0);
    }
    const u32 nb = 512 + n2w * 4 + ((n3 + 7) >> 3);
    for (u32 o = lane * 16; o < nb; o += 1024) st_sys(blk + 2048 + o, *(lu32x4*)(side + o));
    CBAR();
    return ((2048 + nb - 1) >> 7) | (n2w << 5);
}
template <class Done>
__device__ __forceinline__ void op_unpack(u32x4* v, u32 base4, u32 n2w, const lbyte* blk, lbyte* buf, const lbyte* lutp,
                                          int lane, Done done) {
    const u32 lut = (u32)(uintptr_t)lutp;
    const u32x4 N0 = *(const lu32x4*)(blk + lane * 16), N1 = *(const lu32x4*)(blk + 1024 + lane * 16);
    const u32x2 S = *(const lu32x2*)(blk + 2048 + lane * 8);
    const lbyte* const l3 = blk + 2560 + n2w * 4;
    u32 n3 = 0;
    for (u32 w0 = 0; w0 < n2w; w0 += 64) {
        u32 W = *(const lu32*)(blk + 2560 + (w0 + lane) * 4);
        if (w0 + lane >= n2w) W = 0;
        const u32 c3t = __builtin_popcount(W & (W >> 1) & 0x55555555u);
        const u32 i3 = wave_scan_add(c3t);
        const u32 o3 = n3 + i3 - c3t;
        n3 += __builtin_amdgcn_readlane(i3, 63);
        const lbyte* const p = l3 + ((o3 >> 3) << 2);
        const u32 d0 = *(const lu32*)p, d1 = *(const lu32*)(p + 4), d2 = *(const lu32*)(p + 8);
        const u64 str = ((u64)__builtin_amdgcn_alignbit(d2, d1, o3 << 2) << 32) | __builtin_amdgcn_alignbit(d1, d0, o3 << 2);
        u32x4 B;
        u32 c3 = 0;
#pragma unroll
        for (int k = 0; k < 4; ++k) {
            const u32 f = (W >> (8 * k)) & 255u;
            const u32 t = (f | (f << 12)) & 0x000F000Fu;
            const u32 code = (t | (t << 6)) & 0x03030303u;
            const u32 e = __builtin_amdgcn_perm(0u, 0xff000000u, code);
            const u32 x = __builtin_amdgcn_perm(0u, (u32)(str >> c3), 0x01010000u);
            const u32 s4 = bfi_s(0x00FF00FFu, x << 2, x >> 2) & 0x3C3C3C3Cu;
            const u32 u4 = s4 + and_or(ge13(s4 + 0x05050505u), 0x28282828u, 0x04040404u);
            B[k] = __builtin_amdgcn_perm(0u, u4, ff_lut(e, lut)) | __builtin_amdgcn_perm(0u, 0x0000706Cu, code);
            c3 = (u32)__builtin_amdgcn_sdot4((int)e, (int)0xfcfcfcfcu, (int)c3, true);
        }
        *(lu32x4*)(buf + (w0 + lane) * 16) = B;
    }
    done();
    u32 nib[16], ef[16], sel[16], cnt[4] = {0u, 0u, 0u, 0u}, pos[4];
#pragma unroll
    for (int i = 0; i < 16; ++i) {
        const u32 n = i < 8 ? N0[i >> 1] : N1[(i >> 1) - 4];
        nib[i] = ((i & 1) ? n >> 4 : n) & 0x0F0F0F0Fu;
        ef[i] = ge13(nib[i] | 0x01010101u);
        sel[i] = ff_lut(ef[i], lut);
        cnt[i >> 2] = ff_count(ef[i], cnt[i >> 2]);
    }
    chunk_starts(cnt, pos, (u32)(uintptr_t)buf);
    CBAR();
    u32 ps[16], q0[16], q1[16];
#pragma unroll
    for (int i = 0; i < 16; ++i) {
        const int j = (i & 3) * 4 + (i >> 2);
        ps[j] = (j & 3) ? ff_count(ef[j - 1], ps[j - 1]) : pos[j >> 2];
        const lu32* const q = (const lu32*)(uintptr_t)(ps[j] & ~3u);
        q0[j] = q[0]; q1[j] = q[1];
    }
    __builtin_amdgcn_sched_barrier(0);
#pragma unroll
    for (int i = 0; i < 16; ++i) {
        const u32 y = nib[i] + base4 + __builtin_amdgcn_perm(0u, __builtin_amdgcn_alignbyte(q1[i], q0[i], ps[i]), sel[i]);
        v[i >> 2][i & 3] = bfi_s(0x7F7F7F7Fu, y, S[i >> 3] << (7 - (i & 7)));
    }
    CBAR();
}

__device__ __forceinline__ int group_of(int r, int sg) {
    if (sg < 3 * FG) {
        const int s = sg / FG;
        return 16 * (2 * (((r >> 1) + 1 + s) & 3) + (r & 1)) + sg - s * FG;
    }
    if (sg < XGREM) {
        const int j = (sg - 3 * FG) / GG;
        return 16 * (2 * j + 1 - (r & 1)) + FG + sg - 3 * FG - j * GG;
    }
    return 16 * r + sg - XGREM;
}

__device__ __forceinline__ int sent_group(int r, int p, int i) {
    const bool same = ((r ^ p) & 1) == 0;
    const int at = same ? FG * (((p >> 1) - (r >> 1) - 1) & 3) : 3 * FG + GG * (p >> 1);
    return i < at ? i : i + (same ? FG : GG);
}

struct Unit {
    u32 slot;
    u32 flag;
    u32 dst;
    u32 kind;
    u32 aux;
};
constexpr int NF = NFWD;
__device__ __forceinline__ int operand_units(bool same) {
    return 2 * (XGREM - (same ? FG : GG)) - NPUSH - NPP + 256 + 32 + (same ? -4 * NF : 3 * NF);
}
__device__ __forceinline__ Unit unit_of(int n, int r, int p) {
    const bool same = ((r ^ p) & 1) == 0;
    const int nx = XGREM - (same ? FG : GG);
    const int U1 = nx - NPUSH - NPP;
    const int U2 = U1 + 16 * WB;
    const int U3 = U2 + nx;
    const int U4 = U3 + 16 * (2 * I8A - WB);
    const int UA = U4 + 32 * (5 - I8A);
    const int UB = UA + (same ? 0 : 4 * NF);
    const int U5 = UB + 96 - 4 * NF;
    const int UC = U5 + 32;
    const int U6 = UC + (same ? 0 : 3 * NF);
    Unit u;
    if (n >= U6) {
        const int v = n - U6, wn = v & 1, h = v >> 1;
        int i8, part, rg0, j;
        if (same) {
            i8 = h / (4 * FG);
            const int t = h - i8 * (4 * FG);
            part = t & 3; j = t >> 2;
            rg0 = FG * (((r >> 1) - (p >> 1) - 1) & 3);
        } else {
            i8 = h / (3 * GG);
            const int t = h - i8 * (3 * GG), jr = r >> 1;
            j = t / 3;
            part = t - 3 * j + (t - 3 * j >= jr);
            rg0 = 3 * FG + (jr > part ? jr - 1 : jr) * GG;
        }
        const int e0 = rg0 & 1, ne = ((same ? FG : GG) - e0 + 1) >> 1;
        const int g = j < ne ? e0 + 2 * j : 1 - e0 + 2 * (j - ne);
        const int rg = rg0 + g, lg = same ? g : FG + g;
        const int id = ((part * 8 + i8) * 48 + rg) * 2 + wn, gct = block_of(p, part) * 8 + i8;
        u.slot = SL_O + id; u.kind = 2;
        u.flag = ((FL_O + ((part * 8 + i8) * 6 + (rg >> 3)) * 2 + (rg & 1)) * 16 + (rg & 6) + wn) | FW_WIDE;
        u.dst = (u32)(lg * 32 * NN + gct * 128 + wn * 64) * 4u;
        u.aux = i8;
        return u;
    }
    if (n < U1 || (n >= U2 && n < U3) || (n >= U5 && n < UC)) {
        int sg, kp;
        if (n < U1) { sg = sent_group(r, p, n + NPUSH + NPP); kp = 0; }
        else if (n < U3) { sg = sent_group(r, p, n - U2); kp = 1; }
        else { sg = XGREM + ((n - U5) & 15); kp = (n - U5) >> 4; }
        const int gg = group_of(r, sg);
        u.slot = SL_X + gg * 2 + kp; u.kind = 0;
        u.flag = kp ? ((FL_X1 + (gg >> 3)) * 16 + (gg & 7)) | FW_WIDE : ((FL_X + gg) * 16) | FW_SPLIT;
        u.dst = (u32)(WS_GX + (size_t)(p * 4 + 2 * kp) * XSLAB) + (u32)sg * 4096u;
        u.aux = (u32)(sg < XGREM ? WS_XREM : WS_XOWN) + (u32)(p * 2 + kp) * 128u;
        return u;
    }
    int i8, kh, part, src = p;
    if (n < U4) {
        const int v = n < U2 ? n - U1 : n - U3 + 16 * WB, o = (W_ORDER >> (v >> 4 << 2)) & 15;
        i8 = o & 3; kh = (o & 4) + ((v >> 2) & 3); part = v & 3;
    }
    else if (n < UA) { const int v = n - U4; i8 = I8A + (v >> 5); kh = (v >> 2) & 7; part = v & 3; }
    else if (n < UB) { const int v = n - UA; i8 = 7; kh = 8 - NF + (v >> 2); part = v & 3; }
    else if (n < U5) { const int v = n - UB; i8 = 5 + (v >> 5); kh = (v >> 2) & 7; part = v & 3; }
    else {
        const int v = n - UC, f = v / 3;
        i8 = 7; kh = 8 - NF + f; part = p >> 1;
        src = 2 * (((r >> 1) + 1 + v - 3 * f) & 3) + (r & 1);
    }
    const int kc = kh >> 1, half = kh & 1, ct = part * 8 + i8, step = src * 4 + kc;
    u.slot = SL_W + ((kc * 128 + block_of(r, part) * 8 + i8) * 2 + half);
    u.flag = i8 >= I8A ? ((FL_L + ct) * 16 + kh) | FW_WIDE : ((FL_W + (kc * 32 + ct) * 2 + half) * 16) | (kc < 2 ? FW_SPLIT : 0u);
    u.kind = 1 | (half << 2);
    if (n >= UA && n < UB && part != (p >> 1)) u.kind |= 8 | ((kh & 3) << 4) | (part << 6);
    if (n >= UC) { u.slot = SL_F + ((r >> 1) * 8 + src) * 4 + (kh & 3); u.flag = ((FL_F + src * 4 + (kh & 3)) * 16) | FW_SPLIT; }
    u.dst = (u32)WS_GW + (u32)(step * 32 + ct) * 16384u + (u32)half * 8192u;
    u.aux = (u32)WS_WCNT + (u32)(ct * 32 + step) * 128u;
    return u;
}

#define STAMP(k) do { } while (0)

__global__ void __launch_bounds__(NTHR, 2) fused_kernel(Params P) {
    __shared__ __attribute__((aligned(1024))) uint8_t lds_[LDS_CTL + 576];
    lbyte* const lds = (lbyte*)lds_;
    const int tid = threadIdx.x;
    const int lane = tid & 63;
    const int wave = __builtin_amdgcn_readfirstlane(tid >> 6);
    const int b = blockIdx.x;
    const int rank = P.rank;
    uint8_t* const myws = P.myws;
    const uint8_t* const xin = P.x;
    const uint8_t* const win = P.w;
    asm volatile("" :: "s"(rank), "s"(myws), "s"(xin), "s"(win));
    u32* const tmo = (u32*)(myws + WS_TMO);

    STAMP(0);
    u32* const cntp = (u32*)(myws + WS_CNT + (size_t)(b * 8 + wave) * 128);
#define LOAD_CALL() \
    const u32 ncalls = __hip_atomic_load(cntp, __ATOMIC_RELAXED, __HIP_MEMORY_SCOPE_AGENT)
#define GET_CALL() \
    const u32 call = __builtin_amdgcn_readfirstlane(ncalls) + 1; \
    const size_t pkoff = WS_PK + (size_t)(call & 1) * PK_BYTES
#define PUT_CALL() \
    if (lane == 0) __hip_atomic_store(cntp, call, __ATOMIC_RELAXED, __HIP_MEMORY_SCOPE_AGENT)

    if (b < NSEND) {
        if (b >= NENG) {
            if (b != NENG || wave != 0) return;
            LOAD_CALL();
            GET_CALL();
            PUT_CALL();
            const int src = (lane >> 1) & 7;
            const u32 need = call * (lane >= 16 ? 32u : src == rank ? 2u * XGREM : 2u * (XGREM - (((src ^ rank) & 1) ? GG : FG)));
            uint8_t* const cnt = myws + WS_XREM + (lane & 31) * 128;
            const u64 t0 = __builtin_amdgcn_s_memrealtime();
            bool open = lane < 32;
            for (u32 it = 1; __any(open); ++it) {
                if (open && __hip_atomic_load((const u32*)cnt, __ATOMIC_RELAXED, __HIP_MEMORY_SCOPE_SYSTEM) >= need) {
                    __hip_atomic_store((u32*)(cnt + X_DONE), call, __ATOMIC_RELAXED, __HIP_MEMORY_SCOPE_SYSTEM);
                    open = false;
                }
                if ((it & 7) == 0 && (__hip_atomic_load(tmo, __ATOMIC_RELAXED, __HIP_MEMORY_SCOPE_AGENT) == call ||
                                      __builtin_amdgcn_s_memrealtime() - t0 > DEADLINE)) {
                    if (lane == 0) __hip_atomic_store(tmo, call, __ATOMIC_RELAXED, __HIP_MEMORY_SCOPE_AGENT);
                    break;
                }
                __builtin_amdgcn_s_sleep(MON_SLEEP);
            }
            return;
        }
        LOAD_CALL();
        const int p = (rank + 1 + b % 7) & 7;
        const bool same = ((rank ^ p) & 1) == 0;
        const int nop = operand_units(same);
        const int nend = nop + (same ? 64 * FG : 48 * GG);
        for (int n = tid; n < nend + NPP; n += NTHR) {
            const Unit u = unit_of(n < nend ? n : n - nend - NPP, rank, p);
            *(lu32x4*)(lds + EL_TAB + n * 16) = u32x4{u.slot | (u.kind << 16), u.flag, u.dst, u.aux};
        }
        if (tid < (EL_END - EL_GO) / 4) {
            const int fo = EL_GO + tid * 4 - EL_FREE;
            *(lu32*)(lds + EL_GO + tid * 4) = fo >= (NS - NPAIR) * 8 && fo < NS * 8 ? (u32)-NS : 0u;
        }
        *(lu32x4*)(lds + EL_PLANE + tid * 16) = u32x4{0u, 0u, 0u, 0u};
        if (tid < 16) op_lut_init(lds + LDS_LUT, tid);
        __syncthreads();
        GET_CALL();
        PUT_CALL();
        auto lds_get = [&](int off) {
            return (u32)__builtin_amdgcn_readfirstlane(
                __hip_atomic_load((lu32*)(lds + off), __ATOMIC_RELAXED, __HIP_MEMORY_SCOPE_WORKGROUP));
        };
        auto lds_put = [&](int off, u32 v) {
            CBAR();
            if (lane == 0) __hip_atomic_store((lu32*)(lds + off), v, __ATOMIC_RELAXED, __HIP_MEMORY_SCOPE_WORKGROUP);
            CBAR();
        };

        if (wave == 0) {
            __builtin_amdgcn_s_setprio(3);
            __builtin_amdgcn_s_waitcnt(0);
            const uint8_t* const pk = P.ws[p] + pkoff;
            const u32 lds_addr = (u32)(uintptr_t)lds;
            const u32 vo0 = lane * 16, vo1 = lane * 16 + 4096;
            static_assert(WIN >= 5 && WIN <= 9 && WIN_OP >= 5 && WIN_OP <= 9, "window");
#define F_WAIT(W, J) "s_waitcnt vmcnt(" #W "-1)\n\t.if (" #W "+5)%%6 == " #J "\n\ts_mov_b64 exec, 1\n\tds_write_b64 %8, %9\n\ts_mov_b64 exec, -1\n\t.endif\n\t"
#define F_LD(V, OFF) "global_load_lds_dwordx4 " V ", %2 offset:" #OFF " sc0 sc1\n\t"
#define F_ARGS :: "v"(vo0), "v"(vo1), "s"(src), "s"(la), "s"(la + 4096), "s"(mk0), "s"(mk1), "n"(0), "v"(lds_addr + EL_ISS), "v"(st) : "memory"
#define F_OP(W) asm volatile("s_mov_b32 m0, %3\n\t" F_WAIT(W, 0) F_LD("%0", 0) F_WAIT(W, 1) F_LD("%0", 1024) F_WAIT(W, 2) \
                "s_mov_b64 exec, %5\n\t" F_LD("%0", 2048) "s_mov_b64 exec, -1\n\ts_mov_b32 m0, %4\n\t" \
                F_WAIT(W, 3) F_LD("%1", 0) F_WAIT(W, 4) F_LD("%1", 1024) F_WAIT(W, 5) \
                "s_mov_b64 exec, %6\n\t" F_LD("%1", 2048) "s_mov_b64 exec, -1" F_ARGS)
#define F_ACC(W) asm volatile("s_mov_b32 m0, %4\n\t" F_WAIT(W, 0) "s_mov_b64 exec, %6\n\t" F_LD("%1", 1024) "s_mov_b64 exec, -1\n\t" \
                F_WAIT(W, 1) F_LD("%1", 0) "s_mov_b32 m0, %3\n\t" F_WAIT(W, 2) F_LD("%0", 0) F_WAIT(W, 3) F_LD("%0", 1024) \
                F_WAIT(W, 4) F_LD("%0", 2048) F_WAIT(W, 5) F_LD("%0", 3072) F_ARGS)
#define F_MORE(V, M0V, OFF, MK) asm volatile("s_mov_b32 m0, %2\n\ts_waitcnt vmcnt(5)\n\ts_mov_b64 exec, %3\n\t" \
                "global_load_lds_dwordx4 %0, %1 offset:" #OFF " sc0 sc1\n\ts_mov_b64 exec, -1" :: "v"(V), "s"(src), "s"(M0V), "s"(MK) : "memory")
#define F_X(M, W) M(W)
            auto lanes = [](u32 lines) { return lines >= 8 ? ~0ull : (1ull << (8 * lines)) - 1; };
            auto get_go = [&](u32 sl) { CBAR(); const u32x4 r = *(volatile lu32x4*)(lds + EL_GO + sl * 16); CBAR(); return r; };
            u32 sl = 0, pub = 0, k = 0;
            bool six = false;
            u32x4 rec = get_go(0);
            for (;; ++k) {
                u32 go = __builtin_amdgcn_readfirstlane(rec[0]);
                for (u32 it = 0; (go >> 16) != k + 1; ++it) {
                    if (it == STARVE) { WAIT_VM(0); lds_put(EL_FULL, pub = k); }
                    __builtin_amdgcn_s_sleep(1);
                    rec = get_go(sl);
                    go = __builtin_amdgcn_readfirstlane(rec[0]);
                }
                if ((go & 0xffffu) == UNIT_END) break;
                const u32 fw = __builtin_amdgcn_readfirstlane(rec[3]);
                const u32 l0 = (fw >> 16) & 255u, l1 = fw >> 24;
                const uint8_t* src = pk + (size_t)(fw & 0xffffu) * SLOT;
                const u32 la = lds_addr + sl * SLOT;
                sl = sl == NS - 1 ? 0 : sl + 1;
                const bool isop = (int)(go & 0xffffu) < nop;
                pub = (isop ? WIN_OP : WIN) < 7 ? k : max(pub + 1, k) - 1;
                const u32x2 st = {k + 1, pub};
                if (isop) {
                    const u64 mk0 = lanes(l0 - 16), mk1 = lanes(l1 - 16);
                    F_X(F_OP, WIN_OP);
                    if (l0 > 24) F_MORE(vo0, la, 3072, lanes(l0 - 24));
                } else {
                    const u64 mk0 = 0, mk1 = lanes(l1 - 8);
                    F_X(F_ACC, WIN);
                    if (l1 > 16) F_MORE(vo1, la + 4096, 2048, lanes(l1 - 16));
                }
                if (l1 > 24) F_MORE(vo1, la + 4096, 3072, lanes(l1 - 24));
                six = !isop && l1 <= 16;
                rec = get_go(sl);
            }
            WAIT_VM(6);
            if (k) lds_put(EL_FULL, max(pub, k - 1));
            WAIT_VM(4);
            if (six) lds_put(EL_HEAD, k);
            WAIT_VM(0);
            lds_put(EL_FULL, pub = k);
            STAMP(4);
            return;
        }
        if (wave == 1) {
            const u64* const flags = (const u64*)(myws + WS_FLAG) + (size_t)p * 4096 * 16;
            u32* const tick = (u32*)(myws + WS_TICK) + p * 32;
            static_assert(BATCH % 2 == 1 && BATCH <= 7, "a bijection needs BATCH odd");
            constexpr int TC = 8 * (BATCH - 1) / BATCH;
            const u32 ntick = (nend + BATCH - 1) / BATCH + TC, tbase = (call - 1) * (ntick - 16) - 24;
            static_assert(4 * BATCH <= 32, "lanes");
            u32 tick_no = b / 7;
            bool dead = false;
            u32 partials = 0;
            auto give_up = [&](u64 t0) {
                if (__hip_atomic_load(tmo, __ATOMIC_RELAXED, __HIP_MEMORY_SCOPE_AGENT) == call ||
                    __builtin_amdgcn_s_memrealtime() - t0 > DEADLINE) {
                    if (lane == 0) __hip_atomic_store(tmo, call, __ATOMIC_RELAXED, __HIP_MEMORY_SCOPE_AGENT);
                    dead = true;
                    lds_put(EL_DEAD, 1);
                    return true;
                }
                __builtin_amdgcn_s_sleep(2);
                return false;
            };
            auto post = [&](u32 kk, u32 sl, u32 unit, u32 h0, u32 h1, u32 slot) {
                const u32 l0 = (int)unit < nop ? max((h0 & 31u) + 1, 17u) : 32u;
                const u32 l1 = (int)unit < nop ? max(((h1 >> 13) & 31u) + 1, 17u) : min(max(h0 & 127u, 41u), 64u) - 32u;
                for (;;) {
                    CBAR();
                    const u32x2 fr = *(volatile lu32x2*)(lds + EL_FREE + sl * 8);
                    if (__builtin_amdgcn_readfirstlane(min(fr[0] + NS, fr[1] + NS)) >= kk + 1) break;
                    __builtin_amdgcn_s_sleep(1);
                }
                CBAR();
                if (lane == 0) *(volatile lu32x4*)(lds + EL_GO + sl * 16) = u32x4{((kk + 1) << 16) | unit, h0, h1, slot | (l0 << 16) | (l1 << 24)};
                CBAR();
            };
            u32 k = 0, sl = 0;
            while (tick_no < ntick) {
                const int u0 = BATCH * ((int)tick_no - TC);
                while (lds_get(EL_ISS) + LOOK < k) __builtin_amdgcn_s_sleep(SC_SLEEP);
                const int nb = tick_no < 8 ? 2 * BATCH : BATCH;
                const int ul = min(max(u0 + 8 * min((lane >> 1) & 15, nb - 1), 0), nend - 1);
                const u32x2 te = *(lu32x2*)(lds + EL_TAB + ul * 16);
                const u32 fw = te[1];
                const u64* const fp = flags + (fw & 0xffffu) + (call & 1) * ((fw & FW_WIDE) ? 8 : 2) + ((fw & FW_SPLIT) ? lane & 1 : 0);
                u32 tk;
                u64 fv;
                asm volatile("s_mov_b64 exec, 1\n\tglobal_atomic_add %0, %2, %3, off sc0\n\ts_mov_b64 exec, %5\n\t"
                             "global_load_dwordx2 %1, %4, off sc0 sc1\n\ts_mov_b64 exec, -1\n\ts_waitcnt vmcnt(0)"
                             : "=&v"(tk), "=&v"(fv) : "v"(tick), "v"(tick_no < 8 ? 0u : 1u), "v"(fp), "s"((1ull << (2 * nb)) - 1) : "memory");
                for (int j = 0; j < nb; ++j) {
                    const int un = u0 + 8 * j;
                    if (un < 0 || un >= nend) continue;
                    if (!dead && ((__ballot((u32)fv >= call) >> (2 * j)) & 3) != 3) {
                        while (lds_get(EL_ISS) + LOOK < k) __builtin_amdgcn_s_sleep(SC_SLEEP);
                        const u64 t0 = __builtin_amdgcn_s_memrealtime();
                        for (u32 it = 1;; ++it) {
                            if (lane >= 2 * j && lane < 2 * nb) fv = __hip_atomic_load(fp, __ATOMIC_RELAXED, __HIP_MEMORY_SCOPE_SYSTEM);
                            if (((__ballot((u32)fv >= call) >> (2 * j)) & 3) == 3 || ((it & 7) == 0 && give_up(t0))) break;
                            __builtin_amdgcn_s_sleep(1);
                        }
                    }
                    if (un >= nop && partials != 255u) {
                        const u32 bit = 1u << lds_get(EL_TAB + un * 16 + 12);
                        if (!(partials & bit)) {
                            while (lds_get(EL_ISS) + LOOK < k) __builtin_amdgcn_s_sleep(SC_SLEEP);
                            const u64 t0 = __builtin_amdgcn_s_memrealtime();
                            while (!dead) {
                                partials = (u32)__ballot(__hip_atomic_load((const u32*)(myws + WS_PCNT) + (lane & 7) * 32, __ATOMIC_RELAXED,
                                                                           __HIP_MEMORY_SCOPE_SYSTEM) >= 12u * call) & 255u;
                                if ((partials & bit) || give_up(t0)) break;
                            }
                        }
                    }
                    const u32 hi = (u32)(fv >> 32);
                    post(k, sl, un, __builtin_amdgcn_readlane(hi, 2 * j), __builtin_amdgcn_readlane(hi, 2 * j + 1),
                         __builtin_amdgcn_readlane(te[0], 2 * j) & 0xffffu);
                    sl = sl == NS - 1 ? 0 : sl + 1;
                    ++k;
                }
                tick_no = tick_no < 8 ? tick_no + 16 : __builtin_amdgcn_readfirstlane(tk) - tbase;
            }
            for (u32 kk = k; kk < k + NPAIR; ++kk) {
                post(kk, sl, UNIT_END, 0, 0, 0);
                sl = sl == NS - 1 ? 0 : sl + 1;
            }
            return;
        }
        lbyte* const plane = lds + EL_PLANE + wave * 1024;
        const float scale0 = P.sx[0] * P.sw[0];
        const u32 hf = wave & 1;
        const __amdgpu_buffer_rsrc_t outrs = __builtin_amdgcn_make_buffer_rsrc(P.out, 0, MS * NN * 4, 0x00020000);
        const u32 olane = (hf * 16 + (lane >> 4) * 4) * (NN * 4) + (lane & 15) * 16;
        u32 owed = 0;
        auto settle = [&]() {
            if (!owed) return;
            WAIT_VM(0);
            if (lane == 0) count_in(myws + owed, 1u);
            owed = 0;
        };
        u32 sl = ((wave - 2) >> 1) + NS - NPAIR;
        for (u32 k = ((wave - 2) >> 1) - NPAIR;; k += NPAIR) {
            lbyte* const s = lds + sl * SLOT;
            const bool pushed = (int)k < 0;
            const u32 fin = pushed ? 0u : k + 1;
            u32 go, hdr;
            if (pushed) {
                const u32 j = b / 7 * NPAIR + k + NPAIR;
                go = nend + j;
                const u64* const fp = (const u64*)(myws + WS_FLAG) + (size_t)(p * 4096 + FL_P + j) * 16 + (call & 1) * 2 + hf;
                const u64 t0 = __builtin_amdgcn_s_memrealtime();
                u64 fv;
                for (u32 it = 1; (u32)(fv = __hip_atomic_load(fp, __ATOMIC_RELAXED, __HIP_MEMORY_SCOPE_SYSTEM)) < call; ++it) {
                    if ((it & 7) == 0 && (__hip_atomic_load(tmo, __ATOMIC_RELAXED, __HIP_MEMORY_SCOPE_AGENT) == call ||
                                          __builtin_amdgcn_s_memrealtime() - t0 > DEADLINE)) {
                        if (lane == 0) __hip_atomic_store(tmo, call, __ATOMIC_RELAXED, __HIP_MEMORY_SCOPE_AGENT);
                        lds_put(EL_DEAD, 1);
                        break;
                    }
                    __builtin_amdgcn_s_sleep(4);
                }
                hdr = __builtin_amdgcn_readfirstlane((u32)(fv >> 32));
                asm volatile("s_mov_b32 m0, %1\n\ts_nop 0\n\tglobal_load_lds_dwordx4 %0, off sc0 sc1\n\t"
                             "global_load_lds_dwordx4 %0, off offset:1024 sc0 sc1\n\tglobal_load_lds_dwordx4 %0, off offset:2048 sc0 sc1\n\t"
                             "global_load_lds_dwordx4 %0, off offset:3072 sc0 sc1\n\ts_waitcnt vmcnt(0)"
                             :: "v"(myws + pkoff + (size_t)(SL_P + p * NPP + j) * SLOT + hf * 4096 + lane * 16),
                                "s"((u32)(uintptr_t)s + hf * 4096) : "memory");
            } else {
                while (((go = lds_get(EL_GO + sl * 16)) >> 16) != k + 1) { settle(); __builtin_amdgcn_s_sleep(1); }
                if ((go & 0xffffu) == UNIT_END) break;
                CBAR();
                hdr = lds_get(EL_GO + sl * 16 + 4 + hf * 4);
            }
            const u32x4 te = *(lu32x4*)(lds + EL_TAB + (go & 0xffffu) * 16);
            const u32 kind = __builtin_amdgcn_readfirstlane(te[0]) >> 16;
            const u32 dst = __builtin_amdgcn_readfirstlane(te[2]);
            const u32 aux = __builtin_amdgcn_readfirstlane(te[3]);
            const bool acc = kind == 2;
            u32x4 pa[4];
            uint8_t* const od = (uint8_t*)P.out + dst + olane;
            if (acc) {
                settle();
#pragma unroll
                for (int j = 0; j < 4; ++j)
                    pa[j] = __builtin_bit_cast(u32x4, __builtin_amdgcn_raw_buffer_load_b128(outrs, olane, dst + j * (NN * 4), 17));
            }
            u32x4 st;
            for (;;) {
                CBAR();
                st = *(volatile lu32x4*)(lds + EL_ISS);
                if (__builtin_amdgcn_readfirstlane(st[1]) >= fin || __builtin_amdgcn_readfirstlane(st[2]) == fin + pushed) break;
                settle();
                __builtin_amdgcn_s_sleep(WK_SLEEP);
            }
            CBAR();
            if (WK_PRIO) __builtin_amdgcn_s_setprio(WK_PRIO);
            if (!acc) {
                u32x4 v[4];
                const lbyte* const blk = s + hf * 4096;
                const u32 bh = hdr >> (13 * hf);
                if (kind & 8) {
                    const u32 fp = (kind >> 6) & 3u, ff = (kind >> 4) & 3u;
                    uint8_t* const fs = myws + pkoff + (size_t)(SL_F + (fp * 8 + p) * 4 + ff) * SLOT + hf * 4096;
                    for (u32 o = lane * 16; o < ((bh & 31u) + 1) * 128; o += 1024) st_sys(fs + o, *(const lu32x4*)(blk + o));
                    WAIT_VM(0);
                    if (lane == 0) st_flag(P.ws[2 * fp + (p & 1)], rank, FL_F + p * 4 + ff, hf, call, hdr);
                }
                if ((bh & 31u) == 31u) {
#pragma unroll
                    for (int j = 0; j < 4; ++j) v[j] = *(const lu32x4*)(blk + j * 1024 + lane * 16);
                    lds_put(EL_FREE + sl * 8 + hf * 4, fin);
                } else {
                    op_unpack(v, ((hdr >> 26) & 31u) * 0x04040404u, min((bh >> 5) & 255u, 128u), blk,
                              lds + EL_BUF + wave * 2304, lds + LDS_LUT, lane,
                              [&] { lds_put(EL_FREE + sl * 8 + hf * 4, fin); });
                }
                const bool isw = kind & 1;
                uint8_t* d = myws + dst + (isw ? hf * 4096 : (call & 1) * GX_ODD + hf * XSLAB) +
                             ((lane ^ (isw ? (kind & 4u) * 2 + hf * 4 : 0)) << 4);
                settle();
#pragma unroll
                for (int j = 0; j < 4; ++j) st_sys(d + j * 1024, v[j]);
                owed = aux;
            } else {
                const float scale = __builtin_amdgcn_readfirstlane(st[3]) ? __builtin_nanf("") : scale0;
                float a[16], c1 = 1.0f;
                if (hdr & H_RAW) {
#pragma unroll
                    for (int j = 0; j < 4; ++j) {
                        const u32x4 t = *(lu32x4*)(s + hf * 4096 + j * 1024 + lane * 16);
#pragma unroll
                        for (int t4 = 0; t4 < 4; ++t4) a[4 * j + t4] = __uint_as_float(t[t4]);
                    }
                    lds_put(EL_FREE + sl * 8 + hf * 4, fin);
                } else {
                    u32x4 mid = *(lu32x4*)(s + 4096 + lane * 16);
                    const u32 nb = (hdr >> 8) & 0x7ffu;
                    u32 at = ~0u;
                    for (u32 i0 = 0; i0 < nb; i0 += 128) {
                        const u32 i = i0 + 2 * lane;
                        const u32 w2 = *(lu16*)(s + ACC_MAIN + i);
                        u32 d[2], val[2];
#pragma unroll
                        for (int h = 0; h < 2; ++h) {
                            const u32 e = i + h < nb ? (w2 >> (8 * h)) & 255u : 0xe0u;
                            d[h] = e < 0xc0u ? (e & 63u) + 1 : e < 0xe0u ? 64u : 0u;
                            val[h] = e < 0x80u ? 3u + (e >> 6) : e >= 0xe0u ? e & 31u : 0u;
                        }
                        const u32 inc = wave_scan_add(d[0] + d[1]);
#pragma unroll
                        for (int h = 0; h < 2; ++h) {
                            const u32 t = at + inc - (h ? 0 : d[1]);
                            if (val[h] && ((t >> 2) & 1u) == hf) *(plane + (plane_addr(t) & 0x3ffu)) = (uint8_t)val[h];
                        }
                        at += __builtin_amdgcn_readlane(inc, 63);
                    }
                    CBAR();
                    const u32x4 q = *(lu32x4*)(plane + lane * 16);
                    *(lu32x4*)(plane + lane * 16) = u32x4{0u, 0u, 0u, 0u};
                    if (hf) {
                        const u32 m4 = *(lu32*)(s + 5120 + lane * 4), m5 = *(lu32*)(s + 5376 + lane * 4), m6 = *(lu32*)(s + 5632 + lane * 4);
                        u32 m7 = (mid[0] << 1) & 0x02020202u;
                        m7 |= (mid[1] << 2) & 0x04040404u; m7 |= (mid[2] << 3) & 0x08080808u; m7 |= (mid[3] << 4) & 0x10101010u;
                        m7 |= (m4 << 5) & 0x20202020u; m7 |= (m5 << 6) & 0x40404040u; m7 |= (m6 << 7) & 0x80808080u;
                        mid = u32x4{m4, m5, m6, m7};
                    }
                    u32 l0[4], l1[4];
#pragma unroll
                    for (int j = 0; j < 4; ++j) {
                        const u32 mi = bfi(0x01010101u, q[j] >> 2, mid[j]);
                        const u32 lo = (q[j] & 0x03030303u) | ((q[j] << 3) & 0xC0C0C0C0u);
                        l0[j] = __builtin_amdgcn_perm(mi, lo, 0x05010400u);
                        l1[j] = __builtin_amdgcn_perm(mi, lo, 0x07030602u);
                        asm volatile("" : "+v"(l0[j]), "+v"(l1[j]));
                    }
                    if (__builtin_amdgcn_readfirstlane(st[1]) < fin) { while (lds_get(EL_FULL) < fin) { } }
                    CBAR();
                    const u32x4 h0 = *(lu32x4*)(s + hf * 2048 + lane * 16), h1 = *(lu32x4*)(s + hf * 2048 + 1024 + lane * 16);
                    lds_put(EL_FREE + sl * 8 + hf * 4, fin);
                    c1 = __uint_as_float(((hdr >> 19) & 255u) << 23);
#pragma unroll
                    for (int j = 0; j < 4; ++j) {
                        const u32 ha = j < 2 ? h0[2 * j] : h1[2 * j - 4], hb = j < 2 ? h0[2 * j + 1] : h1[2 * j - 3];
                        const u32 T[4] = {__builtin_amdgcn_perm(ha, l0[j], 0x05040100u), __builtin_amdgcn_perm(ha, l0[j], 0x07060302u),
                                          __builtin_amdgcn_perm(hb, l1[j], 0x05040100u), __builtin_amdgcn_perm(hb, l1[j], 0x07060302u)};
#pragma unroll
                        for (int t4 = 0; t4 < 4; ++t4) a[4 * j + t4] = (float)(((int)T[t4] >> 2) ^ (int)(T[t4] << 30));
                    }
                }
#pragma unroll
                for (int j = 0; j < 4; ++j) asm volatile("" : "+v"(pa[j]));
#pragma unroll
                for (int j = 0; j < 4; ++j) {
                    f32x4 y;
#pragma unroll
                    for (int t4 = 0; t4 < 4; ++t4) {
                        const u32 pb = pa[j][t4];
                        y[t4] = silu(fmaf(a[4 * j + t4], c1, __uint_as_float(pb)) * scale);
                    }
                    st_mode<NT_OUT>(od + (size_t)j * (NN * 4), __builtin_bit_cast(u32x4, y));
                }
            }
            if (WK_PRIO) __builtin_amdgcn_s_setprio(0);
            sl = sl >= NS - NPAIR ? sl + NPAIR - NS : sl + NPAIR;
        }
        settle();
        return;
    }

    const int g = b - NSEND;
    lbyte* const scr = lds + wave * 6144;
    constexpr int WFIRST = 64 * I8A, G_X1 = 72, GLATE = G_X1 + 16, GLEND = GLATE + 16 * (8 - I8A);
    constexpr int T_P = 112, T_X = T_P + 14 * NPP, T_W = T_X + 256;
    static_assert(I8A == 3 && T_W + 3 * WFIRST <= 176 * 8, "tasks");
    const bool isx1 = g >= G_X1 && g < GLATE;
    const int task0 = isx1 ? T_X : (g < G_X1 ? g : g - 16) + 176 * wave;
    const int pdst = (rank + 1 + task0 % 7) & 7;
    const int pj = task0 / 7 - T_P / 7;
    const int psg = sent_group(pdst, rank, task0 < T_P ? task0 / 7 : NPUSH + (pj >> 1));
    const int gg = isx1 ? (g - G_X1) * 8 + wave : task0 < T_X ? group_of(pdst, psg) : (task0 - T_X) & 127;
    const int xkc = isx1 ? 2 : task0 < T_P ? 0 : task0 < T_X ? pj & 1 : (task0 - T_X) >> 7;
    u32x4 v[8];
    if (task0 < 7 * NPUSH || (task0 >= T_P && task0 < T_W)) {
        const uint8_t* src = xin + (size_t)(gg * 32 + (lane >> 3)) * KS + xkc * 128 + (lane & 7) * 16;
#pragma unroll
        for (int j = 0; j < 4; ++j) v[j] = *(const u32x4*)(src + j * 8 * KS);
        if (task0 < T_P || isx1) {
#pragma unroll
            for (int j = 0; j < 4; ++j) v[4 + j] = *(const u32x4*)(src + j * 8 * KS + 128);
        }
    }
    u32x2 in[16];
    auto load_w = [&](int kc, int blk, int wi8, int half) {
        const uint8_t* src = win + (size_t)(kc * 128 + 16 * (lane >> 3)) * NN + blk * 1024 + wi8 * 128 + 16 * (lane & 7) + 8 * half;
#pragma unroll
        for (int j = 0; j < 16; ++j) in[j] = *(const u32x2*)(src + (size_t)j * NN);
    };
    const bool isw1 = task0 >= T_W && task0 < T_W + 2 * WFIRST;
    if (isw1) {
        const int wt = task0 - T_W;
        load_w((wt >> 6) & 1, (wt & 7) | ((wt >> 1) & 8), wt >> 7, (wt >> 3) & 1);
    }
    LOAD_CALL();
    const int l7 = (lane >> 2) & 7;
    uint8_t* const wsl = P.ws[l7];
    __builtin_amdgcn_sched_barrier(0);
    if (tid < 3) *(lu32*)(lds + LDS_PUB + tid * 64 + 32) = 0;
    const u64 tstart0 = __builtin_amdgcn_s_memrealtime();
    lbyte* const lut = scr + 4096;
    if (lane < 16) op_lut_init(lut, lane);
    CBAR();
    if (task0 < 7 * NPUSH) {
        uint8_t* const wd = P.ws[pdst];
        uint8_t* d = wd + WS_GX + (size_t)(rank * 4) * XSLAB + (size_t)psg * 4096 + lane * 16;
        GET_CALL();
        d += (call & 1) * GX_ODD;
#pragma unroll
        for (int j = 0; j < 8; ++j) asm volatile("" : "+v"(v[j]));
#pragma unroll
        for (int j = 0; j < 8; ++j) {
            THROTTLE(PUSH_WIN);
            st_sys(d + (j >> 2) * XSLAB + (j & 3) * 1024, v[j]);
        }
        WAIT_VM(0);
        if (lane == 0)
            count_in(wd + WS_XREM + rank * 256, 2u);
    }
    bool isx = task0 >= T_P && task0 < T_W, send = true, keep = true, tell = false;
    int h0 = 0, h1 = 2, flag = 0;
    u32 dbo = 0, swz = 0, slo = 0, cno = 0;
    uint8_t* slotws = myws;
    if (task0 >= T_P && task0 < T_X) {
        h0 = xkc; h1 = h0 + 1;
        keep = false;
        tell = l7 == pdst;
        flag = FL_P + (pj >> 1);
        slo = (u32)(SL_P + rank * NPP + (pj >> 1)) * (u32)SLOT;
        slotws = P.ws[pdst];
    } else if (isx) {
        if (X0_PRIO && !isx1) __builtin_amdgcn_s_setprio(X0_PRIO);
        const int kc = xkc, q = gg >> 4, lg = gg & 15;
        const int fl = (lg >= FG) ^ (q & 1);
        if (!isx1) { h0 = kc; h1 = h0 + 1; }
        send = q != rank;
        keep = q == rank || (rank & 1) == fl;
        if (isx1) {
            tell = send && l7 != rank && (lg < 8 || l7 == q || (l7 & 1) == fl);
            flag = FL_X1 + (gg >> 3);
        } else {
            const int sgl = lg < FG ? FG * (((q >> 1) - (l7 >> 1) - 1) & 3) + lg : 3 * FG + GG * (q >> 1) + lg - FG;
            tell = send && l7 != rank && (l7 == q || ((l7 & 1) == fl && sgl >= sent_group(l7, rank, NPUSH + NPP)));
            flag = FL_X + gg;
        }
        const int sg = q == rank ? XGREM + lg
                     : lg < FG ? FG * (((q >> 1) - (rank >> 1) - 1) & 3) + lg
                               : 3 * FG + GG * (q >> 1) + lg - FG;
        dbo = (u32)WS_GX + (u32)(rank * 4 + (kc & 2)) * (u32)XSLAB + (u32)sg * 4096u;
        slo = (u32)(SL_X + gg * 2 + (kc >> 1)) * (u32)SLOT;
        cno = (u32)(q == rank ? WS_XOWN : WS_XREM) + (u32)(rank * 2 + (kc >> 1)) * 128u;
        asm volatile("" : "+s"(dbo), "+s"(slo), "+s"(cno), "+v"(flag));
    }
    for (int phase = task0 >= T_P && task0 < T_W + 3 * WFIRST ? 0 : 1; phase < (g >= GLATE && g < GLEND ? 2 : 1); ++phase) {
        if (phase || isx1) {
            asm volatile("s_waitcnt lgkmcnt(0)" ::: "memory");
            __builtin_amdgcn_s_barrier();
        }
        if (phase) { isx = false; slotws = myws; }
        if (!isx) {
            const bool first = !phase && isw1;
            const int wt = task0 - T_W, wq = first ? wt >> 6 : (wt - 2 * WFIRST) >> 5;
            h0 = first ? (wt >> 5) & 1 : 0; h1 = first ? h0 + 1 : 2;
            const int half = phase ? wave & 1 : (wt >> 3) & 1;
            const int blk = phase ? (g - GLATE) & 15 : (wt & 7) | ((wt >> 1) & 8);
            const int kc = phase ? wave >> 1 : (first ? 0 : 2) + (wq & 1);
            const int wi8 = phase ? I8A + ((g - GLATE) >> 4) : wq >> 1;
            if (!first) {
                while (__builtin_amdgcn_s_memrealtime() - tstart0 < (u64)(phase ? W_DELAY + LATE_STAG * (wi8 - I8A) : W_DELAY0))
                    __builtin_amdgcn_s_sleep(8);
                load_w(kc, blk, wi8, half);
            }
            __builtin_amdgcn_sched_barrier(0);
            auto needs = [&](int r) { return (r >> 1) == ((r & 1) ? blk & 3 : blk >> 2); };
            auto ctile = [&](int r) { return ((r & 1) ? blk >> 2 : blk & 3) * 8 + wi8; };
            send = keep = true;
            tell = l7 != rank && needs(l7);
            flag = phase ? FL_L + ctile(l7) : FL_W + (kc * 32 + ctile(l7)) * 2 + half;
            const bool mine = needs(rank);
            const int fb = (rank & 1) ? 3 * (blk >> 2) + (blk & 3) - ((blk & 3) > (rank >> 1))
                                      : blk - (blk >= 2 * rank ? 4 : 0);
            const int ft = fb * 8 + wi8;
            dbo = half * 8192u + (mine ? (u32)WS_GW + (u32)((rank * 4 + kc) * 32 + ctile(rank)) * 16384u
                                       : (u32)WS_GW2 + (u32)(((ft >> 5) * 4 + kc) * 32 + (ft & 31)) * 16384u);
            swz = half * 8;
            slo = (u32)(SL_W + (kc * 128 + blk * 8 + wi8) * 2 + half) * (u32)SLOT;
            cno = (u32)WS_WCNT + (u32)((mine ? ctile(rank) : 32 + ft) * 32 + rank * 4 + kc) * 128u;
            asm volatile("" : "+s"(dbo), "+s"(slo), "+s"(cno), "+v"(flag));
#pragma unroll
            for (int d = 0; d < 2; ++d)
#pragma unroll
                for (int e = 0; e < 4; ++e) {
                    const u32 t0 = __builtin_amdgcn_perm(in[4 * e + 1][d], in[4 * e][d], 0x05010400u);
                    const u32 t1 = __builtin_amdgcn_perm(in[4 * e + 1][d], in[4 * e][d], 0x07030602u);
                    const u32 t2 = __builtin_amdgcn_perm(in[4 * e + 3][d], in[4 * e + 2][d], 0x05010400u);
                    const u32 t3 = __builtin_amdgcn_perm(in[4 * e + 3][d], in[4 * e + 2][d], 0x07030602u);
                    v[4 * d + 0][e] = __builtin_amdgcn_perm(t2, t0, 0x05040100u);
                    v[4 * d + 1][e] = __builtin_amdgcn_perm(t2, t0, 0x07060302u);
                    v[4 * d + 2][e] = __builtin_amdgcn_perm(t3, t1, 0x05040100u);
                    v[4 * d + 3][e] = __builtin_amdgcn_perm(t3, t1, 0x07060302u);
                }
            if (h0) {
#pragma unroll
                for (int j = 0; j < 4; ++j) v[j] = v[4 + j];
            }
        }
        GET_CALL();
#pragma unroll
        for (int j = 0; j < 8; ++j) asm volatile("" : "+v"(v[j]));
        uint8_t* const db = myws + dbo + (isx ? (call & 1) * GX_ODD : 0);
        uint8_t* const slot = slotws + pkoff + slo;
        u32 es = 0;
#pragma unroll
        for (int j = 0; j < 8; ++j) es = __builtin_amdgcn_sad_u8(v[j >> 1][(j & 1) * 2] & 0x7C7C7C7Cu, 0u, es);
        const u32 base = (u32)min(max(((int)wave_sum(es) - 1475) >> 13, 0), 29);
        u32 hdr = base << 26;
        for (int hh = h0; hh < h1; ++hh) {
            if (send) hdr |= op_pack(v, base * 0x04040404u, slot + hh * 4096, slotws != myws, scr, lut, lane) << (13 * hh);
            if (keep) {
                uint8_t* d = db + (isx ? hh * XSLAB + lane * 16 : hh * 4096 + ((lane ^ (swz + hh * 4)) << 4));
#pragma unroll
                for (int j = 0; j < 4; ++j) st_sys(d + j * 1024, v[j]);
            }
#pragma unroll
            for (int j = 0; j < 4; ++j) v[j] = v[4 + j];
        }
        WAIT_VM(0);
        if (X0_PRIO) __builtin_amdgcn_s_setprio(0);
        u64 wsq = (u64)wsl;
        asm volatile("" : "+v"(wsq));
        uint8_t* const wsr = (uint8_t*)(__attribute__((address_space(1))) uint8_t*)wsq;
        if (phase || isx1) st_flags8(lds + LDS_PUB, wave, hdr, wsr, tell && lane < 32, rank, flag, call, lane);
        else if (tell && lane < 32 && (lane & 3) == 0) st_flag(wsr, rank, flag, h0, call, hdr);
        if (keep && lane == 0)
            count_in(myws + cno, (u32)(h1 - h0));
    }
    GET_CALL();
    PUT_CALL();
    STAMP(1);

    const int xcd = g & 7, idx = g >> 3;
    const int part = xcd >> 1;
    const int wm = wave >> 1, wn = wave & 1;
    const int c = lane & 15, q4 = lane >> 4;
    const int ra0 = wm * 8192 + c * 128 + ((q4 ^ (c >> 1)) << 4);
    const int ra1 = ra0 ^ 64;
    const int rb0 = 32768 + (c & 3) * 4096 + ((((q4 << 3) | (4 * wn + (c >> 2))) ^ ((c & 3) << 2)) << 4);
    const u32 boff = (u32)wave * 2048u + (u32)lane * 16u;
    const u32 la0 = (u32)(uintptr_t)lds + wave * 4096, lb0 = (u32)(uintptr_t)lds + 32768 + wave * 2048;
    const u32 aoff0 = (u32)(lane >> 3) * 128u + (u32)(((lane & 7) ^ (lane >> 4)) << 4), aoff1 = aoff0 ^ 64u;

    for (int job = idx >= 21 ? -1 : idx >= 12 ? 0 : 2; job < (idx < 8 ? 4 : 3); ++job) {
        const bool help = job < 0;
        const bool pre = job < 2 && !help, local = job == 3, ownrows = pre || local;
        const bool halft = help || (job == 2 && idx < 3);
        const int ft = (g - 96) % 12 * 8 + (g - 96) / 12;
        const int i8 = pre ? ft & 7 : local ? idx : help ? 0 : idx / 3;
        const int rt = pre ? job : local ? (xcd & 1) : 3 * (xcd & 1) + idx % 3;
        const int ct = pre ? 32 + ft : part * 8 + i8;
        const int nstep = pre ? 4 : NSTEP;
        const u32 wbase = pre ? (u32)WS_GW2 + (u32)(((ft >> 5) - rank) * 128 + (ft & 31)) * 16384u : (u32)WS_GW + (u32)ct * 16384u;
        const int rgw = rt * 8 + (halft ? 2 * wm + help : wave);
        const u32 laj = halft ? la0 - wn * 2048 : la0;
        const int sgw = ownrows ? XGREM + rgw : (rgw < 3 * FG + part * GG ? rgw : rgw + GG);
        const int skipsrc = ownrows ? -1 : group_of(rank, sgw) >> 4;
        const u32 xbase = (u32)(WS_GX + (call & 1) * GX_ODD) + (u32)sgw * 4096u + (halft ? wn * 2048u : 0u);

        const u32* const pollp = lane < 32 ? (const u32*)(myws + WS_WCNT) + (ct * 32 + lane) * 32
                                           : (const u32*)(myws + X_DONE + (ownrows ? WS_XOWN : WS_XREM)) + (lane & 15) * 32;
        const u32 pneed = lane < 32 ? 4u * call : call;
        u64 ready = job == 1 ? ~0ull : 0;
        const int tq = ((lane & 31) - 4) / 7;
        const u32 vst = (lane & 31) < 4 ? rank * 4 + (lane & 3) : ((rank + (lane & 31) - 3 - 7 * tq) & 7) * 4 + tq;
        auto stepno = [&](int s) { return (int)__builtin_amdgcn_readlane(vst, s); };
        auto ensure = [&](int s) {
            const int st = stepno(s);
            const u64 want = (1ull << st) | (1ull << (32 + (st >> 1)));
            if ((ready & want) == want) return;
            const u64 t0 = __builtin_amdgcn_s_memrealtime();
            const u64 about = (st >> 2) == rank ? (0xfull << (4 * rank)) | (3ull << (32 + 2 * rank))
                                                : (0x11111111ull << (st & 3)) | (0x5555ull << (32 + ((st >> 1) & 1)));
            for (u32 it = 1;; ++it) {
                u32 v = 0;
                if (((about & ~ready) >> lane) & 1) v = __hip_atomic_load(pollp, __ATOMIC_RELAXED, __HIP_MEMORY_SCOPE_SYSTEM);
                ready |= __ballot(v >= pneed);
                if ((ready & want) == want) break;
                if ((it & 7) == 0 && (__hip_atomic_load(tmo, __ATOMIC_RELAXED, __HIP_MEMORY_SCOPE_AGENT) == call ||
                                      __builtin_amdgcn_s_memrealtime() - t0 > DEADLINE)) {
                    if (lane == 0) __hip_atomic_store(tmo, call, __ATOMIC_RELAXED, __HIP_MEMORY_SCOPE_AGENT);
                    WAIT_VM(0);
                    ready = ~0ull;
                    break;
                }
                __builtin_amdgcn_s_sleep(POLL_SLEEP);
            }
        };
        bool polling = wave == 0;
        int age = 0;
        u64 asked = 0;
        auto peek_issue = [&](int s) {
            if (age) { ++age; return; }
            const int st = stepno(min(s + 3 + PEEK_AHEAD, nstep - 1));
            const u64 want = (1ull << st) | (1ull << (32 + (st >> 1)));
            if ((ready & want) == want) return;
            asked = 0xffffffffffffull & ~ready;
            if ((asked >> lane) & 1) {
                u32 keep;
                asm volatile("s_mov_b32 %0, m0\n\ts_mov_b32 m0, %2\n\ts_nop 0\n\tglobal_load_lds_dword %1, off sc0 sc1\n\ts_mov_b32 m0, %0"
                             : "=&s"(keep) : "v"(pollp), "s"((u32)(uintptr_t)(lds + LDS_CTL)) : "memory");
            }
            age = 1;
        };
        auto peek_read = [&]() {
            if (age != 3) return;
            if (halft) WAIT_VM(4); else WAIT_VM(6);
            ready |= __ballot(*(volatile lu32*)(lds + LDS_CTL + lane * 4) >= pneed) & asked;
            age = 0;
        };
        const u32 vxo = (int)(vst >> 2) == skipsrc ? (u32)WS_ZERO : xbase + vst * (u32)XSLAB;
        const u32 vwo = wbase + vst * (32u * 16384u);
        const uint8_t *xa, *wa;
        auto locate = [&](int s) {
            xa = myws + (u32)__builtin_amdgcn_readlane(vxo, s);
            wa = myws + (u32)__builtin_amdgcn_readlane(vwo, s);
        };
#define DMA(M0V, VOFF, SRC, OFF) \
        asm volatile("s_mov_b32 m0, %2\n\ts_nop 0\n\tglobal_load_lds_dwordx4 %0, %1 offset:" #OFF " sc0 sc1" \
                     :: "v"(VOFF), "s"(SRC), "s"(M0V) : "memory")
        auto issue = [&](u32 so) {
            DMA(so + laj, aoff0, xa, 0); DMA(so + laj, aoff1, xa, 1024);
            if (!halft) { DMA(so + laj, aoff0, xa, 2048); DMA(so + laj, aoff1, xa, 3072); }
            DMA(so + lb0, boff, wa, 0); DMA(so + lb0, boff, wa, 1024);
        };

        f32x4 acc[4][4];
#pragma unroll
        for (int m = 0; m < 4; ++m)
#pragma unroll
            for (int t = 0; t < 4; ++t) acc[m][t] = f32x4{0.f, 0.f, 0.f, 0.f};

        auto rd1 = [&](const lbyte* p0, const lbyte* p1) {
            const u32x4 l = *(const lu32x4*)p0, h = *(const lu32x4*)p1;
            return i32x8{(int)l[0], (int)l[1], (int)l[2], (int)l[3], (int)h[0], (int)h[1], (int)h[2], (int)h[3]};
        };
#define MM(A, B, m, t) acc[m][t] = __builtin_amdgcn_mfma_scale_f32_16x16x128_f8f6f4(A[m], B[t], acc[m][t], 1, 1, 0, 0, 0, 0)
#define SB() __builtin_amdgcn_sched_barrier(0)
#define GTRACE() do { } while (0)
#define KNEXT(ISSUE) \
            ++s; \
            ma = sn + laj; mb = sn + lb0; \
            sn = sn == 2 * STAGE ? 0 : sn + STAGE; \
            pa0 = lds + sn + ra0; pa1 = lds + sn + ra1; pb = lds + sn + rb0; \
            if (ISSUE) locate(s + 3); \
            asm volatile("" : "+s"(ma), "+s"(mb), "+s"(xa), "+s"(wa), "+v"(pa0), "+v"(pa1), "+v"(pb))
#define KSYNC(ISSUE, NVM) \
            if (ISSUE && age) asm volatile("s_waitcnt vmcnt(%0) lgkmcnt(0)" :: "i"(NVM + 1) : "memory"); \
            else asm volatile("s_waitcnt vmcnt(%0) lgkmcnt(0)" :: "i"(NVM) : "memory"); \
            __builtin_amdgcn_s_barrier(); \
            asm volatile("" ::: "memory")
#define KSTEP(HALF, XA, XB, YA, YB, ISSUE, AHEAD) do { \
            if (ISSUE && polling) { peek_read(); ensure(s + 3); } \
            if (!HALF) { \
                KSYNC(ISSUE, 6 * AHEAD); \
                SB(); MM(XA, XB, 0, 0); SB(); YA[0] = rd1(pa0, pa1); \
                SB(); MM(XA, XB, 0, 1); SB(); YB[0] = rd1(pb, pb + 512); \
                SB(); MM(XA, XB, 0, 2); SB(); if (ISSUE) DMA(ma, aoff0, xa, 0); \
                SB(); MM(XA, XB, 0, 3); SB(); YB[1] = rd1(pb + 1024, pb + 1536); \
                SB(); MM(XA, XB, 1, 0); SB(); if (ISSUE) DMA(ma, aoff1, xa, 1024); \
                SB(); MM(XA, XB, 1, 1); SB(); YB[2] = rd1(pb + 2048, pb + 2560); \
                SB(); MM(XA, XB, 1, 2); SB(); if (ISSUE) DMA(ma, aoff0, xa, 2048); \
                SB(); MM(XA, XB, 1, 3); SB(); YB[3] = rd1(pb + 3072, pb + 3584); \
                SB(); MM(XA, XB, 2, 0); SB(); if (ISSUE) DMA(ma, aoff1, xa, 3072); \
                SB(); MM(XA, XB, 2, 1); SB(); YA[1] = rd1(pa0 + 2048, pa1 + 2048); \
                SB(); MM(XA, XB, 2, 2); SB(); if (ISSUE) DMA(mb, boff, wa, 0); \
                SB(); MM(XA, XB, 2, 3); SB(); YA[2] = rd1(pa0 + 4096, pa1 + 4096); \
                SB(); MM(XA, XB, 3, 0); SB(); if (ISSUE) DMA(mb, boff, wa, 1024); \
                YA[3] = rd1(pa0 + 6144, pa1 + 6144); \
                SB(); MM(XA, XB, 3, 1); SB(); \
                KNEXT(ISSUE); \
                SB(); MM(XA, XB, 3, 2); MM(XA, XB, 3, 3); SB(); \
            } else { \
                KSYNC(ISSUE, 4 * AHEAD); \
                SB(); MM(XA, XB, 0, 0); SB(); if (ISSUE) DMA(ma, aoff0, xa, 0); YA[0] = rd1(pa0, pa1); \
                SB(); MM(XA, XB, 0, 1); SB(); if (ISSUE) DMA(ma, aoff1, xa, 1024); YB[0] = rd1(pb, pb + 512); \
                SB(); MM(XA, XB, 0, 2); SB(); if (ISSUE) DMA(mb, boff, wa, 0); YB[1] = rd1(pb + 1024, pb + 1536); \
                SB(); MM(XA, XB, 0, 3); SB(); if (ISSUE) DMA(mb, boff, wa, 1024); YB[2] = rd1(pb + 2048, pb + 2560); \
                SB(); MM(XA, XB, 1, 0); SB(); YB[3] = rd1(pb + 3072, pb + 3584); \
                SB(); MM(XA, XB, 1, 1); SB(); YA[1] = rd1(pa0 + 2048, pa1 + 2048); \
                KNEXT(ISSUE); \
                SB(); MM(XA, XB, 1, 2); MM(XA, XB, 1, 3); SB(); \
            } \
            if (ISSUE && polling) { peek_issue(s - 1); polling = age || (~ready << 16); } \
            GTRACE(); \
        } while (0)

        if (job == 0 && wave == 0) {
            while (__builtin_amdgcn_s_memrealtime() - tstart0 < (u64)(PRE_START + (idx - 12) * PRE_STAG)) __builtin_amdgcn_s_sleep(8);
        }
        if (wave == 0) { ensure(0); ensure(1); ensure(2); }
        asm volatile("s_waitcnt lgkmcnt(0)" ::: "memory");
        __builtin_amdgcn_s_barrier();
        asm volatile("" ::: "memory");
        asm volatile("s_nop 4");
        locate(0); issue(0);
        locate(1); issue(STAGE);
        locate(2); issue(2 * STAGE);
        locate(3);
        if (halft) WAIT_VM(8); else WAIT_VM(12);
        __builtin_amdgcn_s_barrier();
        asm volatile("" ::: "memory");
        u32 tmov;
#define KLOOP(HALF) do { \
            i32x8 A0[4], B0[4], A1[4], B1[4]; \
            _Pragma("unroll") for (int t = 0; t < 4; ++t) { \
                if (!HALF || t < 2) A0[t] = rd1(lds + ra0 + t * 2048, lds + ra1 + t * 2048); \
                B0[t] = rd1(lds + rb0 + t * 1024, lds + rb0 + t * 1024 + 512); \
            } \
            int s = 0; \
            u32 sn = STAGE, ma = laj, mb = lb0; \
            const lbyte *pa0 = lds + STAGE + ra0, *pa1 = lds + STAGE + ra1, *pb = lds + STAGE + rb0; \
            while (s < nstep - 4) { \
                KSTEP(HALF, A0, B0, A1, B1, true, 1); \
                KSTEP(HALF, A1, B1, A0, B0, true, 1); \
            } \
            KSTEP(HALF, A0, B0, A1, B1, true, 1); \
            KSTEP(HALF, A1, B1, A0, B0, false, 1); \
            KSTEP(HALF, A0, B0, A1, B1, false, 0); \
              \
            tmov = __hip_atomic_load(tmo, __ATOMIC_RELAXED, __HIP_MEMORY_SCOPE_AGENT); \
            SB(); \
            _Pragma("unroll") for (int t = 0; t < (HALF ? 8 : 16); ++t) MM(A1, B1, t >> 2, t & 3); \
        } while (0)
        if (halft) KLOOP(true); else KLOOP(false);

        int ln = lane;
        asm volatile("" : "+v"(ln));
        if (local) {
            float scale = tmov == call ? __builtin_nanf("") : P.sx[0] * P.sw[0];
            asm volatile("" : "+v"(scale));
            float* dst = P.out + (size_t)(rt * 256 + wm * 64 + 4 * (ln >> 4)) * NN + block_of(rank, part) * 1024 + i8 * 128 + wn * 64 + 4 * (ln & 15);
#pragma unroll
            for (int m = 0; m < 4; ++m)
#pragma unroll
                for (int r = 0; r < 4; ++r) {
                    f32x4 y;
#pragma unroll
                    for (int t = 0; t < 4; ++t) y[t] = silu(acc[m][t][r] * scale);
                    THROTTLE(LOC_WIN);
                    st_mode<NT_OUT>(dst + (size_t)(m * 16 + r) * NN, __builtin_bit_cast(u32x4, y));
                }
            STAMP(7);
        } else if (pre) {
            const int fb = ft >> 3;
            const int blkf = (rank & 1) ? 4 * (fb / 3) + fb % 3 + (fb % 3 >= (rank >> 1)) : fb + (fb >= 2 * rank ? 4 : 0);
            float* d = P.out + (size_t)(rt * 256 + wm * 64 + 4 * (ln >> 4)) * NN + blkf * 1024 + i8 * 128 + wn * 64 + 4 * (ln & 15);
#pragma unroll
            for (int m = 0; m < 4; ++m)
#pragma unroll
                for (int r = 0; r < 4; ++r) {
                    const f32x4 y = {acc[m][0][r], acc[m][1][r], acc[m][2][r], acc[m][3][r]};
                    st_mode<NT_PRE>(d + (size_t)(m * 16 + r) * NN, __builtin_bit_cast(u32x4, y));
                }
            if (job == 1) {
                WAIT_VM(0);
                __builtin_amdgcn_s_barrier();
                if (tid == 0) __hip_atomic_fetch_add((u32*)(myws + WS_PCNT) + i8 * 32, 1u, __ATOMIC_RELAXED, __HIP_MEMORY_SCOPE_SYSTEM);
                STAMP(7);
            }
        } else {
            const int rl = (ln >> 2) & 7, jl = rl >> 1;
            const int lo = ((rl ^ rank) & 1) ? 3 * FG + (jl > part ? jl - 1 : jl) * GG : FG * ((jl - (rank >> 1) - 1) & 3);
            const int nl = ((rl ^ rank) & 1) ? (jl == part ? 0 : GG) : (rl == rank ? 0 : FG);
            const bool owns = ln < 32 && nl && lo < rt * 8 + 8 && lo + nl > rt * 8;
            u64 qws = (u64)P.ws[rl];
            auto publish = [&](int mp, u32 hdr) {
                WAIT_VM(0);
                st_flags8(lds + LDS_PUB + (halft ? 0 : 64 + mp * 64), wave, hdr, (uint8_t*)qws, owns, rank,
                          FL_O + (ct * 6 + rt) * 2 + mp + help, call, ln);
            };
            float amf = 0.f;
#pragma unroll
            for (int i = 0; i < 64; ++i) amf = fmaxf(amf, fabsf(acc[i >> 4][i & 3][(i >> 2) & 3]));
            const u32 am = wave_umax(__float_as_uint(amf));
            const int ex = min(max((int)(am >> 23), 40), 200);
            u32 votes = 0;
#pragma unroll
            for (int i = 0; i < 8; ++i) {
                const u32 I = (u32)(int)(acc[i >> 1][i & 3][i & 1] * __uint_as_float((u32)(280 - ex) << 23));
                votes += (((I + (1u << 25)) & 0xfc000007u) != 0) + ((((I + (1u << 26)) & 0xf800000fu) != 0) << 10) +
                         (((I & 0x1fu) != 0) << 20);
            }
            votes = wave_sum(votes);
            const u32 n0 = votes & 1023u, n1 = (votes >> 10) & 1023u, n2 = votes >> 20;
            const int dsc = n2 <= n1 && n2 <= n0 ? 2 : n1 <= n0 ? 1 : 0;
            const float up = __uint_as_float((u32)(280 - ex - dsc) << 23);
            u32 hdrs[2];
            asm volatile("" : "+v"(qws), "+v"(tmov));
            const bool bad = __builtin_amdgcn_readfirstlane(tmov) == call;
#pragma unroll
            for (int mp = 0; mp < 2; ++mp) {
                if (mp == 1 && halft) break;
#define ACCV(i) acc[2 * mp + ((i) >> 4)][(i) & 3][((i) >> 2) & 3]
                uint8_t* const slot = myws + pkoff + (size_t)(SL_O + (ct * 48 + rt * 8 + wm * 2 + mp + help) * 2 + wn) * SLOT;
                u32 hi[16], mid[8], q[8], nz = 0, miss = 0;
#pragma unroll
                for (int j = 0; j < 8; ++j) {
                    if (mp == 1 && j == 3) { SB(); publish(0, hdrs[0]); SB(); }
                    u32 t[4];
#pragma unroll
                    for (int t4 = 0; t4 < 4; ++t4) {
                        const float f = ACCV(4 * j + t4) * up;
                        const u32 I = (u32)(int)f;
                        miss |= __float_as_uint(__builtin_amdgcn_fractf(f));
                        t[t4] = __builtin_amdgcn_alignbit(I, I ^ (u32)__builtin_amdgcn_sbfe((int)I, 0, 26), 26);
                    }
                    hi[2 * j] = __builtin_amdgcn_perm(t[1], t[0], 0x07060302u);
                    hi[2 * j + 1] = __builtin_amdgcn_perm(t[3], t[2], 0x07060302u);
                    const u32 l0 = __builtin_amdgcn_perm(t[1], t[0], 0x05010400u), l1 = __builtin_amdgcn_perm(t[3], t[2], 0x05010400u);
                    const u32 lo = __builtin_amdgcn_perm(l1, l0, 0x05040100u);
                    mid[j] = __builtin_amdgcn_perm(l1, l0, 0x07060302u);
                    q[j] = bfi(0x07070707u, bfi(0x03030303u, lo, mid[j] << 2), (lo >> 3) & 0x18181818u);
                    const u32 nb = q[j] + 0x1F1F1F1Fu;
                    nz = bfi(0x01010101u << j, j < 5 ? nb >> (5 - j) : nb << (j - 5), nz);
                }
                const u32 cnt = __builtin_popcount(nz);
                const u32 inc = wave_scan_add(cnt);
                const u32 total = __builtin_amdgcn_readlane(inc, 63);
                if (bad || (int)(am >> 23) > ex || __any(miss != 0) || total > LIST_MAX) {
#pragma unroll
                    for (int j = 0; j < 8; ++j) {
                        f32x4 y = {ACCV(4 * j), ACCV(4 * j + 1), ACCV(4 * j + 2), ACCV(4 * j + 3)};
                        if (bad) y[0] = __builtin_nanf("");
                        st_sys(slot + j * 1024 + ln * 16, __builtin_bit_cast(u32x4, y));
                    }
                    hdrs[mp] = 64u | H_RAW;
                } else {
#pragma unroll
                    for (int j = 0; j < 7; ++j) mid[j] = bfi(0x01010101u, mid[7] >> (j + 1), mid[j]);
#pragma unroll
                    for (int j = 0; j < 4; ++j)
                        st_sys(slot + j * 1024 + ln * 16, u32x4{hi[4 * j], hi[4 * j + 1], hi[4 * j + 2], hi[4 * j + 3]});
                    st_sys(slot + 4096 + ln * 16, u32x4{mid[0], mid[1], mid[2], mid[3]});
                    *(lu32*)(scr + ln * 4) = mid[4];
                    *(lu32*)(scr + 256 + ln * 4) = mid[5];
                    *(lu32*)(scr + 512 + ln * 4) = mid[6];
                    const u32 nb = write_list(q, nz, inc - cnt, total, scr + 768, lds + 2 * STAGE + wave * 4096, ln);
                    for (u32 o = ln * 16; o < 768 + nb; o += 1024) st_sys(slot + 5120 + o, *(lu32x4*)(scr + o));
                    CBAR();
                    hdrs[mp] = ((ACC_MAIN + nb + 127) >> 7) | (nb << 8) | ((u32)(ex + dsc - 30) << 19);
                }
            }
            if (halft) publish(0, hdrs[0]); else publish(1, hdrs[1]);
            STAMP(4);
        }
    }
}

}

extern "C" void kernel_launch(int rank, int nranks, ncclComm_t comm,
                              hipStream_t stream,
                              void* const* d_in, const int* in_sizes, int n_in,
                              void* d_out, int out_size,
                              const int* in_dtypes, int out_dtype,
                              void* const* d_out_all,
                              void* const* d_ws_all, size_t ws_bytes) {
    if (nranks != NR || (unsigned)rank >= (unsigned)NR || n_in != 4 || ws_bytes < WS_NEED ||
        in_sizes[0] != MM * KS || in_sizes[1] != KS * NN || out_size != MS * NN) {
        fprintf(stderr, "kernel_launch: unexpected problem shape\n");
        return;
    }
    Params P;
    P.x = (const uint8_t*)d_in[0];
    P.w = (const uint8_t*)d_in[1];
    P.sx = (const float*)d_in[2];
    P.sw = (const float*)d_in[3];
    P.out = (float*)d_out;
    for (int p = 0; p < NR; ++p) P.ws[p] = (uint8_t*)d_ws_all[p];
    P.myws = P.ws[rank];
    P.rank = rank;
    P.pad = 0;
    hipLaunchKernelGGL(fused_kernel, dim3(GRID), dim3(NTHR), 0, stream, P);
    HIP_CHECK(hipGetLastError());
}
```

```cpp
#include <hip/hip_runtime.h>
#include <rccl/rccl.h>
#include <cstdio>
#include <cstdint>

#define HIP_CHECK(expr) do { hipError_t _e = (expr); if (_e != hipSuccess) { \
    fprintf(stderr, "HIP error %s at %s:%d\n", hipGetErrorString(_e), __FILE__, __LINE__); \
    return; } } while (0)

typedef unsigned u32;
typedef unsigned long long u64;
typedef unsigned u32x4 __attribute__((ext_vector_type(4)));
typedef unsigned u32x2 __attribute__((ext_vector_type(2)));
typedef int i32x8 __attribute__((ext_vector_type(8)));
typedef float f32x4 __attribute__((ext_vector_type(4)));
typedef __attribute__((address_space(1))) const void gvoid;
typedef __attribute__((address_space(3))) void lvoid;
typedef __attribute__((address_space(3))) uint8_t lbyte;
typedef __attribute__((address_space(3))) u32 lu32;
typedef __attribute__((address_space(3))) unsigned short lu16;
typedef __attribute__((address_space(3))) u32x4 lu32x4;
typedef __attribute__((address_space(3))) u32x2 lu32x2;

namespace {

constexpr int NR = 8;
constexpr int MM = 4096, KK = 4096, NN = 16384;
constexpr int KS = KK / NR;
constexpr int MS = MM / NR;
constexpr int FG = 7, GG = 9;
constexpr int F = 32 * FG, G = 32 * GG;
static_assert(F + G == MS, "rows");
constexpr int XGREM = 3 * FG + 4 * GG;
constexpr int XREM = 32 * XGREM;
constexpr int XROWS = XREM + MS;
constexpr int NSTEP = 32;

constexpr int GRID = 256, NSEND = 64, NTHR = 512;
constexpr int NENG = 56;
#define WIN 6
#define WIN_OP 7
#define STARVE 8
#define BATCH 5
#define FCAP 8
#define MARK_AT 30
#define PLAN_GAIN 8
#define SAME_MORE 1500
#define PP_PACE 1
#define W_ORDER 0x652410
#define WB 2
#define MON_SLEEP 4
#define X0_PRIO 2
#define SC_SLEEP 4
#define WK_SLEEP 1
#define WK_PRIO 2
#define LOOK 4
#define NPUSH 5
#define PUSH_WIN 8
#define W_DELAY 500
#define PRE_STAG 140
#define PRE_START 1700
#define LATE_STAG 120
#define LOC_WIN 32
#define PEEK_AHEAD 6
#define W_DELAY0 200
#define NT_PRE 0
#define NT_OUT 0
#define POLL_SLEEP 16
#define I8A 3

constexpr int SLOT = 8192;
constexpr int NPP = 24;
constexpr u32 H_RAW = 1u << 7;
constexpr u32 ACC_MAIN = 5888;
constexpr int SL_X = 0, SL_W = 256, SL_O = SL_W + 1024, SL_F = SL_O + 32 * 48 * 2, SL_P = SL_F + 256, SL_END = SL_P + NR * NPP;
constexpr int FL_X = 0, FL_X1 = 128, FL_W = FL_X1 + 16, FL_L = FL_W + 256, FL_O = FL_L + 32, FL_F = FL_O + 32 * 6 * 2,
              FL_P = FL_F + 64, FL_END = FL_P + NPP;
constexpr u32 FW_SPLIT = 1u << 16, FW_WIDE = 1u << 17;
constexpr u32 FW_RELAYED = 1u << 19;
static_assert(FL_END <= 4096, "flags");

constexpr size_t WS_CNT = 262144;
constexpr size_t WS_ZERO = 0;
constexpr size_t WS_TMO = 32768;
constexpr size_t WS_TICK = 49152;
constexpr size_t WS_XREM = 65536;
constexpr size_t WS_XOWN = WS_XREM + 2048;
constexpr size_t WS_MARK = 53248;
constexpr size_t WS_PLAN = 57344;
constexpr size_t X_DONE = 8192;
constexpr size_t WS_PCNT = WS_XOWN + 2048;
constexpr size_t WS_WCNT = 27u << 20;
constexpr size_t WS_FLAG = 28u << 20;
constexpr size_t WS_DBG = 524288;
constexpr size_t WS_GW = 1u << 20;
constexpr size_t WS_GX = WS_GW + (size_t)NSTEP * 32 * 16384;
constexpr size_t XSLAB = (size_t)XROWS * 128;
constexpr size_t WS_PK = 32u << 20;
constexpr size_t PK_BYTES = 40u << 20;
static_assert(WS_GX + NSTEP * XSLAB <= WS_WCNT && WS_WCNT + 128 * 32 * 128 <= WS_FLAG && WS_FLAG + NR * 4096 * 128 <= WS_PK, "staging");
static_assert((size_t)SL_END * SLOT <= PK_BYTES, "packed");
constexpr size_t GX_ODD = WS_PK + 2 * PK_BYTES - WS_GX;
constexpr size_t WS_GW2 = 124u << 20;
static_assert(WS_GX + GX_ODD + NSTEP * XSLAB <= WS_GW2, "staging");
constexpr size_t WS_NEED = WS_GW2 + (size_t)3 * 128 * 16384;

constexpr int STAGE = 49152;
constexpr int LDS_CTL = 3 * STAGE;
constexpr int LDS_LUT = LDS_CTL + 256;
constexpr int LDS_PUB = LDS_LUT + 128;
constexpr int NS = 12;
constexpr int NPAIR = 3;
constexpr int EL_PLANE = NS * SLOT;
constexpr int EL_BUF = EL_PLANE + 8 * 1024;
constexpr int EL_TAB = EL_BUF + 8 * 2304;
constexpr int EL_GO = EL_TAB + 1024 * 16;
constexpr int EL_FREE = EL_GO + 256;
constexpr int EL_ISS = EL_FREE + 128;
constexpr int EL_FULL = EL_ISS + 4;
constexpr int EL_HEAD = EL_ISS + 8;
constexpr int EL_DEAD = EL_ISS + 12;
constexpr int EL_LIVE = EL_ISS + 64;
constexpr int EL_END = EL_LIVE + 320;
static_assert(EL_END + 1024 <= LDS_CTL && NS % NPAIR == 0 && NS % 2 == 0, "engine LDS");
constexpr u32 UNIT_END = 0xffffu;
constexpr u64 DEADLINE = 40000000ull;

static_assert(7 * NPUSH <= 112 && NPUSH + NPP <= XGREM - GG, "push");

struct Params {
    const uint8_t* x;
    uint8_t* myws;
    int rank;
    int pad;
    const uint8_t* w;
    const float* sx;
    const float* sw;
    float* out;
    uint8_t* ws[NR];
};

__device__ __forceinline__ void st_sys(void* p, u32x4 v) {
    asm volatile("global_store_dwordx4 %0, %1, off sc0 sc1\n\ts_nop 1" :: "v"(p), "v"(v) : "memory");
}
template <int M> __device__ __forceinline__ void st_mode(void* p, u32x4 v) {
    if (M == 0) asm volatile("global_store_dwordx4 %0, %1, off sc0 sc1\n\ts_nop 1" :: "v"(p), "v"(v) : "memory");
    if (M == 1) asm volatile("global_store_dwordx4 %0, %1, off sc0 sc1 nt\n\ts_nop 1" :: "v"(p), "v"(v) : "memory");
    if (M == 2) asm volatile("global_store_dwordx4 %0, %1, off nt\n\ts_nop 1" :: "v"(p), "v"(v) : "memory");
    if (M == 3) asm volatile("global_store_dwordx4 %0, %1, off sc1\n\ts_nop 1" :: "v"(p), "v"(v) : "memory");
}
#define WAIT_VM(n) asm volatile("s_waitcnt vmcnt(" #n ")" ::: "memory")
#define THROTTLE(n) asm volatile("s_waitcnt vmcnt(%0)" :: "i"((n) - 1) : "memory")
#define CBAR() asm volatile("" ::: "memory")
#define DPP(v, ctl, rm) (u32)__builtin_amdgcn_update_dpp(0, (int)(v), ctl, rm, 0xf, false)

__device__ __forceinline__ u32 wave_scan_add(u32 v) {
    v += DPP(v, 0x111, 0xf); v += DPP(v, 0x112, 0xf); v += DPP(v, 0x114, 0xf); v += DPP(v, 0x118, 0xf);
    v += DPP(v, 0x142, 0xa); v += DPP(v, 0x143, 0xc);
    return v;
}
__device__ __forceinline__ u32 wave_sum(u32 v) { return __builtin_amdgcn_readlane(wave_scan_add(v), 63); }
__device__ __forceinline__ u32 wave_umax(u32 v) {
    v = max(v, DPP(v, 0x111, 0xf)); v = max(v, DPP(v, 0x112, 0xf)); v = max(v, DPP(v, 0x114, 0xf));
    v = max(v, DPP(v, 0x118, 0xf)); v = max(v, DPP(v, 0x142, 0xa)); v = max(v, DPP(v, 0x143, 0xc));
    return __builtin_amdgcn_readlane(v, 63);
}
__device__ __forceinline__ int block_of(int r, int part) {
    return (r & 1) ? 4 * part + (r >> 1) : 4 * (r >> 1) + part;
}
__device__ __forceinline__ float silu(float v) {
    const float z = __builtin_amdgcn_fmed3f(v, -60.f, 60.f);
    const float e = __builtin_amdgcn_exp2f(z * -1.4426950408889634f);
    return v * __builtin_amdgcn_rcpf(1.0f + e);
}

constexpr u32 LIST_MAX = 960;
__device__ __forceinline__ u32 plane_addr(u32 t) { return ((t & 4u) << 8) | ((t >> 5) << 4) | ((t & 3u) << 2) | ((t >> 3) & 3u); }
__device__ __forceinline__ u32 write_list(const u32 (&q)[8], u32 nz, u32 pos, u32 total,
                                          lbyte* lst, lbyte* tmp, int lane) {
    *(lu32x4*)(tmp + lane * 16) = u32x4{q[0], q[1], q[2], q[3]};
    *(lu32x4*)(tmp + 1024 + lane * 16) = u32x4{q[4], q[5], q[6], q[7]};
    for (u32 o = lane * 16; o < 2 * total + 32; o += 1024) *(lu32x4*)(lst + o) = u32x4{0xc0c0c0c0u, 0xc0c0c0c0u, 0xc0c0c0c0u, 0xc0c0c0c0u};
    lbyte* const pl = tmp + 2048;
    if (lane == 0) *(lu16*)(pl + 2) = 0xffffu;
    lbyte* wp = pl + 4 + 2 * pos;
    for (u32 m = nz; m; m &= m - 1) {
        *(lu16*)wp = (unsigned short)(((u32)lane << 5) | __builtin_ctz(m));
        wp += 2;
    }
    CBAR();
    u32 nb = 0;
    for (u32 i0 = 0; i0 < total; i0 += 128) {
        const u32 i = i0 + 2 * lane;
        const u32 pa = *(lu32*)(pl + 2 * i), pb = *(lu32*)(pl + 2 * i + 4);
        const u32 t[3] = {pa >> 16, pb & 0xffffu, pb >> 16};
        u32 n[2], e[2], val[2];
#pragma unroll
        for (int h = 0; h < 2; ++h) {
            val[h] = *(tmp + plane_addr(t[h + 1] & 2047u));
            const u32 gap = (t[h + 1] - t[h] - 1) & 0xffffu, ty = min(val[h] - 3u, 2u);
            e[h] = (ty << 6) | (gap & 63u);
            n[h] = i + h < total ? (gap >> 6) + 1 + (ty >> 1) : 0;
        }
        const u32 inc = wave_scan_add(n[0] + n[1]);
        lbyte* const end = lst + nb + inc;
#pragma unroll
        for (int h = 0; h < 2; ++h) {
            lbyte* const le = end - (h ? 0 : n[1]) - 1;
            if (n[h]) {
                *(le - (e[h] >> 7)) = (uint8_t)e[h];
                if (e[h] >> 7) *le = (uint8_t)(0xe0u | val[h]);
            }
        }
        nb += __builtin_amdgcn_readlane(inc, 63);
    }
    CBAR();
    return nb;
}
__device__ __forceinline__ void st_flags8(lbyte* pub, int word, u32 hdr, uint8_t* ws, bool to, int src, int line,
                                          u32 call, int lane) {
    u32 n = 0;
    if (lane == 0) {
        *(lu32*)(pub + word * 4) = hdr;
        CBAR();
        n = __hip_atomic_fetch_add((lu32*)(pub + 32), 1u, __ATOMIC_RELAXED, __HIP_MEMORY_SCOPE_WORKGROUP);
    }
    if (__builtin_amdgcn_readfirstlane(n) != 7) return;
    CBAR();
    const u32x2 h = *(lu32x2*)(pub + (lane & 3) * 8);
    if (to) st_sys(ws + WS_FLAG + (size_t)(src * 4096 + line) * 128 + (call & 1) * 64 + (lane & 3) * 16, u32x4{call, h[0], call, h[1]});
}
__device__ __forceinline__ void st_flag(uint8_t* ws, int src, int flag, int half, u32 call, u32 hdr) {
    __hip_atomic_store((u64*)(ws + WS_FLAG) + (size_t)(src * 4096 + flag) * 16 + (call & 1) * 2 + half,
                       ((u64)hdr << 32) | call, __ATOMIC_RELAXED, __HIP_MEMORY_SCOPE_SYSTEM);
}
__device__ __forceinline__ void count_in(uint8_t* cnt, u32 n) {
    __hip_atomic_fetch_add((u32*)cnt, n, __ATOMIC_RELAXED, __HIP_MEMORY_SCOPE_SYSTEM);
}
__device__ __forceinline__ u32 bfi(u32 m, u32 a, u32 b) { return (a & m) | (b & ~m); }

__device__ __forceinline__ void op_lut_init(lbyte* lut, int m) {
    u32 e = 0, c = 0x0c0c0c0cu, n = 0;
#pragma unroll
    for (u32 i = 0; i < 4; ++i) {
        const bool on = (m >> i) & 1;
        e |= (on ? n : 0x0cu) << (8 * i);
        if (on) c = (c & ~(0xffu << (8 * n))) | (i << (8 * n));
        n += on;
    }
    *(lu32*)(lut + m * 4) = e;
    *(lu32*)(lut + 64 + m * 4) = c;
}
__device__ __forceinline__ u32 and_or(u32 a, u32 m, u32 b) {
    u32 r; asm("v_and_or_b32 %0, %1, %2, %3" : "=v"(r) : "v"(a), "s"(m), "v"(b)); return r;
}
__device__ __forceinline__ u32 bfi_s(u32 m, u32 a, u32 b) {
    u32 r; asm("v_bfi_b32 %0, %1, %2, %3" : "=v"(r) : "s"(m), "v"(a), "v"(b)); return r;
}
__device__ __forceinline__ u32 ff_count(u32 ff, u32 acc) { return (u32)__builtin_amdgcn_sdot4((int)ff, -1, (int)acc, true); }
__device__ __forceinline__ u32 ff_lut(u32 ff, u32 lut) {
    return *(const lu32*)(uintptr_t)(u32)__builtin_amdgcn_sdot4((int)ff, (int)0xe0f0f8fcu, (int)lut, true);
}
__device__ __forceinline__ u32 ge13(u32 x) { return __builtin_amdgcn_perm(0u, 0u, x); }
__device__ __forceinline__ void lds_or(u32 p, u32 v) {
    __hip_atomic_fetch_or((lu32*)(uintptr_t)p, v, __ATOMIC_RELAXED, __HIP_MEMORY_SCOPE_WAVEFRONT);
}
__device__ __forceinline__ u32 chunk_starts(const u32 (&c)[4], u32 (&p)[4], u32 at) {
    const u32 a = c[0] | (c[1] << 16), b = c[2] | (c[3] << 16);
    const u32 ia = wave_scan_add(a), ib = wave_scan_add(b);
    const u32 ta = __builtin_amdgcn_readlane(ia, 63), tb = __builtin_amdgcn_readlane(ib, 63);
    const u32 t1 = ta & 0xffffu, t2 = t1 + (ta >> 16), t3 = t2 + (tb & 0xffffu);
    p[0] = ((ia - a) & 0xffffu) + at;
    p[1] = ((ia - a) >> 16) + (at + t1);
    p[2] = ((ib - b) & 0xffffu) + (at + t2);
    p[3] = ((ib - b) >> 16) + (at + t3);
    return t3 + (tb >> 16);
}
__device__ __forceinline__ u32 op_pack(const u32x4* v, u32 base4, uint8_t* blk, bool paced, lbyte* scr, const lbyte* lutp, int lane) {
    lbyte* const side = scr + 2560;
    const u32 lut = (u32)(uintptr_t)lutp + 64, eb = (u32)(uintptr_t)scr + 16;
    const u32x4 z4 = {0u, 0u, 0u, 0u};
    *(lu32x4*)(scr + lane * 16) = z4;
    *(lu32x4*)(scr + 1024 + lane * 16) = z4;
    *(lu32x4*)(scr + 2048 + lane * 16) = z4;
    *(lu32x4*)(scr + 3072 + lane * 16) = z4;
    u32 ub[16], ef[16], sel[16], N[8], S[2] = {0u, 0u}, cnt[4] = {0u, 0u, 0u, 0u}, pos[4];
#pragma unroll
    for (int i = 0; i < 16; ++i) {
        const u32 d = v[i >> 2][i & 3];
        ub[i] = (d | 0x80808080u) - base4;
        ef[i] = ge13(and_or(ub[i], 0x7C7C7C7Cu, 0x01010101u));
        sel[i] = ff_lut(ef[i], lut);
        const u32 x = and_or(ef[i], 0x0C0C0C0Cu, ub[i]);
        N[i >> 1] = (i & 1) ? bfi_s(0x0F0F0F0Fu, N[i >> 1], x << 4) : x;
        S[i >> 3] = and_or(d >> (7 - (i & 7)), 0x01010101u << (i & 7), S[i >> 3]);
        cnt[i >> 2] = ff_count(ef[i], cnt[i >> 2]);
    }
    const u32 n2 = chunk_starts(cnt, pos, eb - 1), n2w = (n2 + 15) >> 4;
    bool raw = n2 > 2048;
    u32 n3 = 0;
    if (!raw) {
        st_sys(blk + lane * 16, u32x4{N[0], N[1], N[2], N[3]});
        if (!paced) st_sys(blk + 1024 + lane * 16, u32x4{N[4], N[5], N[6], N[7]});
        CBAR();
        *(lu32x2*)(side + lane * 8) = u32x2{S[0], S[1]};
#pragma unroll
        for (int i = 0; i < 16; ++i) {
            const u32 c = __builtin_amdgcn_perm(0u, ub[i], sel[i]), p = pos[i >> 2];
            lu32* const q = (lu32*)(uintptr_t)(p & ~3u);
            __hip_atomic_fetch_or(q, __builtin_amdgcn_alignbyte(c, 0u, ~p), __ATOMIC_RELAXED, __HIP_MEMORY_SCOPE_WAVEFRONT);
            __hip_atomic_fetch_or(q + 1, __builtin_amdgcn_alignbyte(0u, c, ~p), __ATOMIC_RELAXED, __HIP_MEMORY_SCOPE_WAVEFRONT);
            pos[i >> 2] = ff_count(ef[i], p);
        }
        CBAR();
        const u32 l3 = (u32)(uintptr_t)side + 512 + n2w * 4;
        u32 bad = 0;
        for (u32 w0 = 0; w0 < n2w; w0 += 64) {
            const u32x4 B = *(lu32x4*)(scr + 16 + (w0 + lane) * 16);
            u32 W = 0, c3 = 0;
            u64 str = 0;
#pragma unroll
            for (int k = 0; k < 4; ++k) {
                const u32 u = (((B[k] & 0x7C7C7C7Cu) + 0x08080808u) >> 2) & 0x1F1F1F1Fu;
                const u32 code = __builtin_amdgcn_perm(0x83038200u, 0x80008100u, u) & 0x03030303u;
                W |= __builtin_amdgcn_udot4(code, 0x40100401u, 0u, false) << (8 * k);
                bad |= (u + 0x78787878u) & ~(u + 0x6E6E6E6Eu);
                const u32 e = __builtin_amdgcn_perm(0u, 0xff000000u, code);
                const u32 sym = u & (and_or(ge13(u), 0x0E0E0E0Eu, 0x01010101u));
                const u32 c = __builtin_amdgcn_perm(0u, sym, ff_lut(e, lut));
                str |= (u64)__builtin_amdgcn_perm(0u, c | (c >> 4), 0x0c0c0200u) << c3;
                c3 = (u32)__builtin_amdgcn_sdot4((int)e, (int)0xfcfcfcfcu, (int)c3, true);
            }
            const u32 i3 = wave_scan_add(c3);
            const u32 o3 = n3 + i3 - c3;
            n3 += __builtin_amdgcn_readlane(i3, 63);
            if (n2w * 4 + (n3 >> 3) > 1000) { raw = true; break; }
            if (w0 + lane < n2w) *(lu32*)(side + 512 + (w0 + lane) * 4) = W;
            const u64 lo = (u64)(u32)str << (o3 & 31u), hi = (str >> 32) << (o3 & 31u);
            const u32 p = l3 + ((o3 >> 5) << 2);
            if (c3) lds_or(p, (u32)lo);
            if ((o3 & 31u) + c3 > 32) {
                lds_or(p + 4, (u32)(lo >> 32) | (u32)hi);
                lds_or(p + 8, (u32)(hi >> 32));
            }
        }
        raw = raw || __any(bad & 0x80808080u);
        CBAR();
    }
    if (raw) {
        WAIT_VM(0);
#pragma unroll
        for (int j = 0; j < 4; ++j) st_sys(blk + j * 1024 + lane * 16, v[j]);
        return 31u;
    }
    if (paced) {
        WAIT_VM(0);
        st_sys(blk + 1024 + lane * 16, u32x4{N[4], N[5], N[6], N[7]});
        if (PP_PACE) WAIT_VM(0);
    }
    const u32 nb = 512 + n2w * 4 + ((n3 + 7) >> 3);
    for (u32 o = lane * 16; o < nb; o += 1024) st_sys(blk + 2048 + o, *(lu32x4*)(side + o));
    CBAR();
    return ((2048 + nb - 1) >> 7) | (n2w << 5);
}
template <class Done>
__device__ __forceinline__ void op_unpack(u32x4* v, u32 base4, u32 n2w, const lbyte* blk, lbyte* buf, const lbyte* lutp,
                                          int lane, Done done) {
    const u32 lut = (u32)(uintptr_t)lutp;
    const u32x4 N0 = *(const lu32x4*)(blk + lane * 16), N1 = *(const lu32x4*)(blk + 1024 + lane * 16);
    const u32x2 S = *(const lu32x2*)(blk + 2048 + lane * 8);
    const lbyte* const l3 = blk + 2560 + n2w * 4;
    u32 n3 = 0;
    for (u32 w0 = 0; w0 < n2w; w0 += 64) {
        u32 W = *(const lu32*)(blk + 2560 + (w0 + lane) * 4);
        if (w0 + lane >= n2w) W = 0;
        const u32 c3t = __builtin_popcount(W & (W >> 1) & 0x55555555u);
        const u32 i3 = wave_scan_add(c3t);
        const u32 o3 = n3 + i3 - c3t;
        n3 += __builtin_amdgcn_readlane(i3, 63);
        const lbyte* const p = l3 + ((o3 >> 3) << 2);
        const u32 d0 = *(const lu32*)p, d1 = *(const lu32*)(p + 4), d2 = *(const lu32*)(p + 8);
        const u64 str = ((u64)__builtin_amdgcn_alignbit(d2, d1, o3 << 2) << 32) | __builtin_amdgcn_alignbit(d1, d0, o3 << 2);
        u32x4 B;
        u32 c3 = 0;
#pragma unroll
        for (int k = 0; k < 4; ++k) {
            const u32 f = (W >> (8 * k)) & 255u;
            const u32 t = (f | (f << 12)) & 0x000F000Fu;
            const u32 code = (t | (t << 6)) & 0x03030303u;
            const u32 e = __builtin_amdgcn_perm(0u, 0xff000000u, code);
            const u32 x = __builtin_amdgcn_perm(0u, (u32)(str >> c3), 0x01010000u);
            const u32 s4 = bfi_s(0x00FF00FFu, x << 2, x >> 2) & 0x3C3C3C3Cu;
            const u32 u4 = s4 + and_or(ge13(s4 + 0x05050505u), 0x28282828u, 0x04040404u);
            B[k] = __builtin_amdgcn_perm(0u, u4, ff_lut(e, lut)) | __builtin_amdgcn_perm(0u, 0x0000706Cu, code);
            c3 = (u32)__builtin_amdgcn_sdot4((int)e, (int)0xfcfcfcfcu, (int)c3, true);
        }
        *(lu32x4*)(buf + (w0 + lane) * 16) = B;
    }
    done();
    u32 nib[16], ef[16], sel[16], cnt[4] = {0u, 0u, 0u, 0u}, pos[4];
#pragma unroll
    for (int i = 0; i < 16; ++i) {
        const u32 n = i < 8 ? N0[i >> 1] : N1[(i >> 1) - 4];
        nib[i] = ((i & 1) ? n >> 4 : n) & 0x0F0F0F0Fu;
        ef[i] = ge13(nib[i] | 0x01010101u);
        sel[i] = ff_lut(ef[i], lut);
        cnt[i >> 2] = ff_count(ef[i], cnt[i >> 2]);
    }
    chunk_starts(cnt, pos, (u32)(uintptr_t)buf);
    CBAR();
    u32 ps[16], q0[16], q1[16];
#pragma unroll
    for (int i = 0; i < 16; ++i) {
        const int j = (i & 3) * 4 + (i >> 2);
        ps[j] = (j & 3) ? ff_count(ef[j - 1], ps[j - 1]) : pos[j >> 2];
        const lu32* const q = (const lu32*)(uintptr_t)(ps[j] & ~3u);
        q0[j] = q[0]; q1[j] = q[1];
    }
    __builtin_amdgcn_sched_barrier(0);
#pragma unroll
    for (int i = 0; i < 16; ++i) {
        const u32 y = nib[i] + base4 + __builtin_amdgcn_perm(0u, __builtin_amdgcn_alignbyte(q1[i], q0[i], ps[i]), sel[i]);
        v[i >> 2][i & 3] = bfi_s(0x7F7F7F7Fu, y, S[i >> 3] << (7 - (i & 7)));
    }
    CBAR();
}

__device__ __forceinline__ int group_of(int r, int sg) {
    if (sg < 3 * FG) {
        const int s = sg / FG;
        return 16 * (2 * (((r >> 1) + 1 + s) & 3) + (r & 1)) + sg - s * FG;
    }
    if (sg < XGREM) {
        const int j = (sg - 3 * FG) / GG;
        return 16 * (2 * j + 1 - (r & 1)) + FG + sg - 3 * FG - j * GG;
    }
    return 16 * r + sg - XGREM;
}

__device__ __forceinline__ int sent_group(int r, int p, int i) {
    const bool same = ((r ^ p) & 1) == 0;
    const int at = same ? FG * (((p >> 1) - (r >> 1) - 1) & 3) : 3 * FG + GG * (p >> 1);
    return i < at ? i : i + (same ? FG : GG);
}

struct Unit {
    u32 slot;
    u32 flag;
    u32 dst;
    u32 kind;
    u32 aux;
};
__device__ __forceinline__ int operand_units(bool same) {
    return 2 * (XGREM - (same ? FG : GG)) - NPUSH - NPP + 256 + 32 + (same ? 0 : 48);
}
__device__ __forceinline__ Unit unit_of(int n, int r, int p) {
    const bool same = ((r ^ p) & 1) == 0;
    const int nx = XGREM - (same ? FG : GG);
    const int U1 = nx - NPUSH - NPP;
    const int U2 = U1 + 16 * WB;
    const int U3 = U2 + nx;
    const int U4 = U3 + 16 * (2 * I8A - WB);
    const int UA = U4 + 32 * (5 - I8A);
    const int UB = UA + 32;
    const int U5 = UB + 32;
    const int UC = U5 + 32;
    const int UD = UC + 32;
    const int U6 = UD + (same ? 0 : 48);
    Unit u;
    if (n >= U6) {
        const int v = n - U6, wn = v & 1, h = v >> 1;
        int i8, part, rg0, j;
        if (same) {
            i8 = h / (4 * FG);
            const int t = h - i8 * (4 * FG);
            part = t & 3; j = t >> 2;
            rg0 = FG * (((r >> 1) - (p >> 1) - 1) & 3);
        } else {
            i8 = h / (3 * GG);
            const int t = h - i8 * (3 * GG), jr = r >> 1;
            j = t / 3;
            part = t - 3 * j + (t - 3 * j >= jr);
            rg0 = 3 * FG + (jr > part ? jr - 1 : jr) * GG;
        }
        const int e0 = rg0 & 1, ne = ((same ? FG : GG) - e0 + 1) >> 1;
        const int g = j < ne ? e0 + 2 * j : 1 - e0 + 2 * (j - ne);
        const int rg = rg0 + g, lg = same ? g : FG + g;
        const int id = ((part * 8 + i8) * 48 + rg) * 2 + wn, gct = block_of(p, part) * 8 + i8;
        u.slot = SL_O + id; u.kind = 2;
        u.flag = ((FL_O + ((part * 8 + i8) * 6 + (rg >> 3)) * 2 + (rg & 1)) * 16 + (rg & 6) + wn) | FW_WIDE;
        u.dst = (u32)(lg * 32 * NN + gct * 128 + wn * 64) * 4u;
        u.aux = i8;
        return u;
    }
    if (n < U1 || (n >= U2 && n < U3) || (n >= U5 && n < UC)) {
        int sg, kp;
        if (n < U1) { sg = sent_group(r, p, n + NPUSH + NPP); kp = 0; }
        else if (n < U3) { sg = sent_group(r, p, n - U2); kp = 1; }
        else { sg = XGREM + ((n - U5) & 15); kp = (n - U5) >> 4; }
        const int gg = group_of(r, sg);
        u.slot = SL_X + gg * 2 + kp; u.kind = 0;
        u.flag = kp ? ((FL_X1 + (gg >> 3)) * 16 + (gg & 7)) | FW_WIDE : ((FL_X + gg) * 16) | FW_SPLIT;
        u.dst = (u32)(WS_GX + (size_t)(p * 4 + 2 * kp) * XSLAB) + (u32)sg * 4096u;
        u.aux = (u32)(sg < XGREM ? WS_XREM : WS_XOWN) + (u32)(p * 2 + kp) * 128u;
        return u;
    }
    int i8, kh, part, src = p;
    if (n < U4) {
        const int v = n < U2 ? n - U1 : n - U3 + 16 * WB, o = (W_ORDER >> (v >> 4 << 2)) & 15;
        i8 = o & 3; kh = (o & 4) + ((v >> 2) & 3); part = v & 3;
    }
    else if (n < UA) { const int v = n - U4; i8 = I8A + (v >> 5); kh = (v >> 2) & 7; part = v & 3; }
    else if (n < UD) {
        const int v = (n - UA) & 31;
        i8 = n < UB ? 7 - same : n < U5 ? 5 : 6 + same; kh = v >> 2; part = v & 3;
    } else {
        const int v = n - UD;
        kh = v / 6; part = p >> 1;
        src = v - 6 * kh;
        src += src >= min(r, p);
        src += src >= max(r, p);
        i8 = 7 - ((src ^ r) & 1);
    }
    const int kc = kh >> 1, half = kh & 1, ct = part * 8 + i8, step = src * 4 + kc;
    u.slot = SL_W + ((kc * 128 + block_of(r, part) * 8 + i8) * 2 + half);
    u.flag = i8 >= I8A ? ((FL_L + ct) * 16 + kh) | FW_WIDE : ((FL_W + (kc * 32 + ct) * 2 + half) * 16) | (kc < 2 ? FW_SPLIT : 0u);
    u.kind = 1 | (half << 2);
    if (n >= UA && n < UB && (same || part != (p >> 1)) && kh >= 8 - FCAP) u.kind |= 8 | (kh << 4) | (part << 7);
    if (n >= UD) { u.slot = SL_F + ((r >> 1) * 8 + src) * 8 + kh; u.flag = ((FL_F + src * 8 + kh) * 16) | FW_SPLIT | FW_RELAYED; }
    if (n >= UC) u.flag |= (kh << 20) | ((part * 8 + src) << 23);
    u.dst = (u32)WS_GW + (u32)(step * 32 + ct) * 16384u + (u32)half * 8192u;
    u.aux = (u32)WS_WCNT + (u32)(ct * 32 + step) * 128u;
    return u;
}

#define STAMP(k) do { } while (0)

__global__ void __launch_bounds__(NTHR, 2) fused_kernel(Params P) {
    __shared__ __attribute__((aligned(1024))) uint8_t lds_[LDS_CTL + 576];
    lbyte* const lds = (lbyte*)lds_;
    const int tid = threadIdx.x;
    const int lane = tid & 63;
    const int wave = __builtin_amdgcn_readfirstlane(tid >> 6);
    const int b = blockIdx.x;
    const int rank = P.rank;
    uint8_t* const myws = P.myws;
    const uint8_t* const xin = P.x;
    const uint8_t* const win = P.w;
    asm volatile("" :: "s"(rank), "s"(myws), "s"(xin), "s"(win));
    u32* const tmo = (u32*)(myws + WS_TMO);

    STAMP(0);
    u32* const cntp = (u32*)(myws + WS_CNT + (size_t)(b * 8 + wave) * 128);
#define LOAD_CALL() \
    const u32 ncalls = __hip_atomic_load(cntp, __ATOMIC_RELAXED, __HIP_MEMORY_SCOPE_AGENT)
#define GET_CALL() \
    const u32 call = __builtin_amdgcn_readfirstlane(ncalls) + 1; \
    const size_t pkoff = WS_PK + (size_t)(call & 1) * PK_BYTES
#define PUT_CALL() \
    if (lane == 0) __hip_atomic_store(cntp, call, __ATOMIC_RELAXED, __HIP_MEMORY_SCOPE_AGENT)

    if (b < NSEND) {
        if (b >= NENG) {
            if (b != NENG || wave > 1) return;
            LOAD_CALL();
            GET_CALL();
            PUT_CALL();
            if (wave == 1) {
                const u64 t0 = __builtin_amdgcn_s_memrealtime();
                const int ls = lane >> 3;
                u64 mv;
                for (u32 it = 1;; ++it) {
                    mv = __hip_atomic_load((const u64*)(myws + WS_MARK) + ls * 16 + (lane & 7), __ATOMIC_RELAXED, __HIP_MEMORY_SCOPE_AGENT);
                    if (__all(ls == rank || (u32)mv == call)) break;
                    if ((it & 7) == 0 && (__hip_atomic_load(tmo, __ATOMIC_RELAXED, __HIP_MEMORY_SCOPE_AGENT) == call ||
                                          __builtin_amdgcn_s_memrealtime() - t0 > DEADLINE)) {
                        if (lane == 0) __hip_atomic_store(tmo, call, __ATOMIC_RELAXED, __HIP_MEMORY_SCOPE_AGENT);
                        break;
                    }
                    __builtin_amdgcn_s_sleep(16);
                }
                constexpr int UNIT = 78;
                int ld = min((u32)(mv >> 32) - (u32)t0, 1u << 20);
                ld += __shfl_xor(ld, 1); ld += __shfl_xor(ld, 2); ld += __shfl_xor(ld, 4);
                ld = ld * PLAN_GAIN / 8 + (((ls ^ rank) & 1) ? 0 : SAME_MORE);
                const int ps = lane & 7, pr = 2 * ((lane >> 3) & 3) + 1 - (rank & 1);
                int lsrc = __shfl(ld, ps * 8), lrel = __shfl(ld, pr * 8);
                u32 f = 0;
                const bool can = lane < 32 && ps != rank && ps != pr;
                for (int it = 0; it < 128; ++it) {
                    const u32 best = wave_umax(can && f < FCAP && lsrc - lrel > UNIT ? ((u32)(lsrc - lrel) << 5) | (u32)lane : 0u);
                    if (!best) break;
                    const int bs = best & 7, br = 2 * ((best >> 3) & 3) + 1 - (rank & 1);
                    f += (u32)lane == (best & 31u);
                    lsrc += ps == br ? UNIT : ps == bs ? -UNIT : 0;
                    lrel += pr == br ? UNIT : pr == bs ? -UNIT : 0;
                }
                if (lane < 32) __hip_atomic_store((u64*)(myws + WS_PLAN) + lane, ((u64)f << 32) | call, __ATOMIC_RELAXED, __HIP_MEMORY_SCOPE_AGENT);
                STAMP(1);
                return;
            }
            const int src = (lane >> 1) & 7;
            const u32 need = call * (lane >= 16 ? 32u : src == rank ? 2u * XGREM : 2u * (XGREM - (((src ^ rank) & 1) ? GG : FG)));
            uint8_t* const cnt = myws + WS_XREM + (lane & 31) * 128;
            const u64 t0 = __builtin_amdgcn_s_memrealtime();
            bool open = lane < 32;
            for (u32 it = 1; __any(open); ++it) {
                if (open && __hip_atomic_load((const u32*)cnt, __ATOMIC_RELAXED, __HIP_MEMORY_SCOPE_SYSTEM) >= need) {
                    __hip_atomic_store((u32*)(cnt + X_DONE), call, __ATOMIC_RELAXED, __HIP_MEMORY_SCOPE_SYSTEM);
                    open = false;
                }
                if ((it & 7) == 0 && (__hip_atomic_load(tmo, __ATOMIC_RELAXED, __HIP_MEMORY_SCOPE_AGENT) == call ||
                                      __builtin_amdgcn_s_memrealtime() - t0 > DEADLINE)) {
                    if (lane == 0) __hip_atomic_store(tmo, call, __ATOMIC_RELAXED, __HIP_MEMORY_SCOPE_AGENT);
                    break;
                }
                __builtin_amdgcn_s_sleep(MON_SLEEP);
            }
            return;
        }
        LOAD_CALL();
        const int p = (rank + 1 + b % 7) & 7;
        const bool same = ((rank ^ p) & 1) == 0;
        const int nop = operand_units(same);
        const int nend = nop + (same ? 64 * FG : 48 * GG);
        for (int n = tid; n < nend + NPP; n += NTHR) {
            const Unit u = unit_of(n < nend ? n : n - nend - NPP, rank, p);
            *(lu32x4*)(lds + EL_TAB + n * 16) = u32x4{u.slot | (u.kind << 16), u.flag, u.dst, u.aux};
        }
        if (tid < (EL_END - EL_GO) / 4) {
            const int fo = EL_GO + tid * 4 - EL_FREE;
            *(lu32*)(lds + EL_GO + tid * 4) = fo >= (NS - NPAIR) * 8 && fo < NS * 8 ? (u32)-NS : 0u;
        }
        *(lu32x4*)(lds + EL_PLANE + tid * 16) = u32x4{0u, 0u, 0u, 0u};
        if (tid < 16) op_lut_init(lds + LDS_LUT, tid);
        __syncthreads();
        GET_CALL();
        PUT_CALL();
        auto lds_get = [&](int off) {
            return (u32)__builtin_amdgcn_readfirstlane(
                __hip_atomic_load((lu32*)(lds + off), __ATOMIC_RELAXED, __HIP_MEMORY_SCOPE_WORKGROUP));
        };
        auto lds_put = [&](int off, u32 v) {
            CBAR();
            if (lane == 0) __hip_atomic_store((lu32*)(lds + off), v, __ATOMIC_RELAXED, __HIP_MEMORY_SCOPE_WORKGROUP);
            CBAR();
        };

        if (wave == 0) {
            __builtin_amdgcn_s_setprio(3);
            __builtin_amdgcn_s_waitcnt(0);
            const uint8_t* const pk = P.ws[p] + pkoff;
            const u32 lds_addr = (u32)(uintptr_t)lds;
            const u32 vo0 = lane * 16, vo1 = lane * 16 + 4096;
            static_assert(WIN >= 5 && WIN <= 9 && WIN_OP >= 5 && WIN_OP <= 9, "window");
#define F_WAIT(W, J) "s_waitcnt vmcnt(" #W "-1)\n\t.if (" #W "+5)%%6 == " #J "\n\ts_mov_b64 exec, 1\n\tds_write_b64 %8, %9\n\ts_mov_b64 exec, -1\n\t.endif\n\t"
#define F_LD(V, OFF) "global_load_lds_dwordx4 " V ", %2 offset:" #OFF " sc0 sc1\n\t"
#define F_ARGS :: "v"(vo0), "v"(vo1), "s"(src), "s"(la), "s"(la + 4096), "s"(mk0), "s"(mk1), "n"(0), "v"(lds_addr + EL_ISS), "v"(st) : "memory"
#define F_OP(W) asm volatile("s_mov_b32 m0, %3\n\t" F_WAIT(W, 0) F_LD("%0", 0) F_WAIT(W, 1) F_LD("%0", 1024) F_WAIT(W, 2) \
                "s_mov_b64 exec, %5\n\t" F_LD("%0", 2048) "s_mov_b64 exec, -1\n\ts_mov_b32 m0, %4\n\t" \
                F_WAIT(W, 3) F_LD("%1", 0) F_WAIT(W, 4) F_LD("%1", 1024) F_WAIT(W, 5) \
                "s_mov_b64 exec, %6\n\t" F_LD("%1", 2048) "s_mov_b64 exec, -1" F_ARGS)
#define F_ACC(W) asm volatile("s_mov_b32 m0, %4\n\t" F_WAIT(W, 0) "s_mov_b64 exec, %6\n\t" F_LD("%1", 1024) "s_mov_b64 exec, -1\n\t" \
                F_WAIT(W, 1) F_LD("%1", 0) "s_mov_b32 m0, %3\n\t" F_WAIT(W, 2) F_LD("%0", 0) F_WAIT(W, 3) F_LD("%0", 1024) \
                F_WAIT(W, 4) F_LD("%0", 2048) F_WAIT(W, 5) F_LD("%0", 3072) F_ARGS)
#define F_MORE(V, M0V, OFF, MK) asm volatile("s_mov_b32 m0, %2\n\ts_waitcnt vmcnt(5)\n\ts_mov_b64 exec, %3\n\t" \
                "global_load_lds_dwordx4 %0, %1 offset:" #OFF " sc0 sc1\n\ts_mov_b64 exec, -1" :: "v"(V), "s"(src), "s"(M0V), "s"(MK) : "memory")
#define F_X(M, W) M(W)
            auto lanes = [](u32 lines) { return lines >= 8 ? ~0ull : (1ull << (8 * lines)) - 1; };
            auto get_go = [&](u32 sl) { CBAR(); const u32x4 r = *(volatile lu32x4*)(lds + EL_GO + sl * 16); CBAR(); return r; };
            u32 sl = 0, pub = 0, k = 0;
            bool six = false;
            u32x4 rec = get_go(0);
            for (;; ++k) {
                u32 go = __builtin_amdgcn_readfirstlane(rec[0]);
                for (u32 it = 0; (go >> 16) != k + 1; ++it) {
                    if (it == STARVE) { WAIT_VM(0); lds_put(EL_FULL, pub = k); }
                    __builtin_amdgcn_s_sleep(1);
                    rec = get_go(sl);
                    go = __builtin_amdgcn_readfirstlane(rec[0]);
                }
                if ((go & 0xffffu) == UNIT_END) break;
                const u32 fw = __builtin_amdgcn_readfirstlane(rec[3]);
                const u32 l0 = (fw >> 16) & 255u, l1 = fw >> 24;
                const uint8_t* src = pk + (size_t)(fw & 0xffffu) * SLOT;
                const u32 la = lds_addr + sl * SLOT;
                sl = sl == NS - 1 ? 0 : sl + 1;
                const bool isop = (int)(go & 0xffffu) < nop;
                pub = (isop ? WIN_OP : WIN) < 7 ? k : max(pub + 1, k) - 1;
                const u32x2 st = {k + 1, pub};
                if (isop) {
                    const u64 mk0 = lanes(l0 - 16), mk1 = lanes(l1 - 16);
                    F_X(F_OP, WIN_OP);
                    if (l0 > 24) F_MORE(vo0, la, 3072, lanes(l0 - 24));
                } else {
                    const u64 mk0 = 0, mk1 = lanes(l1 - 8);
                    F_X(F_ACC, WIN);
                    if (l1 > 16) F_MORE(vo1, la + 4096, 2048, lanes(l1 - 16));
                }
                if (l1 > 24) F_MORE(vo1, la + 4096, 3072, lanes(l1 - 24));
                six = !isop && l1 <= 16;
                rec = get_go(sl);
            }
            WAIT_VM(6);
            if (k) lds_put(EL_FULL, max(pub, k - 1));
            WAIT_VM(4);
            if (six) lds_put(EL_HEAD, k);
            WAIT_VM(0);
            lds_put(EL_FULL, pub = k);
            STAMP(4);
            return;
        }
        if (wave == 1) {
            const u64* const flags = (const u64*)(myws + WS_FLAG) + (size_t)p * 4096 * 16;
            u32* const tick = (u32*)(myws + WS_TICK) + p * 32;
            static_assert(BATCH % 2 == 1 && BATCH <= 7, "a bijection needs BATCH odd");
            constexpr int TC = 8 * (BATCH - 1) / BATCH;
            const u32 ntick = (nend + BATCH - 1) / BATCH + TC, tbase = (call - 1) * (ntick - 16) - 24;
            static_assert(4 * BATCH <= 32, "lanes");
            u32 tick_no = b / 7;
            bool dead = false, planned = false;
            const int nflex = nop - (same ? 32 : 80);
            int gone = 0;
            u32 partials = 0;
            auto give_up = [&](u64 t0) {
                if (__hip_atomic_load(tmo, __ATOMIC_RELAXED, __HIP_MEMORY_SCOPE_AGENT) == call ||
                    __builtin_amdgcn_s_memrealtime() - t0 > DEADLINE) {
                    if (lane == 0) __hip_atomic_store(tmo, call, __ATOMIC_RELAXED, __HIP_MEMORY_SCOPE_AGENT);
                    dead = true;
                    lds_put(EL_DEAD, 1);
                    return true;
                }
                __builtin_amdgcn_s_sleep(2);
                return false;
            };
            auto post = [&](u32 kk, u32 sl, u32 unit, u32 h0, u32 h1, u32 slot) {
                const u32 l0 = (int)unit < nop ? max((h0 & 31u) + 1, 17u) : 32u;
                const u32 l1 = (int)unit < nop ? max(((h1 >> 13) & 31u) + 1, 17u) : min(max(h0 & 127u, 41u), 64u) - 32u;
                for (;;) {
                    CBAR();
                    const u32x2 fr = *(volatile lu32x2*)(lds + EL_FREE + sl * 8);
                    if (__builtin_amdgcn_readfirstlane(min(fr[0] + NS, fr[1] + NS)) >= kk + 1) break;
                    __builtin_amdgcn_s_sleep(1);
                }
                CBAR();
                if (lane == 0) *(volatile lu32x4*)(lds + EL_GO + sl * 16) = u32x4{((kk + 1) << 16) | unit, h0, h1, slot | (l0 << 16) | (l1 << 24)};
                CBAR();
            };
            u32 k = 0, sl = 0;
            while (tick_no < ntick) {
                const int u0 = BATCH * ((int)tick_no - TC);
                while (lds_get(EL_ISS) + LOOK < k) __builtin_amdgcn_s_sleep(SC_SLEEP);
                const int nb = tick_no < 8 ? 2 * BATCH : BATCH;
                if (u0 + 8 * (nb - 1) >= nflex && !planned) {
                    planned = true;
                    const u64 t0 = __builtin_amdgcn_s_memrealtime();
                    u64 pv;
                    for (;;) {
                        pv = __hip_atomic_load((const u64*)(myws + WS_PLAN) + (lane & 31), __ATOMIC_RELAXED, __HIP_MEMORY_SCOPE_AGENT);
                        if (__all((u32)pv == call) || dead || give_up(t0)) break;
                    }
                    int live = 0;
                    for (int e0 = nflex; e0 < nop; e0 += 64) {
                        const u32 fe = *(lu32*)(lds + EL_TAB + (e0 + lane) * 16 + 4);
                        const bool relayed = ((fe >> 20) & 7u) + __shfl((u32)(pv >> 32), (fe >> 23) & 31u) >= 8u;
                        const bool mine = e0 + lane < nop && relayed == ((fe & FW_RELAYED) != 0);
                        const u64 m = __ballot(mine);
                        if (mine) *(lu32*)(lds + EL_LIVE + (live + __builtin_popcountll(m & ((1ull << lane) - 1))) * 4) = e0 + lane;
                        live += __builtin_popcountll(m);
                    }
                    CBAR();
                    gone = nop - nflex - live;
                }
                int ul = max(u0 + 8 * min((lane >> 1) & 15, nb - 1), 0);
                if (ul >= nflex) ul = ul + gone >= nop ? min(ul + gone, nend - 1) : *(lu32*)(lds + EL_LIVE + (ul - nflex) * 4);
                const u32x2 te = *(lu32x2*)(lds + EL_TAB + ul * 16);
                const u32 fw = te[1];
                const u64* const fp = flags + (fw & 0xffffu) + (call & 1) * ((fw & FW_WIDE) ? 8 : 2) + ((fw & FW_SPLIT) ? lane & 1 : 0);
                u32 tk;
                u64 fv;
                asm volatile("s_mov_b64 exec, 1\n\tglobal_atomic_add %0, %2, %3, off sc0\n\ts_mov_b64 exec, %5\n\t"
                             "global_load_dwordx2 %1, %4, off sc0 sc1\n\ts_mov_b64 exec, -1\n\ts_waitcnt vmcnt(0)"
                             : "=&v"(tk), "=&v"(fv) : "v"(tick), "v"(tick_no < 8 ? 0u : 1u), "v"(fp), "s"((1ull << (2 * nb)) - 1) : "memory");
                if (tick_no - MARK_AT < 8u && lane == 0)
                    __hip_atomic_store((u64*)(myws + WS_MARK) + p * 16 + (tick_no - MARK_AT),
                                       (__builtin_amdgcn_s_memrealtime() << 32) | call, __ATOMIC_RELAXED, __HIP_MEMORY_SCOPE_AGENT);
                for (int j = 0; j < nb; ++j) {
                    if (u0 + 8 * j < 0 || u0 + 8 * j + gone >= nend) continue;
                    const int un = __builtin_amdgcn_readlane(ul, 2 * j);
                    if (!dead && ((__ballot((u32)fv >= call) >> (2 * j)) & 3) != 3) {
                        while (lds_get(EL_ISS) + LOOK < k) __builtin_amdgcn_s_sleep(SC_SLEEP);
                        const u64 t0 = __builtin_amdgcn_s_memrealtime();
                        for (u32 it = 1;; ++it) {
                            if (lane >= 2 * j && lane < 2 * nb) fv = __hip_atomic_load(fp, __ATOMIC_RELAXED, __HIP_MEMORY_SCOPE_SYSTEM);
                            if (((__ballot((u32)fv >= call) >> (2 * j)) & 3) == 3 || ((it & 7) == 0 && give_up(t0))) break;
                            __builtin_amdgcn_s_sleep(1);
                        }
                    }
                    if (un >= nop && partials != 255u) {
                        const u32 bit = 1u << lds_get(EL_TAB + un * 16 + 12);
                        if (!(partials & bit)) {
                            while (lds_get(EL_ISS) + LOOK < k) __builtin_amdgcn_s_sleep(SC_SLEEP);
                            const u64 t0 = __builtin_amdgcn_s_memrealtime();
                            while (!dead) {
                                partials = (u32)__ballot(__hip_atomic_load((const u32*)(myws + WS_PCNT) + (lane & 7) * 32, __ATOMIC_RELAXED,
                                                                           __HIP_MEMORY_SCOPE_SYSTEM) >= 12u * call) & 255u;
                                if ((partials & bit) || give_up(t0)) break;
                            }
                        }
                    }
                    const u32 hi = (u32)(fv >> 32);
                    post(k, sl, un, __builtin_amdgcn_readlane(hi, 2 * j), __builtin_amdgcn_readlane(hi, 2 * j + 1),
                         __builtin_amdgcn_readlane(te[0], 2 * j) & 0xffffu);
                    sl = sl == NS - 1 ? 0 : sl + 1;
                    ++k;
                }
                tick_no = tick_no < 8 ? tick_no + 16 : __builtin_amdgcn_readfirstlane(tk) - tbase;
            }
            for (u32 kk = k; kk < k + NPAIR; ++kk) {
                post(kk, sl, UNIT_END, 0, 0, 0);
                sl = sl == NS - 1 ? 0 : sl + 1;
            }
            return;
        }
        lbyte* const plane = lds + EL_PLANE + wave * 1024;
        const float scale0 = P.sx[0] * P.sw[0];
        const u32 hf = wave & 1;
        const __amdgpu_buffer_rsrc_t outrs = __builtin_amdgcn_make_buffer_rsrc(P.out, 0, MS * NN * 4, 0x00020000);
        const u32 olane = (hf * 16 + (lane >> 4) * 4) * (NN * 4) + (lane & 15) * 16;
        u32 owed = 0;
        auto settle = [&]() {
            if (!owed) return;
            WAIT_VM(0);
            if (lane == 0) count_in(myws + owed, 1u);
            owed = 0;
        };
        u32 sl = ((wave - 2) >> 1) + NS - NPAIR;
        for (u32 k = ((wave - 2) >> 1) - NPAIR;; k += NPAIR) {
            lbyte* const s = lds + sl * SLOT;
            const bool pushed = (int)k < 0;
            const u32 fin = pushed ? 0u : k + 1;
            u32 go, hdr;
            if (pushed) {
                const u32 j = b / 7 * NPAIR + k + NPAIR;
                go = nend + j;
                const u64* const fp = (const u64*)(myws + WS_FLAG) + (size_t)(p * 4096 + FL_P + j) * 16 + (call & 1) * 2 + hf;
                const u64 t0 = __builtin_amdgcn_s_memrealtime();
                u64 fv;
                for (u32 it = 1; (u32)(fv = __hip_atomic_load(fp, __ATOMIC_RELAXED, __HIP_MEMORY_SCOPE_SYSTEM)) < call; ++it) {
                    if ((it & 7) == 0 && (__hip_atomic_load(tmo, __ATOMIC_RELAXED, __HIP_MEMORY_SCOPE_AGENT) == call ||
                                          __builtin_amdgcn_s_memrealtime() - t0 > DEADLINE)) {
                        if (lane == 0) __hip_atomic_store(tmo, call, __ATOMIC_RELAXED, __HIP_MEMORY_SCOPE_AGENT);
                        lds_put(EL_DEAD, 1);
                        break;
                    }
                    __builtin_amdgcn_s_sleep(4);
                }
                hdr = __builtin_amdgcn_readfirstlane((u32)(fv >> 32));
                asm volatile("s_mov_b32 m0, %1\n\ts_nop 0\n\tglobal_load_lds_dwordx4 %0, off sc0 sc1\n\t"
                             "global_load_lds_dwordx4 %0, off offset:1024 sc0 sc1\n\tglobal_load_lds_dwordx4 %0, off offset:2048 sc0 sc1\n\t"
                             "global_load_lds_dwordx4 %0, off offset:3072 sc0 sc1\n\ts_waitcnt vmcnt(0)"
                             :: "v"(myws + pkoff + (size_t)(SL_P + p * NPP + j) * SLOT + hf * 4096 + lane * 16),
                                "s"((u32)(uintptr_t)s + hf * 4096) : "memory");
            } else {
                while (((go = lds_get(EL_GO + sl * 16)) >> 16) != k + 1) { settle(); __builtin_amdgcn_s_sleep(1); }
                if ((go & 0xffffu) == UNIT_END) break;
                CBAR();
                hdr = lds_get(EL_GO + sl * 16 + 4 + hf * 4);
            }
            const u32x4 te = *(lu32x4*)(lds + EL_TAB + (go & 0xffffu) * 16);
            const u32 kind = __builtin_amdgcn_readfirstlane(te[0]) >> 16;
            const u32 dst = __builtin_amdgcn_readfirstlane(te[2]);
            const u32 aux = __builtin_amdgcn_readfirstlane(te[3]);
            const bool acc = kind == 2;
            u32x4 pa[4];
            uint8_t* const od = (uint8_t*)P.out + dst + olane;
            if (acc) {
                settle();
#pragma unroll
                for (int j = 0; j < 4; ++j)
                    pa[j] = __builtin_bit_cast(u32x4, __builtin_amdgcn_raw_buffer_load_b128(outrs, olane, dst + j * (NN * 4), 17));
            }
            u32x4 st;
            for (;;) {
                CBAR();
                st = *(volatile lu32x4*)(lds + EL_ISS);
                if (__builtin_amdgcn_readfirstlane(st[1]) >= fin || __builtin_amdgcn_readfirstlane(st[2]) == fin + pushed) break;
                settle();
                __builtin_amdgcn_s_sleep(WK_SLEEP);
            }
            CBAR();
            if (WK_PRIO) __builtin_amdgcn_s_setprio(WK_PRIO);
            if (!acc) {
                u32x4 v[4];
                const lbyte* const blk = s + hf * 4096;
                const u32 bh = hdr >> (13 * hf);
                if (kind & 8) {
                    const u32 fp = (kind >> 7) & 3u, ff = (kind >> 4) & 7u;
                    uint8_t* const fs = myws + pkoff + (size_t)(SL_F + (fp * 8 + p) * 8 + ff) * SLOT + hf * 4096;
                    for (u32 o = lane * 16; o < ((bh & 31u) + 1) * 128; o += 1024) st_sys(fs + o, *(const lu32x4*)(blk + o));
                    WAIT_VM(0);
                    if (lane == 0) st_flag(P.ws[2 * fp + 1 - (rank & 1)], rank, FL_F + p * 8 + ff, hf, call, hdr);
                }
                if ((bh & 31u) == 31u) {
#pragma unroll
                    for (int j = 0; j < 4; ++j) v[j] = *(const lu32x4*)(blk + j * 1024 + lane * 16);
                    lds_put(EL_FREE + sl * 8 + hf * 4, fin);
                } else {
                    op_unpack(v, ((hdr >> 26) & 31u) * 0x04040404u, min((bh >> 5) & 255u, 128u), blk,
                              lds + EL_BUF + wave * 2304, lds + LDS_LUT, lane,
                              [&] { lds_put(EL_FREE + sl * 8 + hf * 4, fin); });
                }
                const bool isw = kind & 1;
                uint8_t* d = myws + dst + (isw ? hf * 4096 : (call & 1) * GX_ODD + hf * XSLAB) +
                             ((lane ^ (isw ? (kind & 4u) * 2 + hf * 4 : 0)) << 4);
                settle();
#pragma unroll
                for (int j = 0; j < 4; ++j) st_sys(d + j * 1024, v[j]);
                owed = aux;
            } else {
                const float scale = __builtin_amdgcn_readfirstlane(st[3]) ? __builtin_nanf("") : scale0;
                float a[16], c1 = 1.0f;
                if (hdr & H_RAW) {
#pragma unroll
                    for (int j = 0; j < 4; ++j) {
                        const u32x4 t = *(lu32x4*)(s + hf * 4096 + j * 1024 + lane * 16);
#pragma unroll
                        for (int t4 = 0; t4 < 4; ++t4) a[4 * j + t4] = __uint_as_float(t[t4]);
                    }
                    lds_put(EL_FREE + sl * 8 + hf * 4, fin);
                } else {
                    u32x4 mid = *(lu32x4*)(s + 4096 + lane * 16);
                    const u32 nb = (hdr >> 8) & 0x7ffu;
                    u32 at = ~0u;
                    for (u32 i0 = 0; i0 < nb; i0 += 128) {
                        const u32 i = i0 + 2 * lane;
                        const u32 w2 = *(lu16*)(s + ACC_MAIN + i);
                        u32 d[2], val[2];
#pragma unroll
                        for (int h = 0; h < 2; ++h) {
                            const u32 e = i + h < nb ? (w2 >> (8 * h)) & 255u : 0xe0u;
                            d[h] = e < 0xc0u ? (e & 63u) + 1 : e < 0xe0u ? 64u : 0u;
                            val[h] = e < 0x80u ? 3u + (e >> 6) : e >= 0xe0u ? e & 31u : 0u;
                        }
                        const u32 inc = wave_scan_add(d[0] + d[1]);
#pragma unroll
                        for (int h = 0; h < 2; ++h) {
                            const u32 t = at + inc - (h ? 0 : d[1]);
                            if (val[h] && ((t >> 2) & 1u) == hf) *(plane + (plane_addr(t) & 0x3ffu)) = (uint8_t)val[h];
                        }
                        at += __builtin_amdgcn_readlane(inc, 63);
                    }
                    CBAR();
                    const u32x4 q = *(lu32x4*)(plane + lane * 16);
                    *(lu32x4*)(plane + lane * 16) = u32x4{0u, 0u, 0u, 0u};
                    if (hf) {
                        const u32 m4 = *(lu32*)(s + 5120 + lane * 4), m5 = *(lu32*)(s + 5376 + lane * 4), m6 = *(lu32*)(s + 5632 + lane * 4);
                        u32 m7 = (mid[0] << 1) & 0x02020202u;
                        m7 |= (mid[1] << 2) & 0x04040404u; m7 |= (mid[2] << 3) & 0x08080808u; m7 |= (mid[3] << 4) & 0x10101010u;
                        m7 |= (m4 << 5) & 0x20202020u; m7 |= (m5 << 6) & 0x40404040u; m7 |= (m6 << 7) & 0x80808080u;
                        mid = u32x4{m4, m5, m6, m7};
                    }
                    u32 l0[4], l1[4];
#pragma unroll
                    for (int j = 0; j < 4; ++j) {
                        const u32 mi = bfi(0x01010101u, q[j] >> 2, mid[j]);
                        const u32 lo = (q[j] & 0x03030303u) | ((q[j] << 3) & 0xC0C0C0C0u);
                        l0[j] = __builtin_amdgcn_perm(mi, lo, 0x05010400u);
                        l1[j] = __builtin_amdgcn_perm(mi, lo, 0x07030602u);
                        asm volatile("" : "+v"(l0[j]), "+v"(l1[j]));
                    }
                    if (__builtin_amdgcn_readfirstlane(st[1]) < fin) { while (lds_get(EL_FULL) < fin) { } }
                    CBAR();
                    const u32x4 h0 = *(lu32x4*)(s + hf * 2048 + lane * 16), h1 = *(lu32x4*)(s + hf * 2048 + 1024 + lane * 16);
                    lds_put(EL_FREE + sl * 8 + hf * 4, fin);
                    c1 = __uint_as_float(((hdr >> 19) & 255u) << 23);
#pragma unroll
                    for (int j = 0; j < 4; ++j) {
                        const u32 ha = j < 2 ? h0[2 * j] : h1[2 * j - 4], hb = j < 2 ? h0[2 * j + 1] : h1[2 * j - 3];
                        const u32 T[4] = {__builtin_amdgcn_perm(ha, l0[j], 0x05040100u), __builtin_amdgcn_perm(ha, l0[j], 0x07060302u),
                                          __builtin_amdgcn_perm(hb, l1[j], 0x05040100u), __builtin_amdgcn_perm(hb, l1[j], 0x07060302u)};
#pragma unroll
                        for (int t4 = 0; t4 < 4; ++t4) a[4 * j + t4] = (float)(((int)T[t4] >> 2) ^ (int)(T[t4] << 30));
                    }
                }
#pragma unroll
                for (int j = 0; j < 4; ++j) asm volatile("" : "+v"(pa[j]));
#pragma unroll
                for (int j = 0; j < 4; ++j) {
                    f32x4 y;
#pragma unroll
                    for (int t4 = 0; t4 < 4; ++t4) {
                        const u32 pb = pa[j][t4];
                        y[t4] = silu(fmaf(a[4 * j + t4], c1, __uint_as_float(pb)) * scale);
                    }
                    st_mode<NT_OUT>(od + (size_t)j * (NN * 4), __builtin_bit_cast(u32x4, y));
                }
            }
            if (WK_PRIO) __builtin_amdgcn_s_setprio(0);
            sl = sl >= NS - NPAIR ? sl + NPAIR - NS : sl + NPAIR;
        }
        settle();
        return;
    }

    const int g = b - NSEND;
    lbyte* const scr = lds + wave * 6144;
    constexpr int WFIRST = 64 * I8A, G_X1 = 72, GLATE = G_X1 + 16, GLEND = GLATE + 16 * (8 - I8A);
    constexpr int T_P = 112, T_X = T_P + 14 * NPP, T_W = T_X + 256;
    static_assert(I8A == 3 && T_W + 3 * WFIRST <= 176 * 8, "tasks");
    const bool isx1 = g >= G_X1 && g < GLATE;
    static_assert(T_W + 2 * WFIRST == 176 * 6 + 32 && WFIRST == 2 * 96 && GLEND == 168, "tasks");
    const int task0 = isx1 ? T_X : wave < 6 ? (g < G_X1 ? g : g - 16) + 176 * wave
                    : g < G_X1 || g >= GLEND ? T_W + 2 * WFIRST + (g < G_X1 ? g : g - 96) + 96 * (wave - 6)
                    : wave == 6 && g >= GLEND - 32 ? 176 * 6 + g - (GLEND - 32) : 2047;
    const int pdst = (rank + 1 + task0 % 7) & 7;
    const int pj = task0 / 7 - T_P / 7;
    const int psg = sent_group(pdst, rank, task0 < T_P ? task0 / 7 : NPUSH + (pj >> 1));
    const int gg = isx1 ? (g - G_X1) * 8 + wave : task0 < T_X ? group_of(pdst, psg) : (task0 - T_X) & 127;
    const int xkc = isx1 ? 2 : task0 < T_P ? 0 : task0 < T_X ? pj & 1 : (task0 - T_X) >> 7;
    u32x4 v[8];
    if (task0 < 7 * NPUSH || (task0 >= T_P && task0 < T_W)) {
        const uint8_t* src = xin + (size_t)(gg * 32 + (lane >> 3)) * KS + xkc * 128 + (lane & 7) * 16;
#pragma unroll
        for (int j = 0; j < 4; ++j) v[j] = *(const u32x4*)(src + j * 8 * KS);
        if (task0 < T_P || isx1) {
#pragma unroll
            for (int j = 0; j < 4; ++j) v[4 + j] = *(const u32x4*)(src + j * 8 * KS + 128);
        }
    }
    u32x2 in[16];
    auto load_w = [&](int kc, int blk, int wi8, int half) {
        const uint8_t* src = win + (size_t)(kc * 128 + 16 * (lane >> 3)) * NN + blk * 1024 + wi8 * 128 + 16 * (lane & 7) + 8 * half;
#pragma unroll
        for (int j = 0; j < 16; ++j) in[j] = *(const u32x2*)(src + (size_t)j * NN);
    };
    const bool isw1 = task0 >= T_W && task0 < T_W + 2 * WFIRST;
    if (isw1) {
        const int wt = task0 - T_W;
        load_w((wt >> 6) & 1, (wt & 7) | ((wt >> 1) & 8), wt >> 7, (wt >> 3) & 1);
    }
    LOAD_CALL();
    const int l7 = (lane >> 2) & 7;
    uint8_t* const wsl = P.ws[l7];
    __builtin_amdgcn_sched_barrier(0);
    if (tid < 3) *(lu32*)(lds + LDS_PUB + tid * 64 + 32) = 0;
    const u64 tstart0 = __builtin_amdgcn_s_memrealtime();
    lbyte* const lut = scr + 4096;
    if (lane < 16) op_lut_init(lut, lane);
    CBAR();
    if (task0 < 7 * NPUSH) {
        uint8_t* const wd = P.ws[pdst];
        uint8_t* d = wd + WS_GX + (size_t)(rank * 4) * XSLAB + (size_t)psg * 4096 + lane * 16;
        GET_CALL();
        d += (call & 1) * GX_ODD;
#pragma unroll
        for (int j = 0; j < 8; ++j) asm volatile("" : "+v"(v[j]));
#pragma unroll
        for (int j = 0; j < 8; ++j) {
            THROTTLE(PUSH_WIN);
            st_sys(d + (j >> 2) * XSLAB + (j & 3) * 1024, v[j]);
        }
        WAIT_VM(0);
        if (lane == 0)
            count_in(wd + WS_XREM + rank * 256, 2u);
    }
    bool isx = task0 >= T_P && task0 < T_W, send = true, keep = true, tell = false;
    int h0 = 0, h1 = 2, flag = 0;
    u32 dbo = 0, swz = 0, slo = 0, cno = 0;
    uint8_t* slotws = myws;
    if (task0 >= T_P && task0 < T_X) {
        h0 = xkc; h1 = h0 + 1;
        keep = false;
        tell = l7 == pdst;
        flag = FL_P + (pj >> 1);
        slo = (u32)(SL_P + rank * NPP + (pj >> 1)) * (u32)SLOT;
        slotws = P.ws[pdst];
    } else if (isx) {
        if (X0_PRIO && !isx1) __builtin_amdgcn_s_setprio(X0_PRIO);
        const int kc = xkc, q = gg >> 4, lg = gg & 15;
        const int fl = (lg >= FG) ^ (q & 1);
        if (!isx1) { h0 = kc; h1 = h0 + 1; }
        send = q != rank;
        keep = q == rank || (rank & 1) == fl;
        if (isx1) {
            tell = send && l7 != rank && (lg < 8 || l7 == q || (l7 & 1) == fl);
            flag = FL_X1 + (gg >> 3);
        } else {
            const int sgl = lg < FG ? FG * (((q >> 1) - (l7 >> 1) - 1) & 3) + lg : 3 * FG + GG * (q >> 1) + lg - FG;
            tell = send && l7 != rank && (l7 == q || ((l7 & 1) == fl && sgl >= sent_group(l7, rank, NPUSH + NPP)));
            flag = FL_X + gg;
        }
        const int sg = q == rank ? XGREM + lg
                     : lg < FG ? FG * (((q >> 1) - (rank >> 1) - 1) & 3) + lg
                               : 3 * FG + GG * (q >> 1) + lg - FG;
        dbo = (u32)WS_GX + (u32)(rank * 4 + (kc & 2)) * (u32)XSLAB + (u32)sg * 4096u;
        slo = (u32)(SL_X + gg * 2 + (kc >> 1)) * (u32)SLOT;
        cno = (u32)(q == rank ? WS_XOWN : WS_XREM) + (u32)(rank * 2 + (kc >> 1)) * 128u;
        asm volatile("" : "+s"(dbo), "+s"(slo), "+s"(cno), "+v"(flag));
    }
    for (int phase = task0 >= T_P && task0 < T_W + 3 * WFIRST ? 0 : 1; phase < (g >= GLATE && g < GLEND ? 2 : 1); ++phase) {
        if (phase || isx1) {
            asm volatile("s_waitcnt lgkmcnt(0)" ::: "memory");
            __builtin_amdgcn_s_barrier();
        }
        if (phase) { isx = false; slotws = myws; }
        if (!isx) {
            const bool first = !phase && isw1;
            const int wt = task0 - T_W, wq = first ? wt >> 6 : (wt - 2 * WFIRST) >> 5;
            h0 = first ? (wt >> 5) & 1 : 0; h1 = first ? h0 + 1 : 2;
            const int half = phase ? wave & 1 : (wt >> 3) & 1;
            const int blk = phase ? (g - GLATE) & 15 : (wt & 7) | ((wt >> 1) & 8);
            const int kc = phase ? wave >> 1 : (first ? 0 : 2) + (wq & 1);
            const int wi8 = phase ? I8A + ((g - GLATE) >> 4) : wq >> 1;
            if (!first) {
                while (__builtin_amdgcn_s_memrealtime() - tstart0 < (u64)(phase ? W_DELAY + LATE_STAG * ((0x32410 >> (4 * (wi8 - I8A))) & 7) : W_DELAY0))
                    __builtin_amdgcn_s_sleep(8);
                load_w(kc, blk, wi8, half);
            }
            __builtin_amdgcn_sched_barrier(0);
            auto needs = [&](int r) { return (r >> 1) == ((r & 1) ? blk & 3 : blk >> 2); };
            auto ctile = [&](int r) { return ((r & 1) ? blk >> 2 : blk & 3) * 8 + wi8; };
            send = keep = true;
            tell = l7 != rank && needs(l7);
            flag = phase ? FL_L + ctile(l7) : FL_W + (kc * 32 + ctile(l7)) * 2 + half;
            const bool mine = needs(rank);
            const int fb = (rank & 1) ? 3 * (blk >> 2) + (blk & 3) - ((blk & 3) > (rank >> 1))
                                      : blk - (blk >= 2 * rank ? 4 : 0);
            const int ft = fb * 8 + wi8;
            dbo = half * 8192u + (mine ? (u32)WS_GW + (u32)((rank * 4 + kc) * 32 + ctile(rank)) * 16384u
                                       : (u32)WS_GW2 + (u32)(((ft >> 5) * 4 + kc) * 32 + (ft & 31)) * 16384u);
            swz = half * 8;
            slo = (u32)(SL_W + (kc * 128 + blk * 8 + wi8) * 2 + half) * (u32)SLOT;
            cno = (u32)WS_WCNT + (u32)((mine ? ctile(rank) : 32 + ft) * 32 + rank * 4 + kc) * 128u;
            asm volatile("" : "+s"(dbo), "+s"(slo), "+s"(cno), "+v"(flag));
#pragma unroll
            for (int d = 0; d < 2; ++d)
#pragma unroll
                for (int e = 0; e < 4; ++e) {
                    const u32 t0 = __builtin_amdgcn_perm(in[4 * e + 1][d], in[4 * e][d], 0x05010400u);
                    const u32 t1 = __builtin_amdgcn_perm(in[4 * e + 1][d], in[4 * e][d], 0x07030602u);
                    const u32 t2 = __builtin_amdgcn_perm(in[4 * e + 3][d], in[4 * e + 2][d], 0x05010400u);
                    const u32 t3 = __builtin_amdgcn_perm(in[4 * e + 3][d], in[4 * e + 2][d], 0x07030602u);
                    v[4 * d + 0][e] = __builtin_amdgcn_perm(t2, t0, 0x05040100u);
                    v[4 * d + 1][e] = __builtin_amdgcn_perm(t2, t0, 0x07060302u);
                    v[4 * d + 2][e] = __builtin_amdgcn_perm(t3, t1, 0x05040100u);
                    v[4 * d + 3][e] = __builtin_amdgcn_perm(t3, t1, 0x07060302u);
                }
            if (h0) {
#pragma unroll
                for (int j = 0; j < 4; ++j) v[j] = v[4 + j];
            }
        }
        GET_CALL();
#pragma unroll
        for (int j = 0; j < 8; ++j) asm volatile("" : "+v"(v[j]));
        uint8_t* const db = myws + dbo + (isx ? (call & 1) * GX_ODD : 0);
        uint8_t* const slot = slotws + pkoff + slo;
        u32 es = 0;
#pragma unroll
        for (int j = 0; j < 8; ++j) es = __builtin_amdgcn_sad_u8(v[j >> 1][(j & 1) * 2] & 0x7C7C7C7Cu, 0u, es);
        const u32 base = (u32)min(max(((int)wave_sum(es) - 1475) >> 13, 0), 29);
        u32 hdr = base << 26;
        for (int hh = h0; hh < h1; ++hh) {
            if (send) hdr |= op_pack(v, base * 0x04040404u, slot + hh * 4096, slotws != myws, scr, lut, lane) << (13 * hh);
            if (keep) {
                uint8_t* d = db + (isx ? hh * XSLAB + lane * 16 : hh * 4096 + ((lane ^ (swz + hh * 4)) << 4));
#pragma unroll
                for (int j = 0; j < 4; ++j) st_sys(d + j * 1024, v[j]);
            }
#pragma unroll
            for (int j = 0; j < 4; ++j) v[j] = v[4 + j];
        }
        WAIT_VM(0);
        if (X0_PRIO) __builtin_amdgcn_s_setprio(0);
        u64 wsq = (u64)wsl;
        asm volatile("" : "+v"(wsq));
        uint8_t* const wsr = (uint8_t*)(__attribute__((address_space(1))) uint8_t*)wsq;
        if (phase || isx1) st_flags8(lds + LDS_PUB, wave, hdr, wsr, tell && lane < 32, rank, flag, call, lane);
        else if (tell && lane < 32 && (lane & 3) == 0) st_flag(wsr, rank, flag, h0, call, hdr);
        if (keep && lane == 0)
            count_in(myws + cno, (u32)(h1 - h0));
    }
    GET_CALL();
    PUT_CALL();
    STAMP(1);

    const int xcd = g & 7, idx = g >> 3;
    const int part = xcd >> 1;
    const int wm = wave >> 1, wn = wave & 1;
    const int c = lane & 15, q4 = lane >> 4;
    const int ra0 = wm * 8192 + c * 128 + ((q4 ^ (c >> 1)) << 4);
    const int ra1 = ra0 ^ 64;
    const int rb0 = 32768 + (c & 3) * 4096 + ((((q4 << 3) | (4 * wn + (c >> 2))) ^ ((c & 3) << 2)) << 4);
    const u32 boff = (u32)wave * 2048u + (u32)lane * 16u;
    const u32 la0 = (u32)(uintptr_t)lds + wave * 4096, lb0 = (u32)(uintptr_t)lds + 32768 + wave * 2048;
    const u32 aoff0 = (u32)(lane >> 3) * 128u + (u32)(((lane & 7) ^ (lane >> 4)) << 4), aoff1 = aoff0 ^ 64u;

    for (u32 jobs = idx < 3 ? 0x5324 : idx < 8 ? 0x54 : idx < 9 ? 0x4 : idx < 12 ? 0x324 : idx < 15 ? 0x4 : idx < 21 ? 0x432 : 0x41; jobs; jobs >>= 4) {
        const int job = (int)(jobs & 15) - 2;
        const bool help = job < 0;
        const bool pre = job < 2 && !help, local = job == 3, ownrows = pre || local;
        const bool halft = help || (job == 2 && idx < 3);
        const int pi = g < 24 ? g + 48 : g < 96 ? g : g - 120, ft = pi % 12 * 8 + pi / 12;
        const int i8 = pre ? ft & 7 : local ? idx : help ? 0 : idx / 3;
        const int rt = pre ? job : local ? (xcd & 1) : 3 * (xcd & 1) + idx % 3;
        const int ct = pre ? 32 + ft : part * 8 + i8;
        const int nstep = pre ? 4 : NSTEP;
        const u32 wbase = pre ? (u32)WS_GW2 + (u32)(((ft >> 5) - rank) * 128 + (ft & 31)) * 16384u : (u32)WS_GW + (u32)ct * 16384u;
        const int rgw = rt * 8 + (halft ? 2 * wm + help : wave);
        const u32 laj = halft ? la0 - wn * 2048 : la0;
        const int sgw = ownrows ? XGREM + rgw : (rgw < 3 * FG + part * GG ? rgw : rgw + GG);
        const int skipsrc = ownrows ? -1 : group_of(rank, sgw) >> 4;
        const u32 xbase = (u32)(WS_GX + (call & 1) * GX_ODD) + (u32)sgw * 4096u + (halft ? wn * 2048u : 0u);

        const u32* const pollp = lane < 32 ? (const u32*)(myws + WS_WCNT) + (ct * 32 + lane) * 32
                                           : (const u32*)(myws + X_DONE + (ownrows ? WS_XOWN : WS_XREM)) + (lane & 15) * 32;
        const u32 pneed = lane < 32 ? 4u * call : call;
        u64 ready = job == 1 ? ~0ull : 0;
        const int tq = ((lane & 31) - 4) / 7;
        const u32 vst = (lane & 31) < 4 ? rank * 4 + (lane & 3) : ((rank + (lane & 31) - 3 - 7 * tq) & 7) * 4 + tq;
        auto stepno = [&](int s) { return (int)__builtin_amdgcn_readlane(vst, s); };
        auto ensure = [&](int s) {
            const int st = stepno(s);
            const u64 want = (1ull << st) | (1ull << (32 + (st >> 1)));
            if ((ready & want) == want) return;
            const u64 t0 = __builtin_amdgcn_s_memrealtime();
            const u64 about = (st >> 2) == rank ? (0xfull << (4 * rank)) | (3ull << (32 + 2 * rank))
                                                : (0x11111111ull << (st & 3)) | (0x5555ull << (32 + ((st >> 1) & 1)));
            for (u32 it = 1;; ++it) {
                u32 v = 0;
                if (((about & ~ready) >> lane) & 1) v = __hip_atomic_load(pollp, __ATOMIC_RELAXED, __HIP_MEMORY_SCOPE_SYSTEM);
                ready |= __ballot(v >= pneed);
                if ((ready & want) == want) break;
                if ((it & 7) == 0 && (__hip_atomic_load(tmo, __ATOMIC_RELAXED, __HIP_MEMORY_SCOPE_AGENT) == call ||
                                      __builtin_amdgcn_s_memrealtime() - t0 > DEADLINE)) {
                    if (lane == 0) __hip_atomic_store(tmo, call, __ATOMIC_RELAXED, __HIP_MEMORY_SCOPE_AGENT);
                    WAIT_VM(0);
                    ready = ~0ull;
                    break;
                }
                __builtin_amdgcn_s_sleep(POLL_SLEEP);
            }
        };
        bool polling = wave == 0;
        int age = 0;
        u64 asked = 0;
        auto peek_issue = [&](int s) {
            if (age) { ++age; return; }
            const int st = stepno(min(s + 3 + PEEK_AHEAD, nstep - 1));
            const u64 want = (1ull << st) | (1ull << (32 + (st >> 1)));
            if ((ready & want) == want) return;
            asked = 0xffffffffffffull & ~ready;
            if ((asked >> lane) & 1) {
                u32 keep;
                asm volatile("s_mov_b32 %0, m0\n\ts_mov_b32 m0, %2\n\ts_nop 0\n\tglobal_load_lds_dword %1, off sc0 sc1\n\ts_mov_b32 m0, %0"
                             : "=&s"(keep) : "v"(pollp), "s"((u32)(uintptr_t)(lds + LDS_CTL)) : "memory");
            }
            age = 1;
        };
        auto peek_read = [&]() {
            if (age != 3) return;
            if (halft) WAIT_VM(4); else WAIT_VM(6);
            ready |= __ballot(*(volatile lu32*)(lds + LDS_CTL + lane * 4) >= pneed) & asked;
            age = 0;
        };
        const u32 vxo = (int)(vst >> 2) == skipsrc ? (u32)WS_ZERO : xbase + vst * (u32)XSLAB;
        const u32 vwo = wbase + vst * (32u * 16384u);
        const uint8_t *xa, *wa;
        auto locate = [&](int s) {
            xa = myws + (u32)__builtin_amdgcn_readlane(vxo, s);
            wa = myws + (u32)__builtin_amdgcn_readlane(vwo, s);
        };
#define DMA(M0V, VOFF, SRC, OFF) \
        asm volatile("s_mov_b32 m0, %2\n\ts_nop 0\n\tglobal_load_lds_dwordx4 %0, %1 offset:" #OFF " sc0 sc1" \
                     :: "v"(VOFF), "s"(SRC), "s"(M0V) : "memory")
        auto issue = [&](u32 so) {
            DMA(so + laj, aoff0, xa, 0); DMA(so + laj, aoff1, xa, 1024);
            if (!halft) { DMA(so + laj, aoff0, xa, 2048); DMA(so + laj, aoff1, xa, 3072); }
            DMA(so + lb0, boff, wa, 0); DMA(so + lb0, boff, wa, 1024);
        };

        f32x4 acc[4][4];
#pragma unroll
        for (int m = 0; m < 4; ++m)
#pragma unroll
            for (int t = 0; t < 4; ++t) acc[m][t] = f32x4{0.f, 0.f, 0.f, 0.f};

        auto rd1 = [&](const lbyte* p0, const lbyte* p1) {
            const u32x4 l = *(const lu32x4*)p0, h = *(const lu32x4*)p1;
            return i32x8{(int)l[0], (int)l[1], (int)l[2], (int)l[3], (int)h[0], (int)h[1], (int)h[2], (int)h[3]};
        };
#define MM(A, B, m, t) acc[m][t] = __builtin_amdgcn_mfma_scale_f32_16x16x128_f8f6f4(A[m], B[t], acc[m][t], 1, 1, 0, 0, 0, 0)
#define SB() __builtin_amdgcn_sched_barrier(0)
#define GTRACE() do { } while (0)
#define KNEXT(ISSUE) \
            ++s; \
            ma = sn + laj; mb = sn + lb0; \
            sn = sn == 2 * STAGE ? 0 : sn + STAGE; \
            pa0 = lds + sn + ra0; pa1 = lds + sn + ra1; pb = lds + sn + rb0; \
            if (ISSUE) locate(s + 3); \
            asm volatile("" : "+s"(ma), "+s"(mb), "+s"(xa), "+s"(wa), "+v"(pa0), "+v"(pa1), "+v"(pb))
#define KSYNC(ISSUE, NVM) \
            if (ISSUE && age) asm volatile("s_waitcnt vmcnt(%0) lgkmcnt(0)" :: "i"(NVM + 1) : "memory"); \
            else asm volatile("s_waitcnt vmcnt(%0) lgkmcnt(0)" :: "i"(NVM) : "memory"); \
            __builtin_amdgcn_s_barrier(); \
            asm volatile("" ::: "memory")
#define KSTEP(HALF, XA, XB, YA, YB, ISSUE, AHEAD) do { \
            if (ISSUE && polling) { peek_read(); ensure(s + 3); } \
            if (!HALF) { \
                KSYNC(ISSUE, 6 * AHEAD); \
                SB(); MM(XA, XB, 0, 0); SB(); YA[0] = rd1(pa0, pa1); \
                SB(); MM(XA, XB, 0, 1); SB(); YB[0] = rd1(pb, pb + 512); \
                SB(); MM(XA, XB, 0, 2); SB(); if (ISSUE) DMA(ma, aoff0, xa, 0); \
                SB(); MM(XA, XB, 0, 3); SB(); YB[1] = rd1(pb + 1024, pb + 1536); \
                SB(); MM(XA, XB, 1, 0); SB(); if (ISSUE) DMA(ma, aoff1, xa, 1024); \
                SB(); MM(XA, XB, 1, 1); SB(); YB[2] = rd1(pb + 2048, pb + 2560); \
                SB(); MM(XA, XB, 1, 2); SB(); if (ISSUE) DMA(ma, aoff0, xa, 2048); \
                SB(); MM(XA, XB, 1, 3); SB(); YB[3] = rd1(pb + 3072, pb + 3584); \
                SB(); MM(XA, XB, 2, 0); SB(); if (ISSUE) DMA(ma, aoff1, xa, 3072); \
                SB(); MM(XA, XB, 2, 1); SB(); YA[1] = rd1(pa0 + 2048, pa1 + 2048); \
                SB(); MM(XA, XB, 2, 2); SB(); if (ISSUE) DMA(mb, boff, wa, 0); \
                SB(); MM(XA, XB, 2, 3); SB(); YA[2] = rd1(pa0 + 4096, pa1 + 4096); \
                SB(); MM(XA, XB, 3, 0); SB(); if (ISSUE) DMA(mb, boff, wa, 1024); \
                YA[3] = rd1(pa0 + 6144, pa1 + 6144); \
                SB(); MM(XA, XB, 3, 1); SB(); \
                KNEXT(ISSUE); \
                SB(); MM(XA, XB, 3, 2); MM(XA, XB, 3, 3); SB(); \
            } else { \
                KSYNC(ISSUE, 4 * AHEAD); \
                SB(); MM(XA, XB, 0, 0); SB(); if (ISSUE) DMA(ma, aoff0, xa, 0); YA[0] = rd1(pa0, pa1); \
                SB(); MM(XA, XB, 0, 1); SB(); if (ISSUE) DMA(ma, aoff1, xa, 1024); YB[0] = rd1(pb, pb + 512); \
                SB(); MM(XA, XB, 0, 2); SB(); if (ISSUE) DMA(mb, boff, wa, 0); YB[1] = rd1(pb + 1024, pb + 1536); \
                SB(); MM(XA, XB, 0, 3); SB(); if (ISSUE) DMA(mb, boff, wa, 1024); YB[2] = rd1(pb + 2048, pb + 2560); \
                SB(); MM(XA, XB, 1, 0); SB(); YB[3] = rd1(pb + 3072, pb + 3584); \
                SB(); MM(XA, XB, 1, 1); SB(); YA[1] = rd1(pa0 + 2048, pa1 + 2048); \
                KNEXT(ISSUE); \
                SB(); MM(XA, XB, 1, 2); MM(XA, XB, 1, 3); SB(); \
            } \
            if (ISSUE && polling) { peek_issue(s - 1); polling = age || (~ready << 16); } \
            GTRACE(); \
        } while (0)

        if (job == 0 && wave == 0 && idx >= 15) {
            while (__builtin_amdgcn_s_memrealtime() - tstart0 < (u64)(PRE_START + (idx - 15) * PRE_STAG)) __builtin_amdgcn_s_sleep(8);
        }
        if (wave == 0) { ensure(0); ensure(1); ensure(2); }
        asm volatile("s_waitcnt lgkmcnt(0)" ::: "memory");
        __builtin_amdgcn_s_barrier();
        asm volatile("" ::: "memory");
        asm volatile("s_nop 4");
        locate(0); issue(0);
        locate(1); issue(STAGE);
        locate(2); issue(2 * STAGE);
        locate(3);
        if (halft) WAIT_VM(8); else WAIT_VM(12);
        __builtin_amdgcn_s_barrier();
        asm volatile("" ::: "memory");
        u32 tmov;
#define KLOOP(HALF) do { \
            i32x8 A0[4], B0[4], A1[4], B1[4]; \
            _Pragma("unroll") for (int t = 0; t < 4; ++t) { \
                if (!HALF || t < 2) A0[t] = rd1(lds + ra0 + t * 2048, lds + ra1 + t * 2048); \
                B0[t] = rd1(lds + rb0 + t * 1024, lds + rb0 + t * 1024 + 512); \
            } \
            int s = 0; \
            u32 sn = STAGE, ma = laj, mb = lb0; \
            const lbyte *pa0 = lds + STAGE + ra0, *pa1 = lds + STAGE + ra1, *pb = lds + STAGE + rb0; \
            while (s < nstep - 4) { \
                KSTEP(HALF, A0, B0, A1, B1, true, 1); \
                KSTEP(HALF, A1, B1, A0, B0, true, 1); \
            } \
            KSTEP(HALF, A0, B0, A1, B1, true, 1); \
            KSTEP(HALF, A1, B1, A0, B0, false, 1); \
            KSTEP(HALF, A0, B0, A1, B1, false, 0); \
              \
            tmov = __hip_atomic_load(tmo, __ATOMIC_RELAXED, __HIP_MEMORY_SCOPE_AGENT); \
            SB(); \
            _Pragma("unroll") for (int t = 0; t < (HALF ? 8 : 16); ++t) MM(A1, B1, t >> 2, t & 3); \
        } while (0)
        if (halft) KLOOP(true); else KLOOP(false);

        int ln = lane;
        asm volatile("" : "+v"(ln));
        if (local) {
            float scale = tmov == call ? __builtin_nanf("") : P.sx[0] * P.sw[0];
            asm volatile("" : "+v"(scale));
            float* dst = P.out + (size_t)(rt * 256 + wm * 64 + 4 * (ln >> 4)) * NN + block_of(rank, part) * 1024 + i8 * 128 + wn * 64 + 4 * (ln & 15);
#pragma unroll
            for (int m = 0; m < 4; ++m)
#pragma unroll
                for (int r = 0; r < 4; ++r) {
                    f32x4 y;
#pragma unroll
                    for (int t = 0; t < 4; ++t) y[t] = silu(acc[m][t][r] * scale);
                    THROTTLE(LOC_WIN);
                    st_mode<NT_OUT>(dst + (size_t)(m * 16 + r) * NN, __builtin_bit_cast(u32x4, y));
                }
            STAMP(7);
        } else if (pre) {
            const int fb = ft >> 3;
            const int blkf = (rank & 1) ? 4 * (fb / 3) + fb % 3 + (fb % 3 >= (rank >> 1)) : fb + (fb >= 2 * rank ? 4 : 0);
            float* d = P.out + (size_t)(rt * 256 + wm * 64 + 4 * (ln >> 4)) * NN + blkf * 1024 + i8 * 128 + wn * 64 + 4 * (ln & 15);
#pragma unroll
            for (int m = 0; m < 4; ++m)
#pragma unroll
                for (int r = 0; r < 4; ++r) {
                    const f32x4 y = {acc[m][0][r], acc[m][1][r], acc[m][2][r], acc[m][3][r]};
                    st_mode<NT_PRE>(d + (size_t)(m * 16 + r) * NN, __builtin_bit_cast(u32x4, y));
                }
            if (job == 1) {
                WAIT_VM(0);
                __builtin_amdgcn_s_barrier();
                if (tid == 0) __hip_atomic_fetch_add((u32*)(myws + WS_PCNT) + i8 * 32, 1u, __ATOMIC_RELAXED, __HIP_MEMORY_SCOPE_SYSTEM);
                STAMP(7);
            }
        } else {
            const int rl = (ln >> 2) & 7, jl = rl >> 1;
            const int lo = ((rl ^ rank) & 1) ? 3 * FG + (jl > part ? jl - 1 : jl) * GG : FG * ((jl - (rank >> 1) - 1) & 3);
            const int nl = ((rl ^ rank) & 1) ? (jl == part ? 0 : GG) : (rl == rank ? 0 : FG);
            const bool owns = ln < 32 && nl && lo < rt * 8 + 8 && lo + nl > rt * 8;
            u64 qws = (u64)P.ws[rl];
            auto publish = [&](int mp, u32 hdr) {
                WAIT_VM(0);
                st_flags8(lds + LDS_PUB + (halft ? 0 : 64 + mp * 64), wave, hdr, (uint8_t*)qws, owns, rank,
                          FL_O + (ct * 6 + rt) * 2 + mp + help, call, ln);
            };
            float amf = 0.f;
#pragma unroll
            for (int i = 0; i < 64; ++i) amf = fmaxf(amf, fabsf(acc[i >> 4][i & 3][(i >> 2) & 3]));
            const u32 am = wave_umax(__float_as_uint(amf));
            const int ex = min(max((int)(am >> 23), 40), 200);
            u32 votes = 0;
#pragma unroll
            for (int i = 0; i < 8; ++i) {
                const u32 I = (u32)(int)(acc[i >> 1][i & 3][i & 1] * __uint_as_float((u32)(280 - ex) << 23));
                votes += (((I + (1u << 25)) & 0xfc000007u) != 0) + ((((I + (1u << 26)) & 0xf800000fu) != 0) << 10) +
                         (((I & 0x1fu) != 0) << 20);
            }
            votes = wave_sum(votes);
            const u32 n0 = votes & 1023u, n1 = (votes >> 10) & 1023u, n2 = votes >> 20;
            const int dsc = n2 <= n1 && n2 <= n0 ? 2 : n1 <= n0 ? 1 : 0;
            const float up = __uint_as_float((u32)(280 - ex - dsc) << 23);
            u32 hdrs[2];
            asm volatile("" : "+v"(qws), "+v"(tmov));
            const bool bad = __builtin_amdgcn_readfirstlane(tmov) == call;
#pragma unroll
            for (int mp = 0; mp < 2; ++mp) {
                if (mp == 1 && halft) break;
#define ACCV(i) acc[2 * mp + ((i) >> 4)][(i) & 3][((i) >> 2) & 3]
                uint8_t* const slot = myws + pkoff + (size_t)(SL_O + (ct * 48 + rt * 8 + wm * 2 + mp + help) * 2 + wn) * SLOT;
                u32 hi[16], mid[8], q[8], nz = 0, miss = 0;
#pragma unroll
                for (int j = 0; j < 8; ++j) {
                    if (mp == 1 && j == 3) { SB(); publish(0, hdrs[0]); SB(); }
                    u32 t[4];
#pragma unroll
                    for (int t4 = 0; t4 < 4; ++t4) {
                        const float f = ACCV(4 * j + t4) * up;
                        const u32 I = (u32)(int)f;
                        miss |= __float_as_uint(__builtin_amdgcn_fractf(f));
                        t[t4] = __builtin_amdgcn_alignbit(I, I ^ (u32)__builtin_amdgcn_sbfe((int)I, 0, 26), 26);
                    }
                    hi[2 * j] = __builtin_amdgcn_perm(t[1], t[0], 0x07060302u);
                    hi[2 * j + 1] = __builtin_amdgcn_perm(t[3], t[2], 0x07060302u);
                    const u32 l0 = __builtin_amdgcn_perm(t[1], t[0], 0x05010400u), l1 = __builtin_amdgcn_perm(t[3], t[2], 0x05010400u);
                    const u32 lo = __builtin_amdgcn_perm(l1, l0, 0x05040100u);
                    mid[j] = __builtin_amdgcn_perm(l1, l0, 0x07060302u);
                    q[j] = bfi(0x07070707u, bfi(0x03030303u, lo, mid[j] << 2), (lo >> 3) & 0x18181818u);
                    const u32 nb = q[j] + 0x1F1F1F1Fu;
                    nz = bfi(0x01010101u << j, j < 5 ? nb >> (5 - j) : nb << (j - 5), nz);
                }
                const u32 cnt = __builtin_popcount(nz);
                const u32 inc = wave_scan_add(cnt);
                const u32 total = __builtin_amdgcn_readlane(inc, 63);
                if (bad || (int)(am >> 23) > ex || __any(miss != 0) || total > LIST_MAX) {
#pragma unroll
                    for (int j = 0; j < 8; ++j) {
                        f32x4 y = {ACCV(4 * j), ACCV(4 * j + 1), ACCV(4 * j + 2), ACCV(4 * j + 3)};
                        if (bad) y[0] = __builtin_nanf("");
                        st_sys(slot + j * 1024 + ln * 16, __builtin_bit_cast(u32x4, y));
                    }
                    hdrs[mp] = 64u | H_RAW;
                } else {
#pragma unroll
                    for (int j = 0; j < 7; ++j) mid[j] = bfi(0x01010101u, mid[7] >> (j + 1), mid[j]);
#pragma unroll
                    for (int j = 0; j < 4; ++j)
                        st_sys(slot + j * 1024 + ln * 16, u32x4{hi[4 * j], hi[4 * j + 1], hi[4 * j + 2], hi[4 * j + 3]});
                    st_sys(slot + 4096 + ln * 16, u32x4{mid[0], mid[1], mid[2], mid[3]});
                    *(lu32*)(scr + ln * 4) = mid[4];
                    *(lu32*)(scr + 256 + ln * 4) = mid[5];
                    *(lu32*)(scr + 512 + ln * 4) = mid[6];
                    const u32 nb = write_list(q, nz, inc - cnt, total, scr + 768, lds + 2 * STAGE + wave * 4096, ln);
                    for (u32 o = ln * 16; o < 768 + nb; o += 1024) st_sys(slot + 5120 + o, *(lu32x4*)(scr + o));
                    CBAR();
                    hdrs[mp] = ((ACC_MAIN + nb + 127) >> 7) | (nb << 8) | ((u32)(ex + dsc - 30) << 19);
                }
            }
            if (halft) publish(0, hdrs[0]); else publish(1, hdrs[1]);
            STAMP(4);
        }
    }
}

}

extern "C" void kernel_launch(int rank, int nranks, ncclComm_t comm,
                              hipStream_t stream,
                              void* const* d_in, const int* in_sizes, int n_in,
                              void* d_out, int out_size,
                              const int* in_dtypes, int out_dtype,
                              void* const* d_out_all,
                              void* const* d_ws_all, size_t ws_bytes) {
    if (nranks != NR || (unsigned)rank >= (unsigned)NR || n_in != 4 || ws_bytes < WS_NEED ||
        in_sizes[0] != MM * KS || in_sizes[1] != KS * NN || out_size != MS * NN) {
        fprintf(stderr, "kernel_launch: unexpected problem shape\n");
        return;
    }
    Params P;
    P.x = (const uint8_t*)d_in[0];
    P.w = (const uint8_t*)d_in[1];
    P.sx = (const float*)d_in[2];
    P.sw = (const float*)d_in[3];
    P.out = (float*)d_out;
    for (int p = 0; p < NR; ++p) P.ws[p] = (uint8_t*)d_ws_all[p];
    P.myws = P.ws[rank];
    P.rank = rank;
    P.pad = 0;
    hipLaunchKernelGGL(fused_kernel, dim3(GRID), dim3(NTHR), 0, stream, P);
    HIP_CHECK(hipGetLastError());
}
```

```cpp
#include <hip/hip_runtime.h>
#include <rccl/rccl.h>
#include <cstdio>
#include <cstdint>

#define HIP_CHECK(expr) do { hipError_t _e = (expr); if (_e != hipSuccess) { \
    fprintf(stderr, "HIP error %s at %s:%d\n", hipGetErrorString(_e), __FILE__, __LINE__); \
    return; } } while (0)

typedef unsigned u32;
typedef unsigned long long u64;
typedef unsigned u32x4 __attribute__((ext_vector_type(4)));
typedef unsigned u32x2 __attribute__((ext_vector_type(2)));
typedef int i32x8 __attribute__((ext_vector_type(8)));
typedef float f32x4 __attribute__((ext_vector_type(4)));
typedef __attribute__((address_space(1))) const void gvoid;
typedef __attribute__((address_space(3))) void lvoid;
typedef __attribute__((address_space(3))) uint8_t lbyte;
typedef __attribute__((address_space(3))) u32 lu32;
typedef __attribute__((address_space(3))) unsigned short lu16;
typedef __attribute__((address_space(3))) u32x4 lu32x4;
typedef __attribute__((address_space(3))) u32x2 lu32x2;

namespace {

constexpr int NR = 8;
constexpr int MM = 4096, KK = 4096, NN = 16384;
constexpr int KS = KK / NR;
constexpr int MS = MM / NR;
constexpr int FG = 7, GG = 9;
constexpr int F = 32 * FG, G = 32 * GG;
static_assert(F + G == MS, "rows");
constexpr int XGREM = 3 * FG + 4 * GG;
constexpr int XREM = 32 * XGREM;
constexpr int XROWS = XREM + MS;
constexpr int NSTEP = 32;

constexpr int GRID = 256, NSEND = 64, NTHR = 512;
constexpr int NENG = 56;
#define WIN 6
#define WIN_OP 7
#define STARVE 8
#define BATCH 5
#define FCAP 8
#define MARK_AT 30
#define PLAN_GAIN 8
#define SAME_MORE 1500
#define PP_PACE 1
#define W_ORDER 0x652410
#define WB 2
#define MON_SLEEP 4
#define X0_PRIO 2
#define SC_SLEEP 8
#define WK_SLEEP 1
#define WK_PRIO 1
#define KORD 0
#define SC_PRIO 3
#define PLAN_STEP 6
#define LOOK 4
#define NPUSH 5
#define PUSH_WIN 8
#define W_DELAY 500
#define PRE_STAG 100
#define PRE_START 1700
#define LATE_STAG 120
#define LOC_WIN 32
#define PEEK_AHEAD 6
#define W_DELAY0 200
#define NT_PRE 0
#define NT_OUT 0
#define POLL_SLEEP 16
#define I8A 3

constexpr int SLOT = 8192;
constexpr int NPP = 24;
constexpr u32 H_RAW = 1u << 7;
constexpr u32 ACC_MAIN = 5888;
constexpr int SL_X = 0, SL_W = 256, SL_O = SL_W + 1024, SL_F = SL_O + 32 * 48 * 2, SL_P = SL_F + 256, SL_END = SL_P + NR * NPP;
constexpr int FL_X = 0, FL_X1 = 128, FL_W = FL_X1 + 16, FL_L = FL_W + 256, FL_O = FL_L + 32, FL_F = FL_O + 32 * 6 * 2,
              FL_P = FL_F + 64, FL_END = FL_P + NPP;
constexpr u32 FW_SPLIT = 1u << 16, FW_WIDE = 1u << 17;
constexpr u32 FW_RELAYED = 1u << 19;
static_assert(FL_END <= 4096, "flags");

constexpr size_t WS_CNT = 262144;
constexpr size_t WS_ZERO = 0;
constexpr size_t WS_TMO = 32768;
constexpr size_t WS_TICK = 49152;
constexpr size_t WS_XREM = 65536;
constexpr size_t WS_XOWN = WS_XREM + 2048;
constexpr size_t WS_MARK = 53248;
constexpr size_t WS_PLAN = 57344;
constexpr size_t X_DONE = 8192;
constexpr size_t WS_PCNT = WS_XOWN + 2048;
constexpr size_t WS_WCNT = 27u << 20;
constexpr size_t WS_FLAG = 28u << 20;
constexpr size_t WS_DBG = 524288;
constexpr size_t WS_GW = 1u << 20;
constexpr size_t WS_GX = WS_GW + (size_t)NSTEP * 32 * 16384;
constexpr size_t XSLAB = (size_t)XROWS * 128;
constexpr size_t WS_PK = 32u << 20;
constexpr size_t PK_BYTES = 40u << 20;
static_assert(WS_GX + NSTEP * XSLAB <= WS_WCNT && WS_WCNT + 128 * 32 * 128 <= WS_FLAG && WS_FLAG + NR * 4096 * 128 <= WS_PK, "staging");
static_assert((size_t)SL_END * SLOT <= PK_BYTES, "packed");
constexpr size_t GX_ODD = WS_PK + 2 * PK_BYTES - WS_GX;
constexpr size_t WS_GW2 = 124u << 20;
static_assert(WS_GX + GX_ODD + NSTEP * XSLAB <= WS_GW2, "staging");
constexpr size_t WS_NEED = WS_GW2 + (size_t)3 * 128 * 16384;

constexpr int STAGE = 49152;
constexpr int LDS_CTL = 3 * STAGE;
constexpr int LDS_LUT = LDS_CTL + 256;
constexpr int LDS_PUB = LDS_LUT + 128;
constexpr int NS = 12;
constexpr int NPAIR = 3;
constexpr int EL_PLANE = NS * SLOT;
constexpr int EL_BUF = EL_PLANE + 8 * 1024;
constexpr int EL_TAB = EL_BUF + 8 * 2304;
constexpr int EL_GO = EL_TAB + 1024 * 16;
constexpr int EL_FREE = EL_GO + 256;
constexpr int EL_ISS = EL_FREE + 128;
constexpr int EL_FULL = EL_ISS + 4;
constexpr int EL_HEAD = EL_ISS + 8;
constexpr int EL_DEAD = EL_ISS + 12;
constexpr int EL_NEXT = EL_ISS + 16;
constexpr int EL_LIVE = EL_ISS + 64;
constexpr int EL_END = EL_LIVE + 320;
static_assert(EL_END + 1024 <= LDS_CTL && NS % NPAIR == 0 && NS % 2 == 0, "engine LDS");
constexpr u32 UNIT_END = 0xffffu;
#define DEADLINE_TICKS 40000000ull
constexpr u64 DEADLINE = DEADLINE_TICKS;

static_assert(7 * NPUSH <= 112 && NPUSH + NPP <= XGREM - GG, "push");

struct Params {
    const uint8_t* x;
    uint8_t* myws;
    int rank;
    int pad;
    const uint8_t* w;
    const float* sx;
    const float* sw;
    float* out;
    uint8_t* ws[NR];
};

__device__ __forceinline__ void st_sys(void* p, u32x4 v) {
    asm volatile("global_store_dwordx4 %0, %1, off sc0 sc1\n\ts_nop 1" :: "v"(p), "v"(v) : "memory");
}
template <int M> __device__ __forceinline__ void st_mode(void* p, u32x4 v) {
    if (M == 0) asm volatile("global_store_dwordx4 %0, %1, off sc0 sc1\n\ts_nop 1" :: "v"(p), "v"(v) : "memory");
    if (M == 1) asm volatile("global_store_dwordx4 %0, %1, off sc0 sc1 nt\n\ts_nop 1" :: "v"(p), "v"(v) : "memory");
    if (M == 2) asm volatile("global_store_dwordx4 %0, %1, off nt\n\ts_nop 1" :: "v"(p), "v"(v) : "memory");
    if (M == 3) asm volatile("global_store_dwordx4 %0, %1, off sc1\n\ts_nop 1" :: "v"(p), "v"(v) : "memory");
}
#define WAIT_VM(n) asm volatile("s_waitcnt vmcnt(" #n ")" ::: "memory")
#define THROTTLE(n) asm volatile("s_waitcnt vmcnt(%0)" :: "i"((n) - 1) : "memory")
#define CBAR() asm volatile("" ::: "memory")
#define DPP(v, ctl, rm) (u32)__builtin_amdgcn_update_dpp(0, (int)(v), ctl, rm, 0xf, false)

__device__ __forceinline__ u32 wave_scan_add(u32 v) {
    v += DPP(v, 0x111, 0xf); v += DPP(v, 0x112, 0xf); v += DPP(v, 0x114, 0xf); v += DPP(v, 0x118, 0xf);
    v += DPP(v, 0x142, 0xa); v += DPP(v, 0x143, 0xc);
    return v;
}
__device__ __forceinline__ u32 wave_sum(u32 v) { return __builtin_amdgcn_readlane(wave_scan_add(v), 63); }
__device__ __forceinline__ u32 wave_umax(u32 v) {
    v = max(v, DPP(v, 0x111, 0xf)); v = max(v, DPP(v, 0x112, 0xf)); v = max(v, DPP(v, 0x114, 0xf));
    v = max(v, DPP(v, 0x118, 0xf)); v = max(v, DPP(v, 0x142, 0xa)); v = max(v, DPP(v, 0x143, 0xc));
    return __builtin_amdgcn_readlane(v, 63);
}
__device__ __forceinline__ int block_of(int r, int part) {
    return (r & 1) ? 4 * part + (r >> 1) : 4 * (r >> 1) + part;
}
__device__ __forceinline__ float silu(float v) {
    const float z = __builtin_amdgcn_fmed3f(v, -60.f, 60.f);
    const float e = __builtin_amdgcn_exp2f(z * -1.4426950408889634f);
    return v * __builtin_amdgcn_rcpf(1.0f + e);
}

constexpr u32 LIST_MAX = 960;
__device__ __forceinline__ u32 plane_addr(u32 t) { return ((t & 4u) << 8) | ((t >> 5) << 4) | ((t & 3u) << 2) | ((t >> 3) & 3u); }
__device__ __forceinline__ u32 write_list(const u32 (&q)[8], u32 nz, u32 pos, u32 total,
                                          lbyte* lst, lbyte* tmp, int lane) {
    *(lu32x4*)(tmp + lane * 16) = u32x4{q[0], q[1], q[2], q[3]};
    *(lu32x4*)(tmp + 1024 + lane * 16) = u32x4{q[4], q[5], q[6], q[7]};
    for (u32 o = lane * 16; o < 2 * total + 32; o += 1024) *(lu32x4*)(lst + o) = u32x4{0xc0c0c0c0u, 0xc0c0c0c0u, 0xc0c0c0c0u, 0xc0c0c0c0u};
    lbyte* const pl = tmp + 2048;
    if (lane == 0) *(lu16*)(pl + 2) = 0xffffu;
    lbyte* wp = pl + 4 + 2 * pos;
    for (u32 m = nz; m; m &= m - 1) {
        *(lu16*)wp = (unsigned short)(((u32)lane << 5) | __builtin_ctz(m));
        wp += 2;
    }
    CBAR();
    u32 nb = 0;
    for (u32 i0 = 0; i0 < total; i0 += 128) {
        const u32 i = i0 + 2 * lane;
        const u32 pa = *(lu32*)(pl + 2 * i), pb = *(lu32*)(pl + 2 * i + 4);
        const u32 t[3] = {pa >> 16, pb & 0xffffu, pb >> 16};
        u32 n[2], e[2], val[2];
#pragma unroll
        for (int h = 0; h < 2; ++h) {
            val[h] = *(tmp + plane_addr(t[h + 1] & 2047u));
            const u32 gap = (t[h + 1] - t[h] - 1) & 0xffffu, ty = min(val[h] - 3u, 2u);
            e[h] = (ty << 6) | (gap & 63u);
            n[h] = i + h < total ? (gap >> 6) + 1 + (ty >> 1) : 0;
        }
        const u32 inc = wave_scan_add(n[0] + n[1]);
        lbyte* const end = lst + nb + inc;
#pragma unroll
        for (int h = 0; h < 2; ++h) {
            lbyte* const le = end - (h ? 0 : n[1]) - 1;
            if (n[h]) {
                *(le - (e[h] >> 7)) = (uint8_t)e[h];
                if (e[h] >> 7) *le = (uint8_t)(0xe0u | val[h]);
            }
        }
        nb += __builtin_amdgcn_readlane(inc, 63);
    }
    CBAR();
    return nb;
}
__device__ __forceinline__ void st_flags8(lbyte* pub, int word, u32 hdr, uint8_t* ws, bool to, int src, int line,
                                          u32 call, int lane) {
    u32 n = 0;
    if (lane == 0) {
        *(lu32*)(pub + word * 4) = hdr;
        CBAR();
        n = __hip_atomic_fetch_add((lu32*)(pub + 32), 1u, __ATOMIC_RELAXED, __HIP_MEMORY_SCOPE_WORKGROUP);
    }
    if (__builtin_amdgcn_readfirstlane(n) != 7) return;
    CBAR();
    const u32x2 h = *(lu32x2*)(pub + (lane & 3) * 8);
    if (to) st_sys(ws + WS_FLAG + (size_t)(src * 4096 + line) * 128 + (call & 1) * 64 + (lane & 3) * 16, u32x4{call, h[0], call, h[1]});
}
__device__ __forceinline__ void st_flag(uint8_t* ws, int src, int flag, int half, u32 call, u32 hdr) {
    __hip_atomic_store((u64*)(ws + WS_FLAG) + (size_t)(src * 4096 + flag) * 16 + (call & 1) * 2 + half,
                       ((u64)hdr << 32) | call, __ATOMIC_RELAXED, __HIP_MEMORY_SCOPE_SYSTEM);
}
__device__ __forceinline__ void count_in(uint8_t* cnt, u32 n) {
    __hip_atomic_fetch_add((u32*)cnt, n, __ATOMIC_RELAXED, __HIP_MEMORY_SCOPE_SYSTEM);
}
__device__ __forceinline__ u32 bfi(u32 m, u32 a, u32 b) { return (a & m) | (b & ~m); }

__device__ __forceinline__ void op_lut_init(lbyte* lut, int m) {
    u32 e = 0, c = 0x0c0c0c0cu, n = 0;
#pragma unroll
    for (u32 i = 0; i < 4; ++i) {
        const bool on = (m >> i) & 1;
        e |= (on ? n : 0x0cu) << (8 * i);
        if (on) c = (c & ~(0xffu << (8 * n))) | (i << (8 * n));
        n += on;
    }
    *(lu32*)(lut + m * 4) = e;
    *(lu32*)(lut + 64 + m * 4) = c;
}
__device__ __forceinline__ u32 and_or(u32 a, u32 m, u32 b) {
    u32 r; asm("v_and_or_b32 %0, %1, %2, %3" : "=v"(r) : "v"(a), "s"(m), "v"(b)); return r;
}
__device__ __forceinline__ u32 bfi_s(u32 m, u32 a, u32 b) {
    u32 r; asm("v_bfi_b32 %0, %1, %2, %3" : "=v"(r) : "s"(m), "v"(a), "v"(b)); return r;
}
__device__ __forceinline__ u32 ff_count(u32 ff, u32 acc) { return (u32)__builtin_amdgcn_sdot4((int)ff, -1, (int)acc, true); }
__device__ __forceinline__ u32 ff_lut(u32 ff, u32 lut) {
    return *(const lu32*)(uintptr_t)(u32)__builtin_amdgcn_sdot4((int)ff, (int)0xe0f0f8fcu, (int)lut, true);
}
__device__ __forceinline__ u32 ge13(u32 x) { return __builtin_amdgcn_perm(0u, 0u, x); }
__device__ __forceinline__ void lds_or(u32 p, u32 v) {
    __hip_atomic_fetch_or((lu32*)(uintptr_t)p, v, __ATOMIC_RELAXED, __HIP_MEMORY_SCOPE_WAVEFRONT);
}
__device__ __forceinline__ u32 chunk_starts(const u32 (&c)[4], u32 (&p)[4], u32 at) {
    const u32 a = c[0] | (c[1] << 16), b = c[2] | (c[3] << 16);
    const u32 ia = wave_scan_add(a), ib = wave_scan_add(b);
    const u32 ta = __builtin_amdgcn_readlane(ia, 63), tb = __builtin_amdgcn_readlane(ib, 63);
    const u32 t1 = ta & 0xffffu, t2 = t1 + (ta >> 16), t3 = t2 + (tb & 0xffffu);
    p[0] = ((ia - a) & 0xffffu) + at;
    p[1] = ((ia - a) >> 16) + (at + t1);
    p[2] = ((ib - b) & 0xffffu) + (at + t2);
    p[3] = ((ib - b) >> 16) + (at + t3);
    return t3 + (tb >> 16);
}
__device__ __forceinline__ u32 op_pack(const u32x4* v, u32 base4, uint8_t* blk, bool paced, lbyte* scr, const lbyte* lutp, int lane) {
    lbyte* const side = scr + 2560;
    const u32 lut = (u32)(uintptr_t)lutp + 64, eb = (u32)(uintptr_t)scr + 16;
    const u32x4 z4 = {0u, 0u, 0u, 0u};
    *(lu32x4*)(scr + lane * 16) = z4;
    *(lu32x4*)(scr + 1024 + lane * 16) = z4;
    *(lu32x4*)(scr + 2048 + lane * 16) = z4;
    *(lu32x4*)(scr + 3072 + lane * 16) = z4;
    u32 ub[16], ef[16], sel[16], N[8], S[2] = {0u, 0u}, cnt[4] = {0u, 0u, 0u, 0u}, pos[4];
#pragma unroll
    for (int i = 0; i < 16; ++i) {
        const u32 d = v[i >> 2][i & 3];
        ub[i] = (d | 0x80808080u) - base4;
        ef[i] = ge13(and_or(ub[i], 0x7C7C7C7Cu, 0x01010101u));
        sel[i] = ff_lut(ef[i], lut);
        const u32 x = and_or(ef[i], 0x0C0C0C0Cu, ub[i]);
        N[i >> 1] = (i & 1) ? bfi_s(0x0F0F0F0Fu, N[i >> 1], x << 4) : x;
        S[i >> 3] = and_or(d >> (7 - (i & 7)), 0x01010101u << (i & 7), S[i >> 3]);
        cnt[i >> 2] = ff_count(ef[i], cnt[i >> 2]);
    }
    const u32 n2 = chunk_starts(cnt, pos, eb - 1), n2w = (n2 + 15) >> 4;
    bool raw = n2 > 2048;
    u32 n3 = 0;
    if (!raw) {
        st_sys(blk + lane * 16, u32x4{N[0], N[1], N[2], N[3]});
        if (!paced) st_sys(blk + 1024 + lane * 16, u32x4{N[4], N[5], N[6], N[7]});
        CBAR();
        *(lu32x2*)(side + lane * 8) = u32x2{S[0], S[1]};
#pragma unroll
        for (int i = 0; i < 16; ++i) {
            const u32 c = __builtin_amdgcn_perm(0u, ub[i], sel[i]), p = pos[i >> 2];
            lu32* const q = (lu32*)(uintptr_t)(p & ~3u);
            __hip_atomic_fetch_or(q, __builtin_amdgcn_alignbyte(c, 0u, ~p), __ATOMIC_RELAXED, __HIP_MEMORY_SCOPE_WAVEFRONT);
            __hip_atomic_fetch_or(q + 1, __builtin_amdgcn_alignbyte(0u, c, ~p), __ATOMIC_RELAXED, __HIP_MEMORY_SCOPE_WAVEFRONT);
            pos[i >> 2] = ff_count(ef[i], p);
        }
        CBAR();
        const u32 l3 = (u32)(uintptr_t)side + 512 + n2w * 4;
        u32 bad = 0;
        for (u32 w0 = 0; w0 < n2w; w0 += 64) {
            const u32x4 B = *(lu32x4*)(scr + 16 + (w0 + lane) * 16);
            u32 W = 0, c3 = 0;
            u64 str = 0;
#pragma unroll
            for (int k = 0; k < 4; ++k) {
                const u32 u = (((B[k] & 0x7C7C7C7Cu) + 0x08080808u) >> 2) & 0x1F1F1F1Fu;
                const u32 code = __builtin_amdgcn_perm(0x83038200u, 0x80008100u, u) & 0x03030303u;
                W |= __builtin_amdgcn_udot4(code, 0x40100401u, 0u, false) << (8 * k);
                bad |= (u + 0x78787878u) & ~(u + 0x6E6E6E6Eu);
                const u32 e = __builtin_amdgcn_perm(0u, 0xff000000u, code);
                const u32 sym = u & (and_or(ge13(u), 0x0E0E0E0Eu, 0x01010101u));
                const u32 c = __builtin_amdgcn_perm(0u, sym, ff_lut(e, lut));
                str |= (u64)__builtin_amdgcn_perm(0u, c | (c >> 4), 0x0c0c0200u) << c3;
                c3 = (u32)__builtin_amdgcn_sdot4((int)e, (int)0xfcfcfcfcu, (int)c3, true);
            }
            const u32 i3 = wave_scan_add(c3);
            const u32 o3 = n3 + i3 - c3;
            n3 += __builtin_amdgcn_readlane(i3, 63);
            if (n2w * 4 + (n3 >> 3) > 1000) { raw = true; break; }
            if (w0 + lane < n2w) *(lu32*)(side + 512 + (w0 + lane) * 4) = W;
            const u64 lo = (u64)(u32)str << (o3 & 31u), hi = (str >> 32) << (o3 & 31u);
            const u32 p = l3 + ((o3 >> 5) << 2);
            if (c3) lds_or(p, (u32)lo);
            if ((o3 & 31u) + c3 > 32) {
                lds_or(p + 4, (u32)(lo >> 32) | (u32)hi);
                lds_or(p + 8, (u32)(hi >> 32));
            }
        }
        raw = raw || __any(bad & 0x80808080u);
        CBAR();
    }
    if (raw) {
        WAIT_VM(0);
#pragma unroll
        for (int j = 0; j < 4; ++j) st_sys(blk + j * 1024 + lane * 16, v[j]);
        return 31u;
    }
    if (paced) {
        WAIT_VM(0);
        st_sys(blk + 1024 + lane * 16, u32x4{N[4], N[5], N[6], N[7]});
        if (PP_PACE) WAIT_VM(0);
    }
    const u32 nb = 512 + n2w * 4 + ((n3 + 7) >> 3);
    for (u32 o = lane * 16; o < nb; o += 1024) st_sys(blk + 2048 + o, *(lu32x4*)(side + o));
    CBAR();
    return ((2048 + nb - 1) >> 7) | (n2w << 5);
}
template <class Done>
__device__ __forceinline__ void op_unpack(u32x4* v, u32 base4, u32 n2w, const lbyte* blk, lbyte* buf, const lbyte* lutp,
                                          int lane, Done done) {
    const u32 lut = (u32)(uintptr_t)lutp;
    const u32x4 N0 = *(const lu32x4*)(blk + lane * 16), N1 = *(const lu32x4*)(blk + 1024 + lane * 16);
    const u32x2 S = *(const lu32x2*)(blk + 2048 + lane * 8);
    const lbyte* const l3 = blk + 2560 + n2w * 4;
    u32 n3 = 0;
    for (u32 w0 = 0; w0 < n2w; w0 += 64) {
        u32 W = *(const lu32*)(blk + 2560 + (w0 + lane) * 4);
        if (w0 + lane >= n2w) W = 0;
        const u32 c3t = __builtin_popcount(W & (W >> 1) & 0x55555555u);
        const u32 i3 = wave_scan_add(c3t);
        const u32 o3 = n3 + i3 - c3t;
        n3 += __builtin_amdgcn_readlane(i3, 63);
        const lbyte* const p = l3 + ((o3 >> 3) << 2);
        const u32 d0 = *(const lu32*)p, d1 = *(const lu32*)(p + 4), d2 = *(const lu32*)(p + 8);
        const u64 str = ((u64)__builtin_amdgcn_alignbit(d2, d1, o3 << 2) << 32) | __builtin_amdgcn_alignbit(d1, d0, o3 << 2);
        u32x4 B;
        u32 c3 = 0;
#pragma unroll
        for (int k = 0; k < 4; ++k) {
            const u32 f = (W >> (8 * k)) & 255u;
            const u32 t = (f | (f << 12)) & 0x000F000Fu;
            const u32 code = (t | (t << 6)) & 0x03030303u;
            const u32 e = __builtin_amdgcn_perm(0u, 0xff000000u, code);
            const u32 x = __builtin_amdgcn_perm(0u, (u32)(str >> c3), 0x01010000u);
            const u32 s4 = bfi_s(0x00FF00FFu, x << 2, x >> 2) & 0x3C3C3C3Cu;
            const u32 u4 = s4 + and_or(ge13(s4 + 0x05050505u), 0x28282828u, 0x04040404u);
            B[k] = __builtin_amdgcn_perm(0u, u4, ff_lut(e, lut)) | __builtin_amdgcn_perm(0u, 0x0000706Cu, code);
            c3 = (u32)__builtin_amdgcn_sdot4((int)e, (int)0xfcfcfcfcu, (int)c3, true);
        }
        *(lu32x4*)(buf + (w0 + lane) * 16) = B;
    }
    done();
    u32 nib[16], ef[16], sel[16], cnt[4] = {0u, 0u, 0u, 0u}, pos[4];
#pragma unroll
    for (int i = 0; i < 16; ++i) {
        const u32 n = i < 8 ? N0[i >> 1] : N1[(i >> 1) - 4];
        nib[i] = ((i & 1) ? n >> 4 : n) & 0x0F0F0F0Fu;
        ef[i] = ge13(nib[i] | 0x01010101u);
        sel[i] = ff_lut(ef[i], lut);
        cnt[i >> 2] = ff_count(ef[i], cnt[i >> 2]);
    }
    chunk_starts(cnt, pos, (u32)(uintptr_t)buf);
    CBAR();
    u32 ps[16], q0[16], q1[16];
#pragma unroll
    for (int i = 0; i < 16; ++i) {
        const int j = (i & 3) * 4 + (i >> 2);
        ps[j] = (j & 3) ? ff_count(ef[j - 1], ps[j - 1]) : pos[j >> 2];
        const lu32* const q = (const lu32*)(uintptr_t)(ps[j] & ~3u);
        q0[j] = q[0]; q1[j] = q[1];
    }
    __builtin_amdgcn_sched_barrier(0);
#pragma unroll
    for (int i = 0; i < 16; ++i) {
        const u32 y = nib[i] + base4 + __builtin_amdgcn_perm(0u, __builtin_amdgcn_alignbyte(q1[i], q0[i], ps[i]), sel[i]);
        v[i >> 2][i & 3] = bfi_s(0x7F7F7F7Fu, y, S[i >> 3] << (7 - (i & 7)));
    }
    CBAR();
}

__device__ __forceinline__ int group_of(int r, int sg) {
    if (sg < 3 * FG) {
        const int s = sg / FG;
        return 16 * (2 * (((r >> 1) + 1 + s) & 3) + (r & 1)) + sg - s * FG;
    }
    if (sg < XGREM) {
        const int j = (sg - 3 * FG) / GG;
        return 16 * (2 * j + 1 - (r & 1)) + FG + sg - 3 * FG - j * GG;
    }
    return 16 * r + sg - XGREM;
}

__device__ __forceinline__ int sent_group(int r, int p, int i) {
    const bool same = ((r ^ p) & 1) == 0;
    const int at = same ? FG * (((p >> 1) - (r >> 1) - 1) & 3) : 3 * FG + GG * (p >> 1);
    return i < at ? i : i + (same ? FG : GG);
}

struct Unit {
    u32 slot;
    u32 flag;
    u32 dst;
    u32 kind;
    u32 aux;
};
__device__ __forceinline__ int operand_units(bool same) {
    return 2 * (XGREM - (same ? FG : GG)) - NPUSH - NPP + 256 + 32 + (same ? 0 : 48);
}
__device__ __forceinline__ Unit unit_of(int n, int r, int p) {
    const bool same = ((r ^ p) & 1) == 0;
    const int nx = XGREM - (same ? FG : GG);
    const int U1 = nx - NPUSH - NPP;
    const int U2 = U1 + 16 * WB;
    const int U3 = U2 + nx;
    const int U4 = U3 + 16 * (2 * I8A - WB);
    const int UA = U4 + 32 * (5 - I8A);
    const int UB = UA + 32;
    const int U5 = UB + 32;
    const int UC = U5 + 32;
    const int UD = UC + 32;
    const int U6 = UD + (same ? 0 : 48);
    Unit u;
    if (n >= U6) {
        const int v = n - U6, wn = v & 1, h = v >> 1;
        int i8, part, rg0, j;
        if (same) {
            i8 = h / (4 * FG);
            const int t = h - i8 * (4 * FG);
            part = t & 3; j = t >> 2;
            rg0 = FG * (((r >> 1) - (p >> 1) - 1) & 3);
        } else {
            i8 = h / (3 * GG);
            const int t = h - i8 * (3 * GG), jr = r >> 1;
            j = t / 3;
            part = t - 3 * j + (t - 3 * j >= jr);
            rg0 = 3 * FG + (jr > part ? jr - 1 : jr) * GG;
        }
        const int e0 = rg0 & 1, ne = ((same ? FG : GG) - e0 + 1) >> 1;
        const int g = j < ne ? e0 + 2 * j : 1 - e0 + 2 * (j - ne);
        const int rg = rg0 + g, lg = same ? g : FG + g;
        const int id = ((part * 8 + i8) * 48 + rg) * 2 + wn, gct = block_of(p, part) * 8 + i8;
        u.slot = SL_O + id; u.kind = 2;
        u.flag = ((FL_O + ((part * 8 + i8) * 6 + (rg >> 3)) * 2 + (rg & 1)) * 16 + (rg & 6) + wn) | FW_WIDE;
        u.dst = (u32)(lg * 32 * NN + gct * 128 + wn * 64) * 4u;
        u.aux = i8;
        return u;
    }
    if (n < U1 || (n >= U2 && n < U3) || (n >= U5 && n < UC)) {
        int sg, kp;
        if (n < U1) { sg = sent_group(r, p, n + NPUSH + NPP); kp = 0; }
        else if (n < U3) { sg = sent_group(r, p, n - U2); kp = 1; }
        else { sg = XGREM + ((n - U5) & 15); kp = (n - U5) >> 4; }
        const int gg = group_of(r, sg);
        u.slot = SL_X + gg * 2 + kp; u.kind = 0;
        u.flag = kp ? ((FL_X1 + (gg >> 3)) * 16 + (gg & 7)) | FW_WIDE : ((FL_X + gg) * 16) | FW_SPLIT;
        u.dst = (u32)(WS_GX + (size_t)(p * 4 + 2 * kp) * XSLAB) + (u32)sg * 4096u;
        u.aux = (u32)(sg < XGREM ? WS_XREM : WS_XOWN) + (u32)(p * 2 + kp) * 128u;
        return u;
    }
    int i8, kh, part, src = p;
    if (n < U4) {
        const int v = n < U2 ? n - U1 : n - U3 + 16 * WB, o = (W_ORDER >> (v >> 4 << 2)) & 15;
        i8 = o & 3; kh = (o & 4) + ((v >> 2) & 3); part = v & 3;
    }
    else if (n < UA) { const int v = n - U4; i8 = I8A + (v >> 5); kh = (v >> 2) & 7; part = v & 3; }
    else if (n < UD) {
        const int v = (n - UA) & 31;
        i8 = n < UB ? 7 - same : n < U5 ? 5 : 6 + same; kh = v >> 2; part = v & 3;
    } else {
        const int v = n - UD;
        kh = v / 6; part = p >> 1;
        src = v - 6 * kh;
        src += src >= min(r, p);
        src += src >= max(r, p);
        i8 = 7 - ((src ^ r) & 1);
    }
    const int kc = kh >> 1, half = kh & 1, ct = part * 8 + i8, step = src * 4 + kc;
    u.slot = SL_W + ((kc * 128 + block_of(r, part) * 8 + i8) * 2 + half);
    u.flag = i8 >= I8A ? ((FL_L + ct) * 16 + kh) | FW_WIDE : ((FL_W + (kc * 32 + ct) * 2 + half) * 16) | (kc < 2 ? FW_SPLIT : 0u);
    u.kind = 1 | (half << 2);
    if (n >= UA && n < UB && (same || part != (p >> 1)) && kh >= 8 - FCAP) u.kind |= 8 | (kh << 4) | (part << 7);
    if (n >= UD) { u.slot = SL_F + ((r >> 1) * 8 + src) * 8 + kh; u.flag = ((FL_F + src * 8 + kh) * 16) | FW_SPLIT | FW_RELAYED; }
    if (n >= UC) u.flag |= (kh << 20) | ((part * 8 + src) << 23);
    u.dst = (u32)WS_GW + (u32)(step * 32 + ct) * 16384u + (u32)half * 8192u;
    u.aux = (u32)WS_WCNT + (u32)(ct * 32 + step) * 128u;
    return u;
}

#define GAVEUP(site, info) do { } while (0)
#define STAMP(k) do { } while (0)

__global__ void __launch_bounds__(NTHR, 2) fused_kernel(Params P) {
    __shared__ __attribute__((aligned(1024))) uint8_t lds_[LDS_CTL + 576];
    lbyte* const lds = (lbyte*)lds_;
    const int tid = threadIdx.x;
    const int lane = tid & 63;
    const int wave = __builtin_amdgcn_readfirstlane(tid >> 6);
    const int b = blockIdx.x;
    const int rank = P.rank;
    uint8_t* const myws = P.myws;
    const uint8_t* const xin = P.x;
    const uint8_t* const win = P.w;
    asm volatile("" :: "s"(rank), "s"(myws), "s"(xin), "s"(win));
    u32* const tmo = (u32*)(myws + WS_TMO);

    STAMP(0);
    u32* const cntp = (u32*)(myws + WS_CNT + (size_t)(b * 8 + wave) * 128);
#define LOAD_CALL() \
    const u32 ncalls = __hip_atomic_load(cntp, __ATOMIC_RELAXED, __HIP_MEMORY_SCOPE_AGENT)
#define GET_CALL() \
    const u32 call = __builtin_amdgcn_readfirstlane(ncalls) + 1; \
    const size_t pkoff = WS_PK + (size_t)(call & 1) * PK_BYTES
#define PUT_CALL() \
    if (lane == 0) __hip_atomic_store(cntp, call, __ATOMIC_RELAXED, __HIP_MEMORY_SCOPE_AGENT)

    if (b < NSEND) {
        if (b >= NENG) {
            if (b != NENG || wave > 1) return;
            LOAD_CALL();
            GET_CALL();
            PUT_CALL();
            if (wave == 1) {
                const u64 t0 = __builtin_amdgcn_s_memrealtime();
                const int ls = lane >> 3;
                u64 mv;
                for (u32 it = 1;; ++it) {
                    mv = __hip_atomic_load((const u64*)(myws + WS_MARK) + ls * 16 + (lane & 7), __ATOMIC_RELAXED, __HIP_MEMORY_SCOPE_AGENT);
                    if (__all(ls == rank || (u32)mv == call)) break;
                    if ((it & 7) == 0 && (__hip_atomic_load(tmo, __ATOMIC_RELAXED, __HIP_MEMORY_SCOPE_AGENT) == call ||
                                          __builtin_amdgcn_s_memrealtime() - t0 > DEADLINE)) {
                        { const u32 gi_ = (u32)__ballot(ls == rank || (u32)mv == call); GAVEUP(1, gi_); }
                        if (lane == 0) __hip_atomic_store(tmo, call, __ATOMIC_RELAXED, __HIP_MEMORY_SCOPE_AGENT);
                        break;
                    }
                    __builtin_amdgcn_s_sleep(16);
                }
                constexpr int UNIT = 78;
                int ld = min((u32)(mv >> 32) - (u32)t0, 1u << 20);
                ld += __shfl_xor(ld, 1); ld += __shfl_xor(ld, 2); ld += __shfl_xor(ld, 4);
                ld = ld * PLAN_GAIN / 8 + (((ls ^ rank) & 1) ? 0 : SAME_MORE);
                const int ps = lane & 7, pr = 2 * ((lane >> 3) & 3) + 1 - (rank & 1);
                int lsrc = __shfl(ld, ps * 8), lrel = __shfl(ld, pr * 8);
                u32 f = 0;
                const bool can = lane < 32 && ps != rank && ps != pr;
                for (int it = 0; it < 48; ++it) {
                    const u32 best = wave_umax(can && f < FCAP && lsrc - lrel > UNIT ? ((u32)(lsrc - lrel) << 5) | (u32)lane : 0u);
                    if (!best) break;
                    const int bs = best & 7, br = 2 * ((best >> 3) & 3) + 1 - (rank & 1);
                    const u32 n = min((best >> 5) / (PLAN_STEP * UNIT) + 1, FCAP - __builtin_amdgcn_readlane(f, best & 31u));
                    const int mvd = n * UNIT;
                    f += (u32)lane == (best & 31u) ? n : 0;
                    lsrc += ps == br ? mvd : ps == bs ? -mvd : 0;
                    lrel += pr == br ? mvd : pr == bs ? -mvd : 0;
                }
                if (lane < 32) __hip_atomic_store((u64*)(myws + WS_PLAN) + lane, ((u64)f << 32) | call, __ATOMIC_RELAXED, __HIP_MEMORY_SCOPE_AGENT);
                STAMP(1);
                return;
            }
            const int src = (lane >> 1) & 7;
            const u32 need = call * (lane >= 16 ? 32u : src == rank ? 2u * XGREM : 2u * (XGREM - (((src ^ rank) & 1) ? GG : FG)));
            uint8_t* const cnt = myws + WS_XREM + (lane & 31) * 128;
            const u64 t0 = __builtin_amdgcn_s_memrealtime();
            bool open = lane < 32;
            for (u32 it = 1; __any(open); ++it) {
                if (open && __hip_atomic_load((const u32*)cnt, __ATOMIC_RELAXED, __HIP_MEMORY_SCOPE_SYSTEM) >= need) {
                    __hip_atomic_store((u32*)(cnt + X_DONE), call, __ATOMIC_RELAXED, __HIP_MEMORY_SCOPE_SYSTEM);
                    open = false;
                }
                if ((it & 7) == 0 && (__hip_atomic_load(tmo, __ATOMIC_RELAXED, __HIP_MEMORY_SCOPE_AGENT) == call ||
                                      __builtin_amdgcn_s_memrealtime() - t0 > DEADLINE)) {
                        { const u32 gi_ = (u32)__ballot(open); GAVEUP(2, gi_); }
                    if (lane == 0) __hip_atomic_store(tmo, call, __ATOMIC_RELAXED, __HIP_MEMORY_SCOPE_AGENT);
                    break;
                }
                __builtin_amdgcn_s_sleep(MON_SLEEP);
            }
            return;
        }
        LOAD_CALL();
        const int p = (rank + 1 + b % 7) & 7;
        const bool same = ((rank ^ p) & 1) == 0;
        const int nop = operand_units(same);
        const int nend = nop + (same ? 64 * FG : 48 * GG);
        for (int n = tid; n < nend + NPP; n += NTHR) {
            const Unit u = unit_of(n < nend ? n : n - nend - NPP, rank, p);
            *(lu32x4*)(lds + EL_TAB + n * 16) = u32x4{u.slot | (u.kind << 16), u.flag, u.dst, u.aux};
        }
        if (tid < (EL_END - EL_GO) / 4) {
            const int fo = EL_GO + tid * 4 - EL_FREE;
            *(lu32*)(lds + EL_GO + tid * 4) = fo >= (NS - NPAIR) * 8 && fo < NS * 8 ? (u32)-NS : 0u;
        }
        *(lu32x4*)(lds + EL_PLANE + tid * 16) = u32x4{0u, 0u, 0u, 0u};
        if (tid < 16) op_lut_init(lds + LDS_LUT, tid);
        __syncthreads();
        GET_CALL();
        PUT_CALL();
        auto lds_get = [&](int off) {
            return (u32)__builtin_amdgcn_readfirstlane(
                __hip_atomic_load((lu32*)(lds + off), __ATOMIC_RELAXED, __HIP_MEMORY_SCOPE_WORKGROUP));
        };
        auto lds_put = [&](int off, u32 v) {
            CBAR();
            if (lane == 0) __hip_atomic_store((lu32*)(lds + off), v, __ATOMIC_RELAXED, __HIP_MEMORY_SCOPE_WORKGROUP);
            CBAR();
        };

        if (wave == 0) {
            __builtin_amdgcn_s_setprio(3);
            __builtin_amdgcn_s_waitcnt(0);
            const uint8_t* const pk = P.ws[p] + pkoff;
            const u32 lds_addr = (u32)(uintptr_t)lds;
            const u32 vo0 = lane * 16, vo1 = lane * 16 + 4096;
            static_assert(WIN >= 5 && WIN <= 9 && WIN_OP >= 5 && WIN_OP <= 9, "window");
#define F_WAIT(W, J) "s_waitcnt vmcnt(" #W "-1)\n\t.if (" #W "+5)%%6 == " #J "\n\ts_mov_b64 exec, 1\n\tds_write_b64 %8, %9\n\ts_mov_b64 exec, -1\n\t.endif\n\t"
#define F_LD(V, OFF) "global_load_lds_dwordx4 " V ", %2 offset:" #OFF " sc0 sc1\n\t"
#define F_ARGS :: "v"(vo0), "v"(vo1), "s"(src), "s"(la), "s"(la + 4096), "s"(mk0), "s"(mk1), "n"(0), "v"(lds_addr + EL_ISS), "v"(st) : "memory"
#define F_OP(W) asm volatile("s_mov_b32 m0, %3\n\t" F_WAIT(W, 0) F_LD("%0", 0) F_WAIT(W, 1) F_LD("%0", 1024) F_WAIT(W, 2) \
                "s_mov_b64 exec, %5\n\t" F_LD("%0", 2048) "s_mov_b64 exec, -1\n\ts_mov_b32 m0, %4\n\t" \
                F_WAIT(W, 3) F_LD("%1", 0) F_WAIT(W, 4) F_LD("%1", 1024) F_WAIT(W, 5) \
                "s_mov_b64 exec, %6\n\t" F_LD("%1", 2048) "s_mov_b64 exec, -1" F_ARGS)
#define F_ACC(W) asm volatile("s_mov_b32 m0, %4\n\t" F_WAIT(W, 0) "s_mov_b64 exec, %6\n\t" F_LD("%1", 1024) "s_mov_b64 exec, -1\n\t" \
                F_WAIT(W, 1) F_LD("%1", 0) "s_mov_b32 m0, %3\n\t" F_WAIT(W, 2) F_LD("%0", 0) F_WAIT(W, 3) F_LD("%0", 1024) \
                F_WAIT(W, 4) F_LD("%0", 2048) F_WAIT(W, 5) F_LD("%0", 3072) F_ARGS)
#define F_MORE(V, M0V, OFF, MK) asm volatile("s_mov_b32 m0, %2\n\ts_waitcnt vmcnt(5)\n\ts_mov_b64 exec, %3\n\t" \
                "global_load_lds_dwordx4 %0, %1 offset:" #OFF " sc0 sc1\n\ts_mov_b64 exec, -1" :: "v"(V), "s"(src), "s"(M0V), "s"(MK) : "memory")
#define F_X(M, W) M(W)
            auto lanes = [](u32 lines) { return lines >= 8 ? ~0ull : (1ull << (8 * lines)) - 1; };
            auto get_go = [&](u32 sl) { CBAR(); const u32x4 r = *(volatile lu32x4*)(lds + EL_GO + sl * 16); CBAR(); return r; };
            u32 sl = 0, pub = 0, k = 0;
            bool six = false;
            u32x4 rec = get_go(0);
            for (;; ++k) {
                u32 go = __builtin_amdgcn_readfirstlane(rec[0]);
                for (u32 it = 0; (go >> 16) != k + 1; ++it) {
                    if (it == STARVE) { WAIT_VM(0); lds_put(EL_FULL, pub = k); }
                    __builtin_amdgcn_s_sleep(1);
                    rec = get_go(sl);
                    go = __builtin_amdgcn_readfirstlane(rec[0]);
                }
                if ((go & 0xffffu) == UNIT_END) break;
                const u32 fw = __builtin_amdgcn_readfirstlane(rec[3]);
                const u32 l0 = (fw >> 16) & 255u, l1 = fw >> 24;
                const uint8_t* src = pk + (size_t)(fw & 0xffffu) * SLOT;
                const u32 la = lds_addr + sl * SLOT;
                sl = sl == NS - 1 ? 0 : sl + 1;
                const bool isop = (int)(go & 0xffffu) < nop;
                pub = (isop ? WIN_OP : WIN) < 7 ? k : max(pub + 1, k) - 1;
                const u32x2 st = {k + 1, pub};
                if (isop) {
                    const u64 mk0 = lanes(l0 - 16), mk1 = lanes(l1 - 16);
                    F_X(F_OP, WIN_OP);
                    if (l0 > 24) F_MORE(vo0, la, 3072, lanes(l0 - 24));
                } else {
                    const u64 mk0 = 0, mk1 = lanes(l1 - 8);
                    F_X(F_ACC, WIN);
                    if (l1 > 16) F_MORE(vo1, la + 4096, 2048, lanes(l1 - 16));
                }
                if (l1 > 24) F_MORE(vo1, la + 4096, 3072, lanes(l1 - 24));
                six = !isop && l1 <= 16;
                rec = get_go(sl);
            }
            WAIT_VM(6);
            if (k) lds_put(EL_FULL, max(pub, k - 1));
            WAIT_VM(4);
            if (six) lds_put(EL_HEAD, k);
            WAIT_VM(0);
            lds_put(EL_FULL, pub = k);
            STAMP(4);
            return;
        }
        if (wave == 1) {
            __builtin_amdgcn_s_setprio(SC_PRIO);
            const u64* const flags = (const u64*)(myws + WS_FLAG) + (size_t)p * 4096 * 16;
            u32* const tick = (u32*)(myws + WS_TICK) + p * 32;
            static_assert(BATCH % 2 == 1 && BATCH <= 7, "a bijection needs BATCH odd");
            constexpr int TC = 8 * (BATCH - 1) / BATCH;
            const u32 ntick = (nend + BATCH - 1) / BATCH + TC, tbase = (call - 1) * (ntick - 16) - 24;
            static_assert(4 * BATCH <= 32, "lanes");
            u32 tick_no = b / 7;
            bool dead = false, planned = false;
            const int nflex = nop - (same ? 32 : 80);
            int gone = 0;
            u32 partials = 0;
            auto give_up = [&](u64 t0, u32 ginfo) {
                if (__hip_atomic_load(tmo, __ATOMIC_RELAXED, __HIP_MEMORY_SCOPE_AGENT) == call ||
                    __builtin_amdgcn_s_memrealtime() - t0 > DEADLINE) {
                        GAVEUP(3, ginfo);
                    if (lane == 0) __hip_atomic_store(tmo, call, __ATOMIC_RELAXED, __HIP_MEMORY_SCOPE_AGENT);
                    dead = true;
                    lds_put(EL_DEAD, 1);
                    return true;
                }
                __builtin_amdgcn_s_sleep(2);
                return false;
            };
            auto post = [&](u32 kk, u32 sl, u32 unit, u32 h0, u32 h1, u32 slot) {
                const u32 l0 = (int)unit < nop ? max((h0 & 31u) + 1, 17u) : 32u;
                const u32 l1 = (int)unit < nop ? max(((h1 >> 13) & 31u) + 1, 17u) : min(max(h0 & 127u, 41u), 64u) - 32u;
                for (;;) {
                    CBAR();
                    const u32x2 fr = *(volatile lu32x2*)(lds + EL_FREE + sl * 8);
                    if (__builtin_amdgcn_readfirstlane(min(fr[0] + NS, fr[1] + NS)) >= kk + 1) break;
                    __builtin_amdgcn_s_sleep(1);
                }
                CBAR();
                if (lane == 0) *(volatile lu32x4*)(lds + EL_GO + sl * 16) = u32x4{((kk + 1) << 16) | unit, h0, h1, slot | (l0 << 16) | (l1 << 24)};
                CBAR();
            };
            u32 k = 0, sl = 0;
            while (tick_no < ntick) {
                const int u0 = BATCH * ((int)tick_no - TC);
                while (lds_get(EL_ISS) + LOOK < k) __builtin_amdgcn_s_sleep(SC_SLEEP);
                const int nb = tick_no < 8 ? 2 * BATCH : BATCH;
                if (u0 + 8 * (nb - 1) >= nflex && !planned) {
                    planned = true;
                    const u64 t0 = __builtin_amdgcn_s_memrealtime();
                    u64 pv;
                    for (;;) {
                        pv = __hip_atomic_load((const u64*)(myws + WS_PLAN) + (lane & 31), __ATOMIC_RELAXED, __HIP_MEMORY_SCOPE_AGENT);
                        if (__all((u32)pv == call) || dead || give_up(t0, 0x10000)) break;
                    }
                    int live = 0;
                    for (int e0 = nflex; e0 < nop; e0 += 64) {
                        const u32 fe = *(lu32*)(lds + EL_TAB + (e0 + lane) * 16 + 4);
                        const bool relayed = ((fe >> 20) & 7u) + __shfl((u32)(pv >> 32), (fe >> 23) & 31u) >= 8u;
                        const bool mine = e0 + lane < nop && relayed == ((fe & FW_RELAYED) != 0);
                        const u64 m = __ballot(mine);
                        if (mine) *(lu32*)(lds + EL_LIVE + (live + __builtin_popcountll(m & ((1ull << lane) - 1))) * 4) = e0 + lane;
                        live += __builtin_popcountll(m);
                    }
                    CBAR();
                    gone = nop - nflex - live;
                }
                int ul = max(u0 + 8 * min((lane >> 1) & 15, nb - 1), 0);
                if (ul >= nflex) ul = ul + gone >= nop ? min(ul + gone, nend - 1) : *(lu32*)(lds + EL_LIVE + (ul - nflex) * 4);
                const u32x2 te = *(lu32x2*)(lds + EL_TAB + ul * 16);
                const u32 fw = te[1];
                const u64* const fp = flags + (fw & 0xffffu) + (call & 1) * ((fw & FW_WIDE) ? 8 : 2) + ((fw & FW_SPLIT) ? lane & 1 : 0);
                u32 tk;
                u64 fv;
                asm volatile("s_mov_b64 exec, 1\n\tglobal_atomic_add %0, %2, %3, off sc0\n\ts_mov_b64 exec, %5\n\t"
                             "global_load_dwordx2 %1, %4, off sc0 sc1\n\ts_mov_b64 exec, -1\n\ts_waitcnt vmcnt(0)"
                             : "=&v"(tk), "=&v"(fv) : "v"(tick), "v"(tick_no < 8 ? 0u : 1u), "v"(fp), "s"((1ull << (2 * nb)) - 1) : "memory");
                if (tick_no - MARK_AT < 8u && lane == 0)
                    __hip_atomic_store((u64*)(myws + WS_MARK) + p * 16 + (tick_no - MARK_AT),
                                       (__builtin_amdgcn_s_memrealtime() << 32) | call, __ATOMIC_RELAXED, __HIP_MEMORY_SCOPE_AGENT);
                for (int j = 0; j < nb; ++j) {
                    if (u0 + 8 * j < 0 || u0 + 8 * j + gone >= nend) continue;
                    const int un = __builtin_amdgcn_readlane(ul, 2 * j);
                    if (!dead && ((__ballot((u32)fv >= call) >> (2 * j)) & 3) != 3) {
                        while (lds_get(EL_ISS) + LOOK < k) __builtin_amdgcn_s_sleep(SC_SLEEP);
                        const u64 t0 = __builtin_amdgcn_s_memrealtime();
                        for (u32 it = 1;; ++it) {
                            if (lane >= 2 * j && lane < 2 * nb) fv = __hip_atomic_load(fp, __ATOMIC_RELAXED, __HIP_MEMORY_SCOPE_SYSTEM);
                            if (((__ballot((u32)fv >= call) >> (2 * j)) & 3) == 3 || ((it & 7) == 0 && give_up(t0, 0x20000 | un))) break;
                            __builtin_amdgcn_s_sleep(1);
                        }
                    }
                    if (un >= nop && partials != 255u) {
                        const u32 bit = 1u << lds_get(EL_TAB + un * 16 + 12);
                        if (!(partials & bit)) {
                            while (lds_get(EL_ISS) + LOOK < k) __builtin_amdgcn_s_sleep(SC_SLEEP);
                            const u64 t0 = __builtin_amdgcn_s_memrealtime();
                            while (!dead) {
                                partials = (u32)__ballot(__hip_atomic_load((const u32*)(myws + WS_PCNT) + (lane & 7) * 32, __ATOMIC_RELAXED,
                                                                           __HIP_MEMORY_SCOPE_SYSTEM) >= 24u * call) & 255u;
                                if ((partials & bit) || give_up(t0, 0x30000 | bit)) break;
                            }
                        }
                    }
                    const u32 hi = (u32)(fv >> 32);
                    post(k, sl, un, __builtin_amdgcn_readlane(hi, 2 * j), __builtin_amdgcn_readlane(hi, 2 * j + 1),
                         __builtin_amdgcn_readlane(te[0], 2 * j) & 0xffffu);
                    sl = sl == NS - 1 ? 0 : sl + 1;
                    ++k;
                }
                tick_no = tick_no < 8 ? tick_no + 16 : __builtin_amdgcn_readfirstlane(tk) - tbase;
            }
            for (u32 kk = k; kk < k + NPAIR; ++kk) {
                post(kk, sl, UNIT_END, 0, 0, 0);
                sl = sl == NS - 1 ? 0 : sl + 1;
            }
            return;
        }
        lbyte* const plane = lds + EL_PLANE + wave * 1024;
        const float scale0 = P.sx[0] * P.sw[0];
        const __amdgpu_buffer_rsrc_t outrs = __builtin_amdgcn_make_buffer_rsrc(P.out, 0, MS * NN * 4, 0x00020000);
        const u32 olane = (lane >> 4) * 4 * (NN * 4) + (lane & 15) * 16;
        u32 owed = 0;
        auto settle = [&]() {
            if (!owed) return;
            WAIT_VM(0);
            if (lane == 0) count_in(myws + owed, 1u);
            owed = 0;
        };
        for (int item = wave - 8;;) {
            const u32 k = item >> 1, hf = item & 1, sl = (k + NS) % NS;
            lbyte* const s = lds + sl * SLOT;
            const bool pushed = (int)k < 0;
            const u32 fin = pushed ? 0u : k + 1;
            u32 go, hdr;
            if (pushed) {
                const u32 j = b / 7 * NPAIR + k + NPAIR;
                go = nend + j;
                const u64* const fp = (const u64*)(myws + WS_FLAG) + (size_t)(p * 4096 + FL_P + j) * 16 + (call & 1) * 2 + hf;
                const u64 t0 = __builtin_amdgcn_s_memrealtime();
                u64 fv;
                for (u32 it = 1; (u32)(fv = __hip_atomic_load(fp, __ATOMIC_RELAXED, __HIP_MEMORY_SCOPE_SYSTEM)) < call; ++it) {
                    if ((it & 7) == 0 && (__hip_atomic_load(tmo, __ATOMIC_RELAXED, __HIP_MEMORY_SCOPE_AGENT) == call ||
                                          __builtin_amdgcn_s_memrealtime() - t0 > DEADLINE)) {
                        GAVEUP(4, j);
                        if (lane == 0) __hip_atomic_store(tmo, call, __ATOMIC_RELAXED, __HIP_MEMORY_SCOPE_AGENT);
                        lds_put(EL_DEAD, 1);
                        break;
                    }
                    __builtin_amdgcn_s_sleep(4);
                }
                hdr = __builtin_amdgcn_readfirstlane((u32)(fv >> 32));
                asm volatile("s_mov_b32 m0, %1\n\ts_nop 0\n\tglobal_load_lds_dwordx4 %0, off sc0 sc1\n\t"
                             "global_load_lds_dwordx4 %0, off offset:1024 sc0 sc1\n\tglobal_load_lds_dwordx4 %0, off offset:2048 sc0 sc1\n\t"
                             "global_load_lds_dwordx4 %0, off offset:3072 sc0 sc1\n\ts_waitcnt vmcnt(0)"
                             :: "v"(myws + pkoff + (size_t)(SL_P + p * NPP + j) * SLOT + hf * 4096 + lane * 16),
                                "s"((u32)(uintptr_t)s + hf * 4096) : "memory");
            } else {
                u32x4 rec;
                for (;;) {
                    CBAR();
                    rec = *(volatile lu32x4*)(lds + EL_GO + sl * 16);
                    go = __builtin_amdgcn_readfirstlane(rec[0]);
                    if ((go >> 16) == k + 1) break;
                    settle();
                    __builtin_amdgcn_s_sleep(1);
                }
                if ((go & 0xffffu) == UNIT_END) break;
                CBAR();
                hdr = __builtin_amdgcn_readfirstlane(hf ? rec[2] : rec[1]);
            }
            const u32x4 te = *(lu32x4*)(lds + EL_TAB + (go & 0xffffu) * 16);
            const u32 kind = __builtin_amdgcn_readfirstlane(te[0]) >> 16;
            const u32 dst = __builtin_amdgcn_readfirstlane(te[2]);
            const u32 aux = __builtin_amdgcn_readfirstlane(te[3]);
            const bool acc = kind == 2;
            u32x4 pa[4];
            const u32 odst = dst + hf * (16 * NN * 4);
            uint8_t* const od = (uint8_t*)P.out + odst + olane;
            if (acc) {
                settle();
#pragma unroll
                for (int j = 0; j < 4; ++j)
                    pa[j] = __builtin_bit_cast(u32x4, __builtin_amdgcn_raw_buffer_load_b128(outrs, olane, odst + j * (NN * 4), 17));
            }
            u32x4 st;
            for (;;) {
                CBAR();
                st = *(volatile lu32x4*)(lds + EL_ISS);
                if (__builtin_amdgcn_readfirstlane(st[1]) >= fin || __builtin_amdgcn_readfirstlane(st[2]) == fin + pushed) break;
                settle();
                __builtin_amdgcn_s_sleep(WK_SLEEP);
            }
            CBAR();
            if (WK_PRIO) __builtin_amdgcn_s_setprio(WK_PRIO);
            if (!acc) {
                u32x4 v[4];
                const lbyte* const blk = s + hf * 4096;
                const u32 bh = hdr >> (13 * hf);
                if (kind & 8) {
                    const u32 fp = (kind >> 7) & 3u, ff = (kind >> 4) & 7u;
                    uint8_t* const fs = myws + pkoff + (size_t)(SL_F + (fp * 8 + p) * 8 + ff) * SLOT + hf * 4096;
                    for (u32 o = lane * 16; o < ((bh & 31u) + 1) * 128; o += 1024) st_sys(fs + o, *(const lu32x4*)(blk + o));
                }
                if ((bh & 31u) == 31u) {
#pragma unroll
                    for (int j = 0; j < 4; ++j) v[j] = *(const lu32x4*)(blk + j * 1024 + lane * 16);
                    lds_put(EL_FREE + sl * 8 + hf * 4, fin);
                } else {
                    op_unpack(v, ((hdr >> 26) & 31u) * 0x04040404u, min((bh >> 5) & 255u, 128u), blk,
                              lds + EL_BUF + wave * 2304, lds + LDS_LUT, lane,
                              [&] { lds_put(EL_FREE + sl * 8 + hf * 4, fin); });
                }
                const bool isw = kind & 1;
                uint8_t* d = myws + dst + (isw ? hf * 4096 : (call & 1) * GX_ODD + hf * XSLAB) +
                             ((lane ^ (isw ? (kind & 4u) * 2 + hf * 4 : 0)) << 4);
                if (kind & 8) WAIT_VM(0);
                settle();
                if ((kind & 8) && lane == 0)
                    st_flag(P.ws[2 * ((kind >> 7) & 3u) + 1 - (rank & 1)], rank, FL_F + p * 8 + ((kind >> 4) & 7u), hf, call, hdr);
#pragma unroll
                for (int j = 0; j < 4; ++j) st_sys(d + j * 1024, v[j]);
                owed = aux;
            } else {
                const float scale = __builtin_amdgcn_readfirstlane(st[3]) ? __builtin_nanf("") : scale0;
                float a[16], c1 = 1.0f;
                if (hdr & H_RAW) {
#pragma unroll
                    for (int j = 0; j < 4; ++j) {
                        const u32x4 t = *(lu32x4*)(s + hf * 4096 + j * 1024 + lane * 16);
#pragma unroll
                        for (int t4 = 0; t4 < 4; ++t4) a[4 * j + t4] = __uint_as_float(t[t4]);
                    }
                    lds_put(EL_FREE + sl * 8 + hf * 4, fin);
                } else {
                    u32x4 mid = *(lu32x4*)(s + 4096 + lane * 16);
                    const u32 nb = (hdr >> 8) & 0x7ffu;
                    u32 at = ~0u;
                    for (u32 i0 = 0; i0 < nb; i0 += 128) {
                        const u32 i = i0 + 2 * lane;
                        const u32 w2 = *(lu16*)(s + ACC_MAIN + i);
                        u32 d[2], val[2];
#pragma unroll
                        for (int h = 0; h < 2; ++h) {
                            const u32 e = i + h < nb ? (w2 >> (8 * h)) & 255u : 0xe0u;
                            d[h] = e < 0xc0u ? (e & 63u) + 1 : e < 0xe0u ? 64u : 0u;
                            val[h] = e < 0x80u ? 3u + (e >> 6) : e >= 0xe0u ? e & 31u : 0u;
                        }
                        const u32 inc = wave_scan_add(d[0] + d[1]);
#pragma unroll
                        for (int h = 0; h < 2; ++h) {
                            const u32 t = at + inc - (h ? 0 : d[1]);
                            if (val[h] && ((t >> 2) & 1u) == hf) *(plane + (plane_addr(t) & 0x3ffu)) = (uint8_t)val[h];
                        }
                        at += __builtin_amdgcn_readlane(inc, 63);
                    }
                    CBAR();
                    const u32x4 q = *(lu32x4*)(plane + lane * 16);
                    *(lu32x4*)(plane + lane * 16) = u32x4{0u, 0u, 0u, 0u};
                    if (hf) {
                        const u32 m4 = *(lu32*)(s + 5120 + lane * 4), m5 = *(lu32*)(s + 5376 + lane * 4), m6 = *(lu32*)(s + 5632 + lane * 4);
                        u32 m7 = (mid[0] << 1) & 0x02020202u;
                        m7 |= (mid[1] << 2) & 0x04040404u; m7 |= (mid[2] << 3) & 0x08080808u; m7 |= (mid[3] << 4) & 0x10101010u;
                        m7 |= (m4 << 5) & 0x20202020u; m7 |= (m5 << 6) & 0x40404040u; m7 |= (m6 << 7) & 0x80808080u;
                        mid = u32x4{m4, m5, m6, m7};
                    }
                    u32 l0[4], l1[4];
#pragma unroll
                    for (int j = 0; j < 4; ++j) {
                        const u32 mi = bfi(0x01010101u, q[j] >> 2, mid[j]);
                        const u32 lo = (q[j] & 0x03030303u) | ((q[j] << 3) & 0xC0C0C0C0u);
                        l0[j] = __builtin_amdgcn_perm(mi, lo, 0x05010400u);
                        l1[j] = __builtin_amdgcn_perm(mi, lo, 0x07030602u);
                        asm volatile("" : "+v"(l0[j]), "+v"(l1[j]));
                    }
                    if (__builtin_amdgcn_readfirstlane(st[1]) < fin) { while (lds_get(EL_FULL) < fin) { } }
                    CBAR();
                    const u32x4 h0 = *(lu32x4*)(s + hf * 2048 + lane * 16), h1 = *(lu32x4*)(s + hf * 2048 + 1024 + lane * 16);
                    lds_put(EL_FREE + sl * 8 + hf * 4, fin);
                    c1 = __uint_as_float(((hdr >> 19) & 255u) << 23);
#pragma unroll
                    for (int j = 0; j < 4; ++j) {
                        const u32 ha = j < 2 ? h0[2 * j] : h1[2 * j - 4], hb = j < 2 ? h0[2 * j + 1] : h1[2 * j - 3];
                        const u32 T[4] = {__builtin_amdgcn_perm(ha, l0[j], 0x05040100u), __builtin_amdgcn_perm(ha, l0[j], 0x07060302u),
                                          __builtin_amdgcn_perm(hb, l1[j], 0x05040100u), __builtin_amdgcn_perm(hb, l1[j], 0x07060302u)};
#pragma unroll
                        for (int t4 = 0; t4 < 4; ++t4) a[4 * j + t4] = (float)(((int)T[t4] >> 2) ^ (int)(T[t4] << 30));
                    }
                }
#pragma unroll
                for (int j = 0; j < 4; ++j) asm volatile("" : "+v"(pa[j]));
#pragma unroll
                for (int j = 0; j < 4; ++j) {
                    f32x4 y;
#pragma unroll
                    for (int t4 = 0; t4 < 4; ++t4) {
                        const u32 pb = pa[j][t4];
                        y[t4] = silu(fmaf(a[4 * j + t4], c1, __uint_as_float(pb)) * scale);
                    }
                    st_mode<NT_OUT>(od + (size_t)j * (NN * 4), __builtin_bit_cast(u32x4, y));
                }
            }
            if (WK_PRIO) __builtin_amdgcn_s_setprio(0);
            u32 nx = 0;
            if (lane == 0) nx = __hip_atomic_fetch_add((lu32*)(lds + EL_NEXT), 1u, __ATOMIC_RELAXED, __HIP_MEMORY_SCOPE_WORKGROUP);
            item = __builtin_amdgcn_readfirstlane(nx);
        }
        settle();
        return;
    }

    const int g = b - NSEND;
    lbyte* const scr = lds + wave * 6144;
    constexpr int WFIRST = 64 * I8A, G_X1 = 72, GLATE = G_X1 + 16, GLEND = GLATE + 16 * (8 - I8A);
    constexpr int T_P = 112, T_X = T_P + 14 * NPP, T_W = T_X + 256;
    static_assert(I8A == 3 && T_W + 3 * WFIRST <= 176 * 8, "tasks");
    const bool isx1 = g >= G_X1 && g < GLATE;
    static_assert(T_W + 2 * WFIRST == 176 * 6 + 32 && WFIRST == 2 * 96 && GLEND == 168, "tasks");
    const int task0 = isx1 ? T_X : wave < 6 ? (g < G_X1 ? g : g - 16) + 176 * wave
                    : g < G_X1 || g >= GLEND ? T_W + 2 * WFIRST + (g < G_X1 ? g : g - 96) + 96 * (wave - 6)
                    : wave == 6 && g >= GLEND - 32 ? 176 * 6 + g - (GLEND - 32) : 2047;
    const int pdst = (rank + 1 + task0 % 7) & 7;
    const int pj = task0 / 7 - T_P / 7;
    const int psg = sent_group(pdst, rank, task0 < T_P ? task0 / 7 : NPUSH + (pj >> 1));
    const int gg = isx1 ? (g - G_X1) * 8 + wave : task0 < T_X ? group_of(pdst, psg) : (task0 - T_X) & 127;
    const int xkc = isx1 ? 2 : task0 < T_P ? 0 : task0 < T_X ? pj & 1 : (task0 - T_X) >> 7;
    u32x4 v[8];
    if (task0 < 7 * NPUSH || (task0 >= T_P && task0 < T_W)) {
        const uint8_t* src = xin + (size_t)(gg * 32 + (lane >> 3)) * KS + xkc * 128 + (lane & 7) * 16;
#pragma unroll
        for (int j = 0; j < 4; ++j) v[j] = *(const u32x4*)(src + j * 8 * KS);
        if (task0 < T_P || isx1) {
#pragma unroll
            for (int j = 0; j < 4; ++j) v[4 + j] = *(const u32x4*)(src + j * 8 * KS + 128);
        }
    }
    u32x2 in[16];
    auto load_w = [&](int kc, int blk, int wi8, int half) {
        const uint8_t* src = win + (size_t)(kc * 128 + 16 * (lane >> 3)) * NN + blk * 1024 + wi8 * 128 + 16 * (lane & 7) + 8 * half;
#pragma unroll
        for (int j = 0; j < 16; ++j) in[j] = *(const u32x2*)(src + (size_t)j * NN);
    };
    const bool isw1 = task0 >= T_W && task0 < T_W + 2 * WFIRST;
    if (isw1) {
        const int wt = task0 - T_W;
        load_w((wt >> 6) & 1, (wt & 7) | ((wt >> 1) & 8), wt >> 7, (wt >> 3) & 1);
    }
    LOAD_CALL();
    const int l7 = (lane >> 2) & 7;
    uint8_t* const wsl = P.ws[l7];
    __builtin_amdgcn_sched_barrier(0);
    if (tid < 3) *(lu32*)(lds + LDS_PUB + tid * 64 + 32) = 0;
    const u64 tstart0 = __builtin_amdgcn_s_memrealtime();
    lbyte* const lut = scr + 4096;
    if (lane < 16) op_lut_init(lut, lane);
    CBAR();
    if (task0 < 7 * NPUSH) {
        uint8_t* const wd = P.ws[pdst];
        uint8_t* d = wd + WS_GX + (size_t)(rank * 4) * XSLAB + (size_t)psg * 4096 + lane * 16;
        GET_CALL();
        d += (call & 1) * GX_ODD;
#pragma unroll
        for (int j = 0; j < 8; ++j) asm volatile("" : "+v"(v[j]));
#pragma unroll
        for (int j = 0; j < 8; ++j) {
            THROTTLE(PUSH_WIN);
            st_sys(d + (j >> 2) * XSLAB + (j & 3) * 1024, v[j]);
        }
        WAIT_VM(0);
        if (lane == 0)
            count_in(wd + WS_XREM + rank * 256, 2u);
    }
    bool isx = task0 >= T_P && task0 < T_W, send = true, keep = true, tell = false;
    int h0 = 0, h1 = 2, flag = 0;
    u32 dbo = 0, swz = 0, slo = 0, cno = 0;
    uint8_t* slotws = myws;
    if (task0 >= T_P && task0 < T_X) {
        h0 = xkc; h1 = h0 + 1;
        keep = false;
        tell = l7 == pdst;
        flag = FL_P + (pj >> 1);
        slo = (u32)(SL_P + rank * NPP + (pj >> 1)) * (u32)SLOT;
        slotws = P.ws[pdst];
    } else if (isx) {
        if (X0_PRIO && !isx1) __builtin_amdgcn_s_setprio(X0_PRIO);
        const int kc = xkc, q = gg >> 4, lg = gg & 15;
        const int fl = (lg >= FG) ^ (q & 1);
        if (!isx1) { h0 = kc; h1 = h0 + 1; }
        send = q != rank;
        keep = q == rank || (rank & 1) == fl;
        if (isx1) {
            tell = send && l7 != rank && (lg < 8 || l7 == q || (l7 & 1) == fl);
            flag = FL_X1 + (gg >> 3);
        } else {
            const int sgl = lg < FG ? FG * (((q >> 1) - (l7 >> 1) - 1) & 3) + lg : 3 * FG + GG * (q >> 1) + lg - FG;
            tell = send && l7 != rank && (l7 == q || ((l7 & 1) == fl && sgl >= sent_group(l7, rank, NPUSH + NPP)));
            flag = FL_X + gg;
        }
        const int sg = q == rank ? XGREM + lg
                     : lg < FG ? FG * (((q >> 1) - (rank >> 1) - 1) & 3) + lg
                               : 3 * FG + GG * (q >> 1) + lg - FG;
        dbo = (u32)WS_GX + (u32)(rank * 4 + (kc & 2)) * (u32)XSLAB + (u32)sg * 4096u;
        slo = (u32)(SL_X + gg * 2 + (kc >> 1)) * (u32)SLOT;
        cno = (u32)(q == rank ? WS_XOWN : WS_XREM) + (u32)(rank * 2 + (kc >> 1)) * 128u;
        asm volatile("" : "+s"(dbo), "+s"(slo), "+s"(cno), "+v"(flag));
    }
    for (int phase = task0 >= T_P && task0 < T_W + 3 * WFIRST ? 0 : 1; phase < (g >= GLATE && g < GLEND ? 2 : 1); ++phase) {
        if (phase || isx1) {
            asm volatile("s_waitcnt lgkmcnt(0)" ::: "memory");
            __builtin_amdgcn_s_barrier();
        }
        if (phase) { isx = false; slotws = myws; }
        if (!isx) {
            const bool first = !phase && isw1;
            const int wt = task0 - T_W, wq = first ? wt >> 6 : (wt - 2 * WFIRST) >> 5;
            h0 = first ? (wt >> 5) & 1 : 0; h1 = first ? h0 + 1 : 2;
            const int half = phase ? wave & 1 : (wt >> 3) & 1;
            const int blk = phase ? (g - GLATE) & 15 : (wt & 7) | ((wt >> 1) & 8);
            const int kc = phase ? wave >> 1 : (first ? 0 : 2) + (wq & 1);
            const int wi8 = phase ? I8A + ((g - GLATE) >> 4) : wq >> 1;
            if (!first) {
                while (__builtin_amdgcn_s_memrealtime() - tstart0 < (u64)(phase ? W_DELAY + LATE_STAG * ((0x32410 >> (4 * (wi8 - I8A))) & 7) : W_DELAY0))
                    __builtin_amdgcn_s_sleep(8);
                load_w(kc, blk, wi8, half);
            }
            __builtin_amdgcn_sched_barrier(0);
            auto needs = [&](int r) { return (r >> 1) == ((r & 1) ? blk & 3 : blk >> 2); };
            auto ctile = [&](int r) { return ((r & 1) ? blk >> 2 : blk & 3) * 8 + wi8; };
            send = keep = true;
            tell = l7 != rank && needs(l7);
            flag = phase ? FL_L + ctile(l7) : FL_W + (kc * 32 + ctile(l7)) * 2 + half;
            const bool mine = needs(rank);
            const int fb = (rank & 1) ? 3 * (blk >> 2) + (blk & 3) - ((blk & 3) > (rank >> 1))
                                      : blk - (blk >= 2 * rank ? 4 : 0);
            const int ft = fb * 8 + wi8;
            dbo = half * 8192u + (mine ? (u32)WS_GW + (u32)((rank * 4 + kc) * 32 + ctile(rank)) * 16384u
                                       : (u32)WS_GW2 + (u32)(((ft >> 5) * 4 + kc) * 32 + (ft & 31)) * 16384u);
            swz = half * 8;
            slo = (u32)(SL_W + (kc * 128 + blk * 8 + wi8) * 2 + half) * (u32)SLOT;
            cno = (u32)WS_WCNT + (u32)((mine ? ctile(rank) : 32 + ft) * 32 + rank * 4 + kc) * 128u;
            asm volatile("" : "+s"(dbo), "+s"(slo), "+s"(cno), "+v"(flag));
#pragma unroll
            for (int d = 0; d < 2; ++d)
#pragma unroll
                for (int e = 0; e < 4; ++e) {
                    const u32 t0 = __builtin_amdgcn_perm(in[4 * e + 1][d], in[4 * e][d], 0x05010400u);
                    const u32 t1 = __builtin_amdgcn_perm(in[4 * e + 1][d], in[4 * e][d], 0x07030602u);
                    const u32 t2 = __builtin_amdgcn_perm(in[4 * e + 3][d], in[4 * e + 2][d], 0x05010400u);
                    const u32 t3 = __builtin_amdgcn_perm(in[4 * e + 3][d], in[4 * e + 2][d], 0x07030602u);
                    v[4 * d + 0][e] = __builtin_amdgcn_perm(t2, t0, 0x05040100u);
                    v[4 * d + 1][e] = __builtin_amdgcn_perm(t2, t0, 0x07060302u);
                    v[4 * d + 2][e] = __builtin_amdgcn_perm(t3, t1, 0x05040100u);
                    v[4 * d + 3][e] = __builtin_amdgcn_perm(t3, t1, 0x07060302u);
                }
            if (h0) {
#pragma unroll
                for (int j = 0; j < 4; ++j) v[j] = v[4 + j];
            }
        }
        GET_CALL();
#pragma unroll
        for (int j = 0; j < 8; ++j) asm volatile("" : "+v"(v[j]));
        uint8_t* const db = myws + dbo + (isx ? (call & 1) * GX_ODD : 0);
        uint8_t* const slot = slotws + pkoff + slo;
        u32 es = 0;
#pragma unroll
        for (int j = 0; j < 8; ++j) es = __builtin_amdgcn_sad_u8(v[j >> 1][(j & 1) * 2] & 0x7C7C7C7Cu, 0u, es);
        const u32 base = (u32)min(max(((int)wave_sum(es) - 1475) >> 13, 0), 29);
        u32 hdr = base << 26;
        for (int hh = h0; hh < h1; ++hh) {
            if (send) hdr |= op_pack(v, base * 0x04040404u, slot + hh * 4096, slotws != myws, scr, lut, lane) << (13 * hh);
            if (keep) {
                uint8_t* d = db + (isx ? hh * XSLAB + lane * 16 : hh * 4096 + ((lane ^ (swz + hh * 4)) << 4));
#pragma unroll
                for (int j = 0; j < 4; ++j) st_sys(d + j * 1024, v[j]);
            }
#pragma unroll
            for (int j = 0; j < 4; ++j) v[j] = v[4 + j];
        }
        WAIT_VM(0);
        if (X0_PRIO) __builtin_amdgcn_s_setprio(0);
        u64 wsq = (u64)wsl;
        asm volatile("" : "+v"(wsq));
        uint8_t* const wsr = (uint8_t*)(__attribute__((address_space(1))) uint8_t*)wsq;
        if (phase || isx1) st_flags8(lds + LDS_PUB, wave, hdr, wsr, tell && lane < 32, rank, flag, call, lane);
        else if (tell && lane < 32 && (lane & 3) == 0) st_flag(wsr, rank, flag, h0, call, hdr);
        if (keep && lane == 0)
            count_in(myws + cno, (u32)(h1 - h0));
    }
    GET_CALL();
    PUT_CALL();
    STAMP(1);

    const int xcd = g & 7, idx = g >> 3;
    const int part = xcd >> 1;
    const int wm = wave >> 1, wn = wave & 1;
    const int c = lane & 15, q4 = lane >> 4;
    const int ra0 = wm * 8192 + c * 128 + ((q4 ^ (c >> 1)) << 4);
    const int ra1 = ra0 ^ 64;
    const int rb0 = 32768 + (c & 3) * 4096 + ((((q4 << 3) | (4 * wn + (c >> 2))) ^ ((c & 3) << 2)) << 4);
    const u32 boff = (u32)wave * 2048u + (u32)lane * 16u;
    const u32 la0 = (u32)(uintptr_t)lds + wave * 4096, lb0 = (u32)(uintptr_t)lds + 32768 + wave * 2048;
    const u32 aoff0 = (u32)(lane >> 3) * 128u + (u32)(((lane & 7) ^ (lane >> 4)) << 4), aoff1 = aoff0 ^ 64u;

    for (u32 jobs = idx < 3 ? 0x5324 : idx < 8 ? 0x54 : idx < 9 ? 0x4 : idx < 12 ? 0x34 : idx < 15 ? 0x42
                  : idx < 18 ? 0x43 : idx < 21 ? 0x432 : 0x421; jobs; jobs >>= 4) {
        const int job = (int)(jobs & 15) - 2;
        const bool help = job < 0;
        const bool pre = job < 2 && !help, local = job == 3, ownrows = pre || local;
        const bool halft = help || (job == 2 && idx < 3);
        const bool both = idx < 3 || (idx >= 18 && idx < 21);
        const int pi = g < 24 ? g + 48 : g < 96 ? g : g < 120 || g >= 168 ? g - 96 : g - 120, ft = pi % 12 * 8 + pi / 12;
        const int i8 = pre ? ft & 7 : local ? idx : help ? 0 : idx / 3;
        const int rt = pre ? job : local ? (xcd & 1) : 3 * (xcd & 1) + idx % 3;
        const int ct = pre ? 32 + ft : part * 8 + i8;
        const int nstep = pre ? 4 : NSTEP;
        const u32 wbase = pre ? (u32)WS_GW2 + (u32)(((ft >> 5) - rank) * 128 + (ft & 31)) * 16384u : (u32)WS_GW + (u32)ct * 16384u;
        const int rgw = rt * 8 + (halft ? 2 * wm + help : wave);
        const u32 laj = halft ? la0 - wn * 2048 : la0;
        const int sgw = ownrows ? XGREM + rgw : (rgw < 3 * FG + part * GG ? rgw : rgw + GG);
        const int skipsrc = ownrows ? -1 : group_of(rank, sgw) >> 4;
        const u32 xbase = (u32)(WS_GX + (call & 1) * GX_ODD) + (u32)sgw * 4096u + (halft ? wn * 2048u : 0u);

        const u32* const pollp = lane < 32 ? (const u32*)(myws + WS_WCNT) + (ct * 32 + lane) * 32
                                           : (const u32*)(myws + X_DONE + (ownrows ? WS_XOWN : WS_XREM)) + (lane & 15) * 32;
        const u32 pneed = lane < 32 ? 4u * call : call;
        u64 ready = job == 1 && both ? ~0ull : 0;
        const int tq = ((lane & 31) - 4) / 7;
        const u32 vst = (lane & 31) < 4 ? rank * 4 + (lane & 3) : ((rank + (lane & 31) - 3 - 7 * tq) & 7) * 4 + tq;
        auto stepno = [&](int s) { return (int)__builtin_amdgcn_readlane(vst, s); };
        auto ensure = [&](int s) {
            const int st = stepno(s);
            const u64 want = (1ull << st) | (1ull << (32 + (st >> 1)));
            if ((ready & want) == want) return;
            const u64 t0 = __builtin_amdgcn_s_memrealtime();
            const u64 about = (st >> 2) == rank ? (0xfull << (4 * rank)) | (3ull << (32 + 2 * rank))
                                                : (0x11111111ull << (st & 3)) | (0x5555ull << (32 + ((st >> 1) & 1)));
            for (u32 it = 1;; ++it) {
                u32 v = 0;
                if (((about & ~ready) >> lane) & 1) v = __hip_atomic_load(pollp, __ATOMIC_RELAXED, __HIP_MEMORY_SCOPE_SYSTEM);
                ready |= __ballot(v >= pneed);
                if ((ready & want) == want) break;
                if ((it & 7) == 0 && (__hip_atomic_load(tmo, __ATOMIC_RELAXED, __HIP_MEMORY_SCOPE_AGENT) == call ||
                                      __builtin_amdgcn_s_memrealtime() - t0 > DEADLINE)) {
                        GAVEUP(5, (u32)(st | (job << 8) | (ct << 16)));
                    if (lane == 0) __hip_atomic_store(tmo, call, __ATOMIC_RELAXED, __HIP_MEMORY_SCOPE_AGENT);
                    WAIT_VM(0);
                    ready = ~0ull;
                    break;
                }
                __builtin_amdgcn_s_sleep(POLL_SLEEP);
            }
        };
        bool polling = wave == 0;
        int age = 0;
        u64 asked = 0;
        auto peek_issue = [&](int s) {
            if (age) { ++age; return; }
            const int st = stepno(min(s + 3 + PEEK_AHEAD, nstep - 1));
            const u64 want = (1ull << st) | (1ull << (32 + (st >> 1)));
            if ((ready & want) == want) return;
            asked = 0xffffffffffffull & ~ready;
            if ((asked >> lane) & 1) {
                u32 keep;
                asm volatile("s_mov_b32 %0, m0\n\ts_mov_b32 m0, %2\n\ts_nop 0\n\tglobal_load_lds_dword %1, off sc0 sc1\n\ts_mov_b32 m0, %0"
                             : "=&s"(keep) : "v"(pollp), "s"((u32)(uintptr_t)(lds + LDS_CTL)) : "memory");
            }
            age = 1;
        };
        auto peek_read = [&]() {
            if (age != 3) return;
            if (halft) WAIT_VM(4); else WAIT_VM(6);
            ready |= __ballot(*(volatile lu32*)(lds + LDS_CTL + lane * 4) >= pneed) & asked;
            age = 0;
        };
        const u32 vxo = (int)(vst >> 2) == skipsrc ? (u32)WS_ZERO : xbase + vst * (u32)XSLAB;
        const u32 vwo = wbase + vst * (32u * 16384u);
        const uint8_t *xa, *wa;
        auto locate = [&](int s) {
            xa = myws + (u32)__builtin_amdgcn_readlane(vxo, s);
            wa = myws + (u32)__builtin_amdgcn_readlane(vwo, s);
        };
#define DMA(M0V, VOFF, SRC, OFF) \
        asm volatile("s_mov_b32 m0, %2\n\ts_nop 0\n\tglobal_load_lds_dwordx4 %0, %1 offset:" #OFF " sc0 sc1" \
                     :: "v"(VOFF), "s"(SRC), "s"(M0V) : "memory")
        auto issue = [&](u32 so) {
            DMA(so + laj, aoff0, xa, 0); DMA(so + laj, aoff1, xa, 1024);
            if (!halft) { DMA(so + laj, aoff0, xa, 2048); DMA(so + laj, aoff1, xa, 3072); }
            DMA(so + lb0, boff, wa, 0); DMA(so + lb0, boff, wa, 1024);
        };

        f32x4 acc[4][4];
#pragma unroll
        for (int m = 0; m < 4; ++m)
#pragma unroll
            for (int t = 0; t < 4; ++t) acc[m][t] = f32x4{0.f, 0.f, 0.f, 0.f};

        auto rd1 = [&](const lbyte* p0, const lbyte* p1) {
            const u32x4 l = *(const lu32x4*)p0, h = *(const lu32x4*)p1;
            return i32x8{(int)l[0], (int)l[1], (int)l[2], (int)l[3], (int)h[0], (int)h[1], (int)h[2], (int)h[3]};
        };
#define MM(A, B, m, t) acc[m][t] = __builtin_amdgcn_mfma_scale_f32_16x16x128_f8f6f4(A[m], B[t], acc[m][t], 1, 1, 0, 0, 0, 0)
#define SB() __builtin_amdgcn_sched_barrier(0)
#define GTRACE() do { } while (0)
#define KNEXT(ISSUE) \
            ++s; \
            ma = sn + laj; mb = sn + lb0; \
            sn = sn == 2 * STAGE ? 0 : sn + STAGE; \
            pa0 = lds + sn + ra0; pa1 = lds + sn + ra1; pb = lds + sn + rb0; \
            if (ISSUE) locate(s + 3); \
            asm volatile("" : "+s"(ma), "+s"(mb), "+s"(xa), "+s"(wa), "+v"(pa0), "+v"(pa1), "+v"(pb))
#define KSYNC(ISSUE, NVM) \
            if (ISSUE && age) asm volatile("s_waitcnt vmcnt(%0) lgkmcnt(0)" :: "i"(NVM + 1) : "memory"); \
            else asm volatile("s_waitcnt vmcnt(%0) lgkmcnt(0)" :: "i"(NVM) : "memory"); \
            __builtin_amdgcn_s_barrier(); \
            asm volatile("" ::: "memory")
#define KBODY(XA, XB, YA, YB, ISSUE) \
                SB(); MM(XA, XB, 0, 0); SB(); YA[0] = rd1(pa0, pa1); \
                SB(); MM(XA, XB, 0, 1); SB(); YB[0] = rd1(pb, pb + 512); \
                SB(); MM(XA, XB, 0, 2); SB(); if (ISSUE) DMA(ma, aoff0, xa, 0); \
                SB(); MM(XA, XB, 0, 3); SB(); YB[1] = rd1(pb + 1024, pb + 1536); \
                SB(); MM(XA, XB, 1, 0); SB(); if (ISSUE) DMA(ma, aoff1, xa, 1024); \
                SB(); MM(XA, XB, 1, 1); SB(); YB[2] = rd1(pb + 2048, pb + 2560); \
                SB(); MM(XA, XB, 1, 2); SB(); if (ISSUE) DMA(ma, aoff0, xa, 2048); \
                SB(); MM(XA, XB, 1, 3); SB(); YB[3] = rd1(pb + 3072, pb + 3584); \
                SB(); MM(XA, XB, 2, 0); SB(); if (ISSUE) DMA(ma, aoff1, xa, 3072); \
                SB(); MM(XA, XB, 2, 1); SB(); YA[1] = rd1(pa0 + 2048, pa1 + 2048); \
                SB(); MM(XA, XB, 2, 2); SB(); if (ISSUE) DMA(mb, boff, wa, 0); \
                SB(); MM(XA, XB, 2, 3); SB(); YA[2] = rd1(pa0 + 4096, pa1 + 4096); \
                SB(); MM(XA, XB, 3, 0); SB(); if (ISSUE) DMA(mb, boff, wa, 1024); \
                YA[3] = rd1(pa0 + 6144, pa1 + 6144)
#define KSTEP(HALF, XA, XB, YA, YB, ISSUE, AHEAD) do { \
            if (ISSUE && polling) { peek_read(); ensure(s + 3); } \
            if (!HALF) { \
                KSYNC(ISSUE, 6 * AHEAD); \
                KBODY(XA, XB, YA, YB, ISSUE); \
                SB(); MM(XA, XB, 3, 1); SB(); \
                KNEXT(ISSUE); \
                SB(); MM(XA, XB, 3, 2); MM(XA, XB, 3, 3); SB(); \
            } else { \
                KSYNC(ISSUE, 4 * AHEAD); \
                SB(); MM(XA, XB, 0, 0); SB(); if (ISSUE) DMA(ma, aoff0, xa, 0); YA[0] = rd1(pa0, pa1); \
                SB(); MM(XA, XB, 0, 1); SB(); if (ISSUE) DMA(ma, aoff1, xa, 1024); YB[0] = rd1(pb, pb + 512); \
                SB(); MM(XA, XB, 0, 2); SB(); if (ISSUE) DMA(mb, boff, wa, 0); YB[1] = rd1(pb + 1024, pb + 1536); \
                SB(); MM(XA, XB, 0, 3); SB(); if (ISSUE) DMA(mb, boff, wa, 1024); YB[2] = rd1(pb + 2048, pb + 2560); \
                SB(); MM(XA, XB, 1, 0); SB(); YB[3] = rd1(pb + 3072, pb + 3584); \
                SB(); MM(XA, XB, 1, 1); SB(); YA[1] = rd1(pa0 + 2048, pa1 + 2048); \
                KNEXT(ISSUE); \
                SB(); MM(XA, XB, 1, 2); MM(XA, XB, 1, 3); SB(); \
            } \
            if (ISSUE && polling) { peek_issue(s - 1); polling = age || (~ready << 16); } \
            GTRACE(); \
        } while (0)

        if (pre && wave == 0 && idx >= 12) {
            while (__builtin_amdgcn_s_memrealtime() - tstart0 < (u64)(PRE_START + (idx - 12) * PRE_STAG)) __builtin_amdgcn_s_sleep(8);
        }
        if (wave == 0) { ensure(0); ensure(1); ensure(2); }
        asm volatile("s_waitcnt lgkmcnt(0)" ::: "memory");
        __builtin_amdgcn_s_barrier();
        asm volatile("" ::: "memory");
        asm volatile("s_nop 4");
        locate(0); issue(0);
        locate(1); issue(STAGE);
        locate(2); issue(2 * STAGE);
        locate(3);
        if (halft) WAIT_VM(8); else WAIT_VM(12);
        __builtin_amdgcn_s_barrier();
        asm volatile("" ::: "memory");
        u32 tmov;
#define KLOOP(HALF) do { \
            i32x8 A0[4], B0[4], A1[4], B1[4]; \
            _Pragma("unroll") for (int t = 0; t < 4; ++t) { \
                if (!HALF || t < 2) A0[t] = rd1(lds + ra0 + t * 2048, lds + ra1 + t * 2048); \
                B0[t] = rd1(lds + rb0 + t * 1024, lds + rb0 + t * 1024 + 512); \
            } \
            int s = 0; \
            u32 sn = STAGE, ma = laj, mb = lb0; \
            const lbyte *pa0 = lds + STAGE + ra0, *pa1 = lds + STAGE + ra1, *pb = lds + STAGE + rb0; \
            while (s < nstep - 4) { \
                KSTEP(HALF, A0, B0, A1, B1, true, 1); \
                KSTEP(HALF, A1, B1, A0, B0, true, 1); \
            } \
            KSTEP(HALF, A0, B0, A1, B1, true, 1); \
            KSTEP(HALF, A1, B1, A0, B0, false, 1); \
            KSTEP(HALF, A0, B0, A1, B1, false, 0); \
              \
            tmov = __hip_atomic_load(tmo, __ATOMIC_RELAXED, __HIP_MEMORY_SCOPE_AGENT); \
            SB(); \
            _Pragma("unroll") for (int t = 0; t < (HALF ? 8 : 16); ++t) MM(A1, B1, t >> 2, t & 3); \
        } while (0)
        if (halft) KLOOP(true); else KLOOP(false);

        int ln = lane;
        asm volatile("" : "+v"(ln));
        if (local) {
            float scale = tmov == call ? __builtin_nanf("") : P.sx[0] * P.sw[0];
            asm volatile("" : "+v"(scale));
            float* dst = P.out + (size_t)(rt * 256 + wm * 64 + 4 * (ln >> 4)) * NN + block_of(rank, part) * 1024 + i8 * 128 + wn * 64 + 4 * (ln & 15);
#pragma unroll
            for (int m = 0; m < 4; ++m)
#pragma unroll
                for (int r = 0; r < 4; ++r) {
                    f32x4 y;
#pragma unroll
                    for (int t = 0; t < 4; ++t) y[t] = silu(acc[m][t][r] * scale);
                    THROTTLE(LOC_WIN);
                    st_mode<NT_OUT>(dst + (size_t)(m * 16 + r) * NN, __builtin_bit_cast(u32x4, y));
                }
            STAMP(7);
        } else if (pre) {
            const int fb = ft >> 3;
            const int blkf = (rank & 1) ? 4 * (fb / 3) + fb % 3 + (fb % 3 >= (rank >> 1)) : fb + (fb >= 2 * rank ? 4 : 0);
            float* d = P.out + (size_t)(rt * 256 + wm * 64 + 4 * (ln >> 4)) * NN + blkf * 1024 + i8 * 128 + wn * 64 + 4 * (ln & 15);
#pragma unroll
            for (int m = 0; m < 4; ++m)
#pragma unroll
                for (int r = 0; r < 4; ++r) {
                    const f32x4 y = {acc[m][0][r], acc[m][1][r], acc[m][2][r], acc[m][3][r]};
                    st_mode<NT_PRE>(d + (size_t)(m * 16 + r) * NN, __builtin_bit_cast(u32x4, y));
                }
            if (job == 1 || !both) {
                WAIT_VM(0);
                __builtin_amdgcn_s_barrier();
                if (tid == 0) __hip_atomic_fetch_add((u32*)(myws + WS_PCNT) + i8 * 32, both ? 2u : 1u, __ATOMIC_RELAXED, __HIP_MEMORY_SCOPE_SYSTEM);
                STAMP(7);
            }
        } else {
            const int rl = (ln >> 2) & 7, jl = rl >> 1;
            const int lo = ((rl ^ rank) & 1) ? 3 * FG + (jl > part ? jl - 1 : jl) * GG : FG * ((jl - (rank >> 1) - 1) & 3);
            const int nl = ((rl ^ rank) & 1) ? (jl == part ? 0 : GG) : (rl == rank ? 0 : FG);
            const bool owns = ln < 32 && nl && lo < rt * 8 + 8 && lo + nl > rt * 8;
            u64 qws = (u64)P.ws[rl];
            auto publish = [&](int mp, u32 hdr) {
                WAIT_VM(0);
                st_flags8(lds + LDS_PUB + (halft ? 0 : 64 + mp * 64), wave, hdr, (uint8_t*)qws, owns, rank,
                          FL_O + (ct * 6 + rt) * 2 + mp + help, call, ln);
            };
            float amf = 0.f;
#pragma unroll
            for (int i = 0; i < 64; ++i) amf = fmaxf(amf, fabsf(acc[i >> 4][i & 3][(i >> 2) & 3]));
            const u32 am = wave_umax(__float_as_uint(amf));
            const int ex = min(max((int)(am >> 23), 40), 200);
            u32 votes = 0;
#pragma unroll
            for (int i = 0; i < 8; ++i) {
                const u32 I = (u32)(int)(acc[i >> 1][i & 3][i & 1] * __uint_as_float((u32)(280 - ex) << 23));
                votes += (((I + (1u << 25)) & 0xfc000007u) != 0) + ((((I + (1u << 26)) & 0xf800000fu) != 0) << 10) +
                         (((I & 0x1fu) != 0) << 20);
            }
            votes = wave_sum(votes);
            const u32 n0 = votes & 1023u, n1 = (votes >> 10) & 1023u, n2 = votes >> 20;
            const int dsc = n2 <= n1 && n2 <= n0 ? 2 : n1 <= n0 ? 1 : 0;
            const float up = __uint_as_float((u32)(280 - ex - dsc) << 23);
            u32 hdrs[2];
            asm volatile("" : "+v"(qws), "+v"(tmov));
            const bool bad = __builtin_amdgcn_readfirstlane(tmov) == call;
#pragma unroll
            for (int mp = 0; mp < 2; ++mp) {
                if (mp == 1 && halft) break;
#define ACCV(i) acc[2 * mp + ((i) >> 4)][(i) & 3][((i) >> 2) & 3]
                uint8_t* const slot = myws + pkoff + (size_t)(SL_O + (ct * 48 + rt * 8 + wm * 2 + mp + help) * 2 + wn) * SLOT;
                u32 hi[16], mid[8], q[8], nz = 0, miss = 0;
#pragma unroll
                for (int j = 0; j < 8; ++j) {
                    if (mp == 1 && j == 3) { SB(); publish(0, hdrs[0]); SB(); }
                    u32 t[4];
#pragma unroll
                    for (int t4 = 0; t4 < 4; ++t4) {
                        const float f = ACCV(4 * j + t4) * up;
                        const u32 I = (u32)(int)f;
                        miss |= __float_as_uint(__builtin_amdgcn_fractf(f));
                        t[t4] = __builtin_amdgcn_alignbit(I, I ^ (u32)__builtin_amdgcn_sbfe((int)I, 0, 26), 26);
                    }
                    hi[2 * j] = __builtin_amdgcn_perm(t[1], t[0], 0x07060302u);
                    hi[2 * j + 1] = __builtin_amdgcn_perm(t[3], t[2], 0x07060302u);
                    const u32 l0 = __builtin_amdgcn_perm(t[1], t[0], 0x05010400u), l1 = __builtin_amdgcn_perm(t[3], t[2], 0x05010400u);
                    const u32 lo = __builtin_amdgcn_perm(l1, l0, 0x05040100u);
                    mid[j] = __builtin_amdgcn_perm(l1, l0, 0x07060302u);
                    q[j] = bfi(0x07070707u, bfi(0x03030303u, lo, mid[j] << 2), (lo >> 3) & 0x18181818u);
                    const u32 nb = q[j] + 0x1F1F1F1Fu;
                    nz = bfi(0x01010101u << j, j < 5 ? nb >> (5 - j) : nb << (j - 5), nz);
                }
                const u32 cnt = __builtin_popcount(nz);
                const u32 inc = wave_scan_add(cnt);
                const u32 total = __builtin_amdgcn_readlane(inc, 63);
                if (bad || (int)(am >> 23) > ex || __any(miss != 0) || total > LIST_MAX) {
#pragma unroll
                    for (int j = 0; j < 8; ++j) {
                        f32x4 y = {ACCV(4 * j), ACCV(4 * j + 1), ACCV(4 * j + 2), ACCV(4 * j + 3)};
                        if (bad) y[0] = __builtin_nanf("");
                        st_sys(slot + j * 1024 + ln * 16, __builtin_bit_cast(u32x4, y));
                    }
                    hdrs[mp] = 64u | H_RAW;
                } else {
#pragma unroll
                    for (int j = 0; j < 7; ++j) mid[j] = bfi(0x01010101u, mid[7] >> (j + 1), mid[j]);
#pragma unroll
                    for (int j = 0; j < 4; ++j)
                        st_sys(slot + j * 1024 + ln * 16, u32x4{hi[4 * j], hi[4 * j + 1], hi[4 * j + 2], hi[4 * j + 3]});
                    st_sys(slot + 4096 + ln * 16, u32x4{mid[0], mid[1], mid[2], mid[3]});
                    *(lu32*)(scr + ln * 4) = mid[4];
                    *(lu32*)(scr + 256 + ln * 4) = mid[5];
                    *(lu32*)(scr + 512 + ln * 4) = mid[6];
                    const u32 nb = write_list(q, nz, inc - cnt, total, scr + 768, lds + 2 * STAGE + wave * 4096, ln);
                    for (u32 o = ln * 16; o < 768 + nb; o += 1024) st_sys(slot + 5120 + o, *(lu32x4*)(scr + o));
                    CBAR();
                    hdrs[mp] = ((ACC_MAIN + nb + 127) >> 7) | (nb << 8) | ((u32)(ex + dsc - 30) << 19);
                }
            }
            if (halft) publish(0, hdrs[0]); else publish(1, hdrs[1]);
            STAMP(4);
        }
    }
}

}

extern "C" void kernel_launch(int rank, int nranks, ncclComm_t comm,
                              hipStream_t stream,
                              void* const* d_in, const int* in_sizes, int n_in,
                              void* d_out, int out_size,
                              const int* in_dtypes, int out_dtype,
                              void* const* d_out_all,
                              void* const* d_ws_all, size_t ws_bytes) {
    if (nranks != NR || (unsigned)rank >= (unsigned)NR || n_in != 4 || ws_bytes < WS_NEED ||
        in_sizes[0] != MM * KS || in_sizes[1] != KS * NN || out_size != MS * NN) {
        fprintf(stderr, "kernel_launch: unexpected problem shape\n");
        return;
    }
    Params P;
    P.x = (const uint8_t*)d_in[0];
    P.w = (const uint8_t*)d_in[1];
    P.sx = (const float*)d_in[2];
    P.sw = (const float*)d_in[3];
    P.out = (float*)d_out;
    for (int p = 0; p < NR; ++p) P.ws[p] = (uint8_t*)d_ws_all[p];
    P.myws = P.ws[rank];
    P.rank = rank;
    P.pad = 0;
    hipLaunchKernelGGL(fused_kernel, dim3(GRID), dim3(NTHR), 0, stream, P);
    HIP_CHECK(hipGetLastError());
}
```
